```python
import numpy as np

kernel_name = "gpu_gemm_rs_m4096_k4096_n4096_e5m2e5m2_dqrelu"


ABSMAX_RTOL = 0.05

COMPUTE_DTYPE = "float32"
EPILOGUE = "dequant_relu"
IN_QUANT = None
OUT_QUANT = None
IN_FP8 = "e5m2e5m2"
IN_DTYPES = {"x": "e5m2", "w_mat": "e5m2",
             "scale_x": "float32", "scale_w": "float32"}
WORLD_SIZE = 8

def _snap_fp8(x, mant_bits, max_abs, min_normal):
    a = np.abs(x.astype(np.float64))
    _m, _e = np.frexp(a)
    q = np.where(a >= min_normal, np.ldexp(1.0, _e - 1 - mant_bits),
                 min_normal * 2.0 ** -mant_bits)
    snapped = np.minimum(np.round(a / q) * q, max_abs)
    return (np.sign(x) * snapped).astype(np.float32)

def setup_inputs():

    x = _snap_fp8(np.random.randn(4096, 4096).astype(np.float32), 2, 57344.0, 2.0 ** -14)
    w_mat = _snap_fp8(np.random.randn(4096, 4096).astype(np.float32), 2, 57344.0, 2.0 ** -14)
    scale_x = (0.02 * (1.0 + np.random.rand(1))).astype(np.float32)
    scale_w = (0.02 * (1.0 + np.random.rand(1))).astype(np.float32)
    return {"x": x, "w_mat": w_mat, "scale_x": scale_x, "scale_w": scale_w}

def reference(x, w_mat, scale_x, scale_w):
    acc = (x @ w_mat).astype(np.float32)
    y = (acc * (scale_x * scale_w)).astype(np.float32)
    return np.maximum(y, 0.0).astype(np.float32)
```

```cpp
#include <hip/hip_runtime.h>
#include <rccl/rccl.h>
#include <cstdio>
#include <cstdint>

#define HIP_CHECK(expr) do { hipError_t _e = (expr); if (_e != hipSuccess) { \
    fprintf(stderr, "HIP error %s at %s:%d\n", hipGetErrorString(_e), __FILE__, __LINE__); \
    return; } } while (0)

typedef unsigned u32;
typedef unsigned long long u64;
typedef u32 u32x4 __attribute__((ext_vector_type(4)));
typedef u32 u32x2 __attribute__((ext_vector_type(2)));
typedef int i32x8 __attribute__((ext_vector_type(8)));
typedef float f32x16 __attribute__((ext_vector_type(16)));
#define GAS __attribute__((address_space(1)))
#define LAS __attribute__((address_space(3)))

#define SENDER_WAVES 1
#define FORCE_DENSE 0

constexpr int NR = 8;
constexpr int GN = 4096;
constexpr int KS = 512;
constexpr int MS = 512;
constexpr int NJ = 64;
constexpr int NKC = 4;
constexpr int NUNIT = NKC * NJ;
constexpr int N_SEND_WG = NUNIT / SENDER_WAVES;
constexpr int N_CONS_WG = 256;
constexpr int GRID = N_SEND_WG + N_CONS_WG;

constexpr u32 WSLOT = 8192;
constexpr u32 OFF_W = 0;
constexpr u32 OFF_X = OFF_W + NR * NUNIT * WSLOT;
constexpr u32 OFF_WF = OFF_X + NR * MS * KS;
constexpr u32 OFF_XF = OFF_WF + NR * NUNIT * 128;
constexpr u32 PAR_BYTES = OFF_XF + NR * 32 * 128;
constexpr u32 OFF_CNT = 2 * PAR_BYTES;
constexpr u32 OFF_TMO = OFF_CNT + 512 * 4 * 4;
constexpr u32 OFF_DBG = OFF_TMO + 128;
constexpr u32 WS_NEED = OFF_DBG + 512 * 4 * 32 * 8;
constexpr u64 DEADLINE_TICKS = 100000000ull;

struct Params {
    const uint8_t* x;
    const uint8_t* w;
    const float* sx;
    const float* sw;
    float* out;
    char* self;
    char* ws[NR];
    int rank;
    int pad;
};

#define STAMP(i) do { } while (0)

__device__ __forceinline__ u32 bfi(u32 m, u32 a, u32 b) { return (m & a) | (~m & b); }

template <int OFF>
__device__ __forceinline__ void st16(const char* base, u32 voff, u32x4 v) {
    asm volatile("global_store_dwordx4 %0, %1, %2 offset:%3 sc0 sc1\n\ts_nop 1"
                 :: "v"(voff), "v"(v), "s"(base), "i"(OFF) : "memory");
}
#define WAIT_VM(n) asm volatile("s_waitcnt vmcnt(" #n ")" ::: "memory")

__device__ __forceinline__ void put_flag(char* p, u32 call, u32 meta, int lane) {
    if (lane == 0)
        __hip_atomic_store((GAS u64*)p, ((u64)call << 32) | meta, __ATOMIC_RELAXED, __HIP_MEMORY_SCOPE_SYSTEM);
}

__device__ __forceinline__ u32 wait_flags(const char* f, u32 call, GAS u32* tmo) {
    u64 t0 = __builtin_amdgcn_s_memrealtime();
    u64 v;
    for (;;) {
        v = __hip_atomic_load((GAS u64*)f, __ATOMIC_RELAXED, __HIP_MEMORY_SCOPE_SYSTEM);
        if (__all((u32)(v >> 32) >= call)) break;
        u32 t = __builtin_amdgcn_readfirstlane(__hip_atomic_load(tmo, __ATOMIC_RELAXED, __HIP_MEMORY_SCOPE_AGENT));
        if (t == call || __builtin_amdgcn_s_memrealtime() - t0 > DEADLINE_TICKS) {
            __hip_atomic_store(tmo, call, __ATOMIC_RELAXED, __HIP_MEMORY_SCOPE_AGENT);
            break;
        }
        __builtin_amdgcn_s_sleep(2);
    }
    asm volatile("" ::: "memory");
    return __builtin_amdgcn_readfirstlane((u32)v);
}

__device__ __forceinline__ u32 row16_sum(u32 v) {
    v += __builtin_amdgcn_update_dpp(0u, v, 0x111, 0xf, 0xf, true);
    v += __builtin_amdgcn_update_dpp(0u, v, 0x112, 0xf, 0xf, true);
    v += __builtin_amdgcn_update_dpp(0u, v, 0x114, 0xf, 0xf, true);
    v += __builtin_amdgcn_update_dpp(0u, v, 0x118, 0xf, 0xf, true);
    return v;
}

#define WIN 2
template <int N> __device__ __forceinline__ void wait_vm() { asm volatile("s_waitcnt vmcnt(%0)" :: "i"(N) : "memory"); }
template <class F>
__device__ __forceinline__ void send_main_win(const char* dst, u32 voff, const u32x4 (&pk)[6], F f) {
    constexpr int D = WIN > 0 ? WIN - 1 : 0;
    wait_vm<D>(); if (D == 0) f(); st16<-4096>(dst, voff, pk[0]);
    wait_vm<D>(); if (D == 1) f(); st16<-3072>(dst, voff, pk[1]);
    wait_vm<D>(); if (D == 2) f(); st16<-2048>(dst, voff, pk[2]);
    wait_vm<D>(); if (D == 3) f(); st16<-1024>(dst, voff, pk[3]);
    wait_vm<D>(); if (D == 4) f(); st16<0>(dst, voff, pk[4]);
    wait_vm<D>(); if (D == 5) f(); st16<1024>(dst, voff, pk[5]);
}
__device__ __forceinline__ void send_main(const char* dst, u32 voff, const u32x4 (&pk)[6]) {
    st16<-4096>(dst, voff, pk[0]); st16<-3072>(dst, voff, pk[1]); st16<-2048>(dst, voff, pk[2]);
    st16<-1024>(dst, voff, pk[3]); st16<0>(dst, voff, pk[4]); st16<1024>(dst, voff, pk[5]);
}
__device__ __forceinline__ void send_extra(const char* dst, u32 voff, u32x4 e0, u32x4 e1, bool dense, u32 n16, int lane) {
    if (dense) { st16<2048>(dst, voff, e0); st16<3072>(dst, voff, e1); }
    else if ((u32)lane < n16) st16<2048>(dst, voff, e0);
}

__device__ __forceinline__ void sender(const Params& P, int u, int wave, int lane, GAS u32* cp, LAS char* scratch) {
    const int rank = P.rank;
    char* const self = P.self;
    u32 cv = __hip_atomic_load(cp, __ATOMIC_RELAXED, __HIP_MEMORY_SCOPE_AGENT);
    const int kc = u >> 6, j = u & 63;
    const int xdest = (u + (u >> 3)) & 7, xu = u >> 3, rb = xu >> 1, hf = xu & 1;
    const u32 voff = lane * 16 + 4096;

    u32x4 xv[8];
    {
        const uint8_t* xp = P.x + (size_t)(xdest * MS + rb * 32 + (lane & 31)) * KS + hf * 256 + (lane >> 5) * 64;
#pragma unroll
        for (int q = 0; q < 8; ++q) xv[q] = *(const GAS u32x4*)(xp + (q >> 2) * 128 + (q & 3) * 16);
    }
    u32x2 wr[16];
    {
        const uint8_t* wp = P.w + (size_t)(kc * 128 + (lane >> 3) * 16) * GN + j * 64 + (lane & 7) * 8;
#pragma unroll
        for (int i = 0; i < 16; ++i) wr[i] = *(const GAS u32x2*)(wp + (size_t)i * GN);
    }
    *(LAS u32x4*)(scratch + lane * 16) = u32x4{0, 0, 0, 0};
#pragma unroll
    for (int i = 0; i < 16; ++i) asm volatile("" : "+v"(wr[i]));
#pragma unroll
    for (int q = 0; q < 8; ++q) asm volatile("" : "+v"(xv[q]));
    const u32 call = __builtin_amdgcn_readfirstlane(cv) + 1;
    const u32 par = (call & 1) * PAR_BYTES;
    {
        const char* xd = P.ws[xdest] + par + OFF_X + (rank * 16 + rb) * 16384 + hf * 8192;
        st16<-4096>(xd, voff, xv[0]); st16<-3072>(xd, voff, xv[1]); st16<-2048>(xd, voff, xv[2]); st16<-1024>(xd, voff, xv[3]);
        st16<0>(xd, voff, xv[4]); st16<1024>(xd, voff, xv[5]); st16<2048>(xd, voff, xv[6]); st16<3072>(xd, voff, xv[7]);
    }
    STAMP(1);

    u32 base;
    {
        u32 s = __builtin_amdgcn_sad_u8(wr[0].x & 0x7F7F7F7Fu, 0u, 0u);
        s = __builtin_amdgcn_sad_u8(wr[0].y & 0x7F7F7F7Fu, 0u, s);
        s = __builtin_amdgcn_sad_u8(wr[1].x & 0x7F7F7F7Fu, 0u, s);
        s = __builtin_amdgcn_sad_u8(wr[1].y & 0x7F7F7F7Fu, 0u, s);
        s = row16_sum(s);
        int tot = __builtin_amdgcn_readlane(s, 15) + __builtin_amdgcn_readlane(s, 31) +
                  __builtin_amdgcn_readlane(s, 47) + __builtin_amdgcn_readlane(s, 63);
        int b = (tot - 18596) >> 12;
        base = b < 0 ? 0 : (b > 24 ? 24 : b);
    }
    const u32 b4 = base * 0x04040404u;

    u32 d[8][4];
#pragma unroll
    for (int hw = 0; hw < 2; ++hw)
#pragma unroll
        for (int dw = 0; dw < 4; ++dw) {
            u32 r0 = wr[4 * dw][hw], r1 = wr[4 * dw + 1][hw], r2 = wr[4 * dw + 2][hw], r3 = wr[4 * dw + 3][hw];
            u32 t0 = __builtin_amdgcn_perm(r1, r0, 0x05010400u), t1 = __builtin_amdgcn_perm(r1, r0, 0x07030602u);
            u32 t2 = __builtin_amdgcn_perm(r3, r2, 0x05010400u), t3 = __builtin_amdgcn_perm(r3, r2, 0x07030602u);
            d[4 * hw + 0][dw] = __builtin_amdgcn_perm(t2, t0, 0x05040100u);
            d[4 * hw + 1][dw] = __builtin_amdgcn_perm(t2, t0, 0x07060302u);
            d[4 * hw + 2][dw] = __builtin_amdgcn_perm(t3, t1, 0x05040100u);
            d[4 * hw + 3][dw] = __builtin_amdgcn_perm(t3, t1, 0x07060302u);
        }
    u32 pkf[24];
    u32 t[8][4];
#pragma unroll
    for (int c = 0; c < 8; ++c) {
#pragma unroll
        for (int dw = 0; dw < 4; ++dw) t[c][dw] = (d[c][dw] | 0x80808080u) - b4;
        u32 ca = bfi(0x80808080u, d[c][0], t[c][0]);
        u32 cb = bfi(0x80808080u, d[c][1], t[c][1]);
        u32 cc = bfi(0x80808080u, d[c][2], t[c][2]);
        pkf[3 * c + 0] = bfi(0x60606060u, t[c][3] << 5, ca);
        pkf[3 * c + 1] = bfi(0x60606060u, t[c][3] << 3, cb);
        pkf[3 * c + 2] = bfi(0x60606060u, bfi(0x40404040u, d[c][3] >> 1, t[c][3] << 1), cc);
    }
    u32x4 pk[6];
#pragma unroll
    for (int i = 0; i < 6; ++i) pk[i] = u32x4{pkf[4 * i], pkf[4 * i + 1], pkf[4 * i + 2], pkf[4 * i + 3]};

    const u32 woff = par + OFF_W + (rank * NUNIT + u) * WSLOT;
    const u32 foff = par + OFF_WF + (rank * NUNIT + u) * 128;
    int pr = j % 7;
    char* cur = P.ws[(rank + 1 + pr) & 7];
    STAMP(2);

    u32 X[8];
#pragma unroll
    for (int c = 0; c < 8; ++c) {
        u32 xa = d[c][0] ^ ((t[c][0] & 0x1F1F1F1Fu) + b4);
        u32 xb = d[c][1] ^ ((t[c][1] & 0x1F1F1F1Fu) + b4);
        u32 xc = d[c][2] ^ ((t[c][2] & 0x1F1F1F1Fu) + b4);
        u32 xd = d[c][3] ^ ((t[c][3] & 0x1F1F1F1Fu) + b4);
        X[c] = bfi(0x03030303u, xa >> 5, bfi(0x0C0C0C0Cu, xb >> 3, bfi(0x30303030u, xc >> 1, xd << 1)));
    }
    u32 cnt = 0;
#pragma unroll
    for (int c = 0; c < 8; ++c) {
        u32 y = X[c];
        const u32 col = (lane & 7) * 8 + c;
        const u32 piece = (lane >> 3) * 64 + (col ^ (col >> 3));
        for (;;) {
            u64 m = __ballot(y != 0);
            if (m == 0 || cnt > 512) break;
            if (y != 0) {
                u32 p = __builtin_ctz(y) & ~3u;
                u32 e = (piece << 7) | (p << 2) | ((y >> p) & 15u);
                y &= ~(15u << p);
                u32 slot = cnt + __builtin_amdgcn_mbcnt_hi((u32)(m >> 32), __builtin_amdgcn_mbcnt_lo((u32)m, 0u));
                if (slot < 512) *(LAS unsigned short*)(scratch + slot * 2) = (unsigned short)e;
            }
            cnt += __builtin_popcountll(m);
        }
    }
    const bool dense = FORCE_DENSE || cnt > 512;
    const u32 n16 = dense ? 0 : (cnt + 7) >> 3;
    u32x4 e0, e1 = u32x4{X[4], X[5], X[6], X[7]};
    if (dense) e0 = u32x4{X[0], X[1], X[2], X[3]};
    else e0 = *(LAS u32x4*)(scratch + lane * 16);
    const u32 meta = base | (dense ? 32u : 0u) | (n16 << 6);

    send_extra(self + woff, voff, e0, e1, dense, n16, lane);
    send_main(self + woff, voff, pk);
    send_extra(cur + woff, voff, e0, e1, dense, n16, lane);
    send_main_win(cur + woff, voff, pk, [&]() {
        put_flag(P.ws[xdest] + par + OFF_XF + (rank * 32 + xu) * 128, call, 0, lane);
        put_flag(self + foff, call, meta, lane);
    });
    STAMP(3);
#pragma unroll
    for (int s = 1; s < 7; ++s) {
        char* prev = cur;
        pr = pr == 6 ? 0 : pr + 1;
        cur = P.ws[(rank + 1 + pr) & 7];
        send_extra(cur + woff, voff, e0, e1, dense, n16, lane);
        send_main_win(cur + woff, voff, pk, [&]() { put_flag(prev + foff, call, meta, lane); });
        STAMP(3 + s);
    }
    WAIT_VM(0);
    put_flag(cur + foff, call, meta, lane);
    if (lane == 0) __hip_atomic_store(cp, call, __ATOMIC_RELAXED, __HIP_MEMORY_SCOPE_AGENT);
    STAMP(10);
}

__device__ __forceinline__ void consumer(const Params& P, int b, int wave, int lane, GAS u32* cp, LAS char* lds) {
    const int rank = P.rank;
    char* const ws = P.self;
    const u32 call = __builtin_amdgcn_readfirstlane(__hip_atomic_load(cp, __ATOMIC_RELAXED, __HIP_MEMORY_SCOPE_AGENT)) + 1;
    const u32 par = (call & 1) * PAR_BYTES;
    const int j = (b & 7) * 8 + ((b >> 3) & 7), mi = b >> 6;
    const int rb = mi * 4 + wave;
    GAS u32* tmo = (GAS u32*)(ws + OFF_TMO);
    const __amdgpu_buffer_rsrc_t rs = __builtin_amdgcn_make_buffer_rsrc(ws, 0, WS_NEED, 0x00020000);
    const float scale = *P.sx * *P.sw;

    f32x16 acc0 = {}, acc1 = {};
    wait_flags(ws + par + OFF_XF + (((lane >> 1) & 7) * 32 + rb * 2 + (lane & 1)) * 128, call, tmo);
    STAMP(1);

    const u32 wpos = (lane >> 3) * 1024 + (lane & 7) * 128;
    const u32 n = lane & 31, h = lane >> 5;
    const u32 rpos0 = h * 4096 + ((n ^ (n >> 3)) * 16);
    const u32 rpos1 = h * 4096 + (((n + 32) ^ ((n + 32) >> 3)) * 16);
    int pr = j % 7;
    u32 tv = 0;
    for (int s = 0; s < 8; ++s) {
        int src = rank;
        if (s > 0) { src = (rank + 7 - pr) & 7; pr = pr == 6 ? 0 : pr + 1; }
        u32x4 a[16];
        {
            const u32 xo = par + OFF_X + (src * 16 + rb) * 16384;
#pragma unroll
            for (int q = 0; q < 16; ++q)
                a[q] = __builtin_amdgcn_raw_buffer_load_b128(rs, lane * 16 + (q & 3) * 1024, xo + (q >> 2) * 4096, 17);
        }
        const u32 tile = src * NUNIT + wave * 64 + j;
        const u32 meta = wait_flags(ws + par + OFF_WF + tile * 128, call, tmo);
        STAMP(2 + 3 * s);
        const u32 wo = par + OFF_W + tile * WSLOT;
        const bool dense = (meta & 32u) != 0;
        const u32 ndw = ((meta >> 6) & 127u) * 4;
        const u32 b4 = (meta & 31u) * 0x04040404u;
        u32x4 x0 = {0, 0, 0, 0}, x1 = {0, 0, 0, 0};
        u32 ex = 0;
        if ((u32)lane < ndw) ex = __builtin_amdgcn_raw_buffer_load_b32(rs, lane * 4 + 2048, wo + 4096, 17);
        if (dense) {
            x0 = __builtin_amdgcn_raw_buffer_load_b128(rs, lane * 16 + 2048, wo + 4096, 17);
            x1 = __builtin_amdgcn_raw_buffer_load_b128(rs, lane * 16 + 3072, wo + 4096, 17);
        }
        u32x4 pk[6];
#pragma unroll
        for (int i = 0; i < 6; ++i)
            pk[i] = __builtin_amdgcn_raw_buffer_load_b128(rs, lane * 16 + (i & 3) * 1024, wo + (i >> 2) * 4096, 17);
        LAS char* buf = lds + (s & 1) * 32768;
        LAS char* tb = buf + wave * 8192;
#pragma unroll
        for (int c = 0; c < 8; ++c) {
            u32 A = pk[(3 * c) >> 2][(3 * c) & 3], B = pk[(3 * c + 1) >> 2][(3 * c + 1) & 3], C = pk[(3 * c + 2) >> 2][(3 * c + 2) & 3];
            u32 D = bfi(0x80808080u, C << 1, bfi(0x0F0F0F0Fu, bfi(0x03030303u, A >> 5, B >> 3), C >> 1));
            u32x4 o = {(A & 0x9F9F9F9Fu) + b4, (B & 0x9F9F9F9Fu) + b4, (C & 0x9F9F9F9Fu) + b4, (D & 0x9F9F9F9Fu) + b4};
            if (dense) {
                u32 X = c < 4 ? x0[c & 3] : x1[c & 3];
                o[0] ^= (X << 5) & 0x60606060u; o[1] ^= (X << 3) & 0x60606060u;
                o[2] ^= (X << 1) & 0x60606060u; o[3] ^= (X >> 1) & 0x60606060u;
            }
            *(LAS u32x4*)(tb + wpos + ((c ^ (lane & 7)) * 16)) = o;
        }
        for (u32 i0 = 0; i0 < ndw; i0 += 64) {
            if (i0 > 0) {
                ex = 0;
                if (i0 + lane < ndw) ex = __builtin_amdgcn_raw_buffer_load_b32(rs, (i0 + lane) * 4 + 2048, wo + 4096, 17);
            }
#pragma unroll
            for (int k = 0; k < 2; ++k) {
                u32 e = k ? ex >> 16 : ex & 0xFFFFu;
                if (e & 15u) {
                    u32 sh = ((e >> 2) & 24u) + 5;
                    u64 v = (u64)((e & 3u) << sh) | ((u64)(((e >> 2) & 3u) << sh) << 32);
                    __hip_atomic_fetch_xor((LAS u64*)(tb + (e >> 7) * 16 + ((e >> 1) & 8u)), v,
                                           __ATOMIC_RELAXED, __HIP_MEMORY_SCOPE_WORKGROUP);
                }
            }
        }
        STAMP(3 + 3 * s);
        __syncthreads();
        tv = __hip_atomic_load(tmo, __ATOMIC_RELAXED, __HIP_MEMORY_SCOPE_AGENT);
#pragma unroll
        for (int kc = 0; kc < 4; ++kc)
#pragma unroll
            for (int s2 = 0; s2 < 2; ++s2) {
                u32x4 a0 = a[kc * 4 + s2 * 2], a1 = a[kc * 4 + s2 * 2 + 1];
                i32x8 af = {(int)a0[0], (int)a0[1], (int)a0[2], (int)a0[3], (int)a1[0], (int)a1[1], (int)a1[2], (int)a1[3]};
                LAS char* p = buf + kc * 8192 + s2 * 2048;
                u32x4 b0 = *(LAS u32x4*)(p + rpos0), b1 = *(LAS u32x4*)(p + rpos0 + 1024);
                u32x4 b2 = *(LAS u32x4*)(p + rpos1), b3 = *(LAS u32x4*)(p + rpos1 + 1024);
                i32x8 bf0 = {(int)b0[0], (int)b0[1], (int)b0[2], (int)b0[3], (int)b1[0], (int)b1[1], (int)b1[2], (int)b1[3]};
                i32x8 bf1 = {(int)b2[0], (int)b2[1], (int)b2[2], (int)b2[3], (int)b3[0], (int)b3[1], (int)b3[2], (int)b3[3]};
                acc0 = __builtin_amdgcn_mfma_scale_f32_32x32x64_f8f6f4(af, bf0, acc0, 1, 1, 0, 0x7F7F7F7F, 0, 0x7F7F7F7F);
                acc1 = __builtin_amdgcn_mfma_scale_f32_32x32x64_f8f6f4(af, bf1, acc1, 1, 1, 0, 0x7F7F7F7F, 0, 0x7F7F7F7F);
            }
        STAMP(4 + 3 * s);
    }
    const bool bad = __builtin_amdgcn_readfirstlane(tv) == call;
    GAS float* o = (GAS float*)P.out + (size_t)(rb * 32 + h * 4) * GN + j * 64 + n;
#pragma unroll
    for (int r = 0; r < 16; ++r) {
        const int row = (r & 3) + 8 * (r >> 2);
        float v0 = acc0[r] * scale, v1 = acc1[r] * scale;
        v0 = v0 < 0.f ? 0.f : v0;
        v1 = v1 < 0.f ? 0.f : v1;
        if (bad) { v0 = __builtin_nanf(""); v1 = v0; }
        o[(size_t)row * GN] = v0;
        o[(size_t)row * GN + 32] = v1;
    }
    if (lane == 0) __hip_atomic_store(cp, call, __ATOMIC_RELAXED, __HIP_MEMORY_SCOPE_AGENT);
    STAMP(26);
}

__global__ void __launch_bounds__(256, 2) gemm_rs_kernel(Params P) {
    __shared__ __attribute__((aligned(16))) char smem[65536];
    const int lane = threadIdx.x & 63;
    const int wave = __builtin_amdgcn_readfirstlane(threadIdx.x >> 6);
    const bool is_sender = blockIdx.x < N_SEND_WG;
    if (is_sender && wave >= SENDER_WAVES) return;
    GAS u32* cp = (GAS u32*)(P.self + OFF_CNT) + blockIdx.x * 4 + wave;
    STAMP(0);
    if (is_sender) sender(P, blockIdx.x * SENDER_WAVES + wave, wave, lane, cp, (LAS char*)smem + wave * 1024);
    else consumer(P, blockIdx.x - N_SEND_WG, wave, lane, cp, (LAS char*)smem);
}

extern "C" void kernel_launch(int rank, int nranks, ncclComm_t comm,
                              hipStream_t stream,
                              void* const* d_in, const int* in_sizes, int n_in,
                              void* d_out, int out_size,
                              const int* in_dtypes, int out_dtype,
                              void* const* d_out_all,
                              void* const* d_ws_all, size_t ws_bytes) {
    if (nranks != NR || n_in != 4 || ws_bytes < WS_NEED || (unsigned)rank >= (unsigned)NR ||
        in_sizes[0] != 4096 * KS || in_sizes[1] != KS * GN || out_size != MS * GN) {
        fprintf(stderr, "kernel_launch: unexpected problem shape\n");
        return;
    }
    Params P;
    P.x = (const uint8_t*)d_in[0];
    P.w = (const uint8_t*)d_in[1];
    P.sx = (const float*)d_in[2];
    P.sw = (const float*)d_in[3];
    P.out = (float*)d_out;
    for (int p = 0; p < NR; ++p) P.ws[p] = (char*)d_ws_all[p];
    P.self = P.ws[rank];
    P.rank = rank;
    P.pad = 0;
    gemm_rs_kernel<<<GRID, 256, 0, stream>>>(P);
    HIP_CHECK(hipGetLastError());
}
```

```cpp
#include <hip/hip_runtime.h>
#include <rccl/rccl.h>
#include <cstdio>
#include <cstdint>

#define HIP_CHECK(expr) do { hipError_t _e = (expr); if (_e != hipSuccess) { \
    fprintf(stderr, "HIP error %s at %s:%d\n", hipGetErrorString(_e), __FILE__, __LINE__); \
    return; } } while (0)

typedef unsigned u32;
typedef unsigned long long u64;
typedef u32 u32x4 __attribute__((ext_vector_type(4)));
typedef u32 u32x2 __attribute__((ext_vector_type(2)));
typedef int i32x8 __attribute__((ext_vector_type(8)));
typedef float f32x4 __attribute__((ext_vector_type(4)));
#define GAS __attribute__((address_space(1)))
#define LAS __attribute__((address_space(3)))

#define PACE_T0 370
#define PACE_TURN 188
#define PACE_STAG 90
#define POLL_SLEEP 1
#define WIN_LO 2
#define WIN_HI 6
#define OUT_AUX 17
#define FORCE_RAW 0

constexpr int NR = 8;
constexpr int GN = 4096;
constexpr int KS = 512;
constexpr int MS = 512;
constexpr int NUNIT = 256;
constexpr int N_SEND_WG = 128;
constexpr int N_CONS_WG = 128;
constexpr int GRID = N_SEND_WG + N_CONS_WG;

constexpr u32 USLOT = 9216;
constexpr u32 SRC_W = 0;
constexpr u32 SRC_X = SRC_W + NUNIT * USLOT;
constexpr u32 SRC_WF = SRC_X + 32 * USLOT;
constexpr u32 SRC_XF = SRC_WF + NUNIT * 128;
constexpr u32 SRC_BYTES = SRC_XF + 32 * 128;
constexpr u32 PAR_BYTES = NR * SRC_BYTES;
constexpr u32 OFF_XD = 2 * PAR_BYTES;
constexpr u32 OFF_LF = OFF_XD + NR * MS * KS;
constexpr u32 OFF_CNT = OFF_LF + NUNIT * 128;
constexpr u32 OFF_TMO = OFF_CNT + GRID * 16 * 4;
constexpr u32 OFF_DBG = OFF_TMO + 128;
constexpr u32 WS_NEED = OFF_DBG + GRID * 16 * 64 * 8;
constexpr u64 DEADLINE_TICKS = 100000000ull;
constexpr u32 OOB = 0x80000000u;
constexpr u32 RAW16 = 64;
constexpr u32 CAP_DW = 192;
constexpr int NBUF = 8;
constexpr u32 L_SYM = NBUF * 8192;
constexpr u32 L_ENC = L_SYM + 8 * 2304;
constexpr u32 L_XIM = L_ENC + 2 * USLOT + 1024;
constexpr u32 L_CNT = L_XIM + 8 * 2048 + 8192;
constexpr int LDS_BYTES = L_CNT + 512;

struct Params {
    const uint8_t* x;
    const uint8_t* w;
    const float* sx;
    const float* sw;
    float* out;
    char* self;
    char* ws[NR];
    int rank;
    int pad;
};

#define STAMP(i) do { } while (0)

template <int OFF>
__device__ __forceinline__ void st16(const char* base, u32 voff, u32x4 v) {
    asm volatile("global_store_dwordx4 %0, %1, %2 offset:%3 sc0 sc1\n\ts_nop 1"
                 :: "v"(voff), "v"(v), "s"(base), "i"(OFF) : "memory");
}
template <int N> __device__ __forceinline__ void wait_vm() { asm volatile("s_waitcnt vmcnt(%0)" :: "i"(N) : "memory"); }

__device__ __forceinline__ void put_flag(char* p, u32 call, u32 meta, int lane) {
    if (lane == 0)
        __hip_atomic_store((GAS u64*)p, ((u64)call << 32) | meta, __ATOMIC_RELAXED, __HIP_MEMORY_SCOPE_SYSTEM);
}

__device__ __forceinline__ u64 wait_flags(const char* f, u64 need, u32 call, GAS u32* tmo) {
    const u64 t0 = __builtin_amdgcn_s_memrealtime();
    u64 v;
    for (;;) {
        v = __hip_atomic_load((GAS u64*)f, __ATOMIC_RELAXED, __HIP_MEMORY_SCOPE_SYSTEM);
        const u32 t = __hip_atomic_load(tmo, __ATOMIC_RELAXED, __HIP_MEMORY_SCOPE_AGENT);
        if ((__ballot((u32)(v >> 32) >= call) & need) == need) break;
        if (__builtin_amdgcn_readfirstlane(t) == call || __builtin_amdgcn_s_memrealtime() - t0 > DEADLINE_TICKS) {
            __hip_atomic_store(tmo, call, __ATOMIC_RELAXED, __HIP_MEMORY_SCOPE_AGENT);
            break;
        }
        __builtin_amdgcn_s_sleep(1);
    }
    asm volatile("" ::: "memory");
    return v;
}

__device__ __forceinline__ u32 wait_flag(const char* f, u32 call, GAS u32* tmo) {
    const u64 t0 = __builtin_amdgcn_s_memrealtime();
    u64 v;
    for (u32 n = 1;; ++n) {
        v = __hip_atomic_load((GAS u64*)f, __ATOMIC_RELAXED, __HIP_MEMORY_SCOPE_SYSTEM);
        if (__builtin_amdgcn_readfirstlane((u32)(v >> 32)) >= call) break;
        if ((n & 15) == 0) {
            const u32 t = __hip_atomic_load(tmo, __ATOMIC_RELAXED, __HIP_MEMORY_SCOPE_AGENT);
            if (__builtin_amdgcn_readfirstlane(t) == call || __builtin_amdgcn_s_memrealtime() - t0 > DEADLINE_TICKS) {
                __hip_atomic_store(tmo, call, __ATOMIC_RELAXED, __HIP_MEMORY_SCOPE_AGENT);
                break;
            }
        }
        __builtin_amdgcn_s_sleep(POLL_SLEEP);
    }
    asm volatile("" ::: "memory");
    return __builtin_amdgcn_readfirstlane((u32)v);
}

__device__ __forceinline__ u32 wave_scan(u32 v) {
    v += __builtin_amdgcn_update_dpp(0u, v, 0x111, 0xf, 0xf, true);
    v += __builtin_amdgcn_update_dpp(0u, v, 0x112, 0xf, 0xf, true);
    v += __builtin_amdgcn_update_dpp(0u, v, 0x114, 0xf, 0xf, true);
    v += __builtin_amdgcn_update_dpp(0u, v, 0x118, 0xf, 0xf, true);
    v += __builtin_amdgcn_update_dpp(0u, v, 0x142, 0xa, 0xf, false);
    v += __builtin_amdgcn_update_dpp(0u, v, 0x143, 0xc, 0xf, false);
    return v;
}
__device__ __forceinline__ u32 bfi(u32 m, u32 a, u32 b) { return (m & a) | (~m & b); }
__device__ __forceinline__ void lds_order() { __builtin_amdgcn_fence(__ATOMIC_ACQ_REL, "wavefront"); }

template <class Early>
__device__ __forceinline__ u32 encode_quarter(const u32 (&d)[8], const u32 (&img)[4], LAS char* unit, int q,
                                              LAS char* lut, int lane, Early codes_out) {
    u32 mag[8], sum = 0;
#pragma unroll
    for (int k = 0; k < 8; ++k) { mag[k] = d[k] & 0x7F7F7F7Fu; sum = __builtin_amdgcn_sad_u8(mag[k], 0u, sum); }
    int b = ((int)__builtin_amdgcn_readlane(wave_scan(sum), 63) + 1024 - 14541) >> 11;
    b = b < 26 ? 26 : b;
    asm volatile("" : "+s"(b));
    const u32 B = b > 108 ? 108 : b;
    const u32 c2 = (126 - B) * 0x01010101u, c16 = (112 - B) * 0x01010101u, c26 = (154 - B) * 0x01010101u, c20 = (108 - B) * 0x01010101u;
    u32 nib[8], w[8], esc[8];
#pragma unroll
    for (int k = 0; k < 8; ++k) {
        const u32 u = mag[k] + c2;
        w[k] = mag[k] + c16;
        const u32 dir = u & ~w[k] & 0x80808080u;
        nib[k] = w[k] & (((dir >> 3) - (dir >> 6)) | 0x01010101u);
        esc[k] = dir ^ 0x80808080u;
    }
    codes_out(u32x4{nib[0] | (nib[1] << 4), nib[2] | (nib[3] << 4), nib[4] | (nib[5] << 4), nib[6] | (nib[7] << 4)}, false);
    u32 sel[8], t[8], off[8], cnt = 0, sg = 0, mb = 0;
#pragma unroll
    for (int k = 0; k < 8; ++k) sel[k] = *(LAS u32*)(lut + (__builtin_amdgcn_udot4(esc[k], 0x08040201u, 0u, false) >> 5));
#pragma unroll
    for (int k = 0; k < 8; ++k) {
        const u32 hi = w[k] & 0x80808080u, q0 = mag[k] + c26;
        t[k] = ((q0 - ((hi >> 3) - (hi >> 6))) >> 1) & 0x0F0F0F0Fu;
        mb |= ((~q0 | (mag[k] + c20)) & 0x80808080u) >> (7 - k);
        off[k] = cnt;
        cnt += __builtin_popcount(esc[k]);
        sg |= (d[k] >> (7 - k)) & (0x01010101u << k);
    }
    const u32 incl = wave_scan(cnt);
    const u32 nsd = (__builtin_amdgcn_readlane(incl, 63) + 7) >> 3;
    LAS char* blk = unit + 5120 + q * 1024;
    *(LAS u32*)(unit + 4096 + q * 256 + lane * 4) = sg;
    u32 ndw = 1 + nsd;
    if (ndw <= CAP_DW && !FORCE_RAW) {
        const u32 base = incl - cnt + 8;
#pragma unroll
        for (int i = 0; i < 4; ++i) {
            u32 c0 = __builtin_amdgcn_perm(0u, t[2 * i], sel[2 * i]), c1 = __builtin_amdgcn_perm(0u, t[2 * i + 1], sel[2 * i + 1]);
            c0 = __builtin_amdgcn_perm(0u, c0 | (c0 >> 4), 0x0C0C0200u);
            c1 = __builtin_amdgcn_perm(0u, c1 | (c1 >> 4), 0x0C0C0200u);
            const u32 pos = base + off[2 * i];
            const u64 v = (u64)(c0 | (c1 << (4 * (off[2 * i + 1] - off[2 * i])))) << (4 * (pos & 7));
            LAS u32* p = (LAS u32*)(blk + ((pos >> 1) & ~3u));
            __hip_atomic_fetch_or(p, (u32)v, __ATOMIC_RELAXED, __HIP_MEMORY_SCOPE_WORKGROUP);
            if ((u32)(v >> 32)) __hip_atomic_fetch_or(p + 1, (u32)(v >> 32), __ATOMIC_RELAXED, __HIP_MEMORY_SCOPE_WORKGROUP);
        }
        if (lane == 0) *(LAS u32*)blk = B | (nsd << 8);
        lds_order();
        for (u64 mk; (mk = __ballot(mb != 0)) != 0; ndw += __builtin_popcountll(mk)) {
            if (mb != 0) {
                const u32 bit = __builtin_ctz(mb);
                mb &= mb - 1;
                const u32 k0 = (u32)((int)(bit << 31) >> 31), k1 = (u32)((int)(bit << 30) >> 31), k2 = (u32)((int)(bit << 29) >> 31);
                const u32 m0 = bfi(k2, mag[4], mag[0]), m1 = bfi(k2, mag[5], mag[1]), m2 = bfi(k2, mag[6], mag[2]), m3 = bfi(k2, mag[7], mag[3]);
                const u32 mg = (bfi(k0, bfi(k1, m3, m1), bfi(k1, m2, m0)) >> (bit & 24u)) & 127u;
                const u32 ad = bfi(k2, bfi(k1, img[3], img[2]), bfi(k1, img[1], img[0])) + (k0 & 4u) + (bit >> 3);
                const u32 z = (mg + 154 - B - (mg >= B + 16 ? 14 : 0)) & 31u;
                const u32 dflt = B - 26 + z + (z >= 28 ? 14 : 0);
                const u32 slot = ndw + __builtin_amdgcn_mbcnt_hi((u32)(mk >> 32), __builtin_amdgcn_mbcnt_lo((u32)mk, 0u));
                *(LAS u32*)(blk + (slot < 255 ? slot : 255) * 4) = (ad << 8) | (mg ^ dflt);
            }
        }
    }
    if (ndw > CAP_DW || FORCE_RAW) {
        lds_order();
        codes_out(u32x4{d[0], d[1], d[2], d[3]}, true);
        *(LAS u32x4*)(blk + lane * 16) = u32x4{d[4], d[5], d[6], d[7]};
        return RAW16;
    }
    return (ndw + 3) >> 2;
}

struct Packed {
    u32x4 n;
    u32 s, b0;
    u32 len, bo;
};
__device__ __forceinline__ u32 end_of(u32 meta, int q) {
    const u32 sh = q <= 0 ? 0 : 8 * q - 1, mask = q < 0 ? 0 : (q == 0 ? 127u : (q == 3 ? 511u : 255u));
    return (meta >> sh) & mask;
}
__device__ __forceinline__ void load_quarter(Packed& p, __amdgpu_buffer_rsrc_t rs, u32 uo, u32 meta, int q, int lane, u32 off = 0) {
    const u32 beg = end_of(meta, q - 1);
    p.len = end_of(meta, q) - beg;
    p.bo = uo + 5120 + 16 * beg;
    const u32 ndw = p.len * 4, l4 = lane * 4 | off;
    p.b0 = __builtin_amdgcn_raw_buffer_load_b32(rs, (u32)lane < ndw ? l4 : OOB, p.bo, 17);
    p.n = __builtin_amdgcn_raw_buffer_load_b128(rs, lane * 16 | off, uo + q * 1024, 17);
    p.s = __builtin_amdgcn_raw_buffer_load_b32(rs, l4, uo + 4096 + q * 256, 17);
}
__device__ __forceinline__ u32 sym_bytes(u32 t, u32 k1, u32 kx) {
    const u32 h = (t + 0x02020202u) & 0x10101010u;
    return (((t << 1) + k1) + (h - (h >> 3))) ^ kx;
}
__device__ __forceinline__ u32x2 sym8(u32 x, u32 k1, u32 kx) {
    const u32 lo = sym_bytes(x & 0x0F0F0F0Fu, k1, kx), hi = sym_bytes((x >> 4) & 0x0F0F0F0Fu, k1, kx);
    return u32x2{__builtin_amdgcn_perm(hi, lo, 0x05010400u), __builtin_amdgcn_perm(hi, lo, 0x07030602u)};
}
__device__ __forceinline__ void patch(u32 e, bool on, LAS char* image) {
    if (on && (e & 127u))
        __hip_atomic_fetch_xor((LAS u32*)(image + ((e >> 8) & 0x1FFCu)), (e & 127u) << ((e >> 5) & 24u),
                               __ATOMIC_RELAXED, __HIP_MEMORY_SCOPE_WORKGROUP);
}
__device__ __forceinline__ void decode_quarter(const Packed& p, __amdgpu_buffer_rsrc_t rs, const u32 (&img)[4],
                                               LAS char* image, LAS char* sym, int lane) {
    const u32 len = p.len;
    if (len > 16 && len < RAW16) {
        u32 h0 = p.b0;
        asm volatile("" : "+v"(h0));
        const u32 B = __builtin_amdgcn_readfirstlane(h0) & 127u;
        const u32 k1 = (B - 26) * 0x01010101u, kx = (0xFFu ^ B) * 0x01010101u;
        const u32 b1 = __builtin_amdgcn_raw_buffer_load_b32(rs, (u32)lane + 64 < len * 4 ? lane * 4 + 256 : OOB, p.bo, 17);
        const u32 b2 = __builtin_amdgcn_raw_buffer_load_b32(rs, (u32)lane + 128 < len * 4 ? lane * 4 + 512 : OOB, p.bo, 17);
        *(LAS u32x2*)(sym + 512 + lane * 8) = sym8(b1, k1, kx);
        *(LAS u32x2*)(sym + 1024 + lane * 8) = sym8(b2, k1, kx);
    }
    u32 nib[8], sel[8], off[8], cnt = 0;
#pragma unroll
    for (int k = 0; k < 8; ++k) {
        nib[k] = (k & 1 ? p.n[k >> 1] >> 4 : p.n[k >> 1]) & 0x0F0F0F0Fu;
        const u32 n10 = (nib[k] + 0x0E0E0E0Eu) & 0x10101010u, e10 = n10 ^ 0x10101010u;
        sel[k] = __umul24(e10, 0x101010u) + n10;
        off[k] = cnt;
        cnt += __builtin_popcount(e10);
    }
    const u32 hdr = __builtin_amdgcn_readfirstlane(p.b0);
    const u32 B = hdr & 127u, nsd = (hdr >> 8) & 255u;
    const u32 k1 = (B - 26) * 0x01010101u, kx = (0xFFu ^ B) * 0x01010101u;
    const u32 sp = (u32)(size_t)sym + 8 + (wave_scan(cnt) - cnt);
    *(LAS u32x2*)(sym + lane * 8) = sym8(p.b0, k1, kx);
    lds_order();
    u32 g0[8], g1[8], o[8];
#pragma unroll
    for (int k = 0; k < 8; ++k) {
        off[k] += sp;
        LAS u32* gp = (LAS u32*)(size_t)(off[k] & ~3u);
        g0[k] = gp[0]; g1[k] = gp[1];
    }
    __builtin_amdgcn_sched_barrier(0);
#pragma unroll
    for (int k = 0; k < 8; ++k) nib[k] |= (p.s << (7 - k)) & 0x80808080u;
    u32 ev = (u32)lane > nsd && len <= 16 ? (p.b0 & 127u) << ((p.b0 >> 5) & 24u) : 0u;
    u32 ea = (u32)(size_t)image + ((p.b0 >> 8) & 0x1FFCu);
    u32 wa[4];
#pragma unroll
    for (int c = 0; c < 4; ++c) wa[c] = (u32)(size_t)image + img[c];
    asm volatile("" : "+v"(ev), "+v"(ea), "+v"(wa[0]), "+v"(wa[1]), "+v"(wa[2]), "+v"(wa[3]));
    __builtin_amdgcn_sched_barrier(0);
#pragma unroll
    for (int c = 0; c < 4; ++c) {
#pragma unroll
        for (int k = 2 * c; k < 2 * c + 2; ++k) {
            const u32 g = __builtin_amdgcn_alignbyte(g1[k], g0[k], off[k]);
            o[k] = (__builtin_amdgcn_perm(g, g, sel[k]) ^ kx) + nib[k];
        }
        *(LAS u32x2*)(size_t)wa[c] = u32x2{o[2 * c], o[2 * c + 1]};
    }
    lds_order();
    if (ev) __hip_atomic_fetch_xor((LAS u32*)(size_t)ea, ev, __ATOMIC_RELAXED, __HIP_MEMORY_SCOPE_WORKGROUP);
    if (len <= 16) {
    } else if (len < RAW16) {
        for (u32 i = lane; i < len * 4; i += 64)
            patch(__builtin_amdgcn_raw_buffer_load_b32(rs, i * 4, p.bo, 17), i > nsd, image);
    } else {
        const u32x4 h = __builtin_amdgcn_raw_buffer_load_b128(rs, lane * 16, p.bo, 17);
        *(LAS u32x2*)(size_t)wa[0] = u32x2{p.n[0], p.n[1]};
        *(LAS u32x2*)(size_t)wa[1] = u32x2{p.n[2], p.n[3]};
        *(LAS u32x2*)(size_t)wa[2] = u32x2{h[0], h[1]};
        *(LAS u32x2*)(size_t)wa[3] = u32x2{h[2], h[3]};
    }
}

__device__ __forceinline__ void w_img(u32 (&img)[4], int q, int lane) {
    const u32 cg = (lane & 7) | ((lane >> 1) & 8), g = 2 * q + (lane >> 5);
#pragma unroll
    for (int c = 0; c < 4; ++c) img[c] = (g * 64 + c * 16 + cg) * 16 + (lane & 8);
}
__device__ __forceinline__ void x_img(u32 (&img)[4], int lane) {
#pragma unroll
    for (int c = 0; c < 4; ++c) img[c] = lane * 16 + (c >> 1) * 1024 + (c & 1) * 8;
}

struct Unit {
    u32x4 p[6];
    u32 meta, tot16;
    LAS char* lds;
    __device__ __forceinline__ u32x4 blk16(u32 i) const {
        const u32 c1 = end_of(meta, 0), c2 = end_of(meta, 1), c3 = end_of(meta, 2);
        const u32 a = i < c1 ? i : (i < c2 ? i - c1 + 64 : (i < c3 ? i - c2 + 128 : i - c3 + 192));
        return *(LAS u32x4*)(lds + 5120 + (a & 255u) * 16);
    }
    __device__ __forceinline__ void fetch(LAS char* unit, LAS u32* lens, int lane) {
        lds = unit;
        const u32x4 l = *(LAS u32x4*)lens;
        const u32 c2 = l[0] + l[1], c3 = c2 + l[2];
        meta = __builtin_amdgcn_readfirstlane(l[0] | (c2 << 7) | (c3 << 15) | ((c3 + l[3]) << 23));
        tot16 = meta >> 23;
#pragma unroll
        for (int i = 0; i < 5; ++i) p[i] = *(LAS u32x4*)(unit + i * 1024 + lane * 16);
        p[5] = blk16(lane);
    }
    __device__ __forceinline__ void send_blocks(const char* dst, u32 voff, int lane) const {
        for (u32 i = 64 + lane; i < tot16; i += 64) st16<1024>(dst, voff + (i - lane) * 16, blk16(i));
        if ((u32)lane < tot16) st16<1024>(dst, voff, p[5]);
    }
    __device__ __forceinline__ void send_all(const char* dst, u32 voff, int lane) const {
        st16<-4096>(dst, voff, p[0]); st16<-3072>(dst, voff, p[1]); st16<-2048>(dst, voff, p[2]);
        st16<-1024>(dst, voff, p[3]); st16<0>(dst, voff, p[4]);
        send_blocks(dst, voff, lane);
    }
};

__device__ __forceinline__ void pace(u32 due) {
    if ((int)((u32)__builtin_amdgcn_s_memrealtime() - due) > 0) wait_vm<WIN_HI - 1>();
    else wait_vm<WIN_LO - 1>();
}
__device__ __forceinline__ void pace1(u32 due) {
    if ((int)((u32)__builtin_amdgcn_s_memrealtime() - due) > 0) wait_vm<WIN_HI>();
    else wait_vm<WIN_LO>();
}
__device__ __forceinline__ void send_turn(const Unit& U, const char* dst, u32 voff, int lane, u32 due,
                                          bool has_prev, char* prev_flag, u32 prev_meta, u32 call) {
    pace(due); st16<-4096>(dst, voff, U.p[0]);
    pace(due + PACE_TURN / 6); st16<-3072>(dst, voff, U.p[1]);
    wait_vm<2>();
    if (has_prev) put_flag(prev_flag, call, prev_meta, lane);
    pace1(due + 2 * PACE_TURN / 6); st16<-2048>(dst, voff, U.p[2]);
    pace(due + 3 * PACE_TURN / 6); st16<-1024>(dst, voff, U.p[3]);
    pace(due + 4 * PACE_TURN / 6); st16<0>(dst, voff, U.p[4]);
    pace(due + 5 * PACE_TURN / 6); U.send_blocks(dst, voff, lane);
}

__device__ __forceinline__ void bar() {
    asm volatile("s_waitcnt lgkmcnt(0)" ::: "memory");
    __builtin_amdgcn_s_barrier();
    asm volatile("" ::: "memory");
}
__device__ __forceinline__ void enc_init(LAS char* lds, int wave, int lane) {
    *(LAS u32x4*)(lds + (wave >> 2) * USLOT + 5120 + (wave & 3) * 1024 + lane * 16) = u32x4{0, 0, 0, 0};
    u32 sel = 0x0C0C0C0Cu, n = 0;
#pragma unroll
    for (int i = 0; i < 4; ++i)
        if ((lane >> i) & 1) { sel = (sel & ~(0xFFu << n)) | ((u32)i << n); n += 8; }
    if (lane < 16) *(LAS u32*)(lds + 2 * USLOT + wave * 64 + lane * 4) = sel;
    lds_order();
}

__device__ __forceinline__ void sender(const Params& P, int b, int wave, int lane, GAS u32* cp, LAS char* lds) {
    const int rank = P.rank;
    char* const self = P.self;
    const u32 tstart = (u32)__builtin_amdgcn_s_memrealtime();
    const u32 cv = __hip_atomic_load(cp, __ATOMIC_RELAXED, __HIP_MEMORY_SCOPE_AGENT);
    const int q = wave & 3, tl = wave >> 2;
    const int j = b & 63, kc0 = b >> 6;
    u32 wr[8];
    {
        const u32 cg = (lane & 7) | ((lane >> 1) & 8), rg = ((lane >> 3) & 1) | ((lane >> 4) & 2);
        const uint8_t* wp = P.w + (size_t)((kc0 + 2 * tl) * 128 + q * 32 + rg * 8) * GN + j * 64 + cg * 4;
#pragma unroll
        for (int i = 0; i < 8; ++i) wr[i] = *(const GAS u32*)(wp + (size_t)i * GN);
    }
    enc_init(lds, wave, lane);
    LAS char* const unit = lds + tl * USLOT;
    LAS u32* const lens = (LAS u32*)(lds + 2 * USLOT + 512);
    u32 img[4], dw[8];
    w_img(img, q, lane);
#pragma unroll
    for (int h = 0; h < 2; ++h) {
        const u32 r0 = wr[4 * h], r1 = wr[4 * h + 1], r2 = wr[4 * h + 2], r3 = wr[4 * h + 3];
        const u32 t0 = __builtin_amdgcn_perm(r1, r0, 0x05010400u), t1 = __builtin_amdgcn_perm(r1, r0, 0x07030602u);
        const u32 t2 = __builtin_amdgcn_perm(r3, r2, 0x05010400u), t3 = __builtin_amdgcn_perm(r3, r2, 0x07030602u);
        dw[0 + h] = __builtin_amdgcn_perm(t2, t0, 0x05040100u);
        dw[2 + h] = __builtin_amdgcn_perm(t2, t0, 0x07060302u);
        dw[4 + h] = __builtin_amdgcn_perm(t3, t1, 0x05040100u);
        dw[6 + h] = __builtin_amdgcn_perm(t3, t1, 0x07060302u);
    }
    const u32 len = encode_quarter(dw, img, unit, q, lds + 2 * USLOT + wave * 64, lane,
                                   [&](u32x4 v, bool) { *(LAS u32x4*)(unit + q * 1024 + lane * 16) = v; });
    if (lane == 0) lens[wave] = len;
    const u32 call = __builtin_amdgcn_readfirstlane(cv) + 1;
    const u32 mine = (call & 1) * PAR_BYTES + rank * SRC_BYTES;
    const u32 voff = lane * 16 + 4096;
    bar();
    STAMP(3);
    const u32 ua = kc0 * 64 + j, ub = ua + 128;
    const u32 wa = mine + SRC_W + ua * USLOT, wb = mine + SRC_W + ub * USLOT;
    const u32 fa = mine + SRC_WF + ua * 128, fb = mine + SRC_WF + ub * 128;
    if (q == 2) {
        Unit U;
        U.fetch(unit, lens + tl * 4, lane);
        U.send_all(self + (tl ? wb : wa), voff, lane);
        wait_vm<0>();
        put_flag(self + (tl ? fb : fa), call, U.meta, lane);
    } else if (wave == 1) {
        Unit A, Bu;
        A.fetch(lds, lens, lane);
        Bu.fetch(lds + USLOT, lens + 4, lane);
        int pa = (kc0 + j) % 7, pb = (kc0 + 2 + j) % 7;
        char* prev = nullptr;
        u32 due = tstart + PACE_T0 + kc0 * PACE_STAG;
#pragma unroll 1
        for (int s = 0; s < 7; ++s) {
            char* da = P.ws[(rank + 1 + pa) & 7];
            char* db = P.ws[(rank + 1 + pb) & 7];
            pa = pa == 6 ? 0 : pa + 1;
            pb = pb == 6 ? 0 : pb + 1;
            send_turn(A, da + wa, voff, lane, due, s > 0, prev + fb, Bu.meta, call);
            send_turn(Bu, db + wb, voff, lane, due + PACE_TURN, true, da + fa, A.meta, call);
            prev = db;
            due += 2 * PACE_TURN;
            STAMP(5 + s);
        }
        wait_vm<0>();
        put_flag(prev + fb, call, Bu.meta, lane);
        STAMP(12);
    }
    if (lane == 0) __hip_atomic_store(cp, call, __ATOMIC_RELAXED, __HIP_MEMORY_SCOPE_AGENT);
}

__device__ __forceinline__ u32 lds_add(LAS u32* c, int lane) {
    u32 v = 0;
    if (lane == 0) v = __hip_atomic_fetch_add(c, 1u, __ATOMIC_ACQ_REL, __HIP_MEMORY_SCOPE_WORKGROUP);
    return v;
}
__device__ __forceinline__ void lds_wait(LAS u32* c, u32 n) {
    while (__builtin_amdgcn_readfirstlane(__hip_atomic_load(c, __ATOMIC_ACQUIRE, __HIP_MEMORY_SCOPE_WORKGROUP)) < n)
        __builtin_amdgcn_s_sleep(1);
}
__device__ __forceinline__ int src_of(int rank, int j, int t) {
    return t < 4 ? rank : (rank + 7 - ((t >> 2) - 1 + (t & 3) + j) % 7) & 7;
}

__device__ __forceinline__ void mma_waves(const Params& P, int b, int wave, int lane, GAS u32* cp, LAS char* lds) {
    char* const ws = P.self;
    const int rank = P.rank;
    const u32 cv = __hip_atomic_load(cp, __ATOMIC_RELAXED, __HIP_MEMORY_SCOPE_AGENT);
    LAS u32* const cnt = (LAS u32*)(lds + L_CNT);
    u32 call;
    {
        const int q = wave & 3, tl = wave >> 2, ux = 2 * b + tl;
        const int xdest = (ux + (ux >> 3)) & 7, xu = ux >> 3;
        char* const xdst = P.ws[xdest];
        const uint8_t* xp = P.x + (size_t)(xdest * MS + (xu >> 1) * 32 + (q & 1) * 16 + (lane & 15)) * KS + (xu & 1) * 256 +
                            (q >> 1) * 128 + (lane >> 4) * 32;
        const u32x4 a = *(const GAS u32x4*)xp, c = *(const GAS u32x4*)(xp + 16);
        const u32 dx[8] = {a[0], a[1], a[2], a[3], c[0], c[1], c[2], c[3]};
        LAS char* const el = lds + L_ENC;
        enc_init(el, wave, lane);
        LAS u32* const lens = (LAS u32*)(el + 2 * USLOT + 512);
        u32 img[4];
        x_img(img, lane);
        call = __builtin_amdgcn_readfirstlane(cv) + 1;
        const u32 mine = (call & 1) * PAR_BYTES + rank * SRC_BYTES;
        const u32 voff = lane * 16 + 4096;
        const char* const xslot = xdst + mine + SRC_X + xu * USLOT;
        const u32 len = encode_quarter(dx, img, el + tl * USLOT, q, el + 2 * USLOT + wave * 64, lane, [&](u32x4 v, bool again) {
            if (again) wait_vm<0>();
            st16<-4096>(xslot + q * 1024, voff, v);
            STAMP(61);
        });
        if (lane == 0) lens[wave] = len;
        bar();
        if (q == 0) {
            Unit U;
            U.fetch(el + tl * USLOT, lens + tl * 4, lane);
            st16<0>(xslot, voff, U.p[4]);
            U.send_blocks(xslot, voff, lane);
            if (lane == 0) cnt[70 + tl] = U.meta;
            STAMP(62);
        }
        wait_vm<0>();
        if (__builtin_amdgcn_readfirstlane(lds_add(cnt + 66 + tl, lane)) == 3) {
            put_flag(xdst + mine + SRC_XF + xu * 128, call, cnt[70 + tl], lane);
            STAMP(63);
        }
    }
    const int j = (b & 7) * 8 + ((b >> 3) & 7);
    const int rb = (b >> 6) * 8 + wave;
    GAS u32* const tmo = (GAS u32*)(ws + OFF_TMO);
    const __amdgpu_buffer_rsrc_t rs = __builtin_amdgcn_make_buffer_rsrc(ws, 0, WS_NEED, 0x00020000);
    const __amdgpu_buffer_rsrc_t ro = __builtin_amdgcn_make_buffer_rsrc(P.out, 0, MS * GN * 4, 0x00020000);
    const float scale = *P.sx * *P.sw;
    LAS char* const rp = lds + (lane >> 4) * 2048 + (lane & 15) * 16;
    const u32 txo = OFF_XD + (src_of(rank, j, lane & 31) * 16 + rb) * 16384 + (lane & 3) * 4096;
    f32x4 acc[2][4] = {};
    u32x4 ae[4] = {}, ao[4] = {};
    auto load_a = [&](u32x4 (&a)[4], int t) {
        const u32 so = __builtin_amdgcn_readlane(txo, t);
#pragma unroll
        for (int i = 0; i < 4; ++i) a[i] = __builtin_amdgcn_raw_buffer_load_b128(rs, lane * 16 + i * 1024, so, 17);
    };
    auto tile = [&](u32x4 (&a)[4], int t, u32 need, bool more) {
        lds_wait(cnt + t, need);
        __builtin_amdgcn_s_setprio(2);
        LAS char* const bp = rp + (t & (NBUF - 1)) * 8192;
        u32x4 bq[8];
#pragma unroll
        for (int i = 0; i < 8; ++i) bq[i] = *(LAS u32x4*)(bp + (i >> 1) * 256 + (i & 1) * 1024);
        __builtin_amdgcn_sched_barrier(0);
#pragma unroll
        for (int c = 0; c < 4; ++c)
#pragma unroll
            for (int m = 0; m < 2; ++m) {
                const u32x4 a0 = a[2 * m], a1 = a[2 * m + 1], b0 = bq[2 * c], b1 = bq[2 * c + 1];
                const i32x8 af = {(int)a0[0], (int)a0[1], (int)a0[2], (int)a0[3], (int)a1[0], (int)a1[1], (int)a1[2], (int)a1[3]};
                const i32x8 bf = {(int)b0[0], (int)b0[1], (int)b0[2], (int)b0[3], (int)b1[0], (int)b1[1], (int)b1[2], (int)b1[3]};
                acc[m][c] = __builtin_amdgcn_mfma_scale_f32_16x16x128_f8f6f4(af, bf, acc[m][c], 1, 1, 0, 0x7F7F7F7F, 0, 0x7F7F7F7F);
            }
        asm volatile("" : "+v"(acc[0][0]), "+v"(acc[0][1]), "+v"(acc[0][2]), "+v"(acc[0][3]),
                          "+v"(acc[1][0]), "+v"(acc[1][1]), "+v"(acc[1][2]), "+v"(acc[1][3]));
        __builtin_amdgcn_sched_barrier(0);
        if (more) {
            if (lane == 0) __hip_atomic_fetch_add(cnt + 32 + t, 1u, __ATOMIC_RELAXED, __HIP_MEMORY_SCOPE_WORKGROUP);
            load_a(a, t + 2);
        }
        __builtin_amdgcn_s_setprio(0);
    };
#pragma unroll 1
    for (int pass = 0; pass < 2; ++pass) {
        if (pass) {
            wait_flags(ws + OFF_LF + (((lane >> 1) & 7) * 32 + rb * 2 + (lane & 1)) * 128, 0xFFFF, call, tmo);
            STAMP(3);
            load_a(ae, 0);
            __builtin_amdgcn_sched_barrier(0);
            load_a(ao, 1);
#pragma unroll
            for (int i = 0; i < 8; ++i) acc[i >> 2][i & 3] = f32x4{0.f, 0.f, 0.f, 0.f};
#pragma unroll 1
            for (int t = 0; t < 30; t += 2) {
                tile(ae, t, 4, true);
                tile(ao, t + 1, 4, true);
                STAMP(30 + (t >> 1));
            }
        }
        tile(ae, 30, pass * 4, false);
        tile(ao, 31, pass * 4, false);
        const float sc = cnt[72] ? __builtin_nanf("") : scale;
        if (pass) STAMP(59);
        const u32 vo = pass ? (lane >> 4) * 4 * GN * 4 + (lane & 15) * 16 : OOB;
        const u32 so = (rb * 32 * GN + j * 64) * 4;
#pragma unroll
        for (int m = 0; m < 2; ++m)
#pragma unroll
            for (int r = 0; r < 4; ++r) {
                u32x4 v;
#pragma unroll
                for (int c = 0; c < 4; ++c)
                    v[c] = __builtin_bit_cast(u32, __builtin_elementwise_maximum(acc[m][c][r] * sc, 0.f));
                __builtin_amdgcn_raw_buffer_store_b128(v, ro, vo, so + (m * 16 + r) * GN * 4, OUT_AUX);
                __builtin_amdgcn_sched_barrier(0);
                asm volatile("s_nop 1");
                __builtin_amdgcn_sched_barrier(0);
            }
    }
    if (lane == 0) __hip_atomic_store(cp, call, __ATOMIC_RELAXED, __HIP_MEMORY_SCOPE_AGENT);
    STAMP(60);
}

__device__ __forceinline__ void unpack_waves(const Params& P, int b, int wave, int lane, GAS u32* cp, LAS char* lds) {
    char* const ws = P.self;
    const int rank = P.rank;
    const u32 cv = __hip_atomic_load(cp, __ATOMIC_RELAXED, __HIP_MEMORY_SCOPE_AGENT);
    LAS u32* const cnt = (LAS u32*)(lds + L_CNT);
    if (wave == 8) { cnt[lane] = 0; cnt[64 + lane] = 0; }
    bar();
    const u32 call = __builtin_amdgcn_readfirstlane(cv) + 1;
    const u32 par = (call & 1) * PAR_BYTES;
    const int q = wave & 3, tl = (wave >> 2) & 1;
    const int j = (b & 7) * 8 + ((b >> 3) & 7);
    GAS u32* const tmo = (GAS u32*)(ws + OFF_TMO);
    const __amdgpu_buffer_rsrc_t rs = __builtin_amdgcn_make_buffer_rsrc(ws, 0, WS_NEED, 0x00020000);
    LAS char* const sym = lds + L_SYM + (wave & 7) * 2304;
    const int ux = 2 * b + tl, xsrc = ux & 7, xu = ux >> 3;
    constexpr int NJOB = 17;
    u32 juo, jfo;
    {
        const int i = lane < NJOB ? lane : 0, t = i ? 2 * (i - 1) + tl : 0;
        const u32 so = par + (i ? src_of(rank, j, t) : xsrc) * SRC_BYTES;
        juo = so + (i ? SRC_W + ((t & 3) * 64 + j) * USLOT : SRC_X + xu * USLOT);
        jfo = so + (i ? SRC_WF + ((t & 3) * 64 + j) * 128 : SRC_XF + xu * 128);
    }
    u32 img[4];
    x_img(img, lane);
    LAS char* image = lds + L_XIM + (wave & 7) * 2048;
    __builtin_amdgcn_s_setprio(1);
    Packed nx;
    u64 look = 0;
    bool have = false;
#pragma unroll 1
    for (int i = 0; i < NJOB; ++i) {
        const int t = 2 * (i - 1) + tl;
        if (t >= NBUF) lds_wait(cnt + 32 + t - NBUF, 8);
        const u64 up = __ballot((u32)(look >> 32) >= call);
        if (!have) {
            u32 meta = __builtin_amdgcn_readlane((u32)look, i);
            if (!((up >> i) & 1)) meta = wait_flag(ws + __builtin_amdgcn_readlane(jfo, i), call, tmo);
            asm volatile("" ::: "memory");
            load_quarter(nx, rs, __builtin_amdgcn_readlane(juo, i), meta, q, lane);
        }
        if (i >= 9) STAMP(4 + 3 * (i - 9));
        have = i + 1 < NJOB && ((up >> (i + 1)) & 1);
        const u32 nuo = __builtin_amdgcn_readlane(juo, i + 1), nmeta = __builtin_amdgcn_readlane((u32)look, i + 1);
        asm volatile("" ::: "memory");
        Packed pk = nx;
        asm volatile("" : "+v"(pk.n), "+v"(pk.s), "+v"(pk.b0));
        const u32 tv = __hip_atomic_load(tmo, __ATOMIC_RELAXED, __HIP_MEMORY_SCOPE_AGENT);
        if (have) load_quarter(nx, rs, nuo, nmeta, q, lane);
        look = __hip_atomic_load((GAS u64*)(ws + jfo), __ATOMIC_RELAXED, __HIP_MEMORY_SCOPE_SYSTEM);
        if (i >= 9) STAMP(5 + 3 * (i - 9));
        decode_quarter(pk, rs, img, image, sym, lane);
        if (tv == call) cnt[72] = 1;
        if (i) {
            if (lane == 0) __hip_atomic_fetch_add(cnt + t, 1u, __ATOMIC_RELEASE, __HIP_MEMORY_SCOPE_WORKGROUP);
        } else {
            lds_order();
            const u32x4 v0 = *(LAS u32x4*)(image + lane * 16), v1 = *(LAS u32x4*)(image + 1024 + lane * 16);
            const char* xd = ws + OFF_XD + (xsrc * 32 + xu) * 8192 + q * 2048;
            st16<0>(xd, lane * 16, v0);
            st16<1024>(xd, lane * 16, v1);
            wait_vm<0>();
            if (__builtin_amdgcn_readfirstlane(lds_add(cnt + 68 + tl, lane)) == 3)
                put_flag(ws + OFF_LF + (xsrc * 32 + xu) * 128, call, 0, lane);
            w_img(img, q, lane);
            STAMP(2);
        }
        image = lds + ((t + 2) & (NBUF - 1)) * 8192;
        if (i >= 9) STAMP(6 + 3 * (i - 9));
        else STAMP(48 + i);
    }
    if (lane == 0) __hip_atomic_store(cp, call, __ATOMIC_RELAXED, __HIP_MEMORY_SCOPE_AGENT);
}

__global__ void __launch_bounds__(1024, 4) gemm_rs_kernel(Params P) {
    __shared__ __attribute__((aligned(16))) char smem[LDS_BYTES];
    const int lane = threadIdx.x & 63;
    const int wave = __builtin_amdgcn_readfirstlane(threadIdx.x >> 6);
    GAS u32* cp = (GAS u32*)(P.self + OFF_CNT) + blockIdx.x * 16 + wave;
    STAMP(0);
    if (blockIdx.x < N_SEND_WG) {
        if (wave < 8) sender(P, blockIdx.x, wave, lane, cp, (LAS char*)smem);
    } else if (wave < 8) mma_waves(P, blockIdx.x - N_SEND_WG, wave, lane, cp, (LAS char*)smem);
    else unpack_waves(P, blockIdx.x - N_SEND_WG, wave, lane, cp, (LAS char*)smem);
}

extern "C" void kernel_launch(int rank, int nranks, ncclComm_t comm,
                              hipStream_t stream,
                              void* const* d_in, const int* in_sizes, int n_in,
                              void* d_out, int out_size,
                              const int* in_dtypes, int out_dtype,
                              void* const* d_out_all,
                              void* const* d_ws_all, size_t ws_bytes) {
    if (nranks != NR || n_in != 4 || ws_bytes < WS_NEED || (unsigned)rank >= (unsigned)NR ||
        in_sizes[0] != 4096 * KS || in_sizes[1] != KS * GN || out_size != MS * GN) {
        fprintf(stderr, "kernel_launch: unexpected problem shape\n");
        return;
    }
    Params P;
    P.x = (const uint8_t*)d_in[0];
    P.w = (const uint8_t*)d_in[1];
    P.sx = (const float*)d_in[2];
    P.sw = (const float*)d_in[3];
    P.out = (float*)d_out;
    for (int p = 0; p < NR; ++p) P.ws[p] = (char*)d_ws_all[p];
    P.self = P.ws[rank];
    P.rank = rank;
    P.pad = 0;
    gemm_rs_kernel<<<GRID, 1024, 0, stream>>>(P);
    HIP_CHECK(hipGetLastError());
}
```

```cpp
#include <hip/hip_runtime.h>
#include <rccl/rccl.h>
#include <cstdio>
#include <cstdint>

#define HIP_CHECK(expr) do { hipError_t _e = (expr); if (_e != hipSuccess) { \
    fprintf(stderr, "HIP error %s at %s:%d\n", hipGetErrorString(_e), __FILE__, __LINE__); \
    return; } } while (0)

typedef unsigned u32;
typedef unsigned long long u64;
typedef u32 u32x4 __attribute__((ext_vector_type(4)));
typedef u32 u32x2 __attribute__((ext_vector_type(2)));
typedef int i32x8 __attribute__((ext_vector_type(8)));
typedef float f32x4 __attribute__((ext_vector_type(4)));
#define GAS __attribute__((address_space(1)))
#define LAS __attribute__((address_space(3)))

#define PACE_T0 330
#define PACE_SLOT 380
#define PACE_KC 16
#define PACE_STEP 0
#define POLL_SLEEP 1
#define OUT_AUX 17
#define FORCE_RAW 0

constexpr int NR = 8;
constexpr int GN = 4096;
constexpr int KS = 512;
constexpr int MS = 512;
constexpr int NUNIT = 256;
constexpr int N_SEND_WG = 128;
constexpr int N_CONS_WG = 128;
constexpr int GRID = N_SEND_WG + N_CONS_WG;

constexpr u32 USLOT = 9216;
constexpr u32 SRC_W = 0;
constexpr u32 SRC_X = SRC_W + NUNIT * USLOT;
constexpr u32 SRC_WF = SRC_X + 32 * USLOT;
constexpr u32 SRC_XF = SRC_WF + NUNIT * 128;
constexpr u32 SRC_BYTES = SRC_XF + 32 * 128;
constexpr u32 PAR_BYTES = NR * SRC_BYTES;
constexpr u32 OFF_XD = 2 * PAR_BYTES;
constexpr u32 OFF_LF = OFF_XD + NR * MS * KS;
constexpr u32 OFF_CNT = OFF_LF + NUNIT * 128;
constexpr u32 OFF_TMO = OFF_CNT + GRID * 16 * 4;
constexpr u32 OFF_ANC = OFF_TMO + 128;
constexpr u32 OFF_DBG = OFF_ANC + 16 * 128;
constexpr u32 WS_NEED = OFF_DBG + GRID * 16 * 64 * 8;
constexpr u32 DEADLINE_TICKS = 100000000u;
constexpr u32 OOB = 0x80000000u;
constexpr u32 RAW16 = 64;
constexpr u32 CAP_DW = 192;
constexpr int NBUF = 8;
constexpr u32 L_SYM = NBUF * 8192;
constexpr u32 L_XIM = 4 * 8192;
constexpr u32 L_ENC = L_SYM + 8 * 2304;
constexpr u32 L_STA = L_ENC;
constexpr u32 L_CNT = L_STA + 8 * 8192;
constexpr int LDS_BYTES = L_CNT + 512;

struct Params {
    const uint8_t* x;
    const uint8_t* w;
    const float* sx;
    const float* sw;
    float* out;
    char* self;
    char* ws[NR];
    int rank;
    int pad;
};

#define SJ 16
#define STAMP(i) do { } while (0)

template <int OFF>
__device__ __forceinline__ void st16(const char* base, u32 voff, u32x4 v) {
    asm volatile("global_store_dwordx4 %0, %1, %2 offset:%3 sc0 sc1\n\ts_nop 1"
                 :: "v"(voff), "v"(v), "s"(base), "i"(OFF) : "memory");
}
template <int N> __device__ __forceinline__ void wait_vm() { asm volatile("s_waitcnt vmcnt(%0)" :: "i"(N) : "memory"); }

#define FLAG_LANES 0
__device__ __forceinline__ void put_flag(char* p, u32 call, u32 meta, int lane) {
    if (lane == 0)
        __hip_atomic_store((GAS u64*)p, ((u64)call << 32) | meta, __ATOMIC_RELAXED, __HIP_MEMORY_SCOPE_SYSTEM);
}

__device__ __forceinline__ u64 wait_flags(const char* f, u64 need, u32 call, GAS u32* tmo) {
    const u32 t0 = (u32)__builtin_amdgcn_s_memrealtime();
    u64 v;
    for (;;) {
        v = __hip_atomic_load((GAS u64*)f, __ATOMIC_RELAXED, __HIP_MEMORY_SCOPE_SYSTEM);
        const u32 t = __hip_atomic_load(tmo, __ATOMIC_RELAXED, __HIP_MEMORY_SCOPE_AGENT);
        if ((__ballot((u32)(v >> 32) >= call) & need) == need) break;
        if (__builtin_amdgcn_readfirstlane(t) == call || (u32)__builtin_amdgcn_s_memrealtime() - t0 > DEADLINE_TICKS) {
            __hip_atomic_store(tmo, call, __ATOMIC_RELAXED, __HIP_MEMORY_SCOPE_AGENT);
            break;
        }
        __builtin_amdgcn_s_sleep(1);
    }
    asm volatile("" ::: "memory");
    return v;
}

__device__ __forceinline__ u32 wait_flag(const char* f, u32 call, GAS u32* tmo) {
    const u32 t0 = (u32)__builtin_amdgcn_s_memrealtime();
    u64 v;
    for (u32 n = 1;; ++n) {
        v = __hip_atomic_load((GAS u64*)f, __ATOMIC_RELAXED, __HIP_MEMORY_SCOPE_SYSTEM);
        if (__builtin_amdgcn_readfirstlane((u32)(v >> 32)) >= call) break;
        if ((n & 15) == 0) {
            const u32 t = __hip_atomic_load(tmo, __ATOMIC_RELAXED, __HIP_MEMORY_SCOPE_AGENT);
            if (__builtin_amdgcn_readfirstlane(t) == call || (u32)__builtin_amdgcn_s_memrealtime() - t0 > DEADLINE_TICKS) {
                __hip_atomic_store(tmo, call, __ATOMIC_RELAXED, __HIP_MEMORY_SCOPE_AGENT);
                break;
            }
        }
        __builtin_amdgcn_s_sleep(POLL_SLEEP);
    }
    asm volatile("" ::: "memory");
    return __builtin_amdgcn_readfirstlane((u32)v);
}

__device__ __forceinline__ u32 wave_scan(u32 v) {
    v += __builtin_amdgcn_update_dpp(0u, v, 0x111, 0xf, 0xf, true);
    v += __builtin_amdgcn_update_dpp(0u, v, 0x112, 0xf, 0xf, true);
    v += __builtin_amdgcn_update_dpp(0u, v, 0x114, 0xf, 0xf, true);
    v += __builtin_amdgcn_update_dpp(0u, v, 0x118, 0xf, 0xf, true);
    v += __builtin_amdgcn_update_dpp(0u, v, 0x142, 0xa, 0xf, false);
    v += __builtin_amdgcn_update_dpp(0u, v, 0x143, 0xc, 0xf, false);
    return v;
}
__device__ __forceinline__ u32 bfi(u32 m, u32 a, u32 b) { return (m & a) | (~m & b); }
__device__ __forceinline__ void lds_order() { __builtin_amdgcn_fence(__ATOMIC_ACQ_REL, "wavefront"); }

template <class Early>
__device__ __forceinline__ u32 encode_quarter(const u32 (&d)[8], const u32 (&img)[4], LAS char* unit, int q,
                                              LAS char* lut, int lane, Early codes_out) {
    u32 mag[8], sum = 0;
#pragma unroll
    for (int k = 0; k < 8; ++k) { mag[k] = d[k] & 0x7F7F7F7Fu; sum = __builtin_amdgcn_sad_u8(mag[k], 0u, sum); }
    int b = ((int)__builtin_amdgcn_readlane(wave_scan(sum), 63) + 1024 - 14541) >> 11;
    b = b < 26 ? 26 : b;
    asm volatile("" : "+s"(b));
    const u32 B = b > 108 ? 108 : b;
    const u32 c2 = (126 - B) * 0x01010101u, c16 = (112 - B) * 0x01010101u, c26 = (154 - B) * 0x01010101u, c20 = (108 - B) * 0x01010101u;
    u32 nib[8], w[8], esc[8];
#pragma unroll
    for (int k = 0; k < 8; ++k) {
        const u32 u = mag[k] + c2;
        w[k] = mag[k] + c16;
        const u32 dir = u & ~w[k] & 0x80808080u;
        nib[k] = w[k] & (((dir >> 3) - (dir >> 6)) | 0x01010101u);
        esc[k] = dir ^ 0x80808080u;
    }
    codes_out(u32x4{nib[0] | (nib[1] << 4), nib[2] | (nib[3] << 4), nib[4] | (nib[5] << 4), nib[6] | (nib[7] << 4)}, false);
    u32 sel[8], t[8], off[8], cnt = 0, sg = 0, mb = 0;
#pragma unroll
    for (int k = 0; k < 8; ++k) sel[k] = *(LAS u32*)(lut + (__builtin_amdgcn_udot4(esc[k], 0x08040201u, 0u, false) >> 5));
#pragma unroll
    for (int k = 0; k < 8; ++k) {
        const u32 hi = w[k] & 0x80808080u, q0 = mag[k] + c26;
        t[k] = ((q0 - ((hi >> 3) - (hi >> 6))) >> 1) & 0x0F0F0F0Fu;
        mb |= ((~q0 | (mag[k] + c20)) & 0x80808080u) >> (7 - k);
        off[k] = cnt;
        cnt += __builtin_popcount(esc[k]);
        sg |= (d[k] >> (7 - k)) & (0x01010101u << k);
    }
    const u32 incl = wave_scan(cnt);
    const u32 nsd = (__builtin_amdgcn_readlane(incl, 63) + 7) >> 3;
    LAS char* blk = unit + 5120 + q * 1024;
    *(LAS u32*)(unit + 4096 + q * 256 + lane * 4) = sg;
    u32 ndw = 1 + nsd;
    if (ndw <= CAP_DW && !FORCE_RAW) {
        const u32 base = incl - cnt + 8;
#pragma unroll
        for (int i = 0; i < 4; ++i) {
            u32 c0 = __builtin_amdgcn_perm(0u, t[2 * i], sel[2 * i]), c1 = __builtin_amdgcn_perm(0u, t[2 * i + 1], sel[2 * i + 1]);
            c0 = __builtin_amdgcn_perm(0u, c0 | (c0 >> 4), 0x0C0C0200u);
            c1 = __builtin_amdgcn_perm(0u, c1 | (c1 >> 4), 0x0C0C0200u);
            const u32 pos = base + off[2 * i];
            const u64 v = (u64)(c0 | (c1 << (4 * (off[2 * i + 1] - off[2 * i])))) << (4 * (pos & 7));
            LAS u32* p = (LAS u32*)(blk + ((pos >> 1) & ~3u));
            __hip_atomic_fetch_or(p, (u32)v, __ATOMIC_RELAXED, __HIP_MEMORY_SCOPE_WORKGROUP);
            if ((u32)(v >> 32)) __hip_atomic_fetch_or(p + 1, (u32)(v >> 32), __ATOMIC_RELAXED, __HIP_MEMORY_SCOPE_WORKGROUP);
        }
        if (lane == 0) *(LAS u32*)blk = B | (nsd << 8);
        lds_order();
        for (u64 mk; (mk = __ballot(mb != 0)) != 0; ndw += __builtin_popcountll(mk)) {
            if (mb != 0) {
                const u32 bit = __builtin_ctz(mb);
                mb &= mb - 1;
                const u32 k0 = (u32)((int)(bit << 31) >> 31), k1 = (u32)((int)(bit << 30) >> 31), k2 = (u32)((int)(bit << 29) >> 31);
                const u32 m0 = bfi(k2, mag[4], mag[0]), m1 = bfi(k2, mag[5], mag[1]), m2 = bfi(k2, mag[6], mag[2]), m3 = bfi(k2, mag[7], mag[3]);
                const u32 mg = (bfi(k0, bfi(k1, m3, m1), bfi(k1, m2, m0)) >> (bit & 24u)) & 127u;
                const u32 ad = bfi(k2, bfi(k1, img[3], img[2]), bfi(k1, img[1], img[0])) + (k0 & 4u) + (bit >> 3);
                const u32 z = (mg + 154 - B - (mg >= B + 16 ? 14 : 0)) & 31u;
                const u32 dflt = B - 26 + z + (z >= 28 ? 14 : 0);
                const u32 slot = ndw + __builtin_amdgcn_mbcnt_hi((u32)(mk >> 32), __builtin_amdgcn_mbcnt_lo((u32)mk, 0u));
                *(LAS u32*)(blk + (slot < 255 ? slot : 255) * 4) = (ad << 8) | (mg ^ dflt);
            }
        }
    }
    if (ndw > CAP_DW || FORCE_RAW) {
        lds_order();
        codes_out(u32x4{d[0], d[1], d[2], d[3]}, true);
        *(LAS u32x4*)(blk + lane * 16) = u32x4{d[4], d[5], d[6], d[7]};
        return RAW16;
    }
    return (ndw + 3) >> 2;
}

struct Packed {
    u32x4 n;
    u32 s, b0;
    u32 len, bo;
};
__device__ __forceinline__ u32 end_of(u32 meta, int q) {
    const u32 sh = q <= 0 ? 0 : 8 * q - 1, mask = q < 0 ? 0 : (q == 0 ? 127u : (q == 3 ? 511u : 255u));
    return (meta >> sh) & mask;
}
__device__ __forceinline__ void load_quarter(Packed& p, __amdgpu_buffer_rsrc_t rs, u32 uo, u32 meta, int q, int lane, u32 off = 0) {
    const u32 beg = end_of(meta, q - 1);
    p.len = end_of(meta, q) - beg;
    p.bo = uo + 5120 + 16 * beg;
    const u32 ndw = p.len * 4, l4 = lane * 4 | off;
    p.b0 = __builtin_amdgcn_raw_buffer_load_b32(rs, (u32)lane < ndw ? l4 : OOB, p.bo, 17);
    p.n = __builtin_amdgcn_raw_buffer_load_b128(rs, lane * 16 | off, uo + q * 1024, 17);
    p.s = __builtin_amdgcn_raw_buffer_load_b32(rs, l4, uo + 4096 + q * 256, 17);
}
__device__ __forceinline__ u32 sym_bytes(u32 t, u32 k1, u32 kx) {
    const u32 h = (t + 0x02020202u) & 0x10101010u;
    return (((t << 1) + k1) + (h - (h >> 3))) ^ kx;
}
__device__ __forceinline__ u32x2 sym8(u32 x, u32 k1, u32 kx) {
    const u32 lo = sym_bytes(x & 0x0F0F0F0Fu, k1, kx), hi = sym_bytes((x >> 4) & 0x0F0F0F0Fu, k1, kx);
    return u32x2{__builtin_amdgcn_perm(hi, lo, 0x05010400u), __builtin_amdgcn_perm(hi, lo, 0x07030602u)};
}
__device__ __forceinline__ void patch(u32 e, bool on, LAS char* image) {
    if (on && (e & 127u))
        __hip_atomic_fetch_xor((LAS u32*)(image + ((e >> 8) & 0x1FFCu)), (e & 127u) << ((e >> 5) & 24u),
                               __ATOMIC_RELAXED, __HIP_MEMORY_SCOPE_WORKGROUP);
}
template <class Mid>
__device__ __forceinline__ void decode_quarter(const Packed& p, __amdgpu_buffer_rsrc_t rs, const u32 (&img)[4],
                                               LAS char* image, LAS char* sym, int lane, Mid mid) {
    const u32 len = p.len;
    if (len > 16 && len < RAW16) {
        u32 h0 = p.b0;
        asm volatile("" : "+v"(h0));
        const u32 B = __builtin_amdgcn_readfirstlane(h0) & 127u;
        const u32 k1 = (B - 26) * 0x01010101u, kx = (0xFFu ^ B) * 0x01010101u;
        const u32 b1 = __builtin_amdgcn_raw_buffer_load_b32(rs, (u32)lane + 64 < len * 4 ? lane * 4 + 256 : OOB, p.bo, 17);
        const u32 b2 = __builtin_amdgcn_raw_buffer_load_b32(rs, (u32)lane + 128 < len * 4 ? lane * 4 + 512 : OOB, p.bo, 17);
        *(LAS u32x2*)(sym + 512 + lane * 8) = sym8(b1, k1, kx);
        *(LAS u32x2*)(sym + 1024 + lane * 8) = sym8(b2, k1, kx);
    }
    u32 nib[8], sel[8], off[8], cnt = 0;
#pragma unroll
    for (int k = 0; k < 8; ++k) {
        nib[k] = (k & 1 ? p.n[k >> 1] >> 4 : p.n[k >> 1]) & 0x0F0F0F0Fu;
        const u32 n10 = (nib[k] + 0x0E0E0E0Eu) & 0x10101010u, e10 = n10 ^ 0x10101010u;
        sel[k] = __umul24(e10, 0x101010u) + n10;
        off[k] = cnt;
        cnt += __builtin_popcount(e10);
    }
    const u32 hdr = __builtin_amdgcn_readfirstlane(p.b0);
    const u32 B = hdr & 127u, nsd = (hdr >> 8) & 255u;
    const u32 k1 = (B - 26) * 0x01010101u, kx = (0xFFu ^ B) * 0x01010101u;
    const u32 sp = (u32)(size_t)sym + 8 + (wave_scan(cnt) - cnt);
    *(LAS u32x2*)(sym + lane * 8) = sym8(p.b0, k1, kx);
    lds_order();
    mid();
    u32 g0[8], g1[8], o[8];
#pragma unroll
    for (int k = 0; k < 8; ++k) {
        off[k] += sp;
        LAS u32* gp = (LAS u32*)(size_t)(off[k] & ~3u);
        g0[k] = gp[0]; g1[k] = gp[1];
    }
    __builtin_amdgcn_sched_barrier(0);
#pragma unroll
    for (int k = 0; k < 8; ++k) nib[k] |= (p.s << (7 - k)) & 0x80808080u;
    u32 ev = (u32)lane > nsd && len <= 16 ? (p.b0 & 127u) << ((p.b0 >> 5) & 24u) : 0u;
    u32 ea = (u32)(size_t)image + ((p.b0 >> 8) & 0x1FFCu);
    u32 wa[4];
#pragma unroll
    for (int c = 0; c < 4; ++c) wa[c] = (u32)(size_t)image + img[c];
    asm volatile("" : "+v"(ev), "+v"(ea), "+v"(wa[0]), "+v"(wa[1]), "+v"(wa[2]), "+v"(wa[3]));
    __builtin_amdgcn_sched_barrier(0);
#pragma unroll
    for (int c = 0; c < 4; ++c) {
#pragma unroll
        for (int k = 2 * c; k < 2 * c + 2; ++k) {
            const u32 g = __builtin_amdgcn_alignbyte(g1[k], g0[k], off[k]);
            o[k] = (__builtin_amdgcn_perm(g, g, sel[k]) ^ kx) + nib[k];
        }
        *(LAS u32x2*)(size_t)wa[c] = u32x2{o[2 * c], o[2 * c + 1]};
    }
    lds_order();
    if (ev) __hip_atomic_fetch_xor((LAS u32*)(size_t)ea, ev, __ATOMIC_RELAXED, __HIP_MEMORY_SCOPE_WORKGROUP);
    if (len <= 16) {
    } else if (len < RAW16) {
        for (u32 i = lane; i < len * 4; i += 64)
            patch(__builtin_amdgcn_raw_buffer_load_b32(rs, i * 4, p.bo, 17), i > nsd, image);
    } else {
        const u32x4 h = __builtin_amdgcn_raw_buffer_load_b128(rs, lane * 16, p.bo, 17);
        *(LAS u32x2*)(size_t)wa[0] = u32x2{p.n[0], p.n[1]};
        *(LAS u32x2*)(size_t)wa[1] = u32x2{p.n[2], p.n[3]};
        *(LAS u32x2*)(size_t)wa[2] = u32x2{h[0], h[1]};
        *(LAS u32x2*)(size_t)wa[3] = u32x2{h[2], h[3]};
    }
}

__device__ __forceinline__ void w_img(u32 (&img)[4], int q, int lane) {
    const u32 cg = (lane & 7) | ((lane >> 1) & 8), g = 2 * q + (lane >> 5);
#pragma unroll
    for (int c = 0; c < 4; ++c) img[c] = (g * 64 + c * 16 + cg) * 16 + (lane & 8);
}
__device__ __forceinline__ void x_img(u32 (&img)[4], int lane) {
#pragma unroll
    for (int c = 0; c < 4; ++c) img[c] = lane * 16 + (c >> 1) * 1024 + (c & 1) * 8;
}

struct Unit {
    u32x4 p[6];
    u32 meta, tot16;
    LAS char* lds;
    __device__ __forceinline__ u32x4 blk16(u32 i) const {
        const u32 c1 = end_of(meta, 0), c2 = end_of(meta, 1), c3 = end_of(meta, 2);
        const u32 a = i < c1 ? i : (i < c2 ? i - c1 + 64 : (i < c3 ? i - c2 + 128 : i - c3 + 192));
        return *(LAS u32x4*)(lds + 5120 + (a & 255u) * 16);
    }
    __device__ __forceinline__ void fetch(LAS char* unit, LAS u32* lens, int lane) {
        lds = unit;
        const u32x4 l = *(LAS u32x4*)lens;
        const u32 c2 = l[0] + l[1], c3 = c2 + l[2];
        meta = __builtin_amdgcn_readfirstlane(l[0] | (c2 << 7) | (c3 << 15) | ((c3 + l[3]) << 23));
        tot16 = meta >> 23;
#pragma unroll
        for (int i = 0; i < 5; ++i) p[i] = *(LAS u32x4*)(unit + i * 1024 + lane * 16);
        p[5] = blk16(lane);
    }
    __device__ __forceinline__ void send_blocks(const char* dst, u32 voff, int lane) const {
        for (u32 i = 64 + lane; i < tot16; i += 64) st16<1024>(dst, voff + (i - lane) * 16, blk16(i));
        if ((u32)lane < tot16) st16<1024>(dst, voff, p[5]);
    }
    __device__ __forceinline__ void send_all(const char* dst, u32 voff, int lane) const {
        st16<-4096>(dst, voff, p[0]); st16<-3072>(dst, voff, p[1]); st16<-2048>(dst, voff, p[2]);
        st16<-1024>(dst, voff, p[3]); st16<0>(dst, voff, p[4]);
        send_blocks(dst, voff, lane);
    }
};

__device__ __forceinline__ void pace(u32 due) {
    while ((int)((u32)__builtin_amdgcn_s_memrealtime() - due) < 0) __builtin_amdgcn_s_sleep(1);
}

__device__ __forceinline__ void bar() {
    asm volatile("s_waitcnt lgkmcnt(0)" ::: "memory");
    __builtin_amdgcn_s_barrier();
    asm volatile("" ::: "memory");
}
__device__ __forceinline__ void enc_init(LAS char* lds, int wave, int lane) {
    *(LAS u32x4*)(lds + (wave >> 2) * USLOT + 5120 + (wave & 3) * 1024 + lane * 16) = u32x4{0, 0, 0, 0};
    u32 sel = 0x0C0C0C0Cu, n = 0;
#pragma unroll
    for (int i = 0; i < 4; ++i)
        if ((lane >> i) & 1) { sel = (sel & ~(0xFFu << n)) | ((u32)i << n); n += 8; }
    if (lane < 16) *(LAS u32*)(lds + 2 * USLOT + wave * 64 + lane * 4) = sel;
    lds_order();
}

__device__ __forceinline__ void anchor_wave(const Params& P, int wave, int lane, GAS u32* cp, LAS char* lds, u32 cv) {
    const u32 tstart = (u32)__builtin_amdgcn_s_memrealtime();
    const u32 call = __builtin_amdgcn_readfirstlane(cv) + 1;
    u32 a = (u32)__builtin_amdgcn_s_memrealtime();
    __builtin_amdgcn_s_sleep(16);
    for (int n = 0; n < 6; ++n) {
        const u32 v = __hip_atomic_load((GAS u32*)(P.self + OFF_ANC + (lane >> 2) * 128 + (lane & 3) * 4), __ATOMIC_RELAXED, __HIP_MEMORY_SCOPE_AGENT);
        if (__all((v >> 24) == (call & 255u))) {
            a = tstart - 4096 + (__builtin_amdgcn_readlane(wave_scan((v - tstart + 4096) & 0xFFFFFFu), 63) >> 6);
            break;
        }
        __builtin_amdgcn_s_sleep(4);
    }
    *(LAS u32*)(lds + 2 * USLOT + 512 + 64) = a;
    bar();
    if (lane == 0) __hip_atomic_store(cp, call, __ATOMIC_RELAXED, __HIP_MEMORY_SCOPE_AGENT);
    __builtin_amdgcn_endpgm();
}

__device__ __forceinline__ void sender(const Params& P, int b, int wave, int lane, GAS u32* cp, LAS char* lds, u32 cv) {
    const int rank = P.rank;
    char* const self = P.self;
    const int q = wave & 3, tl = wave >> 2;
    const int j = b & 63, kc0 = b >> 6;
    u32 wr[8];
    {
        const u32 cg = (lane & 7) | ((lane >> 1) & 8), rg = ((lane >> 3) & 1) | ((lane >> 4) & 2);
        const uint8_t* wp = P.w + (size_t)((kc0 + 2 * tl) * 128 + q * 32 + rg * 8) * GN + j * 64 + cg * 4;
#pragma unroll
        for (int i = 0; i < 8; ++i) wr[i] = *(const GAS u32*)(wp + (size_t)i * GN);
    }
    enc_init(lds, wave, lane);
    LAS char* const unit = lds + tl * USLOT;
    LAS u32* const lens = (LAS u32*)(lds + 2 * USLOT + 512);
    u32 img[4], dw[8];
    w_img(img, q, lane);
#pragma unroll
    for (int h = 0; h < 2; ++h) {
        const u32 r0 = wr[4 * h], r1 = wr[4 * h + 1], r2 = wr[4 * h + 2], r3 = wr[4 * h + 3];
        const u32 t0 = __builtin_amdgcn_perm(r1, r0, 0x05010400u), t1 = __builtin_amdgcn_perm(r1, r0, 0x07030602u);
        const u32 t2 = __builtin_amdgcn_perm(r3, r2, 0x05010400u), t3 = __builtin_amdgcn_perm(r3, r2, 0x07030602u);
        dw[0 + h] = __builtin_amdgcn_perm(t2, t0, 0x05040100u);
        dw[2 + h] = __builtin_amdgcn_perm(t2, t0, 0x07060302u);
        dw[4 + h] = __builtin_amdgcn_perm(t3, t1, 0x05040100u);
        dw[6 + h] = __builtin_amdgcn_perm(t3, t1, 0x07060302u);
    }
    const u32 len = encode_quarter(dw, img, unit, q, lds + 2 * USLOT + wave * 64, lane,
                                   [&](u32x4 v, bool) { *(LAS u32x4*)(unit + q * 1024 + lane * 16) = v; });
    if (lane == 0) lens[wave] = len;
    LAS u32* const anchor = lens + 16;
    const u32 call = __builtin_amdgcn_readfirstlane(cv) + 1;
    const u32 mine = (call & 1) * PAR_BYTES + rank * SRC_BYTES;
    const u32 voff = lane * 16 + 4096;
    bar();
    STAMP(3);
    if (q == 1 || q == 2) {
        const int kc = kc0 + 2 * tl, u = kc * 64 + j;
        const u32 wo = mine + SRC_W + u * USLOT, fo = mine + SRC_WF + u * 128;
        Unit U;
        U.fetch(unit, lens + tl * 4, lane);
        if (q == 2) {
            U.send_all(self + wo, voff, lane);
            wait_vm<0>();
            put_flag(self + fo, call, U.meta, lane);
        } else {
            const u32 t0 = __builtin_amdgcn_readfirstlane(*anchor) + PACE_T0;
            int pr = (kc + j) % 7;
#pragma unroll 1
            for (int s = 0; s < 7; ++s) {
                char* const peer = P.ws[(rank + 1 + pr) & 7];
                const char* const dst = peer + wo;
                pr = pr == 6 ? 0 : pr + 1;
                int before = 0;
#pragma unroll
                for (int k = 0; k < 4; ++k) {
                    const int v = 64 * s + j + PACE_KC * (kc - k);
                    before += v < 0 ? 0 : (v > 448 ? 448 : v);
                }
                const u32 due = t0 + ((before * PACE_SLOT) >> 8);
                pace(due); st16<-4096>(dst, voff, U.p[0]);
                pace(due + PACE_STEP); st16<-3072>(dst, voff, U.p[1]);
                pace(due + 2 * PACE_STEP); st16<-2048>(dst, voff, U.p[2]);
                pace(due + 3 * PACE_STEP); st16<-1024>(dst, voff, U.p[3]);
                pace(due + 4 * PACE_STEP); st16<0>(dst, voff, U.p[4]);
                pace(due + 5 * PACE_STEP); U.send_blocks(dst, voff, lane);
                wait_vm<0>();
                put_flag(peer + fo, call, U.meta, lane);
                STAMP(5 + s);
            }
            if (tl) STAMP(12);
        }
    }
    if (lane == 0) __hip_atomic_store(cp, call, __ATOMIC_RELAXED, __HIP_MEMORY_SCOPE_AGENT);
    __builtin_amdgcn_endpgm();
}

__device__ __forceinline__ u32 lds_add(LAS u32* c, int lane) {
    u32 v = 0;
    if (lane == 0) v = __hip_atomic_fetch_add(c, 1u, __ATOMIC_ACQ_REL, __HIP_MEMORY_SCOPE_WORKGROUP);
    return v;
}
__device__ __forceinline__ void lds_wait(LAS u32* c, u32 n) {
    while (__builtin_amdgcn_readfirstlane(__hip_atomic_load(c, __ATOMIC_ACQUIRE, __HIP_MEMORY_SCOPE_WORKGROUP)) < n)
        __builtin_amdgcn_s_sleep(1);
}
__device__ __forceinline__ int src_of(int rank, int j, int t) {
    return t < 4 ? rank : (rank + 7 - ((t >> 2) - 1 + (t & 3) + j) % 7) & 7;
}

__device__ __forceinline__ void mma_waves(const Params& P, int b, int wave, int lane, GAS u32* cp, LAS char* lds, u32x4 a, u32x4 c, u32 cv) {
    char* const ws = P.self;
    const int rank = P.rank;
    u32 cnt_a = (u32)(size_t)(lds + L_CNT);
    asm volatile("" : "+v"(cnt_a));
    LAS u32* const cnt = (LAS u32*)(size_t)cnt_a;
    u32 call;
    {
        const int q = wave & 3, tl = wave >> 2, ux = 2 * b + tl;
        const int xdest = (ux + (ux >> 3)) & 7, xu = ux >> 3;
        char* const xdst = P.ws[xdest];
        u32 dx[8] = {a[0], a[1], a[2], a[3], c[0], c[1], c[2], c[3]};
        LAS char* const el = lds + L_ENC;
        enc_init(el, wave, lane);
        LAS u32* const lens = (LAS u32*)(el + 2 * USLOT + 512);
        u32 img[4];
        x_img(img, lane);
        call = __builtin_amdgcn_readfirstlane(cv) + 1;
        const u32 mine = (call & 1) * PAR_BYTES + rank * SRC_BYTES;
        const u32 voff = lane * 16 + 4096;
        const char* const xslot = xdst + mine + SRC_X + xu * USLOT;
        if ((b & 7) == (b >> 4) && (wave & 1)) {
            asm volatile("" : "+v"(dx[0]), "+v"(dx[7]));
            if (lane == 0)
                __hip_atomic_store((GAS u32*)(ws + OFF_ANC + (b >> 3) * 128 + (wave >> 1) * 4),
                                   (call << 24) | ((u32)__builtin_amdgcn_s_memrealtime() & 0xFFFFFFu), __ATOMIC_RELAXED, __HIP_MEMORY_SCOPE_AGENT);
        }
        const u32 len = encode_quarter(dx, img, el + tl * USLOT, q, el + 2 * USLOT + wave * 64, lane, [&](u32x4 v, bool again) {
            if (again) wait_vm<0>();
            st16<-4096>(xslot + q * 1024, voff, v);
            STAMP(61);
        });
        if (lane == 0) lens[wave] = len;
        bar();
        if (q == 0) {
            Unit U;
            U.fetch(el + tl * USLOT, lens + tl * 4, lane);
            st16<0>(xslot, voff, U.p[4]);
            U.send_blocks(xslot, voff, lane);
            if (lane == 0) cnt[70 + tl] = U.meta;
            STAMP(62);
        }
        wait_vm<0>();
        if (__builtin_amdgcn_readfirstlane(lds_add(cnt + 66 + tl, lane)) == 3) {
            put_flag(xdst + mine + SRC_XF + xu * 128, call, cnt[70 + tl], lane);
            STAMP(63);
        }
    }
    const int j = b & 63;
    const int rb = (b >> 6) * 8 + wave;
    GAS u32* const tmo = (GAS u32*)(ws + OFF_TMO);
    const __amdgpu_buffer_rsrc_t rs = __builtin_amdgcn_make_buffer_rsrc(ws, 0, WS_NEED, 0x00020000);
    const __amdgpu_buffer_rsrc_t ro = __builtin_amdgcn_make_buffer_rsrc(P.out, 0, MS * GN * 4, 0x00020000);
    const float scale = __builtin_bit_cast(float, __builtin_amdgcn_readfirstlane(__builtin_bit_cast(u32, *P.sx * *P.sw)));
    if (lane == 0) __hip_atomic_store(cp, call, __ATOMIC_RELAXED, __HIP_MEMORY_SCOPE_AGENT);
    LAS char* const rp = lds + (lane >> 4) * 2048 + (lane & 15) * 16;
    const u32 txo = OFF_XD + (src_of(rank, j, lane & 31) * 16 + rb) * 16384 + (lane & 3) * 4096;
    f32x4 acc[2][4] = {};
    u32x4 ae[4] = {}, ao[4] = {};
    auto load_a = [&](u32x4 (&a)[4], int t) {
        const u32 so = __builtin_amdgcn_readlane(txo, t);
        u32 l16 = lane * 16;
        asm volatile("" : "+v"(l16));
#pragma unroll
        for (int i = 0; i < 4; ++i) a[i] = __builtin_amdgcn_raw_buffer_load_b128(rs, l16 + i * 1024, so, 17);
    };
    LAS char* const stash = lds + L_STA + wave * 8192 + lane * 16;
    auto tile = [&](u32x4 (&a)[4], int t, u32 need, int more) {
        lds_wait(cnt + t, need);
        __builtin_amdgcn_s_setprio(2);
        LAS char* const bp = rp + (t & (NBUF - 1)) * 8192;
        u32x4 bq[8];
#pragma unroll
        for (int i = 0; i < 8; ++i) bq[i] = *(LAS u32x4*)(bp + (i >> 1) * 256 + (i & 1) * 1024);
        __builtin_amdgcn_sched_barrier(0);
#pragma unroll
        for (int c = 0; c < 4; ++c)
#pragma unroll
            for (int m = 0; m < 2; ++m) {
                const u32x4 a0 = a[2 * m], a1 = a[2 * m + 1], b0 = bq[2 * c], b1 = bq[2 * c + 1];
                const i32x8 af = {(int)a0[0], (int)a0[1], (int)a0[2], (int)a0[3], (int)a1[0], (int)a1[1], (int)a1[2], (int)a1[3]};
                const i32x8 bf = {(int)b0[0], (int)b0[1], (int)b0[2], (int)b0[3], (int)b1[0], (int)b1[1], (int)b1[2], (int)b1[3]};
                acc[m][c] = __builtin_amdgcn_mfma_scale_f32_16x16x128_f8f6f4(af, bf, acc[m][c], 1, 1, 0, 0x7F7F7F7F, 0, 0x7F7F7F7F);
            }
        asm volatile("" : "+v"(acc[0][0]), "+v"(acc[0][1]), "+v"(acc[0][2]), "+v"(acc[0][3]),
                          "+v"(acc[1][0]), "+v"(acc[1][1]), "+v"(acc[1][2]), "+v"(acc[1][3]));
        __builtin_amdgcn_sched_barrier(0);
        if (more == 1) {
            if (lane == 0) __hip_atomic_fetch_add(cnt + 32 + t, 1u, __ATOMIC_RELAXED, __HIP_MEMORY_SCOPE_WORKGROUP);
            load_a(a, t + 2);
        } else if (more == 2) {
#pragma unroll
            for (int i = 0; i < 4; ++i) a[i] = *(LAS u32x4*)(stash + (t & 1) * 4096 + i * 1024);
        }
        __builtin_amdgcn_s_setprio(0);
    };
#pragma unroll 1
    for (int pass = 0; pass < 2; ++pass) {
        if (pass) {
            wait_flags(ws + OFF_LF + (((lane >> 1) & 7) * 32 + rb * 2 + (lane & 1)) * 128, 0xFFFF, call, tmo);
            STAMP(3);
            load_a(ae, 30);
            load_a(ao, 31);
            lds_wait(cnt + 66, 4);
            lds_wait(cnt + 67, 4);
#pragma unroll
            for (int i = 0; i < 4; ++i) { *(LAS u32x4*)(stash + i * 1024) = ae[i]; *(LAS u32x4*)(stash + 4096 + i * 1024) = ao[i]; }
            __builtin_amdgcn_sched_barrier(0);
            load_a(ae, 0);
            __builtin_amdgcn_sched_barrier(0);
            load_a(ao, 1);
#pragma unroll
            for (int i = 0; i < 8; ++i) acc[i >> 2][i & 3] = f32x4{0.f, 0.f, 0.f, 0.f};
#pragma unroll 1
            for (int t = 0; t < 28; t += 2) {
                tile(ae, t, 4, 1);
                tile(ao, t + 1, 4, 1);
                if (wave == 0) STAMP(30 + (t >> 1));
            }
        }
        tile(ae, 28, pass * 4, 2);
        tile(ao, 29, pass * 4, 2);
        if (pass && wave == 0) STAMP(44);
        tile(ae, 30, pass * 4, 0);
        tile(ao, 31, pass * 4, 0);
        const float sc = cnt[72] ? __builtin_nanf("") : scale;
        if (pass) STAMP(59);
        const u32 vo = pass ? (lane >> 4) * 4 * GN * 4 + (lane & 15) * 16 : OOB;
        const u32 so = (rb * 32 * GN + j * 64) * 4;
#pragma unroll
        for (int m = 0; m < 2; ++m)
#pragma unroll
            for (int r = 0; r < 4; ++r) {
                u32x4 v;
#pragma unroll
                for (int c = 0; c < 4; ++c)
                    v[c] = __builtin_bit_cast(u32, __builtin_elementwise_maximum(acc[m][c][r] * sc, 0.f));
                __builtin_amdgcn_raw_buffer_store_b128(v, ro, vo, so + (m * 16 + r) * GN * 4, OUT_AUX);
                __builtin_amdgcn_sched_barrier(0);
                asm volatile("s_nop 1");
                __builtin_amdgcn_sched_barrier(0);
            }
    }
    STAMP(60);
    __builtin_amdgcn_endpgm();
}

__device__ __forceinline__ void unpack_waves(const Params& P, int b, int wave, int lane, GAS u32* cp, LAS char* lds, u32 cv) {
    char* const ws = P.self;
    const int rank = P.rank;
    LAS u32* const cnt = (LAS u32*)(lds + L_CNT);
    if (wave == 8) { cnt[lane] = 0; cnt[64 + lane] = 0; }
    bar();
    const u32 call = __builtin_amdgcn_readfirstlane(cv) + 1;
    const u32 par = (call & 1) * PAR_BYTES;
    const int q = wave & 3, tl = (wave >> 2) & 1;
    const int j = b & 63;
    GAS u32* const tmo = (GAS u32*)(ws + OFF_TMO);
    const __amdgpu_buffer_rsrc_t rs = __builtin_amdgcn_make_buffer_rsrc(ws, 0, WS_NEED, 0x00020000);
    LAS char* const sym = lds + L_SYM + (wave & 7) * 2304;
    const int ux = 2 * b + tl, xsrc = ux & 7, xu = ux >> 3;
    constexpr int NJOB = 17;
    u32 juo, jfo;
    {
        const int i = lane < NJOB ? lane : 0, t = i ? 2 * (i - 1) + tl : 0;
        const u32 so = par + (i ? src_of(rank, j, t) : xsrc) * SRC_BYTES;
        juo = so + (i ? SRC_W + ((t & 3) * 64 + j) * USLOT : SRC_X + xu * USLOT);
        jfo = so + (i ? SRC_WF + ((t & 3) * 64 + j) * 128 : SRC_XF + xu * 128);
    }
    u32 img[4];
    x_img(img, lane);
    LAS char* image = lds + L_XIM + (wave & 7) * 2048;
    __builtin_amdgcn_s_setprio(1);
    Packed nx;
    u64 look = 0;
    bool have = false;
#pragma unroll 1
    for (int i = 0; i < NJOB; ++i) {
        const int t = 2 * (i - 1) + tl;
        if (t >= NBUF) lds_wait(cnt + 32 + t - NBUF, 8);
        else if (t >> 1 == 2) lds_wait(cnt + 68 + tl, 4);
        if (!have) {
            u32 meta = __builtin_amdgcn_readfirstlane((u32)look);
            if (__builtin_amdgcn_readfirstlane((u32)(look >> 32)) < call) meta = wait_flag(ws + __builtin_amdgcn_readlane(jfo, i), call, tmo);
            asm volatile("" ::: "memory");
            look = __hip_atomic_load((GAS u64*)(ws + __builtin_amdgcn_readlane(jfo, i + 1)), __ATOMIC_RELAXED, __HIP_MEMORY_SCOPE_SYSTEM);
            load_quarter(nx, rs, __builtin_amdgcn_readlane(juo, i), meta, q, lane);
            if (i >= SJ) STAMP(4 + 3 * (i - 9));
        }
        Packed pk = nx;
        asm volatile("" : "+v"(pk.n), "+v"(pk.s), "+v"(pk.b0), "+v"(look));
        if (i >= SJ && SJ < 16) STAMP(5 + 3 * (i - 9));
        have = i + 1 < NJOB && __builtin_amdgcn_readfirstlane((u32)(look >> 32)) >= call;
        asm volatile("" ::: "memory");
        if (have) load_quarter(nx, rs, __builtin_amdgcn_readlane(juo, i + 1), __builtin_amdgcn_readfirstlane((u32)look), q, lane);
        const u32 la = i + 1 < NJOB ? __builtin_amdgcn_readlane(jfo, i + 1 + have) : OFF_TMO;
        decode_quarter(pk, rs, img, image, sym, lane, [&]() {
            look = __hip_atomic_load((GAS u64*)(ws + la), __ATOMIC_RELAXED, __HIP_MEMORY_SCOPE_SYSTEM);
        });
        if (i + 1 == NJOB) {
            u32 tv = (u32)look;
            asm volatile("" : "+v"(tv));
            if (tv == call) cnt[72] = 1;
        }
        if (i) {
            if (lane == 0) __hip_atomic_fetch_add(cnt + t, 1u, __ATOMIC_RELEASE, __HIP_MEMORY_SCOPE_WORKGROUP);
        } else {
            lds_order();
            const u32x4 v0 = *(LAS u32x4*)(image + lane * 16), v1 = *(LAS u32x4*)(image + 1024 + lane * 16);
            const char* xd = ws + OFF_XD + (xsrc * 32 + xu) * 8192 + q * 2048;
            st16<0>(xd, lane * 16, v0);
            st16<1024>(xd, lane * 16, v1);
            wait_vm<0>();
            if (__builtin_amdgcn_readfirstlane(lds_add(cnt + 68 + tl, lane)) == 3)
                put_flag(ws + OFF_LF + (xsrc * 32 + xu) * 128, call, 0, lane);
            w_img(img, q, lane);
            STAMP(2);
        }
        image = lds + ((t + 2) & (NBUF - 1)) * 8192;
        if (i >= SJ - 1) STAMP(6 + 3 * (i - 9));
        else if (i < 9 && SJ == 9) STAMP(48 + i);
    }
    if (lane == 0) __hip_atomic_store(cp, call, __ATOMIC_RELAXED, __HIP_MEMORY_SCOPE_AGENT);
    __builtin_amdgcn_endpgm();
}

__global__ void __launch_bounds__(1024, 4) gemm_rs_kernel(Params P) {
    __shared__ __attribute__((aligned(16))) char smem[LDS_BYTES];
    const int lane = threadIdx.x & 63;
    const int wave = __builtin_amdgcn_readfirstlane(threadIdx.x >> 6);
    u32x4 xa, xc;
    {
        const u32 ux = 2 * (blockIdx.x - N_SEND_WG) + (wave >> 2), xdest = (ux + (ux >> 3)) & 7, xu = (ux >> 3) & 31;
        const __amdgpu_buffer_rsrc_t rx = __builtin_amdgcn_make_buffer_rsrc((void*)P.x, 0, 4096 * KS, 0x00020000);
        const u32 so = (xdest * MS + (xu >> 1) * 32 + (wave & 1) * 16) * KS + (xu & 1) * 256 + (wave & 2) * 64;
        const u32 vo = ((lane & 15) * KS + (lane >> 4) * 32) | (blockIdx.x >= N_SEND_WG && wave < 8 ? 0 : OOB);
        xa = __builtin_amdgcn_raw_buffer_load_b128(rx, vo, so, 0);
        xc = __builtin_amdgcn_raw_buffer_load_b128(rx, vo + 16, so, 0);
        asm volatile("" ::: "memory");
    }
    GAS u32* cp = (GAS u32*)(P.self + OFF_CNT) + blockIdx.x * 16 + wave;
    const u32 cv = __hip_atomic_load(cp, __ATOMIC_RELAXED, __HIP_MEMORY_SCOPE_AGENT);
    asm volatile("" ::: "memory");
    STAMP(0);
    if (blockIdx.x >= N_SEND_WG) {
        if (wave < 8) mma_waves(P, blockIdx.x - N_SEND_WG, wave, lane, cp, (LAS char*)smem, xa, xc, cv);
        else unpack_waves(P, blockIdx.x - N_SEND_WG, wave, lane, cp, (LAS char*)smem, cv);
    } else {
        if (wave < 8) sender(P, blockIdx.x, wave, lane, cp, (LAS char*)smem, cv);
        else if (wave == 8) anchor_wave(P, wave, lane, cp, (LAS char*)smem, cv);
    }
}

extern "C" void kernel_launch(int rank, int nranks, ncclComm_t comm,
                              hipStream_t stream,
                              void* const* d_in, const int* in_sizes, int n_in,
                              void* d_out, int out_size,
                              const int* in_dtypes, int out_dtype,
                              void* const* d_out_all,
                              void* const* d_ws_all, size_t ws_bytes) {
    if (nranks != NR || n_in != 4 || ws_bytes < WS_NEED || (unsigned)rank >= (unsigned)NR ||
        in_sizes[0] != 4096 * KS || in_sizes[1] != KS * GN || out_size != MS * GN) {
        fprintf(stderr, "kernel_launch: unexpected problem shape\n");
        return;
    }
    Params P;
    P.x = (const uint8_t*)d_in[0];
    P.w = (const uint8_t*)d_in[1];
    P.sx = (const float*)d_in[2];
    P.sw = (const float*)d_in[3];
    P.out = (float*)d_out;
    for (int p = 0; p < NR; ++p) P.ws[p] = (char*)d_ws_all[p];
    P.self = P.ws[rank];
    P.rank = rank;
    P.pad = 0;
    gemm_rs_kernel<<<GRID, 1024, 0, stream>>>(P);
    HIP_CHECK(hipGetLastError());
}
```

```cpp
#include <hip/hip_runtime.h>
#include <rccl/rccl.h>
#include <cstdio>
#include <cstdint>

#define HIP_CHECK(expr) do { hipError_t _e = (expr); if (_e != hipSuccess) { \
    fprintf(stderr, "HIP error %s at %s:%d\n", hipGetErrorString(_e), __FILE__, __LINE__); \
    return; } } while (0)

typedef unsigned u32;
typedef unsigned long long u64;
typedef u32 u32x4 __attribute__((ext_vector_type(4)));
typedef u32 u32x2 __attribute__((ext_vector_type(2)));
typedef int i32x8 __attribute__((ext_vector_type(8)));
typedef float f32x4 __attribute__((ext_vector_type(4)));
#define GAS __attribute__((address_space(1)))
#define LAS __attribute__((address_space(3)))

#define PACE_T0 322
#define PACE_SLOT 376
#define PACE_KC 16
#define USUAL16 51
#define GAP 2
#define POLL_SLEEP 6
#define OUT_AUX 17
#define FORCE_RAW 0

constexpr int NR = 8;
constexpr int GN = 4096;
constexpr int KS = 512;
constexpr int MS = 512;
constexpr int NUNIT = 256;
constexpr int N_SEND_WG = 128;
constexpr int N_CONS_WG = 128;
constexpr int GRID = N_SEND_WG + N_CONS_WG;

constexpr u32 USLOT = 9216;
constexpr u32 SRC_W = 0;
constexpr u32 SRC_X = SRC_W + NUNIT * USLOT;
constexpr u32 SRC_WF = SRC_X + 32 * USLOT;
constexpr u32 SRC_XF = SRC_WF + NUNIT * 128;
constexpr u32 SRC_BYTES = SRC_XF + 32 * 128;
constexpr u32 PAR_BYTES = NR * SRC_BYTES;
constexpr u32 OFF_XD = 2 * PAR_BYTES;
constexpr u32 OFF_LF = OFF_XD + NR * MS * KS;
constexpr u32 OFF_CNT = OFF_LF + NUNIT * 128;
constexpr u32 OFF_TMO = OFF_CNT + GRID * 16 * 4;
constexpr u32 OFF_ANC = OFF_TMO + 128;
constexpr u32 OFF_DBG = OFF_ANC + 16 * 128;
constexpr u32 WS_NEED = OFF_DBG + GRID * 16 * 64 * 8;
constexpr u32 DEADLINE_TICKS = 100000000u;
constexpr u32 OOB = 0x80000000u;
constexpr u32 RAW_DW = 256;
constexpr u32 CAP_DW = 192;
constexpr int NBUF = 8;
constexpr u32 L_SYM = NBUF * 8192;
constexpr u32 L_XIM = 4 * 8192;
constexpr u32 L_ENC = L_SYM + 8 * 2304;
constexpr u32 L_STA = L_ENC;
constexpr u32 L_CNT = L_STA + 8 * 8192;
constexpr int LDS_BYTES = L_CNT + 512;

struct Params {
    const uint8_t* x;
    const uint8_t* w;
    const float* sx;
    const float* sw;
    float* out;
    char* self;
    char* ws[NR];
    int rank;
    int pad;
};

#define SJ 16
#define STAMP(i) do { } while (0)

template <int OFF>
__device__ __forceinline__ void st16(const char* base, u32 voff, u32x4 v) {
    asm volatile("global_store_dwordx4 %0, %1, %2 offset:%3 sc0 sc1\n\ts_nop 1"
                 :: "v"(voff), "v"(v), "s"(base), "i"(OFF) : "memory");
}
template <int N> __device__ __forceinline__ void wait_vm() { asm volatile("s_waitcnt vmcnt(%0)" :: "i"(N) : "memory"); }

__device__ __forceinline__ void put_flag(char* p, u32 call, u32 meta, int lane) {
    if (lane < 4)
        __hip_atomic_store((GAS u64*)p + lane, ((u64)call << 32) | meta, __ATOMIC_RELAXED, __HIP_MEMORY_SCOPE_SYSTEM);
}

__device__ __forceinline__ u64 wait_flags(const char* f, u64 need, u32 call, GAS u32* tmo) {
    const u32 t0 = (u32)__builtin_amdgcn_s_memrealtime();
    u64 v;
    for (;;) {
        v = __hip_atomic_load((GAS u64*)f, __ATOMIC_RELAXED, __HIP_MEMORY_SCOPE_SYSTEM);
        const u32 t = __hip_atomic_load(tmo, __ATOMIC_RELAXED, __HIP_MEMORY_SCOPE_AGENT);
        if ((__ballot((u32)(v >> 32) >= call) & need) == need) break;
        if (__builtin_amdgcn_readfirstlane(t) == call || (u32)__builtin_amdgcn_s_memrealtime() - t0 > DEADLINE_TICKS) {
            __hip_atomic_store(tmo, call, __ATOMIC_RELAXED, __HIP_MEMORY_SCOPE_AGENT);
            break;
        }
        __builtin_amdgcn_s_sleep(1);
    }
    asm volatile("" ::: "memory");
    return v;
}

__device__ __forceinline__ u32 wait_flag(const char* f, u32 call, GAS u32* tmo) {
    const u32 t0 = (u32)__builtin_amdgcn_s_memrealtime();
    u32 meta;
    u64 va = __hip_atomic_load((GAS u64*)f, __ATOMIC_RELAXED, __HIP_MEMORY_SCOPE_SYSTEM);
    __builtin_amdgcn_s_sleep(POLL_SLEEP);
    for (u32 n = 1;; ++n) {
        const u64 vb = __hip_atomic_load((GAS u64*)f, __ATOMIC_RELAXED, __HIP_MEMORY_SCOPE_SYSTEM);
        meta = __builtin_amdgcn_readfirstlane((u32)va);
        if (__builtin_amdgcn_readfirstlane((u32)(va >> 32)) >= call) break;
        va = __hip_atomic_load((GAS u64*)f, __ATOMIC_RELAXED, __HIP_MEMORY_SCOPE_SYSTEM);
        meta = __builtin_amdgcn_readfirstlane((u32)vb);
        if (__builtin_amdgcn_readfirstlane((u32)(vb >> 32)) >= call) break;
        if ((n & 15) == 0) {
            const u32 t = __hip_atomic_load(tmo, __ATOMIC_RELAXED, __HIP_MEMORY_SCOPE_AGENT);
            if (__builtin_amdgcn_readfirstlane(t) == call || (u32)__builtin_amdgcn_s_memrealtime() - t0 > DEADLINE_TICKS) {
                __hip_atomic_store(tmo, call, __ATOMIC_RELAXED, __HIP_MEMORY_SCOPE_AGENT);
                break;
            }
            va = __hip_atomic_load((GAS u64*)f, __ATOMIC_RELAXED, __HIP_MEMORY_SCOPE_SYSTEM);
            __builtin_amdgcn_s_sleep(POLL_SLEEP);
        }
    }
    asm volatile("" ::: "memory");
    return meta;
}

__device__ __forceinline__ u32 wave_scan(u32 v) {
    v += __builtin_amdgcn_update_dpp(0u, v, 0x111, 0xf, 0xf, true);
    v += __builtin_amdgcn_update_dpp(0u, v, 0x112, 0xf, 0xf, true);
    v += __builtin_amdgcn_update_dpp(0u, v, 0x114, 0xf, 0xf, true);
    v += __builtin_amdgcn_update_dpp(0u, v, 0x118, 0xf, 0xf, true);
    v += __builtin_amdgcn_update_dpp(0u, v, 0x142, 0xa, 0xf, false);
    v += __builtin_amdgcn_update_dpp(0u, v, 0x143, 0xc, 0xf, false);
    return v;
}
__device__ __forceinline__ u32 bfi(u32 m, u32 a, u32 b) { return (m & a) | (~m & b); }
__device__ __forceinline__ void lds_order() { __builtin_amdgcn_fence(__ATOMIC_ACQ_REL, "wavefront"); }

template <class Early>
__device__ __forceinline__ u32 encode_quarter(const u32 (&d)[8], u32 img, LAS char* unit, int q,
                                              LAS char* lut, int lane, Early codes_out) {
    u32 mag[8], sum = 0;
#pragma unroll
    for (int k = 0; k < 8; ++k) { mag[k] = d[k] & 0x7F7F7F7Fu; sum = __builtin_amdgcn_sad_u8(mag[k], 0u, sum); }
    int b = ((int)__builtin_amdgcn_readlane(wave_scan(sum), 63) + 1024 - 14541) >> 11;
    b = b < 26 ? 26 : b;
    asm volatile("" : "+s"(b));
    const u32 B = b > 108 ? 108 : b;
    const u32 c2 = (126 - B) * 0x01010101u, c16 = (112 - B) * 0x01010101u, c26 = (154 - B) * 0x01010101u, c20 = (108 - B) * 0x01010101u;
    u32 nib[8], w[8], esc[8];
#pragma unroll
    for (int k = 0; k < 8; ++k) {
        const u32 u = mag[k] + c2;
        w[k] = mag[k] + c16;
        const u32 dir = u & ~w[k] & 0x80808080u;
        nib[k] = w[k] & (((dir >> 3) - (dir >> 6)) | 0x01010101u);
        esc[k] = dir ^ 0x80808080u;
    }
    codes_out(u32x4{nib[0] | (nib[1] << 4), nib[2] | (nib[3] << 4), nib[4] | (nib[5] << 4), nib[6] | (nib[7] << 4)}, false);
    u32 sel[8], t[8], off[8], cnt = 0, sg = 0, mb = 0;
#pragma unroll
    for (int k = 0; k < 8; ++k) sel[k] = *(LAS u32*)(lut + (__builtin_amdgcn_udot4(esc[k], 0x08040201u, 0u, false) >> 5));
#pragma unroll
    for (int k = 0; k < 8; ++k) {
        const u32 hi = w[k] & 0x80808080u, q0 = mag[k] + c26;
        t[k] = ((q0 - ((hi >> 3) - (hi >> 6))) >> 1) & 0x0F0F0F0Fu;
        mb |= ((~q0 | (mag[k] + c20)) & 0x80808080u) >> (7 - k);
        off[k] = cnt;
        cnt += __builtin_popcount(esc[k]);
        sg |= (d[k] >> (7 - k)) & (0x01010101u << k);
    }
    const u32 incl = wave_scan(cnt);
    const u32 nsd = (__builtin_amdgcn_readlane(incl, 63) + 7) >> 3;
    LAS char* blk = unit + 5120 + q * 1024;
    *(LAS u32*)(unit + 4096 + q * 256 + lane * 4) = sg;
    u32 ndw = nsd;
    if (ndw <= CAP_DW && !FORCE_RAW) {
        const u32 base = incl - cnt;
#pragma unroll
        for (int i = 0; i < 4; ++i) {
            u32 c0 = __builtin_amdgcn_perm(0u, t[2 * i], sel[2 * i]), c1 = __builtin_amdgcn_perm(0u, t[2 * i + 1], sel[2 * i + 1]);
            c0 = __builtin_amdgcn_perm(0u, c0 | (c0 >> 4), 0x0C0C0200u);
            c1 = __builtin_amdgcn_perm(0u, c1 | (c1 >> 4), 0x0C0C0200u);
            const u32 pos = base + off[2 * i];
            const u64 v = (u64)(c0 | (c1 << (4 * (off[2 * i + 1] - off[2 * i])))) << (4 * (pos & 7));
            LAS u32* p = (LAS u32*)(blk + ((pos >> 1) & ~3u));
            __hip_atomic_fetch_or(p, (u32)v, __ATOMIC_RELAXED, __HIP_MEMORY_SCOPE_WORKGROUP);
            if ((u32)(v >> 32)) __hip_atomic_fetch_or(p + 1, (u32)(v >> 32), __ATOMIC_RELAXED, __HIP_MEMORY_SCOPE_WORKGROUP);
        }
        lds_order();
        for (u64 mk; (mk = __ballot(mb != 0)) != 0; ndw += __builtin_popcountll(mk)) {
            if (mb != 0) {
                const u32 bit = __builtin_ctz(mb);
                mb &= mb - 1;
                const u32 k0 = (u32)((int)(bit << 31) >> 31), k1 = (u32)((int)(bit << 30) >> 31), k2 = (u32)((int)(bit << 29) >> 31);
                const u32 m0 = bfi(k2, mag[4], mag[0]), m1 = bfi(k2, mag[5], mag[1]), m2 = bfi(k2, mag[6], mag[2]), m3 = bfi(k2, mag[7], mag[3]);
                const u32 mg = (bfi(k0, bfi(k1, m3, m1), bfi(k1, m2, m0)) >> (bit & 24u)) & 127u;
                const u32 ad = img + (bit & 6u) * 128 + (k0 & 4u) + (bit >> 3);
                const u32 z = (mg + 154 - B - (mg >= B + 16 ? 14 : 0)) & 31u;
                const u32 dflt = B - 26 + z + (z >= 28 ? 14 : 0);
                const u32 slot = ndw + __builtin_amdgcn_mbcnt_hi((u32)(mk >> 32), __builtin_amdgcn_mbcnt_lo((u32)mk, 0u));
                *(LAS u32*)(blk + (slot < 255 ? slot : 255) * 4) = (ad << 8) | (mg ^ dflt);
            }
        }
    }
    if (ndw > CAP_DW || ndw - nsd > 63 || FORCE_RAW) {
        lds_order();
        codes_out(u32x4{d[0], d[1], d[2], d[3]}, true);
        *(LAS u32x4*)(blk + lane * 16) = u32x4{d[4], d[5], d[6], d[7]};
        return RAW_DW;
    }
    return ndw | ((ndw - nsd) << 9) | (B << 15);
}

struct Packed {
    u32x4 n;
    u32 s, b0;
    u32 meta, uo;
};
__device__ __forceinline__ void load_quarter(Packed& p, __amdgpu_buffer_rsrc_t rs, u32 uo, u32 meta, int q, int lane) {
    p.n = __builtin_amdgcn_raw_buffer_load_b128(rs, lane * 16 + q * 1024, uo, 17);
    p.s = __builtin_amdgcn_raw_buffer_load_b32(rs, lane * 4 + 4096 + q * 256, uo, 17);
    __builtin_amdgcn_sched_barrier(0);
    p.meta = meta;
    p.uo = uo;
    p.b0 = __builtin_amdgcn_raw_buffer_load_b32(rs, (u32)lane < (meta & 511u) ? lane * 4 : OOB, uo + 5120 + (meta >> 22) * 4, 17);
}
__device__ __forceinline__ u32 sym_bytes(u32 t, u32 k1, u32 kx) {
    const u32 h = (t + 0x02020202u) & 0x10101010u;
    return (((t << 1) + k1) + (h - (h >> 3))) ^ kx;
}
__device__ __forceinline__ u32x2 sym8(u32 x, u32 k1, u32 kx) {
    const u32 lo = sym_bytes(x & 0x0F0F0F0Fu, k1, kx), hi = sym_bytes((x >> 4) & 0x0F0F0F0Fu, k1, kx);
    return u32x2{__builtin_amdgcn_perm(hi, lo, 0x05010400u), __builtin_amdgcn_perm(hi, lo, 0x07030602u)};
}
__device__ __forceinline__ void patch(u32 e, bool on, LAS char* image) {
    if (on && (e & 127u))
        __hip_atomic_fetch_xor((LAS u32*)(image + ((e >> 8) & 0x1FFCu)), (e & 127u) << ((e >> 5) & 24u),
                               __ATOMIC_RELAXED, __HIP_MEMORY_SCOPE_WORKGROUP);
}
template <class Mid>
__device__ __forceinline__ void decode_quarter(const Packed& p, __amdgpu_buffer_rsrc_t rs, u32 img, int q,
                                               LAS char* image, LAS char* sym, int lane, Mid mid) {
    const u32 len = p.meta & 511u, uo = p.uo, bo = uo + 5120 + (p.meta >> 22) * 4;
    const u32 B = (p.meta >> 15) & 127u, nsd = len - ((p.meta >> 9) & 63u);
    const u32 k1 = (B - 26) * 0x01010101u, kx = (0xFFu ^ B) * 0x01010101u;
    if (len > 64 && len < RAW_DW) {
        const u32 b1 = __builtin_amdgcn_raw_buffer_load_b32(rs, (u32)lane + 64 < len ? lane * 4 + 256 : OOB, bo, 17);
        const u32 b2 = __builtin_amdgcn_raw_buffer_load_b32(rs, (u32)lane + 128 < len ? lane * 4 + 512 : OOB, bo, 17);
        *(LAS u32x2*)(sym + 512 + lane * 8) = sym8(b1, k1, kx);
        *(LAS u32x2*)(sym + 1024 + lane * 8) = sym8(b2, k1, kx);
    }
    u32 nib[8], sel[8], off[8], cnt = 0;
#pragma unroll
    for (int k = 0; k < 8; ++k) {
        nib[k] = (k & 1 ? p.n[k >> 1] >> 4 : p.n[k >> 1]) & 0x0F0F0F0Fu;
        const u32 n10 = (nib[k] + 0x0E0E0E0Eu) & 0x10101010u, e10 = n10 ^ 0x10101010u;
        sel[k] = __umul24(e10, 0x101010u) + n10;
        off[k] = cnt;
        cnt += __builtin_popcount(e10);
    }
    const u32 sp = (u32)(size_t)sym + (wave_scan(cnt) - cnt);
    *(LAS u32x2*)(sym + lane * 8) = sym8(p.b0, k1, kx);
    lds_order();
    u32 g0[8], g1[8], o[8];
#pragma unroll
    for (int k = 0; k < 8; ++k) {
        off[k] += sp;
        LAS u32* gp = (LAS u32*)(size_t)(off[k] & ~3u);
        g0[k] = gp[0]; g1[k] = gp[1];
    }
    __builtin_amdgcn_sched_barrier(0);
#pragma unroll
    for (int k = 0; k < 8; ++k) nib[k] |= (p.s << (7 - k)) & 0x80808080u;
    u32 ev = (u32)lane >= (len <= 64 ? nsd : 64u) ? (p.b0 & 127u) << ((p.b0 >> 5) & 24u) : 0u;
    u32 ea = (u32)(size_t)image + ((p.b0 >> 8) & 0x1FFCu);
    u32 wa = (u32)(size_t)image + img;
    asm volatile("" : "+v"(ev), "+v"(ea), "+v"(wa), "+v"(nib[0]), "+v"(nib[1]), "+v"(nib[2]), "+v"(nib[3]),
                      "+v"(nib[4]), "+v"(nib[5]), "+v"(nib[6]), "+v"(nib[7]));
    mid();
    __builtin_amdgcn_sched_barrier(0);
#pragma unroll
    for (int c = 0; c < 4; ++c) {
#pragma unroll
        for (int k = 2 * c; k < 2 * c + 2; ++k) {
            const u32 g = __builtin_amdgcn_alignbyte(g1[k], g0[k], off[k]);
            o[k] = (__builtin_amdgcn_perm(g, g, sel[k]) ^ kx) + nib[k];
        }
        *(LAS u32x2*)(size_t)(wa + c * 256) = u32x2{o[2 * c], o[2 * c + 1]};
    }
    lds_order();
    if (ev) __hip_atomic_fetch_xor((LAS u32*)(size_t)ea, ev, __ATOMIC_RELAXED, __HIP_MEMORY_SCOPE_WORKGROUP);
    if (len > 64) {
        if (len < RAW_DW) {
            patch(__builtin_amdgcn_raw_buffer_load_b32(rs, nsd + lane < len ? (nsd + lane) * 4 : OOB, bo, 17), true, image);
        } else {
            const u32x4 l = __builtin_amdgcn_raw_buffer_load_b128(rs, lane * 16 + q * 1024, uo, 17);
            const u32x4 h = __builtin_amdgcn_raw_buffer_load_b128(rs, lane * 16, bo, 17);
            *(LAS u32x2*)(size_t)wa = u32x2{l[0], l[1]};
            *(LAS u32x2*)(size_t)(wa + 256) = u32x2{l[2], l[3]};
            *(LAS u32x2*)(size_t)(wa + 512) = u32x2{h[0], h[1]};
            *(LAS u32x2*)(size_t)(wa + 768) = u32x2{h[2], h[3]};
        }
    }
}

__device__ __forceinline__ u32 q_img(int lane) {
    return (lane >> 5) * 1024 + ((lane & 7) | ((lane >> 1) & 8)) * 16 + (lane & 8);
}

struct Unit {
    u32x4 p[6];
    u32 meta;
    u32 c1, c2, c3, tot;
    u32 tot16;
    LAS char* lds;
    __device__ __forceinline__ u32x4 blk16(u32 i) const {
        u32x4 v;
#pragma unroll
        for (int k = 0; k < 4; ++k) {
            const u32 w = 4 * i + k;
            const u32 a = w + (w < c1 ? 0 : (w < c2 ? 256 - c1 : (w < c3 ? 512 - c2 : 768 - c3)));
            v[k] = *(LAS u32*)(lds + 5120 + (a & 1023u) * 4);
        }
        return v;
    }
    __device__ __forceinline__ void fetch(LAS char* unit, LAS u32* lens, int lane) {
        lds = unit;
        const u32 l = lens[lane & 3];
        c1 = __builtin_amdgcn_readlane(l, 0) & 511u;
        c2 = c1 + (__builtin_amdgcn_readlane(l, 1) & 511u);
        c3 = c2 + (__builtin_amdgcn_readlane(l, 2) & 511u);
        tot = c3 + (__builtin_amdgcn_readlane(l, 3) & 511u);
        tot16 = (tot + 3) >> 2;
        meta = l | ((lane == 0 ? 0 : (lane == 1 ? c1 : (lane == 2 ? c2 : c3))) << 22);
#pragma unroll
        for (int i = 0; i < 5; ++i) p[i] = *(LAS u32x4*)(unit + i * 1024 + lane * 16);
        p[5] = blk16(lane);
    }
    __device__ __forceinline__ void send_blocks(const char* dst, u32 voff, int lane) const {
        for (u32 i = 64 + lane; i < tot16; i += 64) st16<1024>(dst, voff + (i - lane) * 16, blk16(i));
        if ((u32)lane < tot16) st16<1024>(dst, voff, p[5]);
    }
    __device__ __forceinline__ void send_all(const char* dst, u32 voff, int lane) const {
        st16<-4096>(dst, voff, p[0]); st16<-3072>(dst, voff, p[1]); st16<-2048>(dst, voff, p[2]);
        st16<-1024>(dst, voff, p[3]); st16<0>(dst, voff, p[4]);
        send_blocks(dst, voff, lane);
    }
};

__device__ __forceinline__ void pace(u32 due) {
    while ((int)((u32)__builtin_amdgcn_s_memrealtime() - due) < 0) __builtin_amdgcn_s_sleep(1);
}

__device__ __forceinline__ void bar() {
    asm volatile("s_waitcnt lgkmcnt(0)" ::: "memory");
    __builtin_amdgcn_s_barrier();
    asm volatile("" ::: "memory");
}
__device__ __forceinline__ void enc_init(LAS char* lds, int wave, int lane) {
    *(LAS u32x4*)(lds + (wave >> 2) * USLOT + 5120 + (wave & 3) * 1024 + lane * 16) = u32x4{0, 0, 0, 0};
    u32 sel = 0x0C0C0C0Cu, n = 0;
#pragma unroll
    for (int i = 0; i < 4; ++i)
        if ((lane >> i) & 1) { sel = (sel & ~(0xFFu << n)) | ((u32)i << n); n += 8; }
    if (lane < 16) *(LAS u32*)(lds + 2 * USLOT + wave * 64 + lane * 4) = sel;
    lds_order();
}

__device__ __forceinline__ void anchor_wave(const char* self, int lane, GAS u32* cp, LAS char* lds, u32 cv) {
    const u32 call = __builtin_amdgcn_readfirstlane(cv) + 1;
    u32 v, now;
    u64 ok;
    int n = 0;
    __builtin_amdgcn_s_sleep(16);
    do {
        __builtin_amdgcn_s_sleep(4);
        v = __hip_atomic_load((GAS u32*)(self + OFF_ANC + (lane >> 2) * 128 + (lane & 3) * 4), __ATOMIC_RELAXED, __HIP_MEMORY_SCOPE_AGENT);
        ok = __ballot((v >> 24) == (call & 255u));
        now = (u32)__builtin_amdgcn_s_memrealtime();
    } while (ok != ~0ull && ++n < 8);
    u32 age = (now - v) & 0xFFFFFFu;
    age = (v >> 24) == (call & 255u) ? (age > 1000 ? 1000 : age) : 0;
    *(LAS u32*)(lds + 2 * USLOT + 512 + 64) = now - (__builtin_amdgcn_readlane(wave_scan(age), 63) >> 6);
    bar();
    if (lane == 0) __hip_atomic_store(cp, call, __ATOMIC_RELAXED, __HIP_MEMORY_SCOPE_AGENT);
    __builtin_amdgcn_endpgm();
}

__device__ __forceinline__ void sender(const Params& P, int b, int wave, int lane, GAS u32* cp, LAS char* lds, u32 cv) {
    const int rank = P.rank;
    char* const self = P.self;
    const int q = wave & 3, tl = wave >> 2;
    const int j = b & 63, kc0 = b >> 6;
    u32 wr[8];
    {
        const u32 cg = (lane & 7) | ((lane >> 1) & 8), rg = ((lane >> 3) & 1) | ((lane >> 4) & 2);
        const uint8_t* wp = P.w + (size_t)((kc0 + 2 * tl) * 128 + q * 32 + rg * 8) * GN + j * 64 + cg * 4;
#pragma unroll
        for (int i = 0; i < 8; ++i) wr[i] = *(const GAS u32*)(wp + (size_t)i * GN);
    }
    enc_init(lds, wave, lane);
    LAS char* const unit = lds + tl * USLOT;
    LAS u32* const lens = (LAS u32*)(lds + 2 * USLOT + 512);
    u32 dw[8];
#pragma unroll
    for (int h = 0; h < 2; ++h) {
        const u32 r0 = wr[4 * h], r1 = wr[4 * h + 1], r2 = wr[4 * h + 2], r3 = wr[4 * h + 3];
        const u32 t0 = __builtin_amdgcn_perm(r1, r0, 0x05010400u), t1 = __builtin_amdgcn_perm(r1, r0, 0x07030602u);
        const u32 t2 = __builtin_amdgcn_perm(r3, r2, 0x05010400u), t3 = __builtin_amdgcn_perm(r3, r2, 0x07030602u);
        dw[0 + h] = __builtin_amdgcn_perm(t2, t0, 0x05040100u);
        dw[2 + h] = __builtin_amdgcn_perm(t2, t0, 0x07060302u);
        dw[4 + h] = __builtin_amdgcn_perm(t3, t1, 0x05040100u);
        dw[6 + h] = __builtin_amdgcn_perm(t3, t1, 0x07060302u);
    }
    const u32 len = encode_quarter(dw, q * 2048 + q_img(lane), unit, q, lds + 2 * USLOT + wave * 64, lane,
                                   [&](u32x4 v, bool) { *(LAS u32x4*)(unit + q * 1024 + lane * 16) = v; });
    if (lane == 0) lens[wave] = len;
    const u32 call = __builtin_amdgcn_readfirstlane(cv) + 1;
    const u32 mine = (call & 1) * PAR_BYTES + rank * SRC_BYTES;
    const u32 voff = lane * 16 + 4096;
    bar();
    STAMP(3);
    if (q == 1 || q == 2) {
        const int kc = kc0 + 2 * tl, u = kc * 64 + j;
        const u32 wo = mine + SRC_W + u * USLOT, fo = mine + SRC_WF + u * 128;
        Unit U;
        U.fetch(unit, lens + tl * 4, lane);
        if (q == 2) {
            U.send_all(self + wo, voff, lane);
            wait_vm<0>();
            put_flag(self + fo, call, U.meta, lane);
        } else {
            const int est = __builtin_amdgcn_readlane(wave_scan(lens[lane & 7] & 511u), 7) / 8 + PACE_SLOT - USUAL16;
            const int slot = est > PACE_SLOT + 3 || est < PACE_SLOT - 4 ? est + 2 : PACE_SLOT;
            const u32 anchor = __builtin_amdgcn_readfirstlane(lens[16]);
            const u32 t0 = anchor + PACE_T0 * slot / PACE_SLOT;
            int pr = (kc + j) % 7;
#pragma unroll 1
            for (int s = 0; s < 7; ++s) {
                char* const peer = P.ws[(rank + 1 + pr) & 7];
                const char* const dst = peer + wo;
                pr = pr == 6 ? 0 : pr + 1;
                int before = 0;
#pragma unroll
                for (int k = 0; k < 4; ++k) {
                    const int v = 64 * s + j + PACE_KC * (kc - k);
                    before += v < 0 ? 0 : (v > 448 ? 448 : v);
                }
                const u32 due = t0 + ((before * slot) >> 8);
                pace(due);
                st16<-4096>(dst, voff, U.p[0]); __builtin_amdgcn_s_sleep(GAP);
                st16<-3072>(dst, voff, U.p[1]); __builtin_amdgcn_s_sleep(GAP);
                st16<-2048>(dst, voff, U.p[2]); __builtin_amdgcn_s_sleep(GAP);
                st16<-1024>(dst, voff, U.p[3]); __builtin_amdgcn_s_sleep(GAP);
                st16<0>(dst, voff, U.p[4]); __builtin_amdgcn_s_sleep(GAP);
                U.send_blocks(dst, voff, lane);
                wait_vm<0>();
                put_flag(peer + fo, call, U.meta, lane);
                STAMP(5 + s);
            }
            if (tl) STAMP(12);
        }
    }
    if (lane == 0) __hip_atomic_store(cp, call, __ATOMIC_RELAXED, __HIP_MEMORY_SCOPE_AGENT);
    __builtin_amdgcn_endpgm();
}

__device__ __forceinline__ u32 lds_add(LAS u32* c, int lane) {
    u32 v = 0;
    if (lane == 0) v = __hip_atomic_fetch_add(c, 1u, __ATOMIC_ACQ_REL, __HIP_MEMORY_SCOPE_WORKGROUP);
    return v;
}
__device__ __forceinline__ void lds_wait(LAS u32* c, u32 n) {
    while (__builtin_amdgcn_readfirstlane(__hip_atomic_load(c, __ATOMIC_ACQUIRE, __HIP_MEMORY_SCOPE_WORKGROUP)) < n)
        __builtin_amdgcn_s_sleep(1);
}
__device__ __forceinline__ int src_of(int rank, int j, int t) {
    return t < 4 ? rank : (rank + 7 - ((t >> 2) - 1 + (t & 3) + j) % 7) & 7;
}

__device__ __forceinline__ void mma_waves(const Params& P, int b, int wave, int lane, GAS u32* cp, LAS char* lds, u32x4 a, u32x4 c, u32 cv) {
    char* const ws = P.self;
    const int rank = P.rank;
    u32 cnt_a = (u32)(size_t)(lds + L_CNT);
    asm volatile("" : "+v"(cnt_a));
    LAS u32* const cnt = (LAS u32*)(size_t)cnt_a;
    u32 call;
    {
        const int q = wave & 3, tl = wave >> 2, ux = 2 * b + tl;
        const int xdest = (ux + (ux >> 3)) & 7, xu = ux >> 3;
        char* const xdst = P.ws[xdest];
        u32 dx[8] = {a[0], a[1], a[2], a[3], c[0], c[1], c[2], c[3]};
        LAS char* const el = lds + L_ENC;
        enc_init(el, wave, lane);
        LAS u32* const lens = (LAS u32*)(el + 2 * USLOT + 512);
        call = __builtin_amdgcn_readfirstlane(cv) + 1;
        const u32 mine = (call & 1) * PAR_BYTES + rank * SRC_BYTES;
        const u32 voff = lane * 16 + 4096;
        const char* const xslot = xdst + mine + SRC_X + xu * USLOT;
        if ((b & 7) == (b >> 4) && (wave & 1)) {
            asm volatile("" : "+v"(dx[0]), "+v"(dx[7]));
            if (lane == 0)
                __hip_atomic_store((GAS u32*)(ws + OFF_ANC + (b >> 3) * 128 + (wave >> 1) * 4),
                                   (call << 24) | ((u32)__builtin_amdgcn_s_memrealtime() & 0xFFFFFFu), __ATOMIC_RELAXED, __HIP_MEMORY_SCOPE_AGENT);
        }
        const u32 len = encode_quarter(dx, q * 2048 + q_img(lane), el + tl * USLOT, q, el + 2 * USLOT + wave * 64, lane, [&](u32x4 v, bool again) {
            if (again) wait_vm<0>();
            st16<-4096>(xslot + q * 1024, voff, v);
            STAMP(61);
        });
        if (lane == 0) lens[wave] = len;
        bar();
        if (q == 0) {
            Unit U;
            U.fetch(el + tl * USLOT, lens + tl * 4, lane);
            st16<0>(xslot, voff, U.p[4]);
            U.send_blocks(xslot, voff, lane);
            if (lane < 4) cnt[76 + tl * 4 + lane] = U.meta;
            STAMP(62);
        }
        wait_vm<0>();
        if (__builtin_amdgcn_readfirstlane(lds_add(cnt + 66 + tl, lane)) == 3) {
            put_flag(xdst + mine + SRC_XF + xu * 128, call, cnt[76 + tl * 4 + (lane & 3)], lane);
            STAMP(63);
        }
    }
    const int j = b & 63;
    const int rb = (b >> 6) * 8 + wave;
    GAS u32* const tmo = (GAS u32*)(ws + OFF_TMO);
    const __amdgpu_buffer_rsrc_t rs = __builtin_amdgcn_make_buffer_rsrc(ws, 0, WS_NEED, 0x00020000);
    const __amdgpu_buffer_rsrc_t ro = __builtin_amdgcn_make_buffer_rsrc(P.out, 0, MS * GN * 4, 0x00020000);
    const float scale = __builtin_bit_cast(float, __builtin_amdgcn_readfirstlane(__builtin_bit_cast(u32, *P.sx * *P.sw)));
    if (lane == 0) __hip_atomic_store(cp, call, __ATOMIC_RELAXED, __HIP_MEMORY_SCOPE_AGENT);
    LAS char* const rp = lds + (lane >> 4) * 2048 + (lane & 15) * 16;
    const u32 txo = OFF_XD + (src_of(rank, j, lane & 31) * 16 + rb) * 16384 + (lane & 3) * 4096;
    f32x4 acc[2][4] = {};
    u32x4 ae[4] = {}, ao[4] = {};
    auto load_a = [&](u32x4 (&a)[4], int t) {
        const u32 so = __builtin_amdgcn_readlane(txo, t);
        u32 l16 = lane * 16;
        asm volatile("" : "+v"(l16));
#pragma unroll
        for (int i = 0; i < 4; ++i) a[i] = __builtin_amdgcn_raw_buffer_load_b128(rs, l16 + i * 1024, so, 17);
    };
    LAS char* const stash = lds + L_STA + wave * 8192 + lane * 16;
    auto tile = [&](u32x4 (&a)[4], int t, u32 need, int more) {
        lds_wait(cnt + t, need);
        __builtin_amdgcn_s_setprio(2);
        LAS char* const bp = rp + (t & (NBUF - 1)) * 8192;
        u32x4 bq[8];
#pragma unroll
        for (int i = 0; i < 8; ++i) bq[i] = *(LAS u32x4*)(bp + (i >> 1) * 256 + (i & 1) * 1024);
        __builtin_amdgcn_sched_barrier(0);
#pragma unroll
        for (int c = 0; c < 4; ++c)
#pragma unroll
            for (int m = 0; m < 2; ++m) {
                const u32x4 a0 = a[2 * m], a1 = a[2 * m + 1], b0 = bq[2 * c], b1 = bq[2 * c + 1];
                const i32x8 af = {(int)a0[0], (int)a0[1], (int)a0[2], (int)a0[3], (int)a1[0], (int)a1[1], (int)a1[2], (int)a1[3]};
                const i32x8 bf = {(int)b0[0], (int)b0[1], (int)b0[2], (int)b0[3], (int)b1[0], (int)b1[1], (int)b1[2], (int)b1[3]};
                acc[m][c] = __builtin_amdgcn_mfma_scale_f32_16x16x128_f8f6f4(af, bf, acc[m][c], 1, 1, 0, 0x7F7F7F7F, 0, 0x7F7F7F7F);
            }
        asm volatile("" : "+v"(acc[0][0]), "+v"(acc[0][1]), "+v"(acc[0][2]), "+v"(acc[0][3]),
                          "+v"(acc[1][0]), "+v"(acc[1][1]), "+v"(acc[1][2]), "+v"(acc[1][3]));
        __builtin_amdgcn_sched_barrier(0);
        if (more == 1) {
            if (lane == 0) __hip_atomic_fetch_add(cnt + 32 + t, 1u, __ATOMIC_RELAXED, __HIP_MEMORY_SCOPE_WORKGROUP);
            load_a(a, t + 2);
        } else if (more == 2) {
#pragma unroll
            for (int i = 0; i < 4; ++i) a[i] = *(LAS u32x4*)(stash + (t & 1) * 4096 + i * 1024);
        }
        __builtin_amdgcn_s_setprio(0);
    };
#pragma unroll 1
    for (int pass = 0; pass < 2; ++pass) {
        if (pass) {
            wait_flags(ws + OFF_LF + (((lane >> 1) & 7) * 32 + rb * 2 + (lane & 1)) * 128, 0xFFFF, call, tmo);
            STAMP(3);
            load_a(ae, 30);
            load_a(ao, 31);
            lds_wait(cnt + 66, 4);
            lds_wait(cnt + 67, 4);
#pragma unroll
            for (int i = 0; i < 4; ++i) { *(LAS u32x4*)(stash + i * 1024) = ae[i]; *(LAS u32x4*)(stash + 4096 + i * 1024) = ao[i]; }
            __builtin_amdgcn_sched_barrier(0);
            load_a(ae, 0);
            __builtin_amdgcn_sched_barrier(0);
            load_a(ao, 1);
#pragma unroll
            for (int i = 0; i < 8; ++i) acc[i >> 2][i & 3] = f32x4{0.f, 0.f, 0.f, 0.f};
#pragma unroll 1
            for (int t = 0; t < 28; t += 2) {
                tile(ae, t, 0x01010101u, 1);
                tile(ao, t + 1, 0x01010101u, 1);
                if (wave == 0) STAMP(30 + (t >> 1));
            }
        }
        tile(ae, 28, pass * 0x01010101u, 2);
        tile(ao, 29, pass * 0x01010101u, 2);
        if (pass && wave == 0) STAMP(44);
        tile(ae, 30, pass * 0x01010101u, 0);
        tile(ao, 31, pass * 0x01010101u, 0);
        const float sc = cnt[72] ? __builtin_nanf("") : scale;
        if (pass) STAMP(59);
        const u32 vo = pass ? (lane >> 4) * 4 * GN * 4 + (lane & 15) * 16 : OOB;
        const u32 so = (rb * 32 * GN + j * 64) * 4;
#pragma unroll
        for (int m = 0; m < 2; ++m)
#pragma unroll
            for (int r = 0; r < 4; ++r) {
                u32x4 v;
#pragma unroll
                for (int c = 0; c < 4; ++c)
                    v[c] = __builtin_bit_cast(u32, __builtin_elementwise_maximum(acc[m][c][r] * sc, 0.f));
                __builtin_amdgcn_raw_buffer_store_b128(v, ro, vo, so + (m * 16 + r) * GN * 4, OUT_AUX);
                __builtin_amdgcn_sched_barrier(0);
                asm volatile("s_nop 1");
                __builtin_amdgcn_sched_barrier(0);
            }
    }
    STAMP(60);
    __builtin_amdgcn_endpgm();
}

__device__ __forceinline__ void unpack_waves(const Params& P, int b, int wave, int lane, GAS u32* cp, LAS char* lds, u32 cv) {
    char* const ws = P.self;
    const int rank = P.rank;
    LAS u32* const cnt = (LAS u32*)(lds + L_CNT);
    if (wave == 8) { cnt[lane] = 0; cnt[64 + lane] = 0; }
    bar();
    const u32 call = __builtin_amdgcn_readfirstlane(cv) + 1;
    const u32 par = (call & 1) * PAR_BYTES;
    const int q = wave & 3, tl = (wave >> 2) & 1;
    const int j = b & 63;
    GAS u32* const tmo = (GAS u32*)(ws + OFF_TMO);
    const __amdgpu_buffer_rsrc_t rs = __builtin_amdgcn_make_buffer_rsrc(ws, 0, WS_NEED, 0x00020000);
    LAS char* const sym = lds + L_SYM + (wave & 7) * 2304;
    const int ux = 2 * b + tl, xsrc = ux & 7, xu = ux >> 3;
    constexpr int NJOB = 17;
    u32 juo, jfo;
    {
        const int i = lane < NJOB ? lane : 0, t = i ? 2 * (i - 1) + tl : 0;
        const u32 so = par + (i ? src_of(rank, j, t) : xsrc) * SRC_BYTES;
        juo = so + (i ? SRC_W + ((t & 3) * 64 + j) * USLOT : SRC_X + xu * USLOT);
        jfo = lane < NJOB ? so + (i ? SRC_WF + ((t & 3) * 64 + j) * 128 : SRC_XF + xu * 128) + q * 8 : OFF_TMO;
    }
    const u32 img = q * 2048 + q_img(lane);
    LAS char* image = lds + L_XIM + tl * 8192;
    __builtin_amdgcn_s_setprio(1);
    Packed pk;
    u64 look = 0;
    bool have = false;
    u32 fre = 8;
#pragma unroll 1
    for (int i = 0; i < NJOB; ++i) {
        const int t = 2 * (i - 1) + tl;
        if (__builtin_amdgcn_readfirstlane(fre) < 8) lds_wait(cnt + 32 + t - NBUF, 8);
        if (!have) {
            u32 uo = __builtin_amdgcn_readlane(juo, i);
            const char* nf = ws + __builtin_amdgcn_readlane(jfo, i + 1);
            asm volatile("" : "+s"(uo), "+s"(nf));
            u32 meta = __builtin_amdgcn_readfirstlane((u32)look);
            if (__builtin_amdgcn_readfirstlane((u32)(look >> 32)) < call) meta = wait_flag(ws + __builtin_amdgcn_readlane(jfo, i), call, tmo);
            asm volatile("" ::: "memory");
            load_quarter(pk, rs, uo, meta, q, lane);
            look = __hip_atomic_load((GAS u64*)nf, __ATOMIC_RELAXED, __HIP_MEMORY_SCOPE_SYSTEM);
            if (i >= SJ) STAMP(4 + 3 * (i - 9));
        }
        decode_quarter(pk, rs, img, q, image, sym, lane, [&]() {
            if (t + 2 >= NBUF) fre = __hip_atomic_load(cnt + 32 + t + 2 - NBUF, __ATOMIC_RELAXED, __HIP_MEMORY_SCOPE_WORKGROUP);
            asm volatile("" : "+v"(look));
            if (i + 1 == NJOB) {
                if ((u32)look == call) cnt[72] = 1;
            } else {
                have = __builtin_amdgcn_readfirstlane((u32)(look >> 32)) >= call;
                asm volatile("" ::: "memory");
                if (have) load_quarter(pk, rs, __builtin_amdgcn_readlane(juo, i + 1), __builtin_amdgcn_readfirstlane((u32)look), q, lane);
                look = __hip_atomic_load((GAS u64*)(ws + __builtin_amdgcn_readlane(jfo, i + 1 + have)), __ATOMIC_RELAXED, __HIP_MEMORY_SCOPE_SYSTEM);
            }
        });
        if (i) {
            __hip_atomic_store((LAS unsigned char*)(cnt + t) + q, (unsigned char)1, __ATOMIC_RELEASE, __HIP_MEMORY_SCOPE_WORKGROUP);
        } else {
            lds_order();
            const u32x2 h0 = *(LAS u32x2*)(image + img), h1 = *(LAS u32x2*)(image + img + 256);
            const u32x2 h2 = *(LAS u32x2*)(image + img + 512), h3 = *(LAS u32x2*)(image + img + 768);
            const u32x4 v0 = {h0[0], h0[1], h1[0], h1[1]}, v1 = {h2[0], h2[1], h3[0], h3[1]};
            const char* xd = ws + OFF_XD + (xsrc * 32 + xu) * 8192 + q * 2048;
            st16<0>(xd, lane * 16, v0);
            st16<1024>(xd, lane * 16, v1);
            wait_vm<0>();
            if (__builtin_amdgcn_readfirstlane(lds_add(cnt + 68 + tl, lane)) == 3)
                put_flag(ws + OFF_LF + (xsrc * 32 + xu) * 128, call, 0, lane);
            STAMP(2);
        }
        image = lds + ((t + 2) & (NBUF - 1)) * 8192;
        if (i >= SJ - 1) STAMP(6 + 3 * (i - 9));
        else if (i < 9 && SJ == 9) STAMP(48 + i);
    }
    if (lane == 0) __hip_atomic_store(cp, call, __ATOMIC_RELAXED, __HIP_MEMORY_SCOPE_AGENT);
    __builtin_amdgcn_endpgm();
}

__global__ void __launch_bounds__(1024, 4) gemm_rs_kernel(Params P) {
    __shared__ __attribute__((aligned(16))) char smem[LDS_BYTES];
    const int lane = threadIdx.x & 63;
    const int wave = __builtin_amdgcn_readfirstlane(threadIdx.x >> 6);
    u32x4 xa, xc;
    {
        const u32 ux = 2 * (blockIdx.x - N_SEND_WG) + (wave >> 2), xdest = (ux + (ux >> 3)) & 7, xu = (ux >> 3) & 31;
        const __amdgpu_buffer_rsrc_t rx = __builtin_amdgcn_make_buffer_rsrc((void*)P.x, 0, 4096 * KS, 0x00020000);
        const u32 so = (xdest * MS + (xu >> 1) * 32 + (wave & 1) * 16) * KS + (xu & 1) * 256 + (wave & 2) * 64;
        const u32 vo = ((lane & 15) * KS + (lane >> 4) * 32) | (blockIdx.x >= N_SEND_WG && wave < 8 ? 0 : OOB);
        xa = __builtin_amdgcn_raw_buffer_load_b128(rx, vo, so, 0);
        xc = __builtin_amdgcn_raw_buffer_load_b128(rx, vo + 16, so, 0);
        asm volatile("" ::: "memory");
    }
    GAS u32* cp = (GAS u32*)(P.self + OFF_CNT) + blockIdx.x * 16 + wave;
    const u32 cv = __hip_atomic_load(cp, __ATOMIC_RELAXED, __HIP_MEMORY_SCOPE_AGENT);
    asm volatile("" ::: "memory");
    STAMP(0);
    if (blockIdx.x >= N_SEND_WG) {
        if (wave < 8) mma_waves(P, blockIdx.x - N_SEND_WG, wave, lane, cp, (LAS char*)smem, xa, xc, cv);
        else unpack_waves(P, blockIdx.x - N_SEND_WG, wave, lane, cp, (LAS char*)smem, cv);
    } else {
        if (wave < 8) sender(P, blockIdx.x, wave, lane, cp, (LAS char*)smem, cv);
        else if (wave == 8) anchor_wave(P.self, lane, cp, (LAS char*)smem, cv);
    }
}

extern "C" void kernel_launch(int rank, int nranks, ncclComm_t comm,
                              hipStream_t stream,
                              void* const* d_in, const int* in_sizes, int n_in,
                              void* d_out, int out_size,
                              const int* in_dtypes, int out_dtype,
                              void* const* d_out_all,
                              void* const* d_ws_all, size_t ws_bytes) {
    if (nranks != NR || n_in != 4 || ws_bytes < WS_NEED || (unsigned)rank >= (unsigned)NR ||
        in_sizes[0] != 4096 * KS || in_sizes[1] != KS * GN || out_size != MS * GN) {
        fprintf(stderr, "kernel_launch: unexpected problem shape\n");
        return;
    }
    Params P;
    P.x = (const uint8_t*)d_in[0];
    P.w = (const uint8_t*)d_in[1];
    P.sx = (const float*)d_in[2];
    P.sw = (const float*)d_in[3];
    P.out = (float*)d_out;
    for (int p = 0; p < NR; ++p) P.ws[p] = (char*)d_ws_all[p];
    P.self = P.ws[rank];
    P.rank = rank;
    P.pad = 0;
    gemm_rs_kernel<<<GRID, 1024, 0, stream>>>(P);
    HIP_CHECK(hipGetLastError());
}
```

```cpp
#include <hip/hip_runtime.h>
#include <rccl/rccl.h>
#include <cstdio>
#include <cstdint>

#define HIP_CHECK(expr) do { hipError_t _e = (expr); if (_e != hipSuccess) { \
    fprintf(stderr, "HIP error %s at %s:%d\n", hipGetErrorString(_e), __FILE__, __LINE__); \
    return; } } while (0)

typedef unsigned u32;
typedef unsigned long long u64;
typedef u32 u32x4 __attribute__((ext_vector_type(4)));
typedef u32 u32x2 __attribute__((ext_vector_type(2)));
typedef int i32x8 __attribute__((ext_vector_type(8)));
typedef float f32x4 __attribute__((ext_vector_type(4)));
#define GAS __attribute__((address_space(1)))
#define LAS __attribute__((address_space(3)))

#define PACE_T0 322
#define PACE_SLOT 376
#define PACE_KC 16
#define USUAL16 51
#define GAP 2
#define POLL_SLEEP 6
#define OUT_AUX 17
#define FORCE_RAW 0

constexpr int NR = 8;
constexpr int GN = 4096;
constexpr int KS = 512;
constexpr int MS = 512;
constexpr int NUNIT = 256;
constexpr int N_SEND_WG = 128;
constexpr int N_CONS_WG = 128;
constexpr int GRID = N_SEND_WG + N_CONS_WG;

constexpr u32 USLOT = 9216;
constexpr u32 GSLOT = 6144, NEAR_DW = 224, G_FLAG = 5120 + NEAR_DW * 4;
__host__ __device__ constexpr u32 w_slot(u32 src, u32 kc, u32 j, u32 call) {
    return (((((j & 7) * 8 + src) * 8 + (j >> 3)) * 4 + kc) * 2 + (call & 1)) * GSLOT;
}
__host__ __device__ constexpr u32 x_slot(u32 src, u32 xu, u32 call) { return (4096 + (src * 32 + xu) * 2 + (call & 1)) * GSLOT; }
constexpr u32 OFF_XD = x_slot(NR, 0, 0);
constexpr u32 OFF_LF = OFF_XD + NR * MS * KS;
constexpr u32 OFF_CNT = OFF_LF + NUNIT * 128;
constexpr u32 OFF_TMO = OFF_CNT + GRID * 16 * 4;
constexpr u32 OFF_ANC = OFF_TMO + 128;
constexpr u32 OFF_DBG = OFF_ANC + 16 * 128;
constexpr u32 FAR = OFF_DBG + GRID * 16 * 64 * 8;
constexpr u32 WS_NEED = FAR + OFF_XD;
constexpr u32 DEADLINE_TICKS = 100000000u;
constexpr u32 OOB = 0x80000000u;
constexpr u32 RAW_DW = 256;
constexpr u32 CAP_DW = 192;
constexpr int NBUF = 8;
constexpr u32 L_SYM = NBUF * 8192;
constexpr u32 L_XIM = 4 * 8192;
constexpr u32 L_ENC = L_SYM + 8 * 2304;
constexpr u32 L_STA = L_ENC;
constexpr u32 L_CNT = L_STA + 8 * 8192;
constexpr int LDS_BYTES = L_CNT + 512;

struct Params {
    const uint8_t* x;
    const uint8_t* w;
    const float* sx;
    const float* sw;
    float* out;
    char* self;
    char* ws[NR];
    int rank;
    int pad;
};

#define SJ 16
#define STAMP(i) do { } while (0)

template <int OFF>
__device__ __forceinline__ void st16(const char* base, u32 voff, u32x4 v) {
    asm volatile("global_store_dwordx4 %0, %1, %2 offset:%3 sc0 sc1\n\ts_nop 1"
                 :: "v"(voff), "v"(v), "s"(base), "i"(OFF) : "memory");
}
template <int N> __device__ __forceinline__ void wait_vm() { asm volatile("s_waitcnt vmcnt(%0)" :: "i"(N) : "memory"); }

__device__ __forceinline__ void put_flag(char* p, u32 call, u32 meta, int lane) {
    if (lane < 4)
        __hip_atomic_store((GAS u64*)p + lane, ((u64)call << 32) | meta, __ATOMIC_RELAXED, __HIP_MEMORY_SCOPE_SYSTEM);
}

__device__ __forceinline__ u64 wait_flags(const char* f, u64 need, u32 call, GAS u32* tmo) {
    const u32 t0 = (u32)__builtin_amdgcn_s_memrealtime();
    u64 v;
    for (;;) {
        v = __hip_atomic_load((GAS u64*)f, __ATOMIC_RELAXED, __HIP_MEMORY_SCOPE_SYSTEM);
        const u32 t = __hip_atomic_load(tmo, __ATOMIC_RELAXED, __HIP_MEMORY_SCOPE_AGENT);
        if ((__ballot((u32)(v >> 32) >= call) & need) == need) break;
        if (__builtin_amdgcn_readfirstlane(t) == call || (u32)__builtin_amdgcn_s_memrealtime() - t0 > DEADLINE_TICKS) {
            __hip_atomic_store(tmo, call, __ATOMIC_RELAXED, __HIP_MEMORY_SCOPE_AGENT);
            break;
        }
        __builtin_amdgcn_s_sleep(1);
    }
    asm volatile("" ::: "memory");
    return v;
}

__device__ __forceinline__ u32 wait_flag(const char* f, u32 call, GAS u32* tmo) {
    const u32 t0 = (u32)__builtin_amdgcn_s_memrealtime();
    u32 meta;
    u64 va = __hip_atomic_load((GAS u64*)f, __ATOMIC_RELAXED, __HIP_MEMORY_SCOPE_SYSTEM);
    __builtin_amdgcn_s_sleep(POLL_SLEEP);
    for (u32 n = 1;; ++n) {
        const u64 vb = __hip_atomic_load((GAS u64*)f, __ATOMIC_RELAXED, __HIP_MEMORY_SCOPE_SYSTEM);
        meta = __builtin_amdgcn_readfirstlane((u32)va);
        if (__builtin_amdgcn_readfirstlane((u32)(va >> 32)) >= call) break;
        va = __hip_atomic_load((GAS u64*)f, __ATOMIC_RELAXED, __HIP_MEMORY_SCOPE_SYSTEM);
        meta = __builtin_amdgcn_readfirstlane((u32)vb);
        if (__builtin_amdgcn_readfirstlane((u32)(vb >> 32)) >= call) break;
        if ((n & 15) == 0) {
            const u32 t = __hip_atomic_load(tmo, __ATOMIC_RELAXED, __HIP_MEMORY_SCOPE_AGENT);
            if (__builtin_amdgcn_readfirstlane(t) == call || (u32)__builtin_amdgcn_s_memrealtime() - t0 > DEADLINE_TICKS) {
                __hip_atomic_store(tmo, call, __ATOMIC_RELAXED, __HIP_MEMORY_SCOPE_AGENT);
                break;
            }
            va = __hip_atomic_load((GAS u64*)f, __ATOMIC_RELAXED, __HIP_MEMORY_SCOPE_SYSTEM);
            __builtin_amdgcn_s_sleep(POLL_SLEEP);
        }
    }
    asm volatile("" ::: "memory");
    return meta;
}

__device__ __forceinline__ u32 wave_scan(u32 v) {
    v += __builtin_amdgcn_update_dpp(0u, v, 0x111, 0xf, 0xf, true);
    v += __builtin_amdgcn_update_dpp(0u, v, 0x112, 0xf, 0xf, true);
    v += __builtin_amdgcn_update_dpp(0u, v, 0x114, 0xf, 0xf, true);
    v += __builtin_amdgcn_update_dpp(0u, v, 0x118, 0xf, 0xf, true);
    v += __builtin_amdgcn_update_dpp(0u, v, 0x142, 0xa, 0xf, false);
    v += __builtin_amdgcn_update_dpp(0u, v, 0x143, 0xc, 0xf, false);
    return v;
}
__device__ __forceinline__ u32 bfi(u32 m, u32 a, u32 b) { return (m & a) | (~m & b); }
__device__ __forceinline__ void lds_order() { __builtin_amdgcn_fence(__ATOMIC_ACQ_REL, "wavefront"); }

template <class Early>
__device__ __forceinline__ u32 encode_quarter(const u32 (&d)[8], u32 img, LAS char* unit, int q,
                                              LAS char* lut, int lane, Early codes_out) {
    u32 mag[8], sum = 0;
#pragma unroll
    for (int k = 0; k < 8; ++k) { mag[k] = d[k] & 0x7F7F7F7Fu; sum = __builtin_amdgcn_sad_u8(mag[k], 0u, sum); }
    int b = ((int)__builtin_amdgcn_readlane(wave_scan(sum), 63) + 1024 - 14541) >> 11;
    b = b < 26 ? 26 : b;
    asm volatile("" : "+s"(b));
    const u32 B = b > 108 ? 108 : b;
    const u32 c2 = (126 - B) * 0x01010101u, c16 = (112 - B) * 0x01010101u, c26 = (154 - B) * 0x01010101u, c20 = (108 - B) * 0x01010101u;
    u32 nib[8], w[8], esc[8];
#pragma unroll
    for (int k = 0; k < 8; ++k) {
        const u32 u = mag[k] + c2;
        w[k] = mag[k] + c16;
        const u32 dir = u & ~w[k] & 0x80808080u;
        nib[k] = w[k] & (((dir >> 3) - (dir >> 6)) | 0x01010101u);
        esc[k] = dir ^ 0x80808080u;
    }
    codes_out(u32x4{nib[0] | (nib[1] << 4), nib[2] | (nib[3] << 4), nib[4] | (nib[5] << 4), nib[6] | (nib[7] << 4)}, false);
    u32 sel[8], t[8], off[8], cnt = 0, sg = 0, mb = 0;
#pragma unroll
    for (int k = 0; k < 8; ++k) sel[k] = *(LAS u32*)(lut + (__builtin_amdgcn_udot4(esc[k], 0x08040201u, 0u, false) >> 5));
#pragma unroll
    for (int k = 0; k < 8; ++k) {
        const u32 hi = w[k] & 0x80808080u, q0 = mag[k] + c26;
        t[k] = ((q0 - ((hi >> 3) - (hi >> 6))) >> 1) & 0x0F0F0F0Fu;
        mb |= ((~q0 | (mag[k] + c20)) & 0x80808080u) >> (7 - k);
        off[k] = cnt;
        cnt += __builtin_popcount(esc[k]);
        sg |= (d[k] >> (7 - k)) & (0x01010101u << k);
    }
    const u32 incl = wave_scan(cnt);
    const u32 nsd = (__builtin_amdgcn_readlane(incl, 63) + 7) >> 3;
    LAS char* blk = unit + 5120 + q * 1024;
    *(LAS u32*)(unit + 4096 + q * 256 + lane * 4) = sg;
    u32 ndw = nsd;
    if (ndw <= CAP_DW && !FORCE_RAW) {
        const u32 base = incl - cnt;
#pragma unroll
        for (int i = 0; i < 4; ++i) {
            u32 c0 = __builtin_amdgcn_perm(0u, t[2 * i], sel[2 * i]), c1 = __builtin_amdgcn_perm(0u, t[2 * i + 1], sel[2 * i + 1]);
            c0 = __builtin_amdgcn_perm(0u, c0 | (c0 >> 4), 0x0C0C0200u);
            c1 = __builtin_amdgcn_perm(0u, c1 | (c1 >> 4), 0x0C0C0200u);
            const u32 pos = base + off[2 * i];
            const u64 v = (u64)(c0 | (c1 << (4 * (off[2 * i + 1] - off[2 * i])))) << (4 * (pos & 7));
            LAS u32* p = (LAS u32*)(blk + ((pos >> 1) & ~3u));
            __hip_atomic_fetch_or(p, (u32)v, __ATOMIC_RELAXED, __HIP_MEMORY_SCOPE_WORKGROUP);
            if ((u32)(v >> 32)) __hip_atomic_fetch_or(p + 1, (u32)(v >> 32), __ATOMIC_RELAXED, __HIP_MEMORY_SCOPE_WORKGROUP);
        }
        lds_order();
        for (u64 mk; (mk = __ballot(mb != 0)) != 0; ndw += __builtin_popcountll(mk)) {
            if (mb != 0) {
                const u32 bit = __builtin_ctz(mb);
                mb &= mb - 1;
                const u32 k0 = (u32)((int)(bit << 31) >> 31), k1 = (u32)((int)(bit << 30) >> 31), k2 = (u32)((int)(bit << 29) >> 31);
                const u32 m0 = bfi(k2, mag[4], mag[0]), m1 = bfi(k2, mag[5], mag[1]), m2 = bfi(k2, mag[6], mag[2]), m3 = bfi(k2, mag[7], mag[3]);
                const u32 mg = (bfi(k0, bfi(k1, m3, m1), bfi(k1, m2, m0)) >> (bit & 24u)) & 127u;
                const u32 ad = img + (bit & 6u) * 128 + (k0 & 4u) + (bit >> 3);
                const u32 z = (mg + 154 - B - (mg >= B + 16 ? 14 : 0)) & 31u;
                const u32 dflt = B - 26 + z + (z >= 28 ? 14 : 0);
                const u32 slot = ndw + __builtin_amdgcn_mbcnt_hi((u32)(mk >> 32), __builtin_amdgcn_mbcnt_lo((u32)mk, 0u));
                *(LAS u32*)(blk + (slot < 255 ? slot : 255) * 4) = (ad << 8) | (mg ^ dflt);
            }
        }
    }
    if (ndw > CAP_DW || ndw - nsd > 31 || FORCE_RAW) {
        lds_order();
        codes_out(u32x4{d[0], d[1], d[2], d[3]}, true);
        *(LAS u32x4*)(blk + lane * 16) = u32x4{d[4], d[5], d[6], d[7]};
        return RAW_DW;
    }
    return ndw | ((ndw - nsd) << 9) | (B << 15);
}

struct Packed {
    u32x4 n;
    u32 s, b0;
    u32 meta, uo;
};
__device__ __forceinline__ u32 blk_off(u32 uo, u32 meta) { return uo + (meta & 0x4000u ? FAR : 5120) + (meta >> 22) * 4; }
__device__ __forceinline__ void load_quarter(Packed& p, __amdgpu_buffer_rsrc_t rs, u32 uo, u32 meta, int q, int lane) {
    p.n = __builtin_amdgcn_raw_buffer_load_b128(rs, lane * 16 + q * 1024, uo, 17);
    p.s = __builtin_amdgcn_raw_buffer_load_b32(rs, lane * 4 + 4096 + q * 256, uo, 17);
    __builtin_amdgcn_sched_barrier(0);
    p.meta = meta;
    p.uo = uo;
    p.b0 = __builtin_amdgcn_raw_buffer_load_b32(rs, (u32)lane < (meta & 511u) ? lane * 4 : OOB, blk_off(uo, meta), 17);
}
__device__ __forceinline__ u32 sym_bytes(u32 t, u32 k1, u32 kx) {
    const u32 h = (t + 0x02020202u) & 0x10101010u;
    return (((t << 1) + k1) + (h - (h >> 3))) ^ kx;
}
__device__ __forceinline__ u32x2 sym8(u32 x, u32 k1, u32 kx) {
    const u32 lo = sym_bytes(x & 0x0F0F0F0Fu, k1, kx), hi = sym_bytes((x >> 4) & 0x0F0F0F0Fu, k1, kx);
    return u32x2{__builtin_amdgcn_perm(hi, lo, 0x05010400u), __builtin_amdgcn_perm(hi, lo, 0x07030602u)};
}
__device__ __forceinline__ void patch(u32 e, bool on, LAS char* image) {
    if (on && (e & 127u))
        __hip_atomic_fetch_xor((LAS u32*)(image + ((e >> 8) & 0x1FFCu)), (e & 127u) << ((e >> 5) & 24u),
                               __ATOMIC_RELAXED, __HIP_MEMORY_SCOPE_WORKGROUP);
}
template <class Mid>
__device__ __forceinline__ void decode_quarter(const Packed& p, __amdgpu_buffer_rsrc_t rs, u32 img, int q,
                                               LAS char* image, LAS char* sym, int lane, Mid mid) {
    const u32 len = p.meta & 511u, uo = p.uo, bo = blk_off(uo, p.meta);
    const u32 B = (p.meta >> 15) & 127u, nsd = len - ((p.meta >> 9) & 31u);
    const u32 k1 = (B - 26) * 0x01010101u, kx = (0xFFu ^ B) * 0x01010101u;
    if (len > 64 && len < RAW_DW) {
        const u32 b1 = __builtin_amdgcn_raw_buffer_load_b32(rs, (u32)lane + 64 < len ? lane * 4 + 256 : OOB, bo, 17);
        const u32 b2 = __builtin_amdgcn_raw_buffer_load_b32(rs, (u32)lane + 128 < len ? lane * 4 + 512 : OOB, bo, 17);
        *(LAS u32x2*)(sym + 512 + lane * 8) = sym8(b1, k1, kx);
        *(LAS u32x2*)(sym + 1024 + lane * 8) = sym8(b2, k1, kx);
    }
    u32 nib[8], sel[8], off[8], cnt = 0;
#pragma unroll
    for (int k = 0; k < 8; ++k) {
        nib[k] = (k & 1 ? p.n[k >> 1] >> 4 : p.n[k >> 1]) & 0x0F0F0F0Fu;
        const u32 n10 = (nib[k] + 0x0E0E0E0Eu) & 0x10101010u, e10 = n10 ^ 0x10101010u;
        sel[k] = __umul24(e10, 0x101010u) + n10;
        off[k] = cnt;
        cnt += __builtin_popcount(e10);
    }
    const u32 sp = (u32)(size_t)sym + (wave_scan(cnt) - cnt);
    *(LAS u32x2*)(sym + lane * 8) = sym8(p.b0, k1, kx);
    lds_order();
    u32 g0[8], g1[8], o[8];
#pragma unroll
    for (int k = 0; k < 8; ++k) {
        off[k] += sp;
        LAS u32* gp = (LAS u32*)(size_t)(off[k] & ~3u);
        g0[k] = gp[0]; g1[k] = gp[1];
    }
    __builtin_amdgcn_sched_barrier(0);
#pragma unroll
    for (int k = 0; k < 8; ++k) nib[k] |= (p.s << (7 - k)) & 0x80808080u;
    u32 ev = (u32)lane >= (len <= 64 ? nsd : 64u) ? (p.b0 & 127u) << ((p.b0 >> 5) & 24u) : 0u;
    u32 ea = (u32)(size_t)image + ((p.b0 >> 8) & 0x1FFCu);
    u32 wa = (u32)(size_t)image + img;
    asm volatile("" : "+v"(ev), "+v"(ea), "+v"(wa), "+v"(nib[0]), "+v"(nib[1]), "+v"(nib[2]), "+v"(nib[3]),
                      "+v"(nib[4]), "+v"(nib[5]), "+v"(nib[6]), "+v"(nib[7]));
    mid();
    __builtin_amdgcn_sched_barrier(0);
#pragma unroll
    for (int c = 0; c < 4; ++c) {
#pragma unroll
        for (int k = 2 * c; k < 2 * c + 2; ++k) {
            const u32 g = __builtin_amdgcn_alignbyte(g1[k], g0[k], off[k]);
            o[k] = (__builtin_amdgcn_perm(g, g, sel[k]) ^ kx) + nib[k];
        }
        *(LAS u32x2*)(size_t)(wa + c * 256) = u32x2{o[2 * c], o[2 * c + 1]};
    }
    lds_order();
    if (ev) __hip_atomic_fetch_xor((LAS u32*)(size_t)ea, ev, __ATOMIC_RELAXED, __HIP_MEMORY_SCOPE_WORKGROUP);
    if (len > 64) {
        if (len < RAW_DW) {
            patch(__builtin_amdgcn_raw_buffer_load_b32(rs, nsd + lane < len ? (nsd + lane) * 4 : OOB, bo, 17), true, image);
        } else {
            const u32x4 l = __builtin_amdgcn_raw_buffer_load_b128(rs, lane * 16 + q * 1024, uo, 17);
            const u32x4 h = __builtin_amdgcn_raw_buffer_load_b128(rs, lane * 16, bo, 17);
            *(LAS u32x2*)(size_t)wa = u32x2{l[0], l[1]};
            *(LAS u32x2*)(size_t)(wa + 256) = u32x2{l[2], l[3]};
            *(LAS u32x2*)(size_t)(wa + 512) = u32x2{h[0], h[1]};
            *(LAS u32x2*)(size_t)(wa + 768) = u32x2{h[2], h[3]};
        }
    }
}

__device__ __forceinline__ u32 q_img(int lane) {
    return (lane >> 5) * 1024 + ((lane & 7) | ((lane >> 1) & 8)) * 16 + (lane & 8);
}

struct Unit {
    u32x4 p[6];
    u32 meta;
    u32 c1, c2, c3, tot;
    u32 tot16;
    LAS char* lds;
    __device__ __forceinline__ u32x4 blk16(u32 i) const {
        u32x4 v;
#pragma unroll
        for (int k = 0; k < 4; ++k) {
            const u32 w = 4 * i + k;
            const u32 a = w + (w < c1 ? 0 : (w < c2 ? 256 - c1 : (w < c3 ? 512 - c2 : 768 - c3)));
            v[k] = *(LAS u32*)(lds + 5120 + (a & 1023u) * 4);
        }
        return v;
    }
    __device__ __forceinline__ void fetch(LAS char* unit, LAS u32* lens, int lane) {
        lds = unit;
        const u32 l = lens[lane & 3];
        c1 = __builtin_amdgcn_readlane(l, 0) & 511u;
        c2 = c1 + (__builtin_amdgcn_readlane(l, 1) & 511u);
        c3 = c2 + (__builtin_amdgcn_readlane(l, 2) & 511u);
        tot = c3 + (__builtin_amdgcn_readlane(l, 3) & 511u);
        tot16 = (tot + 3) >> 2;
        meta = l | (tot > NEAR_DW ? 0x4000u : 0u) | ((lane == 0 ? 0 : (lane == 1 ? c1 : (lane == 2 ? c2 : c3))) << 22);
#pragma unroll
        for (int i = 0; i < 5; ++i) p[i] = *(LAS u32x4*)(unit + i * 1024 + lane * 16);
        p[5] = blk16(lane);
    }
    __device__ __forceinline__ void send_blocks(const char* dst, u32 voff, int lane) const {
        if (tot > NEAR_DW) dst += FAR - 5120;
        for (u32 i = 64 + lane; i < tot16; i += 64) st16<1024>(dst, voff + (i - lane) * 16, blk16(i));
        if ((u32)lane < tot16) st16<1024>(dst, voff, p[5]);
    }
    __device__ __forceinline__ void send_all(const char* dst, u32 voff, int lane) const {
        st16<-4096>(dst, voff, p[0]); st16<-3072>(dst, voff, p[1]); st16<-2048>(dst, voff, p[2]);
        st16<-1024>(dst, voff, p[3]); st16<0>(dst, voff, p[4]);
        send_blocks(dst, voff, lane);
    }
};

__device__ __forceinline__ void pace(u32 due) {
    while ((int)((u32)__builtin_amdgcn_s_memrealtime() - due) < 0) __builtin_amdgcn_s_sleep(1);
}

__device__ __forceinline__ void bar() {
    asm volatile("s_waitcnt lgkmcnt(0)" ::: "memory");
    __builtin_amdgcn_s_barrier();
    asm volatile("" ::: "memory");
}
__device__ __forceinline__ void enc_init(LAS char* lds, int wave, int lane) {
    *(LAS u32x4*)(lds + (wave >> 2) * USLOT + 5120 + (wave & 3) * 1024 + lane * 16) = u32x4{0, 0, 0, 0};
    u32 sel = 0x0C0C0C0Cu, n = 0;
#pragma unroll
    for (int i = 0; i < 4; ++i)
        if ((lane >> i) & 1) { sel = (sel & ~(0xFFu << n)) | ((u32)i << n); n += 8; }
    if (lane < 16) *(LAS u32*)(lds + 2 * USLOT + wave * 64 + lane * 4) = sel;
    lds_order();
}

__device__ __forceinline__ void anchor_wave(const char* self, int lane, GAS u32* cp, LAS char* lds, u32 cv) {
    const u32 call = __builtin_amdgcn_readfirstlane(cv) + 1;
    u32 v, now;
    u64 ok;
    int n = 0;
    __builtin_amdgcn_s_sleep(16);
    do {
        __builtin_amdgcn_s_sleep(4);
        v = __hip_atomic_load((GAS u32*)(self + OFF_ANC + (lane >> 2) * 128 + (lane & 3) * 4), __ATOMIC_RELAXED, __HIP_MEMORY_SCOPE_AGENT);
        ok = __ballot((v >> 24) == (call & 255u));
        now = (u32)__builtin_amdgcn_s_memrealtime();
    } while (ok != ~0ull && ++n < 8);
    u32 age = (now - v) & 0xFFFFFFu;
    age = (v >> 24) == (call & 255u) ? (age > 1000 ? 1000 : age) : 0;
    *(LAS u32*)(lds + 2 * USLOT + 512 + 64) = now - (__builtin_amdgcn_readlane(wave_scan(age), 63) >> 6);
    bar();
    if (lane == 0) __hip_atomic_store(cp, call, __ATOMIC_RELAXED, __HIP_MEMORY_SCOPE_AGENT);
    __builtin_amdgcn_endpgm();
}

__device__ __forceinline__ void sender(const Params& P, int b, int wave, int lane, GAS u32* cp, LAS char* lds, u32 cv) {
    const int rank = P.rank;
    char* const self = P.self;
    const int q = wave & 3, tl = wave >> 2;
    const int j = b & 63, kc0 = b >> 6;
    u32 wr[8];
    {
        const u32 cg = (lane & 7) | ((lane >> 1) & 8), rg = ((lane >> 3) & 1) | ((lane >> 4) & 2);
        const uint8_t* wp = P.w + (size_t)((kc0 + 2 * tl) * 128 + q * 32 + rg * 8) * GN + j * 64 + cg * 4;
#pragma unroll
        for (int i = 0; i < 8; ++i) wr[i] = *(const GAS u32*)(wp + (size_t)i * GN);
    }
    enc_init(lds, wave, lane);
    LAS char* const unit = lds + tl * USLOT;
    LAS u32* const lens = (LAS u32*)(lds + 2 * USLOT + 512);
    u32 dw[8];
#pragma unroll
    for (int h = 0; h < 2; ++h) {
        const u32 r0 = wr[4 * h], r1 = wr[4 * h + 1], r2 = wr[4 * h + 2], r3 = wr[4 * h + 3];
        const u32 t0 = __builtin_amdgcn_perm(r1, r0, 0x05010400u), t1 = __builtin_amdgcn_perm(r1, r0, 0x07030602u);
        const u32 t2 = __builtin_amdgcn_perm(r3, r2, 0x05010400u), t3 = __builtin_amdgcn_perm(r3, r2, 0x07030602u);
        dw[0 + h] = __builtin_amdgcn_perm(t2, t0, 0x05040100u);
        dw[2 + h] = __builtin_amdgcn_perm(t2, t0, 0x07060302u);
        dw[4 + h] = __builtin_amdgcn_perm(t3, t1, 0x05040100u);
        dw[6 + h] = __builtin_amdgcn_perm(t3, t1, 0x07060302u);
    }
    const u32 len = encode_quarter(dw, q * 2048 + q_img(lane), unit, q, lds + 2 * USLOT + wave * 64, lane,
                                   [&](u32x4 v, bool) { *(LAS u32x4*)(unit + q * 1024 + lane * 16) = v; });
    if (lane == 0) lens[wave] = len;
    const u32 call = __builtin_amdgcn_readfirstlane(cv) + 1;
    const u32 voff = lane * 16 + 4096;
    bar();
    STAMP(3);
    if (q == 1 || q == 2) {
        const int kc = kc0 + 2 * tl;
        const u32 wo = w_slot(rank, kc, j, call), fo = wo + G_FLAG;
        Unit U;
        U.fetch(unit, lens + tl * 4, lane);
        if (q == 2) {
            U.send_all(self + wo, voff, lane);
            wait_vm<0>();
            put_flag(self + fo, call, U.meta, lane);
        } else {
            const int est = __builtin_amdgcn_readlane(wave_scan(lens[lane & 7] & 511u), 7) / 8 + PACE_SLOT - USUAL16;
            const int slot = est > PACE_SLOT + 3 || est < PACE_SLOT - 4 ? est + 2 : PACE_SLOT;
            const u32 anchor = __builtin_amdgcn_readfirstlane(lens[16]);
            const u32 t0 = anchor + PACE_T0 * slot / PACE_SLOT;
            int pr = (kc + j) % 7;
#pragma unroll 1
            for (int s = 0; s < 7; ++s) {
                char* const peer = P.ws[(rank + 1 + pr) & 7];
                const char* const dst = peer + wo;
                pr = pr == 6 ? 0 : pr + 1;
                int before = 0;
#pragma unroll
                for (int k = 0; k < 4; ++k) {
                    const int v = 64 * s + j + PACE_KC * (kc - k);
                    before += v < 0 ? 0 : (v > 448 ? 448 : v);
                }
                const u32 due = t0 + ((before * slot) >> 8);
                pace(due);
                st16<-4096>(dst, voff, U.p[0]); __builtin_amdgcn_s_sleep(GAP);
                st16<-3072>(dst, voff, U.p[1]); __builtin_amdgcn_s_sleep(GAP);
                st16<-2048>(dst, voff, U.p[2]); __builtin_amdgcn_s_sleep(GAP);
                st16<-1024>(dst, voff, U.p[3]); __builtin_amdgcn_s_sleep(GAP);
                st16<0>(dst, voff, U.p[4]); __builtin_amdgcn_s_sleep(GAP);
                U.send_blocks(dst, voff, lane);
                wait_vm<0>();
                put_flag(peer + fo, call, U.meta, lane);
                STAMP(5 + s);
            }
            if (tl) STAMP(12);
        }
    }
    if (lane == 0) __hip_atomic_store(cp, call, __ATOMIC_RELAXED, __HIP_MEMORY_SCOPE_AGENT);
    __builtin_amdgcn_endpgm();
}

__device__ __forceinline__ u32 lds_add(LAS u32* c, int lane) {
    u32 v = 0;
    if (lane == 0) v = __hip_atomic_fetch_add(c, 1u, __ATOMIC_ACQ_REL, __HIP_MEMORY_SCOPE_WORKGROUP);
    return v;
}
__device__ __forceinline__ void lds_wait(LAS u32* c, u32 n) {
    while (__builtin_amdgcn_readfirstlane(__hip_atomic_load(c, __ATOMIC_ACQUIRE, __HIP_MEMORY_SCOPE_WORKGROUP)) < n)
        __builtin_amdgcn_s_sleep(1);
}
__device__ __forceinline__ int src_of(int rank, int j, int t) {
    return t < 4 ? rank : (rank + 7 - ((t >> 2) - 1 + (t & 3) + j) % 7) & 7;
}

__device__ __forceinline__ void mma_waves(const Params& P, int b, int wave, int lane, GAS u32* cp, LAS char* lds, u32x4 a, u32x4 c, u32 cv) {
    char* const ws = P.self;
    const int rank = P.rank;
    u32 cnt_a = (u32)(size_t)(lds + L_CNT);
    asm volatile("" : "+v"(cnt_a));
    LAS u32* const cnt = (LAS u32*)(size_t)cnt_a;
    u32 call;
    {
        const int q = wave & 3, tl = wave >> 2, ux = 2 * b + tl;
        const int xdest = (ux + (ux >> 3)) & 7, xu = ux >> 3;
        char* const xdst = P.ws[xdest];
        u32 dx[8] = {a[0], a[1], a[2], a[3], c[0], c[1], c[2], c[3]};
        LAS char* const el = lds + L_ENC;
        enc_init(el, wave, lane);
        LAS u32* const lens = (LAS u32*)(el + 2 * USLOT + 512);
        call = __builtin_amdgcn_readfirstlane(cv) + 1;
        const u32 voff = lane * 16 + 4096;
        char* const xslot = xdst + x_slot(rank, xu, call);
        if ((b & 7) == (b >> 4) && (wave & 1)) {
            asm volatile("" : "+v"(dx[0]), "+v"(dx[7]));
            if (lane == 0)
                __hip_atomic_store((GAS u32*)(ws + OFF_ANC + (b >> 3) * 128 + (wave >> 1) * 4),
                                   (call << 24) | ((u32)__builtin_amdgcn_s_memrealtime() & 0xFFFFFFu), __ATOMIC_RELAXED, __HIP_MEMORY_SCOPE_AGENT);
        }
        const u32 len = encode_quarter(dx, q * 2048 + q_img(lane), el + tl * USLOT, q, el + 2 * USLOT + wave * 64, lane, [&](u32x4 v, bool again) {
            if (again) wait_vm<0>();
            st16<-4096>(xslot + q * 1024, voff, v);
            STAMP(61);
        });
        if (lane == 0) lens[wave] = len;
        bar();
        if (q == 0) {
            Unit U;
            U.fetch(el + tl * USLOT, lens + tl * 4, lane);
            st16<0>(xslot, voff, U.p[4]);
            U.send_blocks(xslot, voff, lane);
            if (lane < 4) cnt[76 + tl * 4 + lane] = U.meta;
            STAMP(62);
        }
        wait_vm<0>();
        if (__builtin_amdgcn_readfirstlane(lds_add(cnt + 66 + tl, lane)) == 3) {
            put_flag(xslot + G_FLAG, call, cnt[76 + tl * 4 + (lane & 3)], lane);
            STAMP(63);
        }
    }
    const int j = b & 63;
    const int rb = (b >> 6) * 8 + wave;
    GAS u32* const tmo = (GAS u32*)(ws + OFF_TMO);
    const __amdgpu_buffer_rsrc_t rs = __builtin_amdgcn_make_buffer_rsrc(ws, 0, WS_NEED, 0x00020000);
    const __amdgpu_buffer_rsrc_t ro = __builtin_amdgcn_make_buffer_rsrc(P.out, 0, MS * GN * 4, 0x00020000);
    const float scale = __builtin_bit_cast(float, __builtin_amdgcn_readfirstlane(__builtin_bit_cast(u32, *P.sx * *P.sw)));
    if (lane == 0) __hip_atomic_store(cp, call, __ATOMIC_RELAXED, __HIP_MEMORY_SCOPE_AGENT);
    LAS char* const rp = lds + (lane >> 4) * 2048 + (lane & 15) * 16;
    const u32 txo = OFF_XD + (src_of(rank, j, lane & 31) * 16 + rb) * 16384 + (lane & 3) * 4096;
    f32x4 acc[2][4] = {};
    u32x4 ae[4] = {}, ao[4] = {};
    auto load_a = [&](u32x4 (&a)[4], int t) {
        const u32 so = __builtin_amdgcn_readlane(txo, t);
        u32 l16 = lane * 16;
        asm volatile("" : "+v"(l16));
#pragma unroll
        for (int i = 0; i < 4; ++i) a[i] = __builtin_amdgcn_raw_buffer_load_b128(rs, l16 + i * 1024, so, 17);
    };
    LAS char* const stash = lds + L_STA + wave * 8192 + lane * 16;
    auto tile = [&](u32x4 (&a)[4], int t, u32 need, int more) {
        lds_wait(cnt + t, need);
        __builtin_amdgcn_s_setprio(2);
        LAS char* const bp = rp + (t & (NBUF - 1)) * 8192;
        u32x4 bq[8];
#pragma unroll
        for (int i = 0; i < 8; ++i) bq[i] = *(LAS u32x4*)(bp + (i >> 1) * 256 + (i & 1) * 1024);
        __builtin_amdgcn_sched_barrier(0);
#pragma unroll
        for (int c = 0; c < 4; ++c)
#pragma unroll
            for (int m = 0; m < 2; ++m) {
                const u32x4 a0 = a[2 * m], a1 = a[2 * m + 1], b0 = bq[2 * c], b1 = bq[2 * c + 1];
                const i32x8 af = {(int)a0[0], (int)a0[1], (int)a0[2], (int)a0[3], (int)a1[0], (int)a1[1], (int)a1[2], (int)a1[3]};
                const i32x8 bf = {(int)b0[0], (int)b0[1], (int)b0[2], (int)b0[3], (int)b1[0], (int)b1[1], (int)b1[2], (int)b1[3]};
                acc[m][c] = __builtin_amdgcn_mfma_scale_f32_16x16x128_f8f6f4(af, bf, acc[m][c], 1, 1, 0, 0x7F7F7F7F, 0, 0x7F7F7F7F);
            }
        asm volatile("" : "+v"(acc[0][0]), "+v"(acc[0][1]), "+v"(acc[0][2]), "+v"(acc[0][3]),
                          "+v"(acc[1][0]), "+v"(acc[1][1]), "+v"(acc[1][2]), "+v"(acc[1][3]));
        __builtin_amdgcn_sched_barrier(0);
        if (more == 1) {
            if (lane == 0) __hip_atomic_fetch_add(cnt + 32 + t, 1u, __ATOMIC_RELAXED, __HIP_MEMORY_SCOPE_WORKGROUP);
            load_a(a, t + 2);
        } else if (more == 2) {
#pragma unroll
            for (int i = 0; i < 4; ++i) a[i] = *(LAS u32x4*)(stash + (t & 1) * 4096 + i * 1024);
        }
        __builtin_amdgcn_s_setprio(0);
    };
#pragma unroll 1
    for (int pass = 0; pass < 2; ++pass) {
        if (pass) {
            wait_flags(ws + OFF_LF + (((lane >> 1) & 7) * 32 + rb * 2 + (lane & 1)) * 128, 0xFFFF, call, tmo);
            STAMP(3);
            load_a(ae, 30);
            load_a(ao, 31);
            lds_wait(cnt + 66, 4);
            lds_wait(cnt + 67, 4);
#pragma unroll
            for (int i = 0; i < 4; ++i) { *(LAS u32x4*)(stash + i * 1024) = ae[i]; *(LAS u32x4*)(stash + 4096 + i * 1024) = ao[i]; }
            __builtin_amdgcn_sched_barrier(0);
            load_a(ae, 0);
            __builtin_amdgcn_sched_barrier(0);
            load_a(ao, 1);
#pragma unroll
            for (int i = 0; i < 8; ++i) acc[i >> 2][i & 3] = f32x4{0.f, 0.f, 0.f, 0.f};
#pragma unroll 1
            for (int t = 0; t < 28; t += 2) {
                tile(ae, t, 0x01010101u, 1);
                tile(ao, t + 1, 0x01010101u, 1);
                if (wave == 0) STAMP(30 + (t >> 1));
            }
        }
        tile(ae, 28, pass * 0x01010101u, 2);
        tile(ao, 29, pass * 0x01010101u, 2);
        if (pass && wave == 0) STAMP(44);
        tile(ae, 30, pass * 0x01010101u, 0);
        tile(ao, 31, pass * 0x01010101u, 0);
        const float sc = cnt[72] ? __builtin_nanf("") : scale;
        if (pass) STAMP(59);
        const u32 vo = pass ? (lane >> 4) * 4 * GN * 4 + (lane & 15) * 16 : OOB;
        const u32 so = (rb * 32 * GN + j * 64) * 4;
#pragma unroll
        for (int m = 0; m < 2; ++m)
#pragma unroll
            for (int r = 0; r < 4; ++r) {
                u32x4 v;
#pragma unroll
                for (int c = 0; c < 4; ++c)
                    v[c] = __builtin_bit_cast(u32, __builtin_elementwise_maximum(acc[m][c][r] * sc, 0.f));
                __builtin_amdgcn_raw_buffer_store_b128(v, ro, vo, so + (m * 16 + r) * GN * 4, OUT_AUX);
                __builtin_amdgcn_sched_barrier(0);
                asm volatile("s_nop 1");
                __builtin_amdgcn_sched_barrier(0);
            }
    }
    STAMP(60);
    __builtin_amdgcn_endpgm();
}

__device__ __forceinline__ void unpack_waves(const Params& P, int b, int wave, int lane, GAS u32* cp, LAS char* lds, u32 cv) {
    char* const ws = P.self;
    const int rank = P.rank;
    LAS u32* const cnt = (LAS u32*)(lds + L_CNT);
    if (wave == 8) { cnt[lane] = 0; cnt[64 + lane] = 0; }
    bar();
    const u32 call = __builtin_amdgcn_readfirstlane(cv) + 1;
    const int q = wave & 3, tl = (wave >> 2) & 1;
    const int j = b & 63;
    GAS u32* const tmo = (GAS u32*)(ws + OFF_TMO);
    const __amdgpu_buffer_rsrc_t rs = __builtin_amdgcn_make_buffer_rsrc(ws, 0, WS_NEED, 0x00020000);
    LAS char* const sym = lds + L_SYM + (wave & 7) * 2304;
    const int ux = 2 * b + tl, xsrc = ux & 7, xu = ux >> 3;
    constexpr int NJOB = 17;
    u32 juo, jfo;
    {
        const int i = lane < NJOB ? lane : 0, t = i ? 2 * (i - 1) + tl : 0;
        juo = i ? w_slot(src_of(rank, j, t), t & 3, j, call) : x_slot(xsrc, xu, call);
        jfo = lane < NJOB ? juo + G_FLAG + q * 8 : OFF_TMO;
    }
    const u32 img = q * 2048 + q_img(lane);
    LAS char* image = lds + L_XIM + tl * 8192;
    __builtin_amdgcn_s_setprio(1);
    Packed pk;
    u64 look = 0;
    bool have = false;
    u32 fre = 8;
#pragma unroll 1
    for (int i = 0; i < NJOB; ++i) {
        const int t = 2 * (i - 1) + tl;
        if (__builtin_amdgcn_readfirstlane(fre) < 8) lds_wait(cnt + 32 + t - NBUF, 8);
        if (!have) {
            u32 uo = __builtin_amdgcn_readlane(juo, i);
            const char* nf = ws + __builtin_amdgcn_readlane(jfo, i + 1);
            asm volatile("" : "+s"(uo), "+s"(nf));
            u32 meta = __builtin_amdgcn_readfirstlane((u32)look);
            if (__builtin_amdgcn_readfirstlane((u32)(look >> 32)) < call) meta = wait_flag(ws + __builtin_amdgcn_readlane(jfo, i), call, tmo);
            asm volatile("" ::: "memory");
            load_quarter(pk, rs, uo, meta, q, lane);
            look = __hip_atomic_load((GAS u64*)nf, __ATOMIC_RELAXED, __HIP_MEMORY_SCOPE_SYSTEM);
            if (i >= SJ) STAMP(4 + 3 * (i - 9));
        }
        decode_quarter(pk, rs, img, q, image, sym, lane, [&]() {
            if (t + 2 >= NBUF) fre = __hip_atomic_load(cnt + 32 + t + 2 - NBUF, __ATOMIC_RELAXED, __HIP_MEMORY_SCOPE_WORKGROUP);
            asm volatile("" : "+v"(look));
            if (i + 1 == NJOB) {
                if ((u32)look == call) cnt[72] = 1;
            } else {
                have = __builtin_amdgcn_readfirstlane((u32)(look >> 32)) >= call;
                asm volatile("" ::: "memory");
                if (have) load_quarter(pk, rs, __builtin_amdgcn_readlane(juo, i + 1), __builtin_amdgcn_readfirstlane((u32)look), q, lane);
                look = __hip_atomic_load((GAS u64*)(ws + __builtin_amdgcn_readlane(jfo, i + 1 + have)), __ATOMIC_RELAXED, __HIP_MEMORY_SCOPE_SYSTEM);
            }
        });
        if (i) {
            __hip_atomic_store((LAS unsigned char*)(cnt + t) + q, (unsigned char)1, __ATOMIC_RELEASE, __HIP_MEMORY_SCOPE_WORKGROUP);
        } else {
            lds_order();
            const u32x2 h0 = *(LAS u32x2*)(image + img), h1 = *(LAS u32x2*)(image + img + 256);
            const u32x2 h2 = *(LAS u32x2*)(image + img + 512), h3 = *(LAS u32x2*)(image + img + 768);
            const u32x4 v0 = {h0[0], h0[1], h1[0], h1[1]}, v1 = {h2[0], h2[1], h3[0], h3[1]};
            const char* xd = ws + OFF_XD + (xsrc * 32 + xu) * 8192 + q * 2048;
            st16<0>(xd, lane * 16, v0);
            st16<1024>(xd, lane * 16, v1);
            wait_vm<0>();
            if (__builtin_amdgcn_readfirstlane(lds_add(cnt + 68 + tl, lane)) == 3)
                put_flag(ws + OFF_LF + (xsrc * 32 + xu) * 128, call, 0, lane);
            STAMP(2);
        }
        image = lds + ((t + 2) & (NBUF - 1)) * 8192;
        if (i >= SJ - 1) STAMP(6 + 3 * (i - 9));
        else if (i < 9 && SJ == 9) STAMP(48 + i);
    }
    if (lane == 0) __hip_atomic_store(cp, call, __ATOMIC_RELAXED, __HIP_MEMORY_SCOPE_AGENT);
    __builtin_amdgcn_endpgm();
}

__global__ void __launch_bounds__(1024, 4) gemm_rs_kernel(Params P) {
    __shared__ __attribute__((aligned(16))) char smem[LDS_BYTES];
    const int lane = threadIdx.x & 63;
    const int wave = __builtin_amdgcn_readfirstlane(threadIdx.x >> 6);
    u32x4 xa, xc;
    {
        const u32 ux = 2 * (blockIdx.x - N_SEND_WG) + (wave >> 2), xdest = (ux + (ux >> 3)) & 7, xu = (ux >> 3) & 31;
        const __amdgpu_buffer_rsrc_t rx = __builtin_amdgcn_make_buffer_rsrc((void*)P.x, 0, 4096 * KS, 0x00020000);
        const u32 so = (xdest * MS + (xu >> 1) * 32 + (wave & 1) * 16) * KS + (xu & 1) * 256 + (wave & 2) * 64;
        const u32 vo = ((lane & 15) * KS + (lane >> 4) * 32) | (blockIdx.x >= N_SEND_WG && wave < 8 ? 0 : OOB);
        xa = __builtin_amdgcn_raw_buffer_load_b128(rx, vo, so, 0);
        xc = __builtin_amdgcn_raw_buffer_load_b128(rx, vo + 16, so, 0);
        asm volatile("" ::: "memory");
    }
    GAS u32* cp = (GAS u32*)(P.self + OFF_CNT) + blockIdx.x * 16 + wave;
    const u32 cv = __hip_atomic_load(cp, __ATOMIC_RELAXED, __HIP_MEMORY_SCOPE_AGENT);
    asm volatile("" ::: "memory");
    STAMP(0);
    if (blockIdx.x >= N_SEND_WG) {
        if (wave < 8) mma_waves(P, blockIdx.x - N_SEND_WG, wave, lane, cp, (LAS char*)smem, xa, xc, cv);
        else unpack_waves(P, blockIdx.x - N_SEND_WG, wave, lane, cp, (LAS char*)smem, cv);
    } else {
        if (wave < 8) sender(P, blockIdx.x, wave, lane, cp, (LAS char*)smem, cv);
        else if (wave == 8) anchor_wave(P.self, lane, cp, (LAS char*)smem, cv);
    }
}

extern "C" void kernel_launch(int rank, int nranks, ncclComm_t comm,
                              hipStream_t stream,
                              void* const* d_in, const int* in_sizes, int n_in,
                              void* d_out, int out_size,
                              const int* in_dtypes, int out_dtype,
                              void* const* d_out_all,
                              void* const* d_ws_all, size_t ws_bytes) {
    if (nranks != NR || n_in != 4 || ws_bytes < WS_NEED || (unsigned)rank >= (unsigned)NR ||
        in_sizes[0] != 4096 * KS || in_sizes[1] != KS * GN || out_size != MS * GN) {
        fprintf(stderr, "kernel_launch: unexpected problem shape\n");
        return;
    }
    Params P;
    P.x = (const uint8_t*)d_in[0];
    P.w = (const uint8_t*)d_in[1];
    P.sx = (const float*)d_in[2];
    P.sw = (const float*)d_in[3];
    P.out = (float*)d_out;
    for (int p = 0; p < NR; ++p) P.ws[p] = (char*)d_ws_all[p];
    P.self = P.ws[rank];
    P.rank = rank;
    P.pad = 0;
    gemm_rs_kernel<<<GRID, 1024, 0, stream>>>(P);
    HIP_CHECK(hipGetLastError());
}
```

```cpp
#include <hip/hip_runtime.h>
#include <rccl/rccl.h>
#include <cstdio>
#include <cstdint>

#define HIP_CHECK(expr) do { hipError_t _e = (expr); if (_e != hipSuccess) { \
    fprintf(stderr, "HIP error %s at %s:%d\n", hipGetErrorString(_e), __FILE__, __LINE__); \
    return; } } while (0)

typedef unsigned u32;
typedef unsigned long long u64;
typedef u32 u32x4 __attribute__((ext_vector_type(4)));
typedef u32 u32x2 __attribute__((ext_vector_type(2)));
typedef int i32x8 __attribute__((ext_vector_type(8)));
typedef float f32x4 __attribute__((ext_vector_type(4)));
#define GAS __attribute__((address_space(1)))
#define LAS __attribute__((address_space(3)))

#define PACE_T0 322
#define PACE_SLOT 372
#define PACE_KC 16
#define USUAL16 51
#define LF_SLEEP 16
#define MID_SLEEP 8
#define EAGER_FROM 15
#define PH 222
#define LAM 70
#define WIN_SLOTS 5
#define SLIP_ON 1
#define GAP 2
#define POLL_SLEEP 6
#define OUT_AUX 17
#define FORCE_RAW 0

constexpr int NR = 8;
constexpr int GN = 4096;
constexpr int KS = 512;
constexpr int MS = 512;
constexpr int NUNIT = 256;
constexpr int N_SEND_WG = 128;
constexpr int N_CONS_WG = 128;
constexpr int GRID = N_SEND_WG + N_CONS_WG;

constexpr u32 USLOT = 9216;
constexpr u32 GSLOT = 6144, NEAR_DW = 224, G_FLAG = 5120 + NEAR_DW * 4;
__host__ __device__ constexpr u32 w_slot(u32 src, u32 kc, u32 j, u32 call) {
    return (((((j & 7) * 8 + src) * 8 + (j >> 3)) * 4 + kc) * 2 + (call & 1)) * GSLOT;
}
__host__ __device__ constexpr u32 x_slot(u32 src, u32 xu, u32 call) { return (4096 + (src * 32 + xu) * 2 + (call & 1)) * GSLOT; }
constexpr u32 OFF_XD = x_slot(NR, 0, 0);
constexpr u32 OFF_LF = OFF_XD + NR * MS * KS;
constexpr u32 OFF_CNT = OFF_LF + NUNIT * 128;
constexpr u32 OFF_TMO = OFF_CNT + GRID * 16 * 4;
constexpr u32 OFF_ANC = OFF_TMO + 128;
constexpr u32 OFF_DBG = OFF_ANC + 16 * 128;
constexpr u32 FAR = OFF_DBG + GRID * 16 * 64 * 8;
constexpr u32 WS_NEED = FAR + OFF_XD;
constexpr u32 DEADLINE_TICKS = 100000000u;
constexpr u32 OOB = 0x80000000u;
constexpr u32 RAW_DW = 256;
constexpr u32 CAP_DW = 192;
constexpr int NBUF = 8;
constexpr u32 L_SYM = NBUF * 8192;
constexpr u32 L_XIM = 4 * 8192;
constexpr u32 L_ENC = L_SYM + 8 * 2304;
constexpr u32 L_STA = L_ENC;
constexpr u32 L_CNT = L_STA + 8 * 8192;
constexpr int LDS_BYTES = L_CNT + 512;

struct Params {
    const uint8_t* x;
    const uint8_t* w;
    const float* sx;
    const float* sw;
    float* out;
    char* self;
    char* ws[NR];
    int rank;
    int pad;
};

#define SJ 16
#define STAMP(i) do { } while (0)

template <int OFF>
__device__ __forceinline__ void st16(const char* base, u32 voff, u32x4 v) {
    asm volatile("global_store_dwordx4 %0, %1, %2 offset:%3 sc0 sc1\n\ts_nop 1"
                 :: "v"(voff), "v"(v), "s"(base), "i"(OFF) : "memory");
}
template <int N> __device__ __forceinline__ void wait_vm() { asm volatile("s_waitcnt vmcnt(%0)" :: "i"(N) : "memory"); }

__device__ __forceinline__ void put_flag(char* p, u32 call, u32 meta, int lane) {
    if (lane < 4)
        __hip_atomic_store((GAS u64*)p + lane, ((u64)call << 32) | meta, __ATOMIC_RELAXED, __HIP_MEMORY_SCOPE_SYSTEM);
}

template <int SLEEP>
__device__ __forceinline__ u64 wait_flags(const char* f, u64 need, u32 call, GAS u32* tmo) {
    const u32 t0 = (u32)__builtin_amdgcn_s_memrealtime();
    u64 v;
    for (u32 n = 0;;) {
        v = __hip_atomic_load((GAS u64*)f, __ATOMIC_RELAXED, __HIP_MEMORY_SCOPE_SYSTEM);
        if ((__ballot((u32)(v >> 32) >= call) & need) == need) break;
        if ((++n & 15) == 0) {
            const u32 t = __hip_atomic_load(tmo, __ATOMIC_RELAXED, __HIP_MEMORY_SCOPE_AGENT);
            if (__builtin_amdgcn_readfirstlane(t) == call || (u32)__builtin_amdgcn_s_memrealtime() - t0 > DEADLINE_TICKS) {
                __hip_atomic_store(tmo, call, __ATOMIC_RELAXED, __HIP_MEMORY_SCOPE_AGENT);
                break;
            }
        }
        __builtin_amdgcn_s_sleep(SLEEP);
    }
    asm volatile("" ::: "memory");
    return v;
}

__device__ __forceinline__ u32 wait_flag(const char* f, u32 call, GAS u32* tmo) {
    const u32 t0 = (u32)__builtin_amdgcn_s_memrealtime();
    u32 meta;
    u64 va = __hip_atomic_load((GAS u64*)f, __ATOMIC_RELAXED, __HIP_MEMORY_SCOPE_SYSTEM);
    __builtin_amdgcn_s_sleep(POLL_SLEEP);
    for (u32 n = 1;; ++n) {
        const u64 vb = __hip_atomic_load((GAS u64*)f, __ATOMIC_RELAXED, __HIP_MEMORY_SCOPE_SYSTEM);
        meta = __builtin_amdgcn_readfirstlane((u32)va);
        if (__builtin_amdgcn_readfirstlane((u32)(va >> 32)) >= call) break;
        va = __hip_atomic_load((GAS u64*)f, __ATOMIC_RELAXED, __HIP_MEMORY_SCOPE_SYSTEM);
        meta = __builtin_amdgcn_readfirstlane((u32)vb);
        if (__builtin_amdgcn_readfirstlane((u32)(vb >> 32)) >= call) break;
        if ((n & 15) == 0) {
            const u32 t = __hip_atomic_load(tmo, __ATOMIC_RELAXED, __HIP_MEMORY_SCOPE_AGENT);
            if (__builtin_amdgcn_readfirstlane(t) == call || (u32)__builtin_amdgcn_s_memrealtime() - t0 > DEADLINE_TICKS) {
                __hip_atomic_store(tmo, call, __ATOMIC_RELAXED, __HIP_MEMORY_SCOPE_AGENT);
                break;
            }
            va = __hip_atomic_load((GAS u64*)f, __ATOMIC_RELAXED, __HIP_MEMORY_SCOPE_SYSTEM);
            __builtin_amdgcn_s_sleep(POLL_SLEEP);
        }
    }
    asm volatile("" ::: "memory");
    return meta;
}

__device__ __forceinline__ u32 wave_scan(u32 v) {
    v += __builtin_amdgcn_update_dpp(0u, v, 0x111, 0xf, 0xf, true);
    v += __builtin_amdgcn_update_dpp(0u, v, 0x112, 0xf, 0xf, true);
    v += __builtin_amdgcn_update_dpp(0u, v, 0x114, 0xf, 0xf, true);
    v += __builtin_amdgcn_update_dpp(0u, v, 0x118, 0xf, 0xf, true);
    v += __builtin_amdgcn_update_dpp(0u, v, 0x142, 0xa, 0xf, false);
    v += __builtin_amdgcn_update_dpp(0u, v, 0x143, 0xc, 0xf, false);
    return v;
}
__device__ __forceinline__ u32 bfi(u32 m, u32 a, u32 b) { return (m & a) | (~m & b); }
__device__ __forceinline__ void lds_order() { __builtin_amdgcn_fence(__ATOMIC_ACQ_REL, "wavefront"); }

template <class Early>
__device__ __forceinline__ u32 encode_quarter(const u32 (&d)[8], u32 img, LAS char* unit, int q,
                                              LAS char* lut, int lane, Early codes_out) {
    u32 mag[8], sum = 0;
#pragma unroll
    for (int k = 0; k < 8; ++k) { mag[k] = d[k] & 0x7F7F7F7Fu; sum = __builtin_amdgcn_sad_u8(mag[k], 0u, sum); }
    int b = ((int)__builtin_amdgcn_readlane(wave_scan(sum), 63) + 1024 - 14541) >> 11;
    b = b < 26 ? 26 : b;
    asm volatile("" : "+s"(b));
    const u32 B = b > 108 ? 108 : b;
    const u32 c2 = (126 - B) * 0x01010101u, c16 = (112 - B) * 0x01010101u, c26 = (154 - B) * 0x01010101u, c20 = (108 - B) * 0x01010101u;
    u32 nib[8], w[8], esc[8];
#pragma unroll
    for (int k = 0; k < 8; ++k) {
        const u32 u = mag[k] + c2;
        w[k] = mag[k] + c16;
        const u32 dir = u & ~w[k] & 0x80808080u;
        nib[k] = w[k] & (((dir >> 3) - (dir >> 6)) | 0x01010101u);
        esc[k] = dir ^ 0x80808080u;
    }
    codes_out(u32x4{nib[0] | (nib[1] << 4), nib[2] | (nib[3] << 4), nib[4] | (nib[5] << 4), nib[6] | (nib[7] << 4)}, false);
    u32 sel[8], t[8], off[8], cnt = 0, sg = 0, mb = 0;
#pragma unroll
    for (int k = 0; k < 8; ++k) sel[k] = *(LAS u32*)(lut + (__builtin_amdgcn_udot4(esc[k], 0x08040201u, 0u, false) >> 5));
#pragma unroll
    for (int k = 0; k < 8; ++k) {
        const u32 hi = w[k] & 0x80808080u, q0 = mag[k] + c26;
        t[k] = ((q0 - ((hi >> 3) - (hi >> 6))) >> 1) & 0x0F0F0F0Fu;
        mb |= ((~q0 | (mag[k] + c20)) & 0x80808080u) >> (7 - k);
        off[k] = cnt;
        cnt += __builtin_popcount(esc[k]);
        sg |= (d[k] >> (7 - k)) & (0x01010101u << k);
    }
    const u32 incl = wave_scan(cnt);
    const u32 nsd = (__builtin_amdgcn_readlane(incl, 63) + 7) >> 3;
    LAS char* blk = unit + 5120 + q * 1024;
    *(LAS u32*)(unit + 4096 + q * 256 + lane * 4) = sg;
    u32 ndw = nsd;
    if (ndw <= CAP_DW && !FORCE_RAW) {
        const u32 base = incl - cnt;
#pragma unroll
        for (int i = 0; i < 4; ++i) {
            u32 c0 = __builtin_amdgcn_perm(0u, t[2 * i], sel[2 * i]), c1 = __builtin_amdgcn_perm(0u, t[2 * i + 1], sel[2 * i + 1]);
            c0 = __builtin_amdgcn_perm(0u, c0 | (c0 >> 4), 0x0C0C0200u);
            c1 = __builtin_amdgcn_perm(0u, c1 | (c1 >> 4), 0x0C0C0200u);
            const u32 pos = base + off[2 * i];
            const u64 v = (u64)(c0 | (c1 << (4 * (off[2 * i + 1] - off[2 * i])))) << (4 * (pos & 7));
            LAS u32* p = (LAS u32*)(blk + ((pos >> 1) & ~3u));
            __hip_atomic_fetch_or(p, (u32)v, __ATOMIC_RELAXED, __HIP_MEMORY_SCOPE_WORKGROUP);
            if ((u32)(v >> 32)) __hip_atomic_fetch_or(p + 1, (u32)(v >> 32), __ATOMIC_RELAXED, __HIP_MEMORY_SCOPE_WORKGROUP);
        }
        lds_order();
        for (u64 mk; (mk = __ballot(mb != 0)) != 0; ndw += __builtin_popcountll(mk)) {
            if (mb != 0) {
                const u32 bit = __builtin_ctz(mb);
                mb &= mb - 1;
                const u32 k0 = (u32)((int)(bit << 31) >> 31), k1 = (u32)((int)(bit << 30) >> 31), k2 = (u32)((int)(bit << 29) >> 31);
                const u32 m0 = bfi(k2, mag[4], mag[0]), m1 = bfi(k2, mag[5], mag[1]), m2 = bfi(k2, mag[6], mag[2]), m3 = bfi(k2, mag[7], mag[3]);
                const u32 mg = (bfi(k0, bfi(k1, m3, m1), bfi(k1, m2, m0)) >> (bit & 24u)) & 127u;
                const u32 ad = img + (bit & 6u) * 128 + (k0 & 4u) + (bit >> 3);
                const u32 z = (mg + 154 - B - (mg >= B + 16 ? 14 : 0)) & 31u;
                const u32 dflt = B - 26 + z + (z >= 28 ? 14 : 0);
                const u32 slot = ndw + __builtin_amdgcn_mbcnt_hi((u32)(mk >> 32), __builtin_amdgcn_mbcnt_lo((u32)mk, 0u));
                *(LAS u32*)(blk + (slot < 255 ? slot : 255) * 4) = (ad << 8) | (mg ^ dflt);
            }
        }
    }
    if (ndw > CAP_DW || ndw - nsd > 31 || FORCE_RAW) {
        lds_order();
        codes_out(u32x4{d[0], d[1], d[2], d[3]}, true);
        *(LAS u32x4*)(blk + lane * 16) = u32x4{d[4], d[5], d[6], d[7]};
        return RAW_DW;
    }
    return ndw | ((ndw - nsd) << 9) | (B << 15);
}

struct Packed {
    u32x4 n;
    u32 s, b0;
    u32 meta, uo;
};
__device__ __forceinline__ u32 blk_off(u32 uo, u32 meta) { return uo + (meta & 0x4000u ? FAR : 5120) + (meta >> 22) * 4; }
__device__ __forceinline__ void load_quarter(Packed& p, __amdgpu_buffer_rsrc_t rs, u32 uo, u32 meta, int q, int lane) {
    p.n = __builtin_amdgcn_raw_buffer_load_b128(rs, lane * 16 + q * 1024, uo, 17);
    p.s = __builtin_amdgcn_raw_buffer_load_b32(rs, lane * 4 + 4096 + q * 256, uo, 17);
    __builtin_amdgcn_sched_barrier(0);
    p.meta = meta;
    p.uo = uo;
    p.b0 = __builtin_amdgcn_raw_buffer_load_b32(rs, (u32)lane < (meta & 511u) ? lane * 4 : OOB, blk_off(uo, meta), 17);
}
__device__ __forceinline__ u32 sym_bytes(u32 t, u32 k1, u32 kx) {
    const u32 h = (t + 0x02020202u) & 0x10101010u;
    return (((t << 1) + k1) + (h - (h >> 3))) ^ kx;
}
__device__ __forceinline__ u32x2 sym8(u32 x, u32 k1, u32 kx) {
    const u32 lo = sym_bytes(x & 0x0F0F0F0Fu, k1, kx), hi = sym_bytes((x >> 4) & 0x0F0F0F0Fu, k1, kx);
    return u32x2{__builtin_amdgcn_perm(hi, lo, 0x05010400u), __builtin_amdgcn_perm(hi, lo, 0x07030602u)};
}
__device__ __forceinline__ void patch(u32 e, bool on, LAS char* image) {
    if (on && (e & 127u))
        __hip_atomic_fetch_xor((LAS u32*)(image + ((e >> 8) & 0x1FFCu)), (e & 127u) << ((e >> 5) & 24u),
                               __ATOMIC_RELAXED, __HIP_MEMORY_SCOPE_WORKGROUP);
}
template <class Mid>
__device__ __forceinline__ void decode_quarter(const Packed& p, __amdgpu_buffer_rsrc_t rs, u32 img, int q,
                                               LAS char* image, LAS char* sym, int lane, Mid mid) {
    const u32 len = p.meta & 511u, uo = p.uo, bo = blk_off(uo, p.meta);
    const u32 B = (p.meta >> 15) & 127u, nsd = len - ((p.meta >> 9) & 31u);
    const u32 k1 = (B - 26) * 0x01010101u, kx = (0xFFu ^ B) * 0x01010101u;
    if (len > 64 && len < RAW_DW) {
        const u32 b1 = __builtin_amdgcn_raw_buffer_load_b32(rs, (u32)lane + 64 < len ? lane * 4 + 256 : OOB, bo, 17);
        const u32 b2 = __builtin_amdgcn_raw_buffer_load_b32(rs, (u32)lane + 128 < len ? lane * 4 + 512 : OOB, bo, 17);
        *(LAS u32x2*)(sym + 512 + lane * 8) = sym8(b1, k1, kx);
        *(LAS u32x2*)(sym + 1024 + lane * 8) = sym8(b2, k1, kx);
    }
    u32 nib[8], sel[8], off[8], cnt = 0;
#pragma unroll
    for (int k = 0; k < 8; ++k) {
        nib[k] = (k & 1 ? p.n[k >> 1] >> 4 : p.n[k >> 1]) & 0x0F0F0F0Fu;
        const u32 n10 = (nib[k] + 0x0E0E0E0Eu) & 0x10101010u, e10 = n10 ^ 0x10101010u;
        sel[k] = __umul24(e10, 0x101010u) + n10;
        off[k] = cnt;
        cnt += __builtin_popcount(e10);
    }
    const u32 sp = (u32)(size_t)sym + (wave_scan(cnt) - cnt);
    *(LAS u32x2*)(sym + lane * 8) = sym8(p.b0, k1, kx);
    lds_order();
    u32 g0[8], g1[8], o[8];
#pragma unroll
    for (int k = 0; k < 8; ++k) {
        off[k] += sp;
        LAS u32* gp = (LAS u32*)(size_t)(off[k] & ~3u);
        g0[k] = gp[0]; g1[k] = gp[1];
    }
    __builtin_amdgcn_sched_barrier(0);
#pragma unroll
    for (int k = 0; k < 8; ++k) nib[k] |= (p.s << (7 - k)) & 0x80808080u;
    u32 ev = (u32)lane >= (len <= 64 ? nsd : 64u) ? (p.b0 & 127u) << ((p.b0 >> 5) & 24u) : 0u;
    u32 ea = (u32)(size_t)image + ((p.b0 >> 8) & 0x1FFCu);
    u32 wa = (u32)(size_t)image + img;
    asm volatile("" : "+v"(ev), "+v"(ea), "+v"(wa), "+v"(nib[0]), "+v"(nib[1]), "+v"(nib[2]), "+v"(nib[3]),
                      "+v"(nib[4]), "+v"(nib[5]), "+v"(nib[6]), "+v"(nib[7]));
    mid();
    __builtin_amdgcn_sched_barrier(0);
#pragma unroll
    for (int c = 0; c < 4; ++c) {
#pragma unroll
        for (int k = 2 * c; k < 2 * c + 2; ++k) {
            const u32 g = __builtin_amdgcn_alignbyte(g1[k], g0[k], off[k]);
            o[k] = (__builtin_amdgcn_perm(g, g, sel[k]) ^ kx) + nib[k];
        }
        *(LAS u32x2*)(size_t)(wa + c * 256) = u32x2{o[2 * c], o[2 * c + 1]};
    }
    lds_order();
    if (ev) __hip_atomic_fetch_xor((LAS u32*)(size_t)ea, ev, __ATOMIC_RELAXED, __HIP_MEMORY_SCOPE_WORKGROUP);
    if (len > 64) {
        if (len < RAW_DW) {
            patch(__builtin_amdgcn_raw_buffer_load_b32(rs, nsd + lane < len ? (nsd + lane) * 4 : OOB, bo, 17), true, image);
        } else {
            const u32x4 l = __builtin_amdgcn_raw_buffer_load_b128(rs, lane * 16 + q * 1024, uo, 17);
            const u32x4 h = __builtin_amdgcn_raw_buffer_load_b128(rs, lane * 16, bo, 17);
            *(LAS u32x2*)(size_t)wa = u32x2{l[0], l[1]};
            *(LAS u32x2*)(size_t)(wa + 256) = u32x2{l[2], l[3]};
            *(LAS u32x2*)(size_t)(wa + 512) = u32x2{h[0], h[1]};
            *(LAS u32x2*)(size_t)(wa + 768) = u32x2{h[2], h[3]};
        }
    }
}

__device__ __forceinline__ u32 q_img(int lane) {
    return (lane >> 5) * 1024 + ((lane & 7) | ((lane >> 1) & 8)) * 16 + (lane & 8);
}

struct Unit {
    u32x4 p[6];
    u32 meta;
    u32 c1, c2, c3, tot;
    u32 tot16;
    LAS char* lds;
    __device__ __forceinline__ u32x4 blk16(u32 i) const {
        u32x4 v;
#pragma unroll
        for (int k = 0; k < 4; ++k) {
            const u32 w = 4 * i + k;
            const u32 a = w + (w < c1 ? 0 : (w < c2 ? 256 - c1 : (w < c3 ? 512 - c2 : 768 - c3)));
            v[k] = *(LAS u32*)(lds + 5120 + (a & 1023u) * 4);
        }
        return v;
    }
    __device__ __forceinline__ void fetch(LAS char* unit, LAS u32* lens, int lane) {
        lds = unit;
        const u32 l = lens[lane & 3];
        c1 = __builtin_amdgcn_readlane(l, 0) & 511u;
        c2 = c1 + (__builtin_amdgcn_readlane(l, 1) & 511u);
        c3 = c2 + (__builtin_amdgcn_readlane(l, 2) & 511u);
        tot = c3 + (__builtin_amdgcn_readlane(l, 3) & 511u);
        tot16 = (tot + 3) >> 2;
        meta = l | (tot > NEAR_DW ? 0x4000u : 0u) | ((lane == 0 ? 0 : (lane == 1 ? c1 : (lane == 2 ? c2 : c3))) << 22);
#pragma unroll
        for (int i = 0; i < 5; ++i) p[i] = *(LAS u32x4*)(unit + i * 1024 + lane * 16);
        p[5] = blk16(lane);
    }
    __device__ __forceinline__ void send_blocks(const char* dst, u32 voff, int lane) const {
        if (tot > NEAR_DW) dst += FAR - 5120;
        for (u32 i = 64 + lane; i < tot16; i += 64) st16<1024>(dst, voff + (i - lane) * 16, blk16(i));
        if ((u32)lane < tot16) st16<1024>(dst, voff, p[5]);
    }
    __device__ __forceinline__ void send_all(const char* dst, u32 voff, int lane) const {
        st16<-4096>(dst, voff, p[0]); st16<-3072>(dst, voff, p[1]); st16<-2048>(dst, voff, p[2]);
        st16<-1024>(dst, voff, p[3]); st16<0>(dst, voff, p[4]);
        send_blocks(dst, voff, lane);
    }
};

__device__ __forceinline__ void pace(u32 due) {
    while ((int)((u32)__builtin_amdgcn_s_memrealtime() - due) < 0) __builtin_amdgcn_s_sleep(1);
}

__device__ __forceinline__ void bar() {
    asm volatile("s_waitcnt lgkmcnt(0)" ::: "memory");
    __builtin_amdgcn_s_barrier();
    asm volatile("" ::: "memory");
}
__device__ __forceinline__ void enc_init(LAS char* lds, int wave, int lane) {
    *(LAS u32x4*)(lds + (wave >> 2) * USLOT + 5120 + (wave & 3) * 1024 + lane * 16) = u32x4{0, 0, 0, 0};
    u32 sel = 0x0C0C0C0Cu, n = 0;
#pragma unroll
    for (int i = 0; i < 4; ++i)
        if ((lane >> i) & 1) { sel = (sel & ~(0xFFu << n)) | ((u32)i << n); n += 8; }
    if (lane < 16) *(LAS u32*)(lds + 2 * USLOT + wave * 64 + lane * 4) = sel;
    lds_order();
}

__device__ __forceinline__ void anchor_wave(const char* self, int lane, GAS u32* cp, LAS char* lds, u32 cv) {
    const u32 call = __builtin_amdgcn_readfirstlane(cv) + 1;
    u32 v, now;
    u64 ok;
    int n = 0;
    __builtin_amdgcn_s_sleep(16);
    do {
        __builtin_amdgcn_s_sleep(4);
        v = __hip_atomic_load((GAS u32*)(self + OFF_ANC + (lane >> 2) * 128 + (lane & 3) * 4), __ATOMIC_RELAXED, __HIP_MEMORY_SCOPE_AGENT);
        ok = __ballot((v >> 24) == (call & 255u));
        now = (u32)__builtin_amdgcn_s_memrealtime();
    } while (ok != ~0ull && ++n < 8);
    u32 age = (now - v) & 0xFFFFFFu;
    age = (v >> 24) == (call & 255u) ? (age > 1000 ? 1000 : age) : 0;
    *(LAS u32*)(lds + 2 * USLOT + 512 + 64) = now - (__builtin_amdgcn_readlane(wave_scan(age), 63) >> 6);
    bar();
    if (lane == 0) __hip_atomic_store(cp, call, __ATOMIC_RELAXED, __HIP_MEMORY_SCOPE_AGENT);
    __builtin_amdgcn_endpgm();
}

__device__ __forceinline__ void sender(const Params& P, int b, int wave, int lane, GAS u32* cp, LAS char* lds, u32 cv) {
    const int rank = P.rank;
    char* const self = P.self;
    const int q = wave & 3, tl = wave >> 2;
    const int j = b & 63, kc0 = b >> 6;
    u32 wr[8];
    {
        const u32 cg = (lane & 7) | ((lane >> 1) & 8), rg = ((lane >> 3) & 1) | ((lane >> 4) & 2);
        const uint8_t* wp = P.w + (size_t)((kc0 + 2 * tl) * 128 + q * 32 + rg * 8) * GN + j * 64 + cg * 4;
#pragma unroll
        for (int i = 0; i < 8; ++i) wr[i] = *(const GAS u32*)(wp + (size_t)i * GN);
    }
    enc_init(lds, wave, lane);
    LAS char* const unit = lds + tl * USLOT;
    LAS u32* const lens = (LAS u32*)(lds + 2 * USLOT + 512);
    u32 dw[8];
#pragma unroll
    for (int h = 0; h < 2; ++h) {
        const u32 r0 = wr[4 * h], r1 = wr[4 * h + 1], r2 = wr[4 * h + 2], r3 = wr[4 * h + 3];
        const u32 t0 = __builtin_amdgcn_perm(r1, r0, 0x05010400u), t1 = __builtin_amdgcn_perm(r1, r0, 0x07030602u);
        const u32 t2 = __builtin_amdgcn_perm(r3, r2, 0x05010400u), t3 = __builtin_amdgcn_perm(r3, r2, 0x07030602u);
        dw[0 + h] = __builtin_amdgcn_perm(t2, t0, 0x05040100u);
        dw[2 + h] = __builtin_amdgcn_perm(t2, t0, 0x07060302u);
        dw[4 + h] = __builtin_amdgcn_perm(t3, t1, 0x05040100u);
        dw[6 + h] = __builtin_amdgcn_perm(t3, t1, 0x07060302u);
    }
    const u32 len = encode_quarter(dw, q * 2048 + q_img(lane), unit, q, lds + 2 * USLOT + wave * 64, lane,
                                   [&](u32x4 v, bool) { *(LAS u32x4*)(unit + q * 1024 + lane * 16) = v; });
    if (lane == 0) lens[wave] = len;
    const u32 call = __builtin_amdgcn_readfirstlane(cv) + 1;
    const u32 voff = lane * 16 + 4096;
    bar();
    STAMP(3);
    if (q == 1 || q == 2) {
        const int kc = kc0 + 2 * tl;
        const u32 wo = w_slot(rank, kc, j, call), fo = wo + G_FLAG;
        Unit U;
        U.fetch(unit, lens + tl * 4, lane);
        if (q == 2) {
            U.send_all(self + wo, voff, lane);
            wait_vm<0>();
            put_flag(self + fo, call, U.meta, lane);
        } else {
            const int est = __builtin_amdgcn_readlane(wave_scan(lens[lane & 7] & 511u), 7) / 8 + PACE_SLOT - USUAL16;
            const int slot = est > PACE_SLOT + 3 || est < PACE_SLOT - 4 ? est + 2 : PACE_SLOT;
            const u32 anchor = __builtin_amdgcn_readfirstlane(lens[16]);
            const u32 t0 = anchor + PACE_T0 * slot / PACE_SLOT;
            u32 slip = 0;
            int pr = (kc + j) % 7;
#pragma unroll 1
            for (int s = 0; s < 7; ++s) {
                char* const peer = P.ws[(rank + 1 + pr) & 7];
                const char* const dst = peer + wo;
                pr = pr == 6 ? 0 : pr + 1;
                int before = 0;
#pragma unroll
                for (int k = 0; k < 4; ++k) {
                    const int v = 64 * s + j + PACE_KC * (kc - k);
                    before += v < 0 ? 0 : (v > 448 ? 448 : v);
                }
                const u32 due = t0 + ((before * slot) >> 8);
                const bool win = s < WIN_SLOTS;
                pace(win ? due - LAM : due + slip);
                st16<-4096>(dst, voff, U.p[0]); __builtin_amdgcn_s_sleep(GAP);
                st16<-3072>(dst, voff, U.p[1]); __builtin_amdgcn_s_sleep(GAP);
                if (win) { wait_vm<0>(); pace(due); }
                st16<-2048>(dst, voff, U.p[2]); __builtin_amdgcn_s_sleep(GAP);
                st16<-1024>(dst, voff, U.p[3]); __builtin_amdgcn_s_sleep(GAP);
                if (win) { wait_vm<0>(); pace(due + LAM); slip = SLIP_ON * ((u32)__builtin_amdgcn_s_memrealtime() - (due + LAM)) / 2; }
                st16<0>(dst, voff, U.p[4]); __builtin_amdgcn_s_sleep(GAP);
                U.send_blocks(dst, voff, lane);
                wait_vm<0>();
                put_flag(peer + fo, call, U.meta, lane);
                STAMP(5 + s);
            }
            if (tl) STAMP(12);
        }
    }
    if (lane == 0) __hip_atomic_store(cp, call, __ATOMIC_RELAXED, __HIP_MEMORY_SCOPE_AGENT);
    __builtin_amdgcn_endpgm();
}

__device__ __forceinline__ u32 lds_add(LAS u32* c, int lane) {
    u32 v = 0;
    if (lane == 0) v = __hip_atomic_fetch_add(c, 1u, __ATOMIC_ACQ_REL, __HIP_MEMORY_SCOPE_WORKGROUP);
    return v;
}
__device__ __forceinline__ void lds_wait(LAS u32* c, u32 n) {
    while (__builtin_amdgcn_readfirstlane(__hip_atomic_load(c, __ATOMIC_ACQUIRE, __HIP_MEMORY_SCOPE_WORKGROUP)) < n)
        __builtin_amdgcn_s_sleep(1);
}
__device__ __forceinline__ int src_of(int rank, int j, int t) {
    return t < 4 ? rank : (rank + 7 - ((t >> 2) - 1 + (t & 3) + j) % 7) & 7;
}

__device__ __forceinline__ void mma_waves(const Params& P, int b, int wave, int lane, GAS u32* cp, LAS char* lds, u32x4 a, u32x4 c, u32 cv) {
    char* const ws = P.self;
    const int rank = P.rank;
    u32 cnt_a = (u32)(size_t)(lds + L_CNT);
    asm volatile("" : "+v"(cnt_a));
    LAS u32* const cnt = (LAS u32*)(size_t)cnt_a;
    u32 call;
    {
        const int q = wave & 3, tl = wave >> 2, ux = 2 * b + tl;
        const int xdest = (ux + (ux >> 3)) & 7, xu = ux >> 3;
        char* const xdst = P.ws[xdest];
        u32 dx[8] = {a[0], a[1], a[2], a[3], c[0], c[1], c[2], c[3]};
        LAS char* const el = lds + L_ENC;
        enc_init(el, wave, lane);
        LAS u32* const lens = (LAS u32*)(el + 2 * USLOT + 512);
        call = __builtin_amdgcn_readfirstlane(cv) + 1;
        const u32 voff = lane * 16 + 4096;
        char* const xslot = xdst + x_slot(rank, xu, call);
        if ((b & 7) == (b >> 4) && (wave & 1)) {
            asm volatile("" : "+v"(dx[0]), "+v"(dx[7]));
            if (lane == 0)
                __hip_atomic_store((GAS u32*)(ws + OFF_ANC + (b >> 3) * 128 + (wave >> 1) * 4),
                                   (call << 24) | ((u32)__builtin_amdgcn_s_memrealtime() & 0xFFFFFFu), __ATOMIC_RELAXED, __HIP_MEMORY_SCOPE_AGENT);
        }
        const u32 len = encode_quarter(dx, q * 2048 + q_img(lane), el + tl * USLOT, q, el + 2 * USLOT + wave * 64, lane, [&](u32x4 v, bool again) {
            if (again) wait_vm<0>();
            st16<-4096>(xslot + q * 1024, voff, v);
            STAMP(61);
        });
        if (lane == 0) lens[wave] = len;
        bar();
        if (q == 0) {
            Unit U;
            U.fetch(el + tl * USLOT, lens + tl * 4, lane);
            st16<0>(xslot, voff, U.p[4]);
            U.send_blocks(xslot, voff, lane);
            if (lane < 4) cnt[76 + tl * 4 + lane] = U.meta;
            STAMP(62);
        }
        wait_vm<0>();
        if (__builtin_amdgcn_readfirstlane(lds_add(cnt + 66 + tl, lane)) == 3) {
            put_flag(xslot + G_FLAG, call, cnt[76 + tl * 4 + (lane & 3)], lane);
            STAMP(63);
        }
    }
    const int j = b & 63;
    const int rb = (b >> 6) * 8 + wave;
    GAS u32* const tmo = (GAS u32*)(ws + OFF_TMO);
    const __amdgpu_buffer_rsrc_t rs = __builtin_amdgcn_make_buffer_rsrc(ws, 0, WS_NEED, 0x00020000);
    const __amdgpu_buffer_rsrc_t ro = __builtin_amdgcn_make_buffer_rsrc(P.out, 0, MS * GN * 4, 0x00020000);
    const float scale = __builtin_bit_cast(float, __builtin_amdgcn_readfirstlane(__builtin_bit_cast(u32, *P.sx * *P.sw)));
    if (lane == 0) __hip_atomic_store(cp, call, __ATOMIC_RELAXED, __HIP_MEMORY_SCOPE_AGENT);
    LAS char* const rp = lds + (lane >> 4) * 2048 + (lane & 15) * 16;
    const u32 txo = OFF_XD + (src_of(rank, j, lane & 31) * 16 + rb) * 16384 + (lane & 3) * 4096;
    f32x4 acc[2][4] = {};
    u32x4 ae[4] = {}, ao[4] = {};
    auto load_a = [&](u32x4 (&a)[4], int t) {
        const u32 so = __builtin_amdgcn_readlane(txo, t);
        u32 l16 = lane * 16;
        asm volatile("" : "+v"(l16));
#pragma unroll
        for (int i = 0; i < 4; ++i) a[i] = __builtin_amdgcn_raw_buffer_load_b128(rs, l16 + i * 1024, so, 17);
    };
    LAS char* const stash = lds + L_STA + wave * 8192 + lane * 16;
    auto tile = [&](u32x4 (&a)[4], int t, u32 need, int more) {
        lds_wait(cnt + t, need);
        __builtin_amdgcn_s_setprio(2);
        LAS char* const bp = rp + (t & (NBUF - 1)) * 8192;
        u32x4 bq[8];
#pragma unroll
        for (int i = 0; i < 8; ++i) bq[i] = *(LAS u32x4*)(bp + (i >> 1) * 256 + (i & 1) * 1024);
        __builtin_amdgcn_sched_barrier(0);
#pragma unroll
        for (int c = 0; c < 4; ++c)
#pragma unroll
            for (int m = 0; m < 2; ++m) {
                const u32x4 a0 = a[2 * m], a1 = a[2 * m + 1], b0 = bq[2 * c], b1 = bq[2 * c + 1];
                const i32x8 af = {(int)a0[0], (int)a0[1], (int)a0[2], (int)a0[3], (int)a1[0], (int)a1[1], (int)a1[2], (int)a1[3]};
                const i32x8 bf = {(int)b0[0], (int)b0[1], (int)b0[2], (int)b0[3], (int)b1[0], (int)b1[1], (int)b1[2], (int)b1[3]};
                acc[m][c] = __builtin_amdgcn_mfma_scale_f32_16x16x128_f8f6f4(af, bf, acc[m][c], 1, 1, 0, 0x7F7F7F7F, 0, 0x7F7F7F7F);
            }
        asm volatile("" : "+v"(acc[0][0]), "+v"(acc[0][1]), "+v"(acc[0][2]), "+v"(acc[0][3]),
                          "+v"(acc[1][0]), "+v"(acc[1][1]), "+v"(acc[1][2]), "+v"(acc[1][3]));
        __builtin_amdgcn_sched_barrier(0);
        if (more == 1) {
            if (lane == 0) __hip_atomic_fetch_add(cnt + 32 + t, 1u, __ATOMIC_RELAXED, __HIP_MEMORY_SCOPE_WORKGROUP);
            load_a(a, t + 2);
        } else if (more == 2) {
#pragma unroll
            for (int i = 0; i < 4; ++i) a[i] = *(LAS u32x4*)(stash + (t & 1) * 4096 + i * 1024);
        }
        __builtin_amdgcn_s_setprio(0);
    };
#pragma unroll 1
    for (int pass = 0; pass < 2; ++pass) {
        if (pass) {
            lds_wait(cnt + 68, 4);
            lds_wait(cnt + 69, 4);
            wait_flags<LF_SLEEP>(ws + OFF_LF + (((lane >> 1) & 7) * 32 + rb * 2 + (lane & 1)) * 128, 0xFFFF, call, tmo);
            STAMP(3);
            load_a(ae, 30);
            load_a(ao, 31);
            lds_wait(cnt + 66, 4);
            lds_wait(cnt + 67, 4);
#pragma unroll
            for (int i = 0; i < 4; ++i) { *(LAS u32x4*)(stash + i * 1024) = ae[i]; *(LAS u32x4*)(stash + 4096 + i * 1024) = ao[i]; }
            __builtin_amdgcn_sched_barrier(0);
            load_a(ae, 0);
            __builtin_amdgcn_sched_barrier(0);
            load_a(ao, 1);
#pragma unroll
            for (int i = 0; i < 8; ++i) acc[i >> 2][i & 3] = f32x4{0.f, 0.f, 0.f, 0.f};
#pragma unroll 1
            for (int t = 0; t < 28; t += 2) {
                tile(ae, t, 0x01010101u, 1);
                tile(ao, t + 1, 0x01010101u, 1);
                if (wave == 0) STAMP(30 + (t >> 1));
            }
        }
        tile(ae, 28, pass * 0x01010101u, 2);
        tile(ao, 29, pass * 0x01010101u, 2);
        if (pass && wave == 0) STAMP(44);
        tile(ae, 30, pass * 0x01010101u, 0);
        tile(ao, 31, pass * 0x01010101u, 0);
        const float sc = cnt[72] ? __builtin_nanf("") : scale;
        if (pass) STAMP(59);
        const u32 vo = pass ? (lane >> 4) * 4 * GN * 4 + (lane & 15) * 16 : OOB;
        const u32 so = (rb * 32 * GN + j * 64) * 4;
#pragma unroll
        for (int m = 0; m < 2; ++m)
#pragma unroll
            for (int r = 0; r < 4; ++r) {
                u32x4 v;
#pragma unroll
                for (int c = 0; c < 4; ++c)
                    v[c] = __builtin_bit_cast(u32, __builtin_elementwise_maximum(acc[m][c][r] * sc, 0.f));
                __builtin_amdgcn_raw_buffer_store_b128(v, ro, vo, so + (m * 16 + r) * GN * 4, OUT_AUX);
                __builtin_amdgcn_sched_barrier(0);
                asm volatile("s_nop 1");
                __builtin_amdgcn_sched_barrier(0);
            }
    }
    STAMP(60);
    __builtin_amdgcn_endpgm();
}

__device__ __forceinline__ void unpack_waves(const Params& P, int b, int wave, int lane, GAS u32* cp, LAS char* lds, u32 cv) {
    char* const ws = P.self;
    const int rank = P.rank;
    LAS u32* const cnt = (LAS u32*)(lds + L_CNT);
    if (wave == 8) { cnt[lane] = 0; cnt[64 + lane] = 0; }
    bar();
    const u32 call = __builtin_amdgcn_readfirstlane(cv) + 1;
    const int q = wave & 3, tl = (wave >> 2) & 1;
    const int j = b & 63;
    GAS u32* const tmo = (GAS u32*)(ws + OFF_TMO);
    const __amdgpu_buffer_rsrc_t rs = __builtin_amdgcn_make_buffer_rsrc(ws, 0, WS_NEED, 0x00020000);
    LAS char* const sym = lds + L_SYM + (wave & 7) * 2304;
    const int ux = 2 * b + tl, xsrc = ux & 7, xu = ux >> 3;
    constexpr int NJOB = 17;
    u32 juo, jfo;
    {
        const int i = lane < NJOB ? lane : 0, t = i ? 2 * (i - 1) + tl : 0;
        juo = i ? w_slot(src_of(rank, j, t), t & 3, j, call) : x_slot(xsrc, xu, call);
        jfo = lane < NJOB ? juo + G_FLAG + q * 8 : OFF_TMO;
    }
    const u32 img = q * 2048 + q_img(lane);
    LAS char* image = lds + L_XIM + tl * 8192;
    __builtin_amdgcn_s_setprio(1);
    Packed pk;
    u64 look = 0;
    bool have = false;
    u32 fre = 8;
#pragma unroll 1
    for (int i = 0; i < NJOB; ++i) {
        const int t = 2 * (i - 1) + tl;
        if (__builtin_amdgcn_readfirstlane(fre) < 8) lds_wait(cnt + 32 + t - NBUF, 8);
        if (!have) {
            u32 uo = __builtin_amdgcn_readlane(juo, i);
            const char* nf = ws + __builtin_amdgcn_readlane(jfo, i + 1);
            asm volatile("" : "+s"(uo), "+s"(nf));
            u32 meta = __builtin_amdgcn_readfirstlane((u32)look);
            if (__builtin_amdgcn_readfirstlane((u32)(look >> 32)) < call) {
                const char* f = ws + __builtin_amdgcn_readlane(jfo, i);
                if (i < EAGER_FROM) meta = __builtin_amdgcn_readfirstlane((u32)wait_flags<MID_SLEEP>(f, 1, call, tmo));
                else meta = wait_flag(f, call, tmo);
            }
            asm volatile("" ::: "memory");
            load_quarter(pk, rs, uo, meta, q, lane);
            look = __hip_atomic_load((GAS u64*)nf, __ATOMIC_RELAXED, __HIP_MEMORY_SCOPE_SYSTEM);
            if (i >= SJ) STAMP(4 + 3 * (i - 9));
        }
        decode_quarter(pk, rs, img, q, image, sym, lane, [&]() {
            if (t + 2 >= NBUF) fre = __hip_atomic_load(cnt + 32 + t + 2 - NBUF, __ATOMIC_RELAXED, __HIP_MEMORY_SCOPE_WORKGROUP);
            asm volatile("" : "+v"(look));
            if (i + 1 == NJOB) {
                if ((u32)look == call) cnt[72] = 1;
            } else {
                have = __builtin_amdgcn_readfirstlane((u32)(look >> 32)) >= call;
                asm volatile("" ::: "memory");
                if (have) load_quarter(pk, rs, __builtin_amdgcn_readlane(juo, i + 1), __builtin_amdgcn_readfirstlane((u32)look), q, lane);
                look = __hip_atomic_load((GAS u64*)(ws + __builtin_amdgcn_readlane(jfo, i + 1 + have)), __ATOMIC_RELAXED, __HIP_MEMORY_SCOPE_SYSTEM);
            }
        });
        if (i) {
            __hip_atomic_store((LAS unsigned char*)(cnt + t) + q, (unsigned char)1, __ATOMIC_RELEASE, __HIP_MEMORY_SCOPE_WORKGROUP);
        } else {
            lds_order();
            const u32x2 h0 = *(LAS u32x2*)(image + img), h1 = *(LAS u32x2*)(image + img + 256);
            const u32x2 h2 = *(LAS u32x2*)(image + img + 512), h3 = *(LAS u32x2*)(image + img + 768);
            const u32x4 v0 = {h0[0], h0[1], h1[0], h1[1]}, v1 = {h2[0], h2[1], h3[0], h3[1]};
            const char* xd = ws + OFF_XD + (xsrc * 32 + xu) * 8192 + q * 2048;
            st16<0>(xd, lane * 16, v0);
            st16<1024>(xd, lane * 16, v1);
            wait_vm<0>();
            if (__builtin_amdgcn_readfirstlane(lds_add(cnt + 68 + tl, lane)) == 3)
                put_flag(ws + OFF_LF + (xsrc * 32 + xu) * 128, call, 0, lane);
            STAMP(2);
        }
        image = lds + ((t + 2) & (NBUF - 1)) * 8192;
        if (i >= SJ - 1) STAMP(6 + 3 * (i - 9));
        else if (i < 9 && SJ == 9) STAMP(48 + i);
    }
    if (lane == 0) __hip_atomic_store(cp, call, __ATOMIC_RELAXED, __HIP_MEMORY_SCOPE_AGENT);
    __builtin_amdgcn_endpgm();
}

__global__ void __launch_bounds__(1024, 4) gemm_rs_kernel(Params P) {
    __shared__ __attribute__((aligned(16))) char smem[LDS_BYTES];
    const int lane = threadIdx.x & 63;
    const int wave = __builtin_amdgcn_readfirstlane(threadIdx.x >> 6);
    u32x4 xa, xc;
    {
        const u32 ux = 2 * (blockIdx.x - N_SEND_WG) + (wave >> 2), xdest = (ux + (ux >> 3)) & 7, xu = (ux >> 3) & 31;
        const __amdgpu_buffer_rsrc_t rx = __builtin_amdgcn_make_buffer_rsrc((void*)P.x, 0, 4096 * KS, 0x00020000);
        const u32 so = (xdest * MS + (xu >> 1) * 32 + (wave & 1) * 16) * KS + (xu & 1) * 256 + (wave & 2) * 64;
        const u32 vo = ((lane & 15) * KS + (lane >> 4) * 32) | (blockIdx.x >= N_SEND_WG && wave < 8 ? 0 : OOB);
        xa = __builtin_amdgcn_raw_buffer_load_b128(rx, vo, so, 0);
        xc = __builtin_amdgcn_raw_buffer_load_b128(rx, vo + 16, so, 0);
        asm volatile("" ::: "memory");
    }
    GAS u32* cp = (GAS u32*)(P.self + OFF_CNT) + blockIdx.x * 16 + wave;
    const u32 cv = __hip_atomic_load(cp, __ATOMIC_RELAXED, __HIP_MEMORY_SCOPE_AGENT);
    asm volatile("" ::: "memory");
    STAMP(0);
    if (blockIdx.x >= N_SEND_WG) {
        if (wave < 8) mma_waves(P, blockIdx.x - N_SEND_WG, wave, lane, cp, (LAS char*)smem, xa, xc, cv);
        else unpack_waves(P, blockIdx.x - N_SEND_WG, wave, lane, cp, (LAS char*)smem, cv);
    } else {
        if (wave < 8) sender(P, blockIdx.x, wave, lane, cp, (LAS char*)smem, cv);
        else if (wave == 8) anchor_wave(P.self, lane, cp, (LAS char*)smem, cv);
    }
}

extern "C" void kernel_launch(int rank, int nranks, ncclComm_t comm,
                              hipStream_t stream,
                              void* const* d_in, const int* in_sizes, int n_in,
                              void* d_out, int out_size,
                              const int* in_dtypes, int out_dtype,
                              void* const* d_out_all,
                              void* const* d_ws_all, size_t ws_bytes) {
    if (nranks != NR || n_in != 4 || ws_bytes < WS_NEED || (unsigned)rank >= (unsigned)NR ||
        in_sizes[0] != 4096 * KS || in_sizes[1] != KS * GN || out_size != MS * GN) {
        fprintf(stderr, "kernel_launch: unexpected problem shape\n");
        return;
    }
    Params P;
    P.x = (const uint8_t*)d_in[0];
    P.w = (const uint8_t*)d_in[1];
    P.sx = (const float*)d_in[2];
    P.sw = (const float*)d_in[3];
    P.out = (float*)d_out;
    for (int p = 0; p < NR; ++p) P.ws[p] = (char*)d_ws_all[p];
    P.self = P.ws[rank];
    P.rank = rank;
    P.pad = 0;
    gemm_rs_kernel<<<GRID, 1024, 0, stream>>>(P);
    HIP_CHECK(hipGetLastError());
}
```

```cpp
#include <hip/hip_runtime.h>
#include <rccl/rccl.h>
#include <cstdio>
#include <cstdint>

#define HIP_CHECK(expr) do { hipError_t _e = (expr); if (_e != hipSuccess) { \
    fprintf(stderr, "HIP error %s at %s:%d\n", hipGetErrorString(_e), __FILE__, __LINE__); \
    return; } } while (0)

typedef unsigned u32;
typedef unsigned long long u64;
typedef u32 u32x4 __attribute__((ext_vector_type(4)));
typedef u32 u32x2 __attribute__((ext_vector_type(2)));
typedef int i32x8 __attribute__((ext_vector_type(8)));
typedef float f32x4 __attribute__((ext_vector_type(4)));
#define GAS __attribute__((address_space(1)))
#define LAS __attribute__((address_space(3)))

#define PACE_T0 318
#define PACE_SLOT 364
#define PACE_KC 16
#define USUAL16 42
#define LF_SLEEP 16
#define MID_SLEEP 8
#define EAGER_FROM 15
#define PH 222
#define LAM 70
#define WIN_SLOTS 5
#define DEAD 25
#define GAP 2
#define POLL_SLEEP 6
#define OUT_AUX 17
#define FORCE_RAW 0

constexpr int NR = 8;
constexpr int GN = 4096;
constexpr int KS = 512;
constexpr int MS = 512;
constexpr int NUNIT = 256;
constexpr int N_SEND_WG = 128;
constexpr int N_CONS_WG = 128;
constexpr int GRID = N_SEND_WG + N_CONS_WG;

constexpr u32 USLOT = 9216;
constexpr u32 GSLOT = 6144, NEAR_DW = 224, G_FLAG = 5120 + NEAR_DW * 4;
__host__ __device__ constexpr u32 w_slot(u32 src, u32 kc, u32 j, u32 call) {
    return (((((j & 7) * 8 + src) * 8 + (j >> 3)) * 4 + kc) * 2 + (call & 1)) * GSLOT;
}
__host__ __device__ constexpr u32 x_slot(u32 src, u32 xu, u32 call) { return (4096 + (src * 32 + xu) * 2 + (call & 1)) * GSLOT; }
constexpr u32 OFF_XD = x_slot(NR, 0, 0);
constexpr u32 OFF_LF = OFF_XD + NR * MS * KS;
constexpr u32 OFF_CNT = OFF_LF + NUNIT * 128;
constexpr u32 OFF_TMO = OFF_CNT + GRID * 16 * 4;
constexpr u32 OFF_ANC = OFF_TMO + 128;
constexpr u32 OFF_DBG = OFF_ANC + 16 * 128;
constexpr u32 FAR = OFF_DBG + GRID * 16 * 64 * 8;
constexpr u32 WS_NEED = FAR + OFF_XD;
constexpr u32 DEADLINE_TICKS = 100000000u;
constexpr u32 OOB = 0x80000000u;
constexpr u32 RAW_DW = 256;
constexpr int NBUF = 8;
constexpr u32 L_SYM = NBUF * 8192;
constexpr u32 L_XIM = 4 * 8192;
constexpr u32 L_ENC = L_SYM + 8 * 2304;
constexpr u32 L_STA = L_ENC;
constexpr u32 L_CNT = L_STA + 8 * 8192;
constexpr int LDS_BYTES = L_CNT + 512;

struct Params {
    const uint8_t* x;
    const uint8_t* w;
    const float* sx;
    const float* sw;
    float* out;
    char* self;
    char* ws[NR];
    int rank;
    int pad;
};

#define SJ 16
#define STAMP(i) do { } while (0)

template <int OFF>
__device__ __forceinline__ void st16(const char* base, u32 voff, u32x4 v) {
    asm volatile("global_store_dwordx4 %0, %1, %2 offset:%3 sc0 sc1\n\ts_nop 1"
                 :: "v"(voff), "v"(v), "s"(base), "i"(OFF) : "memory");
}
template <int N> __device__ __forceinline__ void wait_vm() { asm volatile("s_waitcnt vmcnt(%0)" :: "i"(N) : "memory"); }

__device__ __forceinline__ void put_flag(char* p, u32 call, u32 meta, int lane) {
    if (lane < 4)
        __hip_atomic_store((GAS u64*)p + lane, ((u64)call << 32) | meta, __ATOMIC_RELAXED, __HIP_MEMORY_SCOPE_SYSTEM);
}

template <int SLEEP>
__device__ __forceinline__ u64 wait_flags(const char* f, u64 need, u32 call, GAS u32* tmo) {
    const u32 t0 = (u32)__builtin_amdgcn_s_memrealtime();
    u64 v;
    for (u32 n = 0;;) {
        v = __hip_atomic_load((GAS u64*)f, __ATOMIC_RELAXED, __HIP_MEMORY_SCOPE_SYSTEM);
        if ((__ballot((u32)(v >> 32) >= call) & need) == need) break;
        if ((++n & 15) == 0) {
            const u32 t = __hip_atomic_load(tmo, __ATOMIC_RELAXED, __HIP_MEMORY_SCOPE_AGENT);
            if (__builtin_amdgcn_readfirstlane(t) == call || (u32)__builtin_amdgcn_s_memrealtime() - t0 > DEADLINE_TICKS) {
                __hip_atomic_store(tmo, call, __ATOMIC_RELAXED, __HIP_MEMORY_SCOPE_AGENT);
                break;
            }
        }
        __builtin_amdgcn_s_sleep(SLEEP);
    }
    asm volatile("" ::: "memory");
    return v;
}

__device__ __forceinline__ u32 wait_flag(const char* f, u32 call, GAS u32* tmo) {
    const u32 t0 = (u32)__builtin_amdgcn_s_memrealtime();
    u32 meta;
    u64 va = __hip_atomic_load((GAS u64*)f, __ATOMIC_RELAXED, __HIP_MEMORY_SCOPE_SYSTEM);
    __builtin_amdgcn_s_sleep(POLL_SLEEP);
    for (u32 n = 1;; ++n) {
        const u64 vb = __hip_atomic_load((GAS u64*)f, __ATOMIC_RELAXED, __HIP_MEMORY_SCOPE_SYSTEM);
        meta = __builtin_amdgcn_readfirstlane((u32)va);
        if (__builtin_amdgcn_readfirstlane((u32)(va >> 32)) >= call) break;
        va = __hip_atomic_load((GAS u64*)f, __ATOMIC_RELAXED, __HIP_MEMORY_SCOPE_SYSTEM);
        meta = __builtin_amdgcn_readfirstlane((u32)vb);
        if (__builtin_amdgcn_readfirstlane((u32)(vb >> 32)) >= call) break;
        if ((n & 15) == 0) {
            const u32 t = __hip_atomic_load(tmo, __ATOMIC_RELAXED, __HIP_MEMORY_SCOPE_AGENT);
            if (__builtin_amdgcn_readfirstlane(t) == call || (u32)__builtin_amdgcn_s_memrealtime() - t0 > DEADLINE_TICKS) {
                __hip_atomic_store(tmo, call, __ATOMIC_RELAXED, __HIP_MEMORY_SCOPE_AGENT);
                break;
            }
            va = __hip_atomic_load((GAS u64*)f, __ATOMIC_RELAXED, __HIP_MEMORY_SCOPE_SYSTEM);
            __builtin_amdgcn_s_sleep(POLL_SLEEP);
        }
    }
    asm volatile("" ::: "memory");
    return meta;
}

__device__ __forceinline__ u32 wave_scan(u32 v) {
    v += __builtin_amdgcn_update_dpp(0u, v, 0x111, 0xf, 0xf, true);
    v += __builtin_amdgcn_update_dpp(0u, v, 0x112, 0xf, 0xf, true);
    v += __builtin_amdgcn_update_dpp(0u, v, 0x114, 0xf, 0xf, true);
    v += __builtin_amdgcn_update_dpp(0u, v, 0x118, 0xf, 0xf, true);
    v += __builtin_amdgcn_update_dpp(0u, v, 0x142, 0xa, 0xf, false);
    v += __builtin_amdgcn_update_dpp(0u, v, 0x143, 0xc, 0xf, false);
    return v;
}
__device__ __forceinline__ u32 bfi(u32 m, u32 a, u32 b) { return (m & a) | (~m & b); }
__device__ __forceinline__ void lds_order() { __builtin_amdgcn_fence(__ATOMIC_ACQ_REL, "wavefront"); }

__device__ __forceinline__ u32 spread80(u32 m) { return (m << 1) - (m >> 7); }
template <class Early>
__device__ __forceinline__ u32 encode_quarter(const u32 (&d)[8], u32 img, LAS char* unit, int q, LAS char* stage,
                                              LAS char* lut, int lane, u32& meta, Early codes_out) {
    u32 mag[8], sum = 0;
#pragma unroll
    for (int k = 0; k < 8; ++k) { mag[k] = d[k] & 0x7F7F7F7Fu; sum = __builtin_amdgcn_sad_u8(mag[k], 0u, sum); }
    int b = ((int)__builtin_amdgcn_readlane(wave_scan(sum), 63) + 1024 - 14541) >> 11;
    b = b < 12 ? 12 : b;
    asm volatile("" : "+s"(b));
    const u32 B = b > 108 ? 108 : b;
    const u32 c2 = (126 - B) * 0x01010101u, c16 = (112 - B) * 0x01010101u;
    u32 nib[8], esc[8];
#pragma unroll
    for (int k = 0; k < 8; ++k) {
        const u32 u = mag[k] + c2, w = mag[k] + c16;
        const u32 dir = u & ~w & 0x80808080u;
        nib[k] = w & (((dir >> 3) - (dir >> 6)) | 0x01010101u);
        esc[k] = dir ^ 0x80808080u;
    }
    codes_out(u32x4{nib[0] | (nib[1] << 4), nib[2] | (nib[3] << 4), nib[4] | (nib[5] << 4), nib[6] | (nib[7] << 4)}, false);
    const u32 amax = B >= 40 ? 20 : B >> 1;
    const u32 cz = (129 + B) * 0x01010101u, clo = (128 - B + 2 * amax) * 0x01010101u, c20 = (108 - B) * 0x01010101u;
    u32 sel[8], off[8], cnt = 0, sg = 0, mb = 0;
#pragma unroll
    for (int k = 0; k < 8; ++k) sel[k] = *(LAS u32*)(lut + (__builtin_amdgcn_udot4(esc[k], 0x08040201u, 0u, false) >> 5));
#pragma unroll
    for (int k = 0; k < 8; ++k) {
        mb |= ((~(mag[k] + clo) | (mag[k] + c20)) & 0x80808080u) >> (7 - k);
        off[k] = cnt;
        cnt += __builtin_popcount(esc[k]);
        sg |= (d[k] >> (7 - k)) & (0x01010101u << k);
    }
    const u32 incl = wave_scan(cnt);
    const u32 nrec = (__builtin_amdgcn_readlane(incl, 63) + 7) >> 3, n1 = (3 * nrec + 3) >> 2;
    LAS char* blk = unit + 5120 + q * 1024;
    *(LAS u32*)(unit + 4096 + q * 256 + lane * 4) = sg;
    u32 n2 = 0, ne = 0;
    if (nrec <= 192 && !FORCE_RAW) {
        const u32 base = incl - cnt;
#pragma unroll
        for (int k = 0; k < 8; ++k) {
            const u32 v = ((cz - mag[k]) >> 1) & 0x3F3F3F3Fu, pos = base + off[k];
            const u64 x = (u64)__builtin_amdgcn_perm(0u, v, sel[k]) << (8 * (pos & 3));
            LAS u32* p = (LAS u32*)(stage + (pos & ~3u));
            __hip_atomic_fetch_or(p, (u32)x, __ATOMIC_RELAXED, __HIP_MEMORY_SCOPE_WORKGROUP);
            __hip_atomic_fetch_or(p + 1, (u32)(x >> 32), __ATOMIC_RELAXED, __HIP_MEMORY_SCOPE_WORKGROUP);
        }
        lds_order();
        const u32 cga = (127 - amax) * 0x01010101u;
        u32 nfar = 0;
        for (u32 r0 = 0; r0 < nrec; r0 += 64) {
            const u32x2 vv = *(LAS u32x2*)(stage + (r0 + lane) * 8);
            u32 c1[2], cf[2], far[2];
#pragma unroll
            for (int h = 0; h < 2; ++h) {
                const u32 v = vv[h];
                const u32 ge6 = v + 0x7A7A7A7Au, gta = v + cga, ge55 = v + 0x49494949u, ge56 = v + 0x48484848u;
                far[h] = ((ge6 & ~gta) | (ge55 & ~ge56)) & 0x80808080u;
                c1[h] = ((v & ~spread80(gta & 0x80808080u)) | (spread80(ge56 & 0x80808080u) & 0x06060606u) | spread80(far[h])) & 0x07070707u;
                const u32 t = (ge6 | spread80(ge55 & 0x80808080u)) & 0x0F0F0F0Fu;
                const u32 sl = *(LAS u32*)(lut + (__builtin_amdgcn_udot4(far[h], 0x08040201u, 0u, false) >> 5));
                const u32 y = __builtin_amdgcn_perm(0u, t, sl);
                cf[h] = __builtin_amdgcn_perm(0u, y | (y >> 4), 0x0C0C0200u);
            }
            const u32 x = c1[0] | (c1[1] << 3);
            const u32 rec = x | (__umul24(x >> 24, 0x41040u) & 0xC0C0C0u);
            if (r0 + lane < nrec) {
                LAS unsigned char* rp = (LAS unsigned char*)blk + 4 * n1 - 3 * (r0 + lane + 1);
                rp[0] = rec; rp[1] = rec >> 8; rp[2] = rec >> 16;
            }
            const u32 f0 = __builtin_popcount(far[0]), f = f0 + __builtin_popcount(far[1]), fi = wave_scan(f);
            const u32 pos = nfar + fi - f;
            nfar += __builtin_amdgcn_readlane(fi, 63);
            if (f != 0 && pos <= 256) {
                const u64 z = (u64)(cf[0] | (cf[1] << (4 * f0))) << (4 * (pos & 7));
                LAS u32* p = (LAS u32*)(blk + 4 * n1 + ((pos >> 1) & ~3u));
                __hip_atomic_fetch_or(p, (u32)z, __ATOMIC_RELAXED, __HIP_MEMORY_SCOPE_WORKGROUP);
                __hip_atomic_fetch_or(p + 1, (u32)(z >> 32), __ATOMIC_RELAXED, __HIP_MEMORY_SCOPE_WORKGROUP);
            }
        }
        n2 = (nfar + 7) >> 3;
        lds_order();
        for (u64 mk; (mk = __ballot(mb != 0)) != 0; ne += __builtin_popcountll(mk)) {
            if (mb != 0) {
                const u32 bit = __builtin_ctz(mb);
                mb &= mb - 1;
                const u32 k0 = (u32)((int)(bit << 31) >> 31), k1 = (u32)((int)(bit << 30) >> 31), k2 = (u32)((int)(bit << 29) >> 31);
                const u32 m0 = bfi(k2, mag[4], mag[0]), m1 = bfi(k2, mag[5], mag[1]), m2 = bfi(k2, mag[6], mag[2]), m3 = bfi(k2, mag[7], mag[3]);
                const u32 mg = (bfi(k0, bfi(k1, m3, m1), bfi(k1, m2, m0)) >> (bit & 24u)) & 127u;
                const u32 ad = img + (bit & 6u) * 128 + (k0 & 4u) + (bit >> 3);
                const u32 v = ((129 + B - mg) >> 1) & 63u;
                const u32 dflt = B - (v <= amax ? 2 * v : 0) + ((mg - B) & 1u);
                const u32 slot = n1 + n2 + ne + __builtin_amdgcn_mbcnt_hi((u32)(mk >> 32), __builtin_amdgcn_mbcnt_lo((u32)mk, 0u));
                *(LAS u32*)(blk + (slot < 255 ? slot : 255) * 4) = (ad << 8) | (mg ^ dflt);
            }
        }
    }
    if (nrec > 192 || n2 > 32 || ne > 31 || FORCE_RAW) {
        lds_order();
        codes_out(u32x4{d[0], d[1], d[2], d[3]}, true);
        *(LAS u32x4*)(blk + lane * 16) = u32x4{d[4], d[5], d[6], d[7]};
        meta = 3u << 11;
        return RAW_DW;
    }
    meta = n2 | (ne << 6) | ((nrec > 128 ? 2u : (nrec > 64 ? 1u : 0u)) << 11) | (B << 15) | (n1 << 22);
    return n1 + n2 + ne;
}

struct Packed {
    u32x4 n;
    u32 s, b1, b2, be;
    u32 meta, uo;
};
__device__ __forceinline__ u32 blk_off(u32 uo, u32 meta) { return uo + (meta & 0x4000u ? FAR : 5120) + (meta >> 22) * 4; }
__device__ __forceinline__ void load_quarter(Packed& p, __amdgpu_buffer_rsrc_t rs, u32 uo, u32 meta, int q, int lane) {
    p.n = __builtin_amdgcn_raw_buffer_load_b128(rs, lane * 16 + q * 1024, uo, 17);
    __builtin_amdgcn_sched_barrier(0);
    p.meta = meta;
    p.uo = uo;
    const u32 bo = blk_off(uo, meta), n2 = meta & 63u;
    p.b1 = __builtin_amdgcn_raw_buffer_load_b32(rs, 189 - 3 * lane, bo - 192, 17);
    p.b2 = __builtin_amdgcn_raw_buffer_load_b16(rs, (u32)lane < 2 * n2 ? lane * 2 : OOB, bo, 17);
    p.s = __builtin_amdgcn_raw_buffer_load_b32(rs, lane * 4 + 4096 + q * 256, uo, 17);
    p.be = __builtin_amdgcn_raw_buffer_load_b32(rs, (u32)lane < ((meta >> 6) & 31u) ? lane * 4 : OOB, bo + n2 * 4, 17);
}
struct Rec {
    u32 near[2], sel[2], f0, f;
    __device__ __forceinline__ Rec(u32 rec, u32 lut0, u32 lut1) {
        const u32 x = bfi(0xFF000000u, __umul24(rec & 0xC0C0C0u, 0x41040u), rec);
#pragma unroll
        for (int h = 0; h < 2; ++h) {
            const u32 c = (h ? x >> 3 : x) & 0x07070707u;
            near[h] = __builtin_amdgcn_perm(lut1, lut0, c);
            const u32 e10 = (c + 0x09090909u) & 0x10101010u;
            sel[h] = __umul24(e10, 0x101010u) + (e10 ^ 0x10101010u);
            if (h) f = f0 + __builtin_popcount(e10);
            else f0 = __builtin_popcount(e10);
        }
    }
    u32 ah[2], g0[2], g1[2];
    __device__ __forceinline__ void gather(u32 a) {
#pragma unroll
        for (int h = 0; h < 2; ++h) {
            ah[h] = h ? a + f0 : a;
            LAS u32* gp = (LAS u32*)(size_t)(ah[h] & ~3u);
            g0[h] = gp[0]; g1[h] = gp[1];
        }
    }
    __device__ __forceinline__ u32x2 bytes() const {
        u32x2 o;
#pragma unroll
        for (int h = 0; h < 2; ++h) {
            const u32 g = __builtin_amdgcn_alignbyte(g1[h], g0[h], ah[h]);
            o[h] = near[h] & __builtin_amdgcn_perm(g, g, sel[h]);
        }
        return o;
    }
};
template <class Mid>
__device__ __forceinline__ void decode_quarter(const Packed& p, __amdgpu_buffer_rsrc_t rs, u32 img, int q,
                                               LAS char* image, LAS char* sym, int lane, Mid mid) {
    const u32 uo = p.uo, bo = blk_off(uo, p.meta), cls = (p.meta >> 11) & 3u, B1 = ((p.meta >> 15) & 127u) * 0x01010101u;
    const u32 kx = ~B1, lut0 = (B1 - 0x06040200u) ^ kx, lut1 = ((B1 + 0x00100000u - 0x00000A08u) ^ kx) | 0xFF000000u;
    u32 nib[8], e10[8], off[8], cnt = 0;
#pragma unroll
    for (int k = 0; k < 8; ++k) {
        nib[k] = (k & 1 ? p.n[k >> 1] >> 4 : p.n[k >> 1]) & 0x0F0F0F0Fu;
        e10[k] = ~(nib[k] + 0x0E0E0E0Eu) & 0x10101010u;
        off[k] = cnt;
        cnt += __builtin_popcount(e10[k]);
    }
    __builtin_amdgcn_sched_barrier(0);
    {
        u32 t = (p.b2 | (p.b2 << 8)) & 0x00FF00FFu;
        t = (t | (t << 4)) & 0x0F0F0F0Fu;
        const u32 h = (t + 0x01010101u) & 0x10101010u;
        *(LAS u32*)(sym + 1600 + lane * 4) = (B1 - 0x0C0C0C0Cu + ((h << 2) - (h >> 2)) - (t << 1)) ^ kx;
    }
    Rec R(p.b1, lut0, lut1);
    const u32 both = cnt | (R.f << 16), inc = wave_scan(both), exc = inc - both;
    lds_order();
    R.gather((u32)(size_t)sym + 1600 + (exc >> 16));
    __builtin_amdgcn_sched_barrier(0);
    u32 ga[8], g0[8], g1[8], sel[8], o[8];
#pragma unroll
    for (int k = 0; k < 8; ++k) {
        off[k] += (u32)(size_t)sym + (exc & 0xFFFFu);
        ga[k] = off[k] & ~3u;
        nib[k] |= (p.s << (7 - k)) & 0x80808080u;
    }
    asm volatile("" : "+v"(ga[0]), "+v"(ga[1]), "+v"(ga[2]), "+v"(ga[3]), "+v"(ga[4]), "+v"(ga[5]), "+v"(ga[6]), "+v"(ga[7]),
                      "+v"(nib[0]), "+v"(nib[1]), "+v"(nib[2]), "+v"(nib[3]), "+v"(nib[4]), "+v"(nib[5]), "+v"(nib[6]), "+v"(nib[7]));
    __builtin_amdgcn_sched_barrier(0);
    *(LAS u32x2*)(sym + lane * 8) = R.bytes();
    if (cls == 1 || cls == 2) {
        u32 nf = __builtin_amdgcn_readlane(inc, 63) >> 16;
        for (u32 c = 1; c <= cls; ++c) {
            Rec Rc(__builtin_amdgcn_raw_buffer_load_b32(rs, 189 - 3 * lane, bo - 192 * (c + 1), 17), lut0, lut1);
            const u32 fi = wave_scan(Rc.f);
            Rc.gather((u32)(size_t)sym + 1600 + ((nf + fi - Rc.f) & 511u));
            *(LAS u32x2*)(sym + c * 512 + lane * 8) = Rc.bytes();
            nf += __builtin_amdgcn_readlane(fi, 63);
        }
    }
    lds_order();
#pragma unroll
    for (int k = 0; k < 8; ++k) {
        LAS u32* gp = (LAS u32*)(size_t)ga[k];
        g0[k] = gp[0]; g1[k] = gp[1];
    }
    __builtin_amdgcn_sched_barrier(0);
#pragma unroll
    for (int k = 0; k < 8; ++k)
        sel[k] = __umul24(e10[k], 0x101010u) + (e10[k] ^ 0x10101010u);
    u32 ev = (p.be & 127u) << ((p.be >> 5) & 24u);
    u32 ea = (u32)(size_t)image + ((p.be >> 8) & 0x1FFCu);
    u32 wa = (u32)(size_t)image + img;
    asm volatile("" : "+v"(ev), "+v"(ea), "+v"(wa), "+v"(sel[0]), "+v"(sel[1]), "+v"(sel[2]), "+v"(sel[3]),
                      "+v"(sel[4]), "+v"(sel[5]), "+v"(sel[6]), "+v"(sel[7]));
    mid();
    __builtin_amdgcn_sched_barrier(0);
#pragma unroll
    for (int c = 0; c < 4; ++c) {
#pragma unroll
        for (int k = 2 * c; k < 2 * c + 2; ++k) {
            const u32 g = __builtin_amdgcn_alignbyte(g1[k], g0[k], off[k]);
            o[k] = (__builtin_amdgcn_perm(g, g, sel[k]) ^ kx) + nib[k];
        }
        *(LAS u32x2*)(size_t)(wa + c * 256) = u32x2{o[2 * c], o[2 * c + 1]};
    }
    lds_order();
    if (ev) __hip_atomic_fetch_xor((LAS u32*)(size_t)ea, ev, __ATOMIC_RELAXED, __HIP_MEMORY_SCOPE_WORKGROUP);
    if (cls == 3) {
        const u32x4 l = __builtin_amdgcn_raw_buffer_load_b128(rs, lane * 16 + q * 1024, uo, 17);
        const u32x4 h = __builtin_amdgcn_raw_buffer_load_b128(rs, lane * 16, bo, 17);
        *(LAS u32x2*)(size_t)wa = u32x2{l[0], l[1]};
        *(LAS u32x2*)(size_t)(wa + 256) = u32x2{l[2], l[3]};
        *(LAS u32x2*)(size_t)(wa + 512) = u32x2{h[0], h[1]};
        *(LAS u32x2*)(size_t)(wa + 768) = u32x2{h[2], h[3]};
    }
}

__device__ __forceinline__ u32 q_img(int lane) {
    return (lane >> 5) * 1024 + ((lane & 7) | ((lane >> 1) & 8)) * 16 + (lane & 8);
}

struct Unit {
    u32x4 p[6];
    u32 meta;
    u32 c1, c2, c3, tot;
    u32 tot16;
    LAS char* lds;
    __device__ __forceinline__ u32x4 blk16(u32 i) const {
        u32x4 v;
#pragma unroll
        for (int k = 0; k < 4; ++k) {
            const u32 w = 4 * i + k;
            const u32 a = w + (w < c1 ? 0 : (w < c2 ? 256 - c1 : (w < c3 ? 512 - c2 : 768 - c3)));
            v[k] = *(LAS u32*)(lds + 5120 + (a & 1023u) * 4);
        }
        return v;
    }
    __device__ __forceinline__ void fetch(LAS char* unit, LAS u32* lens, int lane) {
        lds = unit;
        const u32 l = lens[lane & 3];
        c1 = __builtin_amdgcn_readlane(l, 0);
        c2 = c1 + __builtin_amdgcn_readlane(l, 1);
        c3 = c2 + __builtin_amdgcn_readlane(l, 2);
        tot = c3 + __builtin_amdgcn_readlane(l, 3);
        tot16 = (tot + 3) >> 2;
        meta = lens[8 + (lane & 3)] + (tot > NEAR_DW ? 0x4000u : 0u) + ((lane == 0 ? 0 : (lane == 1 ? c1 : (lane == 2 ? c2 : c3))) << 22);
#pragma unroll
        for (int i = 0; i < 5; ++i) p[i] = *(LAS u32x4*)(unit + i * 1024 + lane * 16);
        p[5] = blk16(lane);
    }
    __device__ __forceinline__ void send_blocks(const char* dst, u32 voff, int lane) const {
        if (tot > NEAR_DW) dst += FAR - 5120;
        for (u32 i = 64 + lane; i < tot16; i += 64) st16<1024>(dst, voff + (i - lane) * 16, blk16(i));
        if ((u32)lane < tot16) st16<1024>(dst, voff, p[5]);
    }
    __device__ __forceinline__ void send_all(const char* dst, u32 voff, int lane) const {
        st16<-4096>(dst, voff, p[0]); st16<-3072>(dst, voff, p[1]); st16<-2048>(dst, voff, p[2]);
        st16<-1024>(dst, voff, p[3]); st16<0>(dst, voff, p[4]);
        send_blocks(dst, voff, lane);
    }
};

__device__ __forceinline__ void pace(u32 due) {
    while ((int)((u32)__builtin_amdgcn_s_memrealtime() - due) < 0) __builtin_amdgcn_s_sleep(1);
}

__device__ __forceinline__ void bar() {
    asm volatile("s_waitcnt lgkmcnt(0)" ::: "memory");
    __builtin_amdgcn_s_barrier();
    asm volatile("" ::: "memory");
}
constexpr u32 E_STAGE = 2 * USLOT + 1024;
__device__ __forceinline__ void enc_init(LAS char* lds, int wave, int lane) {
    *(LAS u32x4*)(lds + (wave >> 2) * USLOT + 5120 + (wave & 3) * 1024 + lane * 16) = u32x4{0, 0, 0, 0};
    *(LAS u32x4*)(lds + E_STAGE + wave * 2048 + lane * 16) = u32x4{0, 0, 0, 0};
    *(LAS u32x4*)(lds + E_STAGE + wave * 2048 + 1024 + lane * 16) = u32x4{0, 0, 0, 0};
    u32 sel = 0x0C0C0C0Cu, n = 0;
#pragma unroll
    for (int i = 0; i < 4; ++i)
        if ((lane >> i) & 1) { sel = (sel & ~(0xFFu << n)) | ((u32)i << n); n += 8; }
    if (lane < 16) *(LAS u32*)(lds + 2 * USLOT + wave * 64 + lane * 4) = sel;
    lds_order();
}

__device__ __forceinline__ void anchor_wave(const char* self, int lane, GAS u32* cp, LAS char* lds, u32 cv) {
    const u32 call = __builtin_amdgcn_readfirstlane(cv) + 1;
    u32 v, now;
    u64 ok;
    int n = 0;
    __builtin_amdgcn_s_sleep(16);
    do {
        __builtin_amdgcn_s_sleep(4);
        v = __hip_atomic_load((GAS u32*)(self + OFF_ANC + (lane >> 2) * 128 + (lane & 3) * 4), __ATOMIC_RELAXED, __HIP_MEMORY_SCOPE_AGENT);
        ok = __ballot((v >> 24) == (call & 255u));
        now = (u32)__builtin_amdgcn_s_memrealtime();
    } while (ok != ~0ull && ++n < 8);
    u32 age = (now - v) & 0xFFFFFFu;
    age = (v >> 24) == (call & 255u) ? (age > 1000 ? 1000 : age) : 0;
    *(LAS u32*)(lds + 2 * USLOT + 512 + 64) = now - (__builtin_amdgcn_readlane(wave_scan(age), 63) >> 6);
    bar();
    if (lane == 0) __hip_atomic_store(cp, call, __ATOMIC_RELAXED, __HIP_MEMORY_SCOPE_AGENT);
    __builtin_amdgcn_endpgm();
}

__device__ __forceinline__ void sender(const Params& P, int b, int wave, int lane, GAS u32* cp, LAS char* lds, u32 cv) {
    const int rank = P.rank;
    char* const self = P.self;
    const int q = wave & 3, tl = wave >> 2;
    const int j = b & 63, kc0 = b >> 6;
    u32 wr[8];
    {
        const u32 cg = (lane & 7) | ((lane >> 1) & 8), rg = ((lane >> 3) & 1) | ((lane >> 4) & 2);
        const uint8_t* wp = P.w + (size_t)((kc0 + 2 * tl) * 128 + q * 32 + rg * 8) * GN + j * 64 + cg * 4;
#pragma unroll
        for (int i = 0; i < 8; ++i) wr[i] = *(const GAS u32*)(wp + (size_t)i * GN);
    }
    enc_init(lds, wave, lane);
    LAS char* const unit = lds + tl * USLOT;
    LAS u32* const lens = (LAS u32*)(lds + 2 * USLOT + 512);
    u32 dw[8];
#pragma unroll
    for (int h = 0; h < 2; ++h) {
        const u32 r0 = wr[4 * h], r1 = wr[4 * h + 1], r2 = wr[4 * h + 2], r3 = wr[4 * h + 3];
        const u32 t0 = __builtin_amdgcn_perm(r1, r0, 0x05010400u), t1 = __builtin_amdgcn_perm(r1, r0, 0x07030602u);
        const u32 t2 = __builtin_amdgcn_perm(r3, r2, 0x05010400u), t3 = __builtin_amdgcn_perm(r3, r2, 0x07030602u);
        dw[0 + h] = __builtin_amdgcn_perm(t2, t0, 0x05040100u);
        dw[2 + h] = __builtin_amdgcn_perm(t2, t0, 0x07060302u);
        dw[4 + h] = __builtin_amdgcn_perm(t3, t1, 0x05040100u);
        dw[6 + h] = __builtin_amdgcn_perm(t3, t1, 0x07060302u);
    }
    u32 qmeta;
    const u32 len = encode_quarter(dw, q * 2048 + q_img(lane), unit, q, lds + E_STAGE + wave * 2048, lds + 2 * USLOT + wave * 64, lane, qmeta,
                                   [&](u32x4 v, bool) { *(LAS u32x4*)(unit + q * 1024 + lane * 16) = v; });
    if (lane == 0) { lens[wave] = len; lens[8 + wave] = qmeta; }
    const u32 call = __builtin_amdgcn_readfirstlane(cv) + 1;
    const u32 voff = lane * 16 + 4096;
    bar();
    STAMP(3);
    if (q == 1 || q == 2) {
        const int kc = kc0 + 2 * tl;
        const u32 wo = w_slot(rank, kc, j, call), fo = wo + G_FLAG;
        Unit U;
        U.fetch(unit, lens + tl * 4, lane);
        if (q == 2) {
            U.send_all(self + wo, voff, lane);
            wait_vm<0>();
            put_flag(self + fo, call, U.meta, lane);
        } else {
            const int est = __builtin_amdgcn_readlane(wave_scan(lens[lane & 7]), 7) / 8 + PACE_SLOT - USUAL16;
            const int slot = est > PACE_SLOT + 3 || est < PACE_SLOT - 4 ? est + 2 : PACE_SLOT;
            const u32 anchor = __builtin_amdgcn_readfirstlane(lens[16]);
            const u32 t0 = anchor + PACE_T0 * slot / PACE_SLOT;
            u32 slip = 0;
            int pr = (kc + j) % 7;
#pragma unroll 1
            for (int s = 0; s < 7; ++s) {
                char* const peer = P.ws[(rank + 1 + pr) & 7];
                const char* const dst = peer + wo;
                pr = pr == 6 ? 0 : pr + 1;
                int before = 0;
#pragma unroll
                for (int k = 0; k < 4; ++k) {
                    const int v = 64 * s + j + PACE_KC * (kc - k);
                    before += v < 0 ? 0 : (v > 448 ? 448 : v);
                }
                const u32 due = t0 + ((before * slot) >> 8);
                const bool win = s < WIN_SLOTS;
                pace(win ? due - LAM : due + slip);
                st16<-4096>(dst, voff, U.p[0]); __builtin_amdgcn_s_sleep(GAP);
                st16<-3072>(dst, voff, U.p[1]); __builtin_amdgcn_s_sleep(GAP);
                if (win) { wait_vm<0>(); pace(due); }
                st16<-2048>(dst, voff, U.p[2]); __builtin_amdgcn_s_sleep(GAP);
                st16<-1024>(dst, voff, U.p[3]); __builtin_amdgcn_s_sleep(GAP);
                if (win) { wait_vm<0>(); pace(due + LAM); const int late = (int)((u32)__builtin_amdgcn_s_memrealtime() - (due + LAM + DEAD)); slip = late > 0 ? late / 2 : 0; }
                st16<0>(dst, voff, U.p[4]); __builtin_amdgcn_s_sleep(GAP);
                U.send_blocks(dst, voff, lane);
                wait_vm<0>();
                put_flag(peer + fo, call, U.meta, lane);
                STAMP(5 + s);
            }
            if (tl) STAMP(12);
        }
    }
    if (lane == 0) __hip_atomic_store(cp, call, __ATOMIC_RELAXED, __HIP_MEMORY_SCOPE_AGENT);
    __builtin_amdgcn_endpgm();
}

__device__ __forceinline__ u32 lds_add(LAS u32* c, int lane) {
    u32 v = 0;
    if (lane == 0) v = __hip_atomic_fetch_add(c, 1u, __ATOMIC_ACQ_REL, __HIP_MEMORY_SCOPE_WORKGROUP);
    return v;
}
__device__ __forceinline__ void lds_wait(LAS u32* c, u32 n) {
    while (__builtin_amdgcn_readfirstlane(__hip_atomic_load(c, __ATOMIC_ACQUIRE, __HIP_MEMORY_SCOPE_WORKGROUP)) < n)
        __builtin_amdgcn_s_sleep(1);
}
__device__ __forceinline__ int src_of(int rank, int j, int t) {
    return t < 4 ? rank : (rank + 7 - ((t >> 2) - 1 + (t & 3) + j) % 7) & 7;
}

__device__ __forceinline__ void mma_waves(const Params& P, int b, int wave, int lane, GAS u32* cp, LAS char* lds, u32x4 a, u32x4 c, u32 cv) {
    char* const ws = P.self;
    const int rank = P.rank;
    u32 cnt_a = (u32)(size_t)(lds + L_CNT);
    asm volatile("" : "+v"(cnt_a));
    LAS u32* const cnt = (LAS u32*)(size_t)cnt_a;
    u32 call;
    {
        const int q = wave & 3, tl = wave >> 2, ux = 2 * b + tl;
        const int xdest = (ux + (ux >> 3)) & 7, xu = ux >> 3;
        char* const xdst = P.ws[xdest];
        u32 dx[8] = {a[0], a[1], a[2], a[3], c[0], c[1], c[2], c[3]};
        LAS char* const el = lds + L_ENC;
        enc_init(el, wave, lane);
        LAS u32* const lens = (LAS u32*)(el + 2 * USLOT + 512);
        call = __builtin_amdgcn_readfirstlane(cv) + 1;
        const u32 voff = lane * 16 + 4096;
        char* const xslot = xdst + x_slot(rank, xu, call);
        if ((b & 7) == (b >> 4) && (wave & 1)) {
            asm volatile("" : "+v"(dx[0]), "+v"(dx[7]));
            if (lane == 0)
                __hip_atomic_store((GAS u32*)(ws + OFF_ANC + (b >> 3) * 128 + (wave >> 1) * 4),
                                   (call << 24) | ((u32)__builtin_amdgcn_s_memrealtime() & 0xFFFFFFu), __ATOMIC_RELAXED, __HIP_MEMORY_SCOPE_AGENT);
        }
        u32 qmeta;
        const u32 len = encode_quarter(dx, q * 2048 + q_img(lane), el + tl * USLOT, q, el + E_STAGE + wave * 2048, el + 2 * USLOT + wave * 64, lane, qmeta, [&](u32x4 v, bool again) {
            if (again) wait_vm<0>();
            st16<-4096>(xslot + q * 1024, voff, v);
            STAMP(61);
        });
        if (lane == 0) { lens[wave] = len; lens[8 + wave] = qmeta; }
        bar();
        if (q == 0) {
            Unit U;
            U.fetch(el + tl * USLOT, lens + tl * 4, lane);
            st16<0>(xslot, voff, U.p[4]);
            U.send_blocks(xslot, voff, lane);
            if (lane < 4) cnt[76 + tl * 4 + lane] = U.meta;
            STAMP(62);
        }
        wait_vm<0>();
        if (__builtin_amdgcn_readfirstlane(lds_add(cnt + 66 + tl, lane)) == 3) {
            put_flag(xslot + G_FLAG, call, cnt[76 + tl * 4 + (lane & 3)], lane);
            STAMP(63);
        }
    }
    const int j = b & 63;
    const int rb = (b >> 6) * 8 + wave;
    GAS u32* const tmo = (GAS u32*)(ws + OFF_TMO);
    const __amdgpu_buffer_rsrc_t rs = __builtin_amdgcn_make_buffer_rsrc(ws, 0, WS_NEED, 0x00020000);
    const __amdgpu_buffer_rsrc_t ro = __builtin_amdgcn_make_buffer_rsrc(P.out, 0, MS * GN * 4, 0x00020000);
    const float scale = __builtin_bit_cast(float, __builtin_amdgcn_readfirstlane(__builtin_bit_cast(u32, *P.sx * *P.sw)));
    if (lane == 0) __hip_atomic_store(cp, call, __ATOMIC_RELAXED, __HIP_MEMORY_SCOPE_AGENT);
    LAS char* const rp = lds + (lane >> 4) * 2048 + (lane & 15) * 16;
    const u32 txo = OFF_XD + (src_of(rank, j, lane & 31) * 16 + rb) * 16384 + (lane & 3) * 4096;
    f32x4 acc[2][4] = {};
    u32x4 ae[4] = {}, ao[4] = {};
    auto load_a = [&](u32x4 (&a)[4], int t) {
        const u32 so = __builtin_amdgcn_readlane(txo, t);
        u32 l16 = lane * 16;
        asm volatile("" : "+v"(l16));
#pragma unroll
        for (int i = 0; i < 4; ++i) a[i] = __builtin_amdgcn_raw_buffer_load_b128(rs, l16 + i * 1024, so, 17);
    };
    LAS char* const stash = lds + L_STA + wave * 8192 + lane * 16;
    auto tile = [&](u32x4 (&a)[4], int t, u32 need, int more) {
        lds_wait(cnt + t, need);
        __builtin_amdgcn_s_setprio(2);
        LAS char* const bp = rp + (t & (NBUF - 1)) * 8192;
        u32x4 bq[8];
#pragma unroll
        for (int i = 0; i < 8; ++i) bq[i] = *(LAS u32x4*)(bp + (i >> 1) * 256 + (i & 1) * 1024);
        __builtin_amdgcn_sched_barrier(0);
#pragma unroll
        for (int c = 0; c < 4; ++c)
#pragma unroll
            for (int m = 0; m < 2; ++m) {
                const u32x4 a0 = a[2 * m], a1 = a[2 * m + 1], b0 = bq[2 * c], b1 = bq[2 * c + 1];
                const i32x8 af = {(int)a0[0], (int)a0[1], (int)a0[2], (int)a0[3], (int)a1[0], (int)a1[1], (int)a1[2], (int)a1[3]};
                const i32x8 bf = {(int)b0[0], (int)b0[1], (int)b0[2], (int)b0[3], (int)b1[0], (int)b1[1], (int)b1[2], (int)b1[3]};
                acc[m][c] = __builtin_amdgcn_mfma_scale_f32_16x16x128_f8f6f4(af, bf, acc[m][c], 1, 1, 0, 0x7F7F7F7F, 0, 0x7F7F7F7F);
            }
        asm volatile("" : "+v"(acc[0][0]), "+v"(acc[0][1]), "+v"(acc[0][2]), "+v"(acc[0][3]),
                          "+v"(acc[1][0]), "+v"(acc[1][1]), "+v"(acc[1][2]), "+v"(acc[1][3]));
        __builtin_amdgcn_sched_barrier(0);
        if (more == 1) {
            if (lane == 0) __hip_atomic_fetch_add(cnt + 32 + t, 1u, __ATOMIC_RELAXED, __HIP_MEMORY_SCOPE_WORKGROUP);
            load_a(a, t + 2);
        } else if (more == 2) {
#pragma unroll
            for (int i = 0; i < 4; ++i) a[i] = *(LAS u32x4*)(stash + (t & 1) * 4096 + i * 1024);
        }
        __builtin_amdgcn_s_setprio(0);
    };
#pragma unroll 1
    for (int pass = 0; pass < 2; ++pass) {
        if (pass) {
            lds_wait(cnt + 68, 4);
            lds_wait(cnt + 69, 4);
            wait_flags<LF_SLEEP>(ws + OFF_LF + (((lane >> 1) & 7) * 32 + rb * 2 + (lane & 1)) * 128, 0xFFFF, call, tmo);
            STAMP(3);
            load_a(ae, 30);
            load_a(ao, 31);
            lds_wait(cnt + 66, 4);
            lds_wait(cnt + 67, 4);
#pragma unroll
            for (int i = 0; i < 4; ++i) { *(LAS u32x4*)(stash + i * 1024) = ae[i]; *(LAS u32x4*)(stash + 4096 + i * 1024) = ao[i]; }
            __builtin_amdgcn_sched_barrier(0);
            load_a(ae, 0);
            __builtin_amdgcn_sched_barrier(0);
            load_a(ao, 1);
#pragma unroll
            for (int i = 0; i < 8; ++i) acc[i >> 2][i & 3] = f32x4{0.f, 0.f, 0.f, 0.f};
#pragma unroll 1
            for (int t = 0; t < 28; t += 2) {
                tile(ae, t, 0x01010101u, 1);
                tile(ao, t + 1, 0x01010101u, 1);
                if (wave == 0) STAMP(30 + (t >> 1));
            }
        }
        tile(ae, 28, pass * 0x01010101u, 2);
        tile(ao, 29, pass * 0x01010101u, 2);
        if (pass && wave == 0) STAMP(44);
        tile(ae, 30, pass * 0x01010101u, 0);
        tile(ao, 31, pass * 0x01010101u, 0);
        const float sc = cnt[72] ? __builtin_nanf("") : scale;
        if (pass) STAMP(59);
        const u32 vo = pass ? (lane >> 4) * 4 * GN * 4 + (lane & 15) * 16 : OOB;
        const u32 so = (rb * 32 * GN + j * 64) * 4;
#pragma unroll
        for (int m = 0; m < 2; ++m)
#pragma unroll
            for (int r = 0; r < 4; ++r) {
                u32x4 v;
#pragma unroll
                for (int c = 0; c < 4; ++c)
                    v[c] = __builtin_bit_cast(u32, __builtin_elementwise_maximum(acc[m][c][r] * sc, 0.f));
                __builtin_amdgcn_raw_buffer_store_b128(v, ro, vo, so + (m * 16 + r) * GN * 4, OUT_AUX);
                __builtin_amdgcn_sched_barrier(0);
                asm volatile("s_nop 1");
                __builtin_amdgcn_sched_barrier(0);
            }
    }
    STAMP(60);
    __builtin_amdgcn_endpgm();
}

__device__ __forceinline__ void unpack_waves(const Params& P, int b, int wave, int lane, GAS u32* cp, LAS char* lds, u32 cv) {
    char* const ws = P.self;
    const int rank = P.rank;
    LAS u32* const cnt = (LAS u32*)(lds + L_CNT);
    if (wave == 8) { cnt[lane] = 0; cnt[64 + lane] = 0; }
    bar();
    const u32 call = __builtin_amdgcn_readfirstlane(cv) + 1;
    const int q = wave & 3, tl = (wave >> 2) & 1;
    const int j = b & 63;
    GAS u32* const tmo = (GAS u32*)(ws + OFF_TMO);
    const __amdgpu_buffer_rsrc_t rs = __builtin_amdgcn_make_buffer_rsrc(ws, 0, WS_NEED, 0x00020000);
    LAS char* const sym = lds + L_SYM + (wave & 7) * 2304;
    const int ux = 2 * b + tl, xsrc = ux & 7, xu = ux >> 3;
    constexpr int NJOB = 17;
    u32 juo, jfo;
    {
        const int i = lane < NJOB ? lane : 0, t = i ? 2 * (i - 1) + tl : 0;
        juo = i ? w_slot(src_of(rank, j, t), t & 3, j, call) : x_slot(xsrc, xu, call);
        jfo = lane < NJOB ? juo + G_FLAG + q * 8 : OFF_TMO;
    }
    const u32 img = q * 2048 + q_img(lane);
    LAS char* image = lds + L_XIM + tl * 8192;
    __builtin_amdgcn_s_setprio(1);
    Packed pk;
    u64 look = 0;
    bool have = false;
    u32 fre = 8;
#pragma unroll 1
    for (int i = 0; i < NJOB; ++i) {
        const int t = 2 * (i - 1) + tl;
        if (__builtin_amdgcn_readfirstlane(fre) < 8) lds_wait(cnt + 32 + t - NBUF, 8);
        asm volatile("" : "+v"(look));
        if (!have) {
            u32 uo = __builtin_amdgcn_readlane(juo, i);
            const char* nf = ws + __builtin_amdgcn_readlane(jfo, i + 1);
            asm volatile("" : "+s"(uo), "+s"(nf));
            u32 meta = __builtin_amdgcn_readfirstlane((u32)look);
            if (__builtin_amdgcn_readfirstlane((u32)(look >> 32)) < call) {
                const char* f = ws + __builtin_amdgcn_readlane(jfo, i);
                if (i < EAGER_FROM) meta = __builtin_amdgcn_readfirstlane((u32)wait_flags<MID_SLEEP>(f, 1, call, tmo));
                else meta = wait_flag(f, call, tmo);
            }
            asm volatile("" ::: "memory");
            load_quarter(pk, rs, uo, meta, q, lane);
            look = __hip_atomic_load((GAS u64*)nf, __ATOMIC_RELAXED, __HIP_MEMORY_SCOPE_SYSTEM);
            if (i >= SJ) STAMP(4 + 3 * (i - 9));
        }
        decode_quarter(pk, rs, img, q, image, sym, lane, [&]() {
            if (t + 2 >= NBUF) fre = __hip_atomic_load(cnt + 32 + t + 2 - NBUF, __ATOMIC_RELAXED, __HIP_MEMORY_SCOPE_WORKGROUP);
            asm volatile("" : "+v"(look));
            Packed nx;
            if (i + 1 == NJOB) {
                if ((u32)look == call) cnt[72] = 1;
            } else {
                have = __builtin_amdgcn_readfirstlane((u32)(look >> 32)) >= call;
                asm volatile("" ::: "memory");
                if (have) load_quarter(nx, rs, __builtin_amdgcn_readlane(juo, i + 1), __builtin_amdgcn_readfirstlane((u32)look), q, lane);
                look = __hip_atomic_load((GAS u64*)(ws + __builtin_amdgcn_readlane(jfo, i + 1 + have)), __ATOMIC_RELAXED, __HIP_MEMORY_SCOPE_SYSTEM);
            }
            pk = nx;
        });
        if (i) {
            __hip_atomic_store((LAS unsigned char*)(cnt + t) + q, (unsigned char)1, __ATOMIC_RELEASE, __HIP_MEMORY_SCOPE_WORKGROUP);
        } else {
            lds_order();
            const u32x2 h0 = *(LAS u32x2*)(image + img), h1 = *(LAS u32x2*)(image + img + 256);
            const u32x2 h2 = *(LAS u32x2*)(image + img + 512), h3 = *(LAS u32x2*)(image + img + 768);
            const u32x4 v0 = {h0[0], h0[1], h1[0], h1[1]}, v1 = {h2[0], h2[1], h3[0], h3[1]};
            const char* xd = ws + OFF_XD + (xsrc * 32 + xu) * 8192 + q * 2048;
            st16<0>(xd, lane * 16, v0);
            st16<1024>(xd, lane * 16, v1);
            wait_vm<0>();
            if (__builtin_amdgcn_readfirstlane(lds_add(cnt + 68 + tl, lane)) == 3)
                put_flag(ws + OFF_LF + (xsrc * 32 + xu) * 128, call, 0, lane);
            STAMP(2);
        }
        image = lds + ((t + 2) & (NBUF - 1)) * 8192;
        if (i >= SJ - 1) STAMP(6 + 3 * (i - 9));
        else if (i < 9 && SJ == 9) STAMP(48 + i);
    }
    if (lane == 0) __hip_atomic_store(cp, call, __ATOMIC_RELAXED, __HIP_MEMORY_SCOPE_AGENT);
    __builtin_amdgcn_endpgm();
}

__global__ void __launch_bounds__(1024, 4) gemm_rs_kernel(Params P) {
    __shared__ __attribute__((aligned(16))) char smem[LDS_BYTES];
    const int lane = threadIdx.x & 63;
    const int wave = __builtin_amdgcn_readfirstlane(threadIdx.x >> 6);
    u32x4 xa, xc;
    {
        const u32 ux = 2 * (blockIdx.x - N_SEND_WG) + (wave >> 2), xdest = (ux + (ux >> 3)) & 7, xu = (ux >> 3) & 31;
        const __amdgpu_buffer_rsrc_t rx = __builtin_amdgcn_make_buffer_rsrc((void*)P.x, 0, 4096 * KS, 0x00020000);
        const u32 so = (xdest * MS + (xu >> 1) * 32 + (wave & 1) * 16) * KS + (xu & 1) * 256 + (wave & 2) * 64;
        const u32 vo = ((lane & 15) * KS + (lane >> 4) * 32) | (blockIdx.x >= N_SEND_WG && wave < 8 ? 0 : OOB);
        xa = __builtin_amdgcn_raw_buffer_load_b128(rx, vo, so, 0);
        xc = __builtin_amdgcn_raw_buffer_load_b128(rx, vo + 16, so, 0);
        asm volatile("" ::: "memory");
    }
    GAS u32* cp = (GAS u32*)(P.self + OFF_CNT) + blockIdx.x * 16 + wave;
    const u32 cv = __hip_atomic_load(cp, __ATOMIC_RELAXED, __HIP_MEMORY_SCOPE_AGENT);
    asm volatile("" ::: "memory");
    STAMP(0);
    if (blockIdx.x >= N_SEND_WG) {
        if (wave < 8) mma_waves(P, blockIdx.x - N_SEND_WG, wave, lane, cp, (LAS char*)smem, xa, xc, cv);
        else unpack_waves(P, blockIdx.x - N_SEND_WG, wave, lane, cp, (LAS char*)smem, cv);
    } else {
        if (wave < 8) sender(P, blockIdx.x, wave, lane, cp, (LAS char*)smem, cv);
        else if (wave == 8) anchor_wave(P.self, lane, cp, (LAS char*)smem, cv);
    }
}

extern "C" void kernel_launch(int rank, int nranks, ncclComm_t comm,
                              hipStream_t stream,
                              void* const* d_in, const int* in_sizes, int n_in,
                              void* d_out, int out_size,
                              const int* in_dtypes, int out_dtype,
                              void* const* d_out_all,
                              void* const* d_ws_all, size_t ws_bytes) {
    if (nranks != NR || n_in != 4 || ws_bytes < WS_NEED || (unsigned)rank >= (unsigned)NR ||
        in_sizes[0] != 4096 * KS || in_sizes[1] != KS * GN || out_size != MS * GN) {
        fprintf(stderr, "kernel_launch: unexpected problem shape\n");
        return;
    }
    Params P;
    P.x = (const uint8_t*)d_in[0];
    P.w = (const uint8_t*)d_in[1];
    P.sx = (const float*)d_in[2];
    P.sw = (const float*)d_in[3];
    P.out = (float*)d_out;
    for (int p = 0; p < NR; ++p) P.ws[p] = (char*)d_ws_all[p];
    P.self = P.ws[rank];
    P.rank = rank;
    P.pad = 0;
    gemm_rs_kernel<<<GRID, 1024, 0, stream>>>(P);
    HIP_CHECK(hipGetLastError());
}
```

```cpp
#include <hip/hip_runtime.h>
#include <rccl/rccl.h>
#include <cstdio>
#include <cstdint>

#define HIP_CHECK(expr) do { hipError_t _e = (expr); if (_e != hipSuccess) { \
    fprintf(stderr, "HIP error %s at %s:%d\n", hipGetErrorString(_e), __FILE__, __LINE__); \
    return; } } while (0)

typedef unsigned u32;
typedef unsigned long long u64;
typedef u32 u32x4 __attribute__((ext_vector_type(4)));
typedef u32 u32x2 __attribute__((ext_vector_type(2)));
typedef int i32x8 __attribute__((ext_vector_type(8)));
typedef float f32x4 __attribute__((ext_vector_type(4)));
#define GAS __attribute__((address_space(1)))
#define LAS __attribute__((address_space(3)))

#define PACE_T0 318
#define PACE_SLOT 364
#define PACE_C 4
#define USUAL16 42
#define LF_SLEEP 16
#define MID_SLEEP 8
#define EAGER_FROM 15
#define PH 222
#define LAM 70
#define WIN_SLOTS 5
#define DEAD 25
#define GAP 2
#define POLL_SLEEP 6
#define OUT_AUX 17
#define FORCE_RAW 0

constexpr int NR = 8;
constexpr int GN = 4096;
constexpr int KS = 512;
constexpr int MS = 512;
constexpr int NUNIT = 256;
constexpr int N_SEND_WG = 128;
constexpr int N_CONS_WG = 128;
constexpr int GRID = N_SEND_WG + N_CONS_WG;

constexpr u32 USLOT = 9216;
constexpr u32 GSLOT = 6144, NEAR_DW = 224, G_FLAG = 5120 + NEAR_DW * 4;
__host__ __device__ constexpr u32 w_slot(u32 src, u32 kc, u32 j, u32 call) {
    return (((((j & 7) * 8 + src) * 8 + (j >> 3)) * 4 + kc) * 2 + (call & 1)) * GSLOT;
}
__host__ __device__ constexpr u32 x_slot(u32 src, u32 xu, u32 call) { return (4096 + (src * 32 + xu) * 2 + (call & 1)) * GSLOT; }
constexpr u32 OFF_XD = x_slot(NR, 0, 0);
constexpr u32 OFF_LF = OFF_XD + NR * MS * KS;
constexpr u32 OFF_CNT = OFF_LF + NUNIT * 128;
constexpr u32 OFF_TMO = OFF_CNT + GRID * 16 * 4;
constexpr u32 OFF_ANC = OFF_TMO + 128;
constexpr u32 OFF_DBG = OFF_ANC + 16 * 128;
constexpr u32 FAR = OFF_DBG + GRID * 16 * 64 * 8;
constexpr u32 WS_NEED = FAR + OFF_XD;
constexpr u32 DEADLINE_TICKS = 100000000u;
constexpr u32 OOB = 0x80000000u;
constexpr u32 RAW_DW = 256;
constexpr int NBUF = 8;
constexpr u32 L_SYM = NBUF * 8192;
constexpr u32 L_XIM = 4 * 8192;
constexpr u32 L_ENC = L_SYM + 8 * 2304;
constexpr u32 L_STA = L_ENC;
constexpr u32 L_CNT = L_STA + 8 * 8192;
constexpr int LDS_BYTES = L_CNT + 512;

struct Params {
    const uint8_t* x;
    const uint8_t* w;
    const float* sx;
    const float* sw;
    float* out;
    char* self;
    char* ws[NR];
    int rank;
    int pad;
};

#define SJ 16
#define STAMP(i) do { } while (0)

template <int OFF>
__device__ __forceinline__ void st16(const char* base, u32 voff, u32x4 v) {
    asm volatile("global_store_dwordx4 %0, %1, %2 offset:%3 sc0 sc1\n\ts_nop 1"
                 :: "v"(voff), "v"(v), "s"(base), "i"(OFF) : "memory");
}
template <int N> __device__ __forceinline__ void wait_vm() { asm volatile("s_waitcnt vmcnt(%0)" :: "i"(N) : "memory"); }

__device__ __forceinline__ void put_flag(char* p, u32 call, u32 meta, int lane) {
    if (lane < 4)
        __hip_atomic_store((GAS u64*)p + lane, ((u64)call << 32) | meta, __ATOMIC_RELAXED, __HIP_MEMORY_SCOPE_SYSTEM);
}

template <int SLEEP>
__device__ __forceinline__ u64 wait_flags(const char* f, u64 need, u32 call, GAS u32* tmo) {
    const u32 t0 = (u32)__builtin_amdgcn_s_memrealtime();
    u64 v;
    for (u32 n = 0;;) {
        v = __hip_atomic_load((GAS u64*)f, __ATOMIC_RELAXED, __HIP_MEMORY_SCOPE_SYSTEM);
        if ((__ballot((u32)(v >> 32) >= call) & need) == need) break;
        if ((++n & 15) == 0) {
            const u32 t = __hip_atomic_load(tmo, __ATOMIC_RELAXED, __HIP_MEMORY_SCOPE_AGENT);
            if (__builtin_amdgcn_readfirstlane(t) == call || (u32)__builtin_amdgcn_s_memrealtime() - t0 > DEADLINE_TICKS) {
                __hip_atomic_store(tmo, call, __ATOMIC_RELAXED, __HIP_MEMORY_SCOPE_AGENT);
                break;
            }
        }
        __builtin_amdgcn_s_sleep(SLEEP);
    }
    asm volatile("" ::: "memory");
    return v;
}

__device__ __forceinline__ u32 wait_flag(const char* f, u32 call, GAS u32* tmo) {
    const u32 t0 = (u32)__builtin_amdgcn_s_memrealtime();
    u32 meta;
    u64 va = __hip_atomic_load((GAS u64*)f, __ATOMIC_RELAXED, __HIP_MEMORY_SCOPE_SYSTEM);
    __builtin_amdgcn_s_sleep(POLL_SLEEP);
    for (u32 n = 1;; ++n) {
        const u64 vb = __hip_atomic_load((GAS u64*)f, __ATOMIC_RELAXED, __HIP_MEMORY_SCOPE_SYSTEM);
        meta = __builtin_amdgcn_readfirstlane((u32)va);
        if (__builtin_amdgcn_readfirstlane((u32)(va >> 32)) >= call) break;
        va = __hip_atomic_load((GAS u64*)f, __ATOMIC_RELAXED, __HIP_MEMORY_SCOPE_SYSTEM);
        meta = __builtin_amdgcn_readfirstlane((u32)vb);
        if (__builtin_amdgcn_readfirstlane((u32)(vb >> 32)) >= call) break;
        if ((n & 15) == 0) {
            const u32 t = __hip_atomic_load(tmo, __ATOMIC_RELAXED, __HIP_MEMORY_SCOPE_AGENT);
            if (__builtin_amdgcn_readfirstlane(t) == call || (u32)__builtin_amdgcn_s_memrealtime() - t0 > DEADLINE_TICKS) {
                __hip_atomic_store(tmo, call, __ATOMIC_RELAXED, __HIP_MEMORY_SCOPE_AGENT);
                break;
            }
            va = __hip_atomic_load((GAS u64*)f, __ATOMIC_RELAXED, __HIP_MEMORY_SCOPE_SYSTEM);
            __builtin_amdgcn_s_sleep(POLL_SLEEP);
        }
    }
    asm volatile("" ::: "memory");
    return meta;
}

__device__ __forceinline__ u32 wave_scan(u32 v) {
    v += __builtin_amdgcn_update_dpp(0u, v, 0x111, 0xf, 0xf, true);
    v += __builtin_amdgcn_update_dpp(0u, v, 0x112, 0xf, 0xf, true);
    v += __builtin_amdgcn_update_dpp(0u, v, 0x114, 0xf, 0xf, true);
    v += __builtin_amdgcn_update_dpp(0u, v, 0x118, 0xf, 0xf, true);
    v += __builtin_amdgcn_update_dpp(0u, v, 0x142, 0xa, 0xf, false);
    v += __builtin_amdgcn_update_dpp(0u, v, 0x143, 0xc, 0xf, false);
    return v;
}
__device__ __forceinline__ u32 bfi(u32 m, u32 a, u32 b) { return (m & a) | (~m & b); }
__device__ __forceinline__ void lds_order() { __builtin_amdgcn_fence(__ATOMIC_ACQ_REL, "wavefront"); }

__device__ __forceinline__ u32 spread80(u32 m) { return (m << 1) - (m >> 7); }
template <class Early>
__device__ __forceinline__ u32 encode_quarter(const u32 (&d)[8], u32 img, LAS char* unit, int q, LAS char* stage,
                                              LAS char* lut, int lane, u32& meta, Early codes_out) {
    u32 mag[8], sum = 0;
#pragma unroll
    for (int k = 0; k < 8; ++k) { mag[k] = d[k] & 0x7F7F7F7Fu; sum = __builtin_amdgcn_sad_u8(mag[k], 0u, sum); }
    int b = ((int)__builtin_amdgcn_readlane(wave_scan(sum), 63) + 1024 - 14541) >> 11;
    b = b < 12 ? 12 : b;
    asm volatile("" : "+s"(b));
    const u32 B = b > 108 ? 108 : b;
    const u32 c2 = (126 - B) * 0x01010101u, c16 = (112 - B) * 0x01010101u;
    u32 nib[8], esc[8];
#pragma unroll
    for (int k = 0; k < 8; ++k) {
        const u32 u = mag[k] + c2, w = mag[k] + c16;
        const u32 dir = u & ~w & 0x80808080u;
        nib[k] = w & (((dir >> 3) - (dir >> 6)) | 0x01010101u);
        esc[k] = dir ^ 0x80808080u;
    }
    codes_out(u32x4{nib[0] | (nib[1] << 4), nib[2] | (nib[3] << 4), nib[4] | (nib[5] << 4), nib[6] | (nib[7] << 4)}, false);
    const u32 amax = B >= 40 ? 20 : B >> 1;
    const u32 cz = (129 + B) * 0x01010101u, clo = (128 - B + 2 * amax) * 0x01010101u, c20 = (108 - B) * 0x01010101u;
    u32 sel[8], off[8], cnt = 0, sg = 0, mb = 0;
#pragma unroll
    for (int k = 0; k < 8; ++k) sel[k] = *(LAS u32*)(lut + (__builtin_amdgcn_udot4(esc[k], 0x08040201u, 0u, false) >> 5));
#pragma unroll
    for (int k = 0; k < 8; ++k) {
        mb |= ((~(mag[k] + clo) | (mag[k] + c20)) & 0x80808080u) >> (7 - k);
        off[k] = cnt;
        cnt += __builtin_popcount(esc[k]);
        sg |= (d[k] >> (7 - k)) & (0x01010101u << k);
    }
    const u32 incl = wave_scan(cnt);
    const u32 nrec = (__builtin_amdgcn_readlane(incl, 63) + 7) >> 3, n1 = (3 * nrec + 3) >> 2;
    LAS char* blk = unit + 5120 + q * 1024;
    *(LAS u32*)(unit + 4096 + q * 256 + lane * 4) = sg;
    u32 n2 = 0, ne = 0;
    if (nrec <= 192 && !FORCE_RAW) {
        const u32 base = incl - cnt;
#pragma unroll
        for (int k = 0; k < 8; ++k) {
            const u32 v = ((cz - mag[k]) >> 1) & 0x3F3F3F3Fu, pos = base + off[k];
            const u64 x = (u64)__builtin_amdgcn_perm(0u, v, sel[k]) << (8 * (pos & 3));
            LAS u32* p = (LAS u32*)(stage + (pos & ~3u));
            __hip_atomic_fetch_or(p, (u32)x, __ATOMIC_RELAXED, __HIP_MEMORY_SCOPE_WORKGROUP);
            __hip_atomic_fetch_or(p + 1, (u32)(x >> 32), __ATOMIC_RELAXED, __HIP_MEMORY_SCOPE_WORKGROUP);
        }
        lds_order();
        const u32 cga = (127 - amax) * 0x01010101u;
        u32 nfar = 0;
        for (u32 r0 = 0; r0 < nrec; r0 += 64) {
            const u32x2 vv = *(LAS u32x2*)(stage + (r0 + lane) * 8);
            u32 c1[2], cf[2], far[2];
#pragma unroll
            for (int h = 0; h < 2; ++h) {
                const u32 v = vv[h];
                const u32 ge6 = v + 0x7A7A7A7Au, gta = v + cga, ge55 = v + 0x49494949u, ge56 = v + 0x48484848u;
                far[h] = ((ge6 & ~gta) | (ge55 & ~ge56)) & 0x80808080u;
                c1[h] = ((v & ~spread80(gta & 0x80808080u)) | (spread80(ge56 & 0x80808080u) & 0x06060606u) | spread80(far[h])) & 0x07070707u;
                const u32 t = (ge6 | spread80(ge55 & 0x80808080u)) & 0x0F0F0F0Fu;
                const u32 sl = *(LAS u32*)(lut + (__builtin_amdgcn_udot4(far[h], 0x08040201u, 0u, false) >> 5));
                const u32 y = __builtin_amdgcn_perm(0u, t, sl);
                cf[h] = __builtin_amdgcn_perm(0u, y | (y >> 4), 0x0C0C0200u);
            }
            const u32 x = c1[0] | (c1[1] << 3);
            const u32 rec = x | (__umul24(x >> 24, 0x41040u) & 0xC0C0C0u);
            if (r0 + lane < nrec) {
                LAS unsigned char* rp = (LAS unsigned char*)blk + 4 * n1 - 3 * (r0 + lane + 1);
                rp[0] = rec; rp[1] = rec >> 8; rp[2] = rec >> 16;
            }
            const u32 f0 = __builtin_popcount(far[0]), f = f0 + __builtin_popcount(far[1]), fi = wave_scan(f);
            const u32 pos = nfar + fi - f;
            nfar += __builtin_amdgcn_readlane(fi, 63);
            if (f != 0 && pos <= 256) {
                const u64 z = (u64)(cf[0] | (cf[1] << (4 * f0))) << (4 * (pos & 7));
                LAS u32* p = (LAS u32*)(blk + 4 * n1 + ((pos >> 1) & ~3u));
                __hip_atomic_fetch_or(p, (u32)z, __ATOMIC_RELAXED, __HIP_MEMORY_SCOPE_WORKGROUP);
                __hip_atomic_fetch_or(p + 1, (u32)(z >> 32), __ATOMIC_RELAXED, __HIP_MEMORY_SCOPE_WORKGROUP);
            }
        }
        n2 = (nfar + 7) >> 3;
        lds_order();
        for (u64 mk; (mk = __ballot(mb != 0)) != 0; ne += __builtin_popcountll(mk)) {
            if (mb != 0) {
                const u32 bit = __builtin_ctz(mb);
                mb &= mb - 1;
                const u32 k0 = (u32)((int)(bit << 31) >> 31), k1 = (u32)((int)(bit << 30) >> 31), k2 = (u32)((int)(bit << 29) >> 31);
                const u32 m0 = bfi(k2, mag[4], mag[0]), m1 = bfi(k2, mag[5], mag[1]), m2 = bfi(k2, mag[6], mag[2]), m3 = bfi(k2, mag[7], mag[3]);
                const u32 mg = (bfi(k0, bfi(k1, m3, m1), bfi(k1, m2, m0)) >> (bit & 24u)) & 127u;
                const u32 ad = img + (bit & 6u) * 128 + (k0 & 4u) + (bit >> 3);
                const u32 v = ((129 + B - mg) >> 1) & 63u;
                const u32 dflt = B - (v <= amax ? 2 * v : 0) + ((mg - B) & 1u);
                const u32 slot = n1 + n2 + ne + __builtin_amdgcn_mbcnt_hi((u32)(mk >> 32), __builtin_amdgcn_mbcnt_lo((u32)mk, 0u));
                *(LAS u32*)(blk + (slot < 255 ? slot : 255) * 4) = (ad << 8) | (mg ^ dflt);
            }
        }
    }
    if (nrec > 192 || n2 > 32 || ne > 31 || FORCE_RAW) {
        lds_order();
        codes_out(u32x4{d[0], d[1], d[2], d[3]}, true);
        *(LAS u32x4*)(blk + lane * 16) = u32x4{d[4], d[5], d[6], d[7]};
        meta = 3u << 11;
        return RAW_DW;
    }
    meta = n2 | (ne << 6) | ((nrec > 128 ? 2u : (nrec > 64 ? 1u : 0u)) << 11) | (B << 15) | (n1 << 22);
    return n1 + n2 + ne;
}

struct Packed {
    u32x4 n;
    u32 s, b1, b2, be;
    u32 meta, uo;
};
__device__ __forceinline__ u32 blk_off(u32 uo, u32 meta) { return uo + (meta & 0x4000u ? FAR : 5120) + (meta >> 22) * 4; }
__device__ __forceinline__ void load_quarter(Packed& p, __amdgpu_buffer_rsrc_t rs, u32 uo, u32 meta, int q, int lane) {
    p.n = __builtin_amdgcn_raw_buffer_load_b128(rs, lane * 16 + q * 1024, uo, 17);
    __builtin_amdgcn_sched_barrier(0);
    p.meta = meta;
    p.uo = uo;
    const u32 bo = blk_off(uo, meta), n2 = meta & 63u;
    p.b1 = __builtin_amdgcn_raw_buffer_load_b32(rs, 189 - 3 * lane, bo - 192, 17);
    p.b2 = __builtin_amdgcn_raw_buffer_load_b16(rs, (u32)lane < 2 * n2 ? lane * 2 : OOB, bo, 17);
    p.s = __builtin_amdgcn_raw_buffer_load_b32(rs, lane * 4 + 4096 + q * 256, uo, 17);
    p.be = __builtin_amdgcn_raw_buffer_load_b32(rs, (u32)lane < ((meta >> 6) & 31u) ? lane * 4 : OOB, bo + n2 * 4, 17);
}
struct Rec {
    u32 near[2], sel[2], f0, f;
    __device__ __forceinline__ Rec(u32 rec, u32 lut0, u32 lut1) {
        const u32 x = bfi(0xFF000000u, __umul24(rec & 0xC0C0C0u, 0x41040u), rec);
#pragma unroll
        for (int h = 0; h < 2; ++h) {
            const u32 c = (h ? x >> 3 : x) & 0x07070707u;
            near[h] = __builtin_amdgcn_perm(lut1, lut0, c);
            const u32 e10 = (c + 0x09090909u) & 0x10101010u;
            sel[h] = __umul24(e10, 0x101010u) + (e10 ^ 0x10101010u);
            if (h) f = f0 + __builtin_popcount(e10);
            else f0 = __builtin_popcount(e10);
        }
    }
    u32 ah[2], g0[2], g1[2];
    __device__ __forceinline__ void gather(u32 a) {
#pragma unroll
        for (int h = 0; h < 2; ++h) {
            ah[h] = h ? a + f0 : a;
            LAS u32* gp = (LAS u32*)(size_t)(ah[h] & ~3u);
            g0[h] = gp[0]; g1[h] = gp[1];
        }
    }
    __device__ __forceinline__ u32x2 bytes() const {
        u32x2 o;
#pragma unroll
        for (int h = 0; h < 2; ++h) {
            const u32 g = __builtin_amdgcn_alignbyte(g1[h], g0[h], ah[h]);
            o[h] = near[h] & __builtin_amdgcn_perm(g, g, sel[h]);
        }
        return o;
    }
};
template <class Mid>
__device__ __forceinline__ void decode_quarter(const Packed& p, __amdgpu_buffer_rsrc_t rs, u32 img, int q,
                                               LAS char* image, LAS char* sym, int lane, Mid mid) {
    const u32 uo = p.uo, bo = blk_off(uo, p.meta), cls = (p.meta >> 11) & 3u, B1 = ((p.meta >> 15) & 127u) * 0x01010101u;
    const u32 kx = ~B1, lut0 = (B1 - 0x06040200u) ^ kx, lut1 = ((B1 + 0x00100000u - 0x00000A08u) ^ kx) | 0xFF000000u;
    u32 nib[8], e10[8], off[8], cnt = 0;
#pragma unroll
    for (int k = 0; k < 8; ++k) {
        nib[k] = (k & 1 ? p.n[k >> 1] >> 4 : p.n[k >> 1]) & 0x0F0F0F0Fu;
        e10[k] = ~(nib[k] + 0x0E0E0E0Eu) & 0x10101010u;
        off[k] = cnt;
        cnt += __builtin_popcount(e10[k]);
    }
    __builtin_amdgcn_sched_barrier(0);
    {
        u32 t = (p.b2 | (p.b2 << 8)) & 0x00FF00FFu;
        t = (t | (t << 4)) & 0x0F0F0F0Fu;
        const u32 h = (t + 0x01010101u) & 0x10101010u;
        *(LAS u32*)(sym + 1600 + lane * 4) = (B1 - 0x0C0C0C0Cu + ((h << 2) - (h >> 2)) - (t << 1)) ^ kx;
    }
    Rec R(p.b1, lut0, lut1);
    const u32 both = cnt | (R.f << 16), inc = wave_scan(both), exc = inc - both;
    lds_order();
    R.gather((u32)(size_t)sym + 1600 + (exc >> 16));
    __builtin_amdgcn_sched_barrier(0);
    u32 ga[8], g0[8], g1[8], sel[8], o[8];
#pragma unroll
    for (int k = 0; k < 8; ++k) {
        off[k] += (u32)(size_t)sym + (exc & 0xFFFFu);
        ga[k] = off[k] & ~3u;
        nib[k] |= (p.s << (7 - k)) & 0x80808080u;
    }
    asm volatile("" : "+v"(ga[0]), "+v"(ga[1]), "+v"(ga[2]), "+v"(ga[3]), "+v"(ga[4]), "+v"(ga[5]), "+v"(ga[6]), "+v"(ga[7]),
                      "+v"(nib[0]), "+v"(nib[1]), "+v"(nib[2]), "+v"(nib[3]), "+v"(nib[4]), "+v"(nib[5]), "+v"(nib[6]), "+v"(nib[7]));
    __builtin_amdgcn_sched_barrier(0);
    *(LAS u32x2*)(sym + lane * 8) = R.bytes();
    if (cls == 1 || cls == 2) {
        u32 nf = __builtin_amdgcn_readlane(inc, 63) >> 16;
        for (u32 c = 1; c <= cls; ++c) {
            Rec Rc(__builtin_amdgcn_raw_buffer_load_b32(rs, 189 - 3 * lane, bo - 192 * (c + 1), 17), lut0, lut1);
            const u32 fi = wave_scan(Rc.f);
            Rc.gather((u32)(size_t)sym + 1600 + ((nf + fi - Rc.f) & 511u));
            *(LAS u32x2*)(sym + c * 512 + lane * 8) = Rc.bytes();
            nf += __builtin_amdgcn_readlane(fi, 63);
        }
    }
    lds_order();
#pragma unroll
    for (int k = 0; k < 8; ++k) {
        LAS u32* gp = (LAS u32*)(size_t)ga[k];
        g0[k] = gp[0]; g1[k] = gp[1];
    }
    __builtin_amdgcn_sched_barrier(0);
#pragma unroll
    for (int k = 0; k < 8; ++k)
        sel[k] = __umul24(e10[k], 0x101010u) + (e10[k] ^ 0x10101010u);
    u32 ev = (p.be & 127u) << ((p.be >> 5) & 24u);
    u32 ea = (u32)(size_t)image + ((p.be >> 8) & 0x1FFCu);
    u32 wa = (u32)(size_t)image + img;
    asm volatile("" : "+v"(ev), "+v"(ea), "+v"(wa), "+v"(sel[0]), "+v"(sel[1]), "+v"(sel[2]), "+v"(sel[3]),
                      "+v"(sel[4]), "+v"(sel[5]), "+v"(sel[6]), "+v"(sel[7]));
    mid();
    __builtin_amdgcn_sched_barrier(0);
#pragma unroll
    for (int c = 0; c < 4; ++c) {
#pragma unroll
        for (int k = 2 * c; k < 2 * c + 2; ++k) {
            const u32 g = __builtin_amdgcn_alignbyte(g1[k], g0[k], off[k]);
            o[k] = (__builtin_amdgcn_perm(g, g, sel[k]) ^ kx) + nib[k];
        }
        *(LAS u32x2*)(size_t)(wa + c * 256) = u32x2{o[2 * c], o[2 * c + 1]};
    }
    lds_order();
    if (ev) __hip_atomic_fetch_xor((LAS u32*)(size_t)ea, ev, __ATOMIC_RELAXED, __HIP_MEMORY_SCOPE_WORKGROUP);
    if (cls == 3) {
        const u32x4 l = __builtin_amdgcn_raw_buffer_load_b128(rs, lane * 16 + q * 1024, uo, 17);
        const u32x4 h = __builtin_amdgcn_raw_buffer_load_b128(rs, lane * 16, bo, 17);
        *(LAS u32x2*)(size_t)wa = u32x2{l[0], l[1]};
        *(LAS u32x2*)(size_t)(wa + 256) = u32x2{l[2], l[3]};
        *(LAS u32x2*)(size_t)(wa + 512) = u32x2{h[0], h[1]};
        *(LAS u32x2*)(size_t)(wa + 768) = u32x2{h[2], h[3]};
    }
}

__device__ __forceinline__ u32 q_img(int lane) {
    return (lane >> 5) * 1024 + ((lane & 7) | ((lane >> 1) & 8)) * 16 + (lane & 8);
}

struct Unit {
    u32x4 p[6];
    u32 meta;
    u32 c1, c2, c3, tot;
    u32 tot16;
    LAS char* lds;
    __device__ __forceinline__ u32x4 blk16(u32 i) const {
        u32x4 v;
#pragma unroll
        for (int k = 0; k < 4; ++k) {
            const u32 w = 4 * i + k;
            const u32 a = w + (w < c1 ? 0 : (w < c2 ? 256 - c1 : (w < c3 ? 512 - c2 : 768 - c3)));
            v[k] = *(LAS u32*)(lds + 5120 + (a & 1023u) * 4);
        }
        return v;
    }
    __device__ __forceinline__ void fetch(LAS char* unit, LAS u32* lens, int lane) {
        lds = unit;
        const u32 l = lens[lane & 3];
        c1 = __builtin_amdgcn_readlane(l, 0);
        c2 = c1 + __builtin_amdgcn_readlane(l, 1);
        c3 = c2 + __builtin_amdgcn_readlane(l, 2);
        tot = c3 + __builtin_amdgcn_readlane(l, 3);
        tot16 = (tot + 3) >> 2;
        meta = lens[8 + (lane & 3)] + (tot > NEAR_DW ? 0x4000u : 0u) + ((lane == 0 ? 0 : (lane == 1 ? c1 : (lane == 2 ? c2 : c3))) << 22);
#pragma unroll
        for (int i = 0; i < 5; ++i) p[i] = *(LAS u32x4*)(unit + i * 1024 + lane * 16);
        p[5] = blk16(lane);
    }
    __device__ __forceinline__ void send_blocks(const char* dst, u32 voff, int lane) const {
        if (tot > NEAR_DW) dst += FAR - 5120;
        for (u32 i = 64 + lane; i < tot16; i += 64) st16<1024>(dst, voff + (i - lane) * 16, blk16(i));
        if ((u32)lane < tot16) st16<1024>(dst, voff, p[5]);
    }
    __device__ __forceinline__ void send_all(const char* dst, u32 voff, int lane) const {
        st16<-4096>(dst, voff, p[0]); st16<-3072>(dst, voff, p[1]); st16<-2048>(dst, voff, p[2]);
        st16<-1024>(dst, voff, p[3]); st16<0>(dst, voff, p[4]);
        send_blocks(dst, voff, lane);
    }
};

__device__ __forceinline__ void pace(u32 due) {
    while ((int)((u32)__builtin_amdgcn_s_memrealtime() - due) < 0) __builtin_amdgcn_s_sleep(1);
}

__device__ __forceinline__ void bar() {
    asm volatile("s_waitcnt lgkmcnt(0)" ::: "memory");
    __builtin_amdgcn_s_barrier();
    asm volatile("" ::: "memory");
}
constexpr u32 E_STAGE = 2 * USLOT + 1024;
__device__ __forceinline__ void enc_init(LAS char* lds, int wave, int lane) {
    *(LAS u32x4*)(lds + (wave >> 2) * USLOT + 5120 + (wave & 3) * 1024 + lane * 16) = u32x4{0, 0, 0, 0};
    *(LAS u32x4*)(lds + E_STAGE + wave * 2048 + lane * 16) = u32x4{0, 0, 0, 0};
    *(LAS u32x4*)(lds + E_STAGE + wave * 2048 + 1024 + lane * 16) = u32x4{0, 0, 0, 0};
    u32 sel = 0x0C0C0C0Cu, n = 0;
#pragma unroll
    for (int i = 0; i < 4; ++i)
        if ((lane >> i) & 1) { sel = (sel & ~(0xFFu << n)) | ((u32)i << n); n += 8; }
    if (lane < 16) *(LAS u32*)(lds + 2 * USLOT + wave * 64 + lane * 4) = sel;
    lds_order();
}

__device__ __forceinline__ void anchor_wave(const char* self, int lane, GAS u32* cp, LAS char* lds, u32 cv) {
    const u32 call = __builtin_amdgcn_readfirstlane(cv) + 1;
    u32 v, now;
    u64 ok;
    int n = 0;
    __builtin_amdgcn_s_sleep(16);
    do {
        __builtin_amdgcn_s_sleep(4);
        v = __hip_atomic_load((GAS u32*)(self + OFF_ANC + (lane >> 2) * 128 + (lane & 3) * 4), __ATOMIC_RELAXED, __HIP_MEMORY_SCOPE_AGENT);
        ok = __ballot((v >> 24) == (call & 255u));
        now = (u32)__builtin_amdgcn_s_memrealtime();
    } while (ok != ~0ull && ++n < 8);
    u32 age = (now - v) & 0xFFFFFFu;
    age = (v >> 24) == (call & 255u) ? (age > 1000 ? 1000 : age) : 0;
    *(LAS u32*)(lds + 2 * USLOT + 512 + 64) = now - (__builtin_amdgcn_readlane(wave_scan(age), 63) >> 6);
    bar();
    if (lane == 0) __hip_atomic_store(cp, call, __ATOMIC_RELAXED, __HIP_MEMORY_SCOPE_AGENT);
    __builtin_amdgcn_endpgm();
}

__device__ __forceinline__ void sender(const Params& P, int b, int wave, int lane, GAS u32* cp, LAS char* lds, u32 cv) {
    const int rank = P.rank;
    char* const self = P.self;
    const int q = wave & 3, tl = wave >> 2;
    const int j = b & 63, kc0 = b >> 6;
    u32 wr[8];
    {
        const u32 cg = (lane & 7) | ((lane >> 1) & 8), rg = ((lane >> 3) & 1) | ((lane >> 4) & 2);
        const uint8_t* wp = P.w + (size_t)((kc0 + 2 * tl) * 128 + q * 32 + rg * 8) * GN + j * 64 + cg * 4;
#pragma unroll
        for (int i = 0; i < 8; ++i) wr[i] = *(const GAS u32*)(wp + (size_t)i * GN);
    }
    enc_init(lds, wave, lane);
    LAS char* const unit = lds + tl * USLOT;
    LAS u32* const lens = (LAS u32*)(lds + 2 * USLOT + 512);
    u32 dw[8];
#pragma unroll
    for (int h = 0; h < 2; ++h) {
        const u32 r0 = wr[4 * h], r1 = wr[4 * h + 1], r2 = wr[4 * h + 2], r3 = wr[4 * h + 3];
        const u32 t0 = __builtin_amdgcn_perm(r1, r0, 0x05010400u), t1 = __builtin_amdgcn_perm(r1, r0, 0x07030602u);
        const u32 t2 = __builtin_amdgcn_perm(r3, r2, 0x05010400u), t3 = __builtin_amdgcn_perm(r3, r2, 0x07030602u);
        dw[0 + h] = __builtin_amdgcn_perm(t2, t0, 0x05040100u);
        dw[2 + h] = __builtin_amdgcn_perm(t2, t0, 0x07060302u);
        dw[4 + h] = __builtin_amdgcn_perm(t3, t1, 0x05040100u);
        dw[6 + h] = __builtin_amdgcn_perm(t3, t1, 0x07060302u);
    }
    u32 qmeta;
    const u32 len = encode_quarter(dw, q * 2048 + q_img(lane), unit, q, lds + E_STAGE + wave * 2048, lds + 2 * USLOT + wave * 64, lane, qmeta,
                                   [&](u32x4 v, bool) { *(LAS u32x4*)(unit + q * 1024 + lane * 16) = v; });
    if (lane == 0) { lens[wave] = len; lens[8 + wave] = qmeta; }
    const u32 call = __builtin_amdgcn_readfirstlane(cv) + 1;
    const u32 voff = lane * 16 + 4096;
    bar();
    STAMP(3);
    if (q == 1 || q == 2) {
        const int kc = kc0 + 2 * tl;
        const u32 wo = w_slot(rank, kc, j, call), fo = wo + G_FLAG;
        Unit U;
        U.fetch(unit, lens + tl * 4, lane);
        if (q == 2) {
            U.send_all(self + wo, voff, lane);
            wait_vm<0>();
            put_flag(self + fo, call, U.meta, lane);
        } else {
            const int est = __builtin_amdgcn_readlane(wave_scan(lens[lane & 7]), 7) / 8 + PACE_SLOT - USUAL16;
            const int slot = est > PACE_SLOT + 6 || est < PACE_SLOT - 7 ? est + 2 : PACE_SLOT;
            const u32 anchor = __builtin_amdgcn_readfirstlane(lens[16]);
            const u32 t0 = anchor + PACE_T0 * slot / PACE_SLOT;
            u32 slip = 0;
            int pr = (kc + j) % 7;
#pragma unroll 1
            for (int s = 0; s < 7; ++s) {
                char* const peer = P.ws[(rank + 1 + pr) & 7];
                const char* const dst = peer + wo;
                pr = pr == 6 ? 0 : pr + 1;
                const int m0 = 4 * s + kc;
                int before = (m0 > 3 ? m0 - 3 : 0) * 64 * PACE_C;
#pragma unroll
                for (int k = -3; k < 4; ++k) {
                    const int v = PACE_C * j - 128 * k;
                    if ((u32)(m0 + k) < 28) before += v < 0 ? 0 : (v > 64 * PACE_C ? 64 * PACE_C : v);
                }
                const u32 due = t0 + before * slot / (256 * PACE_C);
                const bool win = s < WIN_SLOTS;
                pace(win ? due - LAM : due + slip);
                st16<-4096>(dst, voff, U.p[0]); __builtin_amdgcn_s_sleep(GAP);
                st16<-3072>(dst, voff, U.p[1]); __builtin_amdgcn_s_sleep(GAP);
                if (win) { wait_vm<0>(); pace(due); }
                st16<-2048>(dst, voff, U.p[2]); __builtin_amdgcn_s_sleep(GAP);
                st16<-1024>(dst, voff, U.p[3]); __builtin_amdgcn_s_sleep(GAP);
                if (win) { wait_vm<0>(); pace(due + LAM); const int late = (int)((u32)__builtin_amdgcn_s_memrealtime() - (due + LAM + DEAD)); slip = late > 0 ? late / 2 : 0; }
                st16<0>(dst, voff, U.p[4]); __builtin_amdgcn_s_sleep(GAP);
                U.send_blocks(dst, voff, lane);
                wait_vm<0>();
                put_flag(peer + fo, call, U.meta, lane);
                STAMP(5 + s);
            }
            if (tl) STAMP(12);
        }
    }
    if (lane == 0) __hip_atomic_store(cp, call, __ATOMIC_RELAXED, __HIP_MEMORY_SCOPE_AGENT);
    __builtin_amdgcn_endpgm();
}

__device__ __forceinline__ u32 lds_add(LAS u32* c, int lane) {
    u32 v = 0;
    if (lane == 0) v = __hip_atomic_fetch_add(c, 1u, __ATOMIC_ACQ_REL, __HIP_MEMORY_SCOPE_WORKGROUP);
    return v;
}
__device__ __forceinline__ void lds_wait(LAS u32* c, u32 n) {
    while (__builtin_amdgcn_readfirstlane(__hip_atomic_load(c, __ATOMIC_ACQUIRE, __HIP_MEMORY_SCOPE_WORKGROUP)) < n)
        __builtin_amdgcn_s_sleep(1);
}
__device__ __forceinline__ int src_of(int rank, int j, int t) {
    return t < 4 ? rank : (rank + 7 - ((t >> 2) - 1 + (t & 3) + j) % 7) & 7;
}

__device__ __forceinline__ void mma_waves(const Params& P, int b, int wave, int lane, GAS u32* cp, LAS char* lds, u32x4 a, u32x4 c, u32 cv) {
    char* const ws = P.self;
    const int rank = P.rank;
    u32 cnt_a = (u32)(size_t)(lds + L_CNT);
    asm volatile("" : "+v"(cnt_a));
    LAS u32* const cnt = (LAS u32*)(size_t)cnt_a;
    u32 call;
    {
        const int q = wave & 3, tl = wave >> 2, ux = 2 * b + tl;
        const int xdest = (ux + (ux >> 3)) & 7, xu = ux >> 3;
        char* const xdst = P.ws[xdest];
        u32 dx[8] = {a[0], a[1], a[2], a[3], c[0], c[1], c[2], c[3]};
        LAS char* const el = lds + L_ENC;
        enc_init(el, wave, lane);
        LAS u32* const lens = (LAS u32*)(el + 2 * USLOT + 512);
        call = __builtin_amdgcn_readfirstlane(cv) + 1;
        const u32 voff = lane * 16 + 4096;
        char* const xslot = xdst + x_slot(rank, xu, call);
        if ((b & 7) == (b >> 4) && (wave & 1)) {
            asm volatile("" : "+v"(dx[0]), "+v"(dx[7]));
            if (lane == 0)
                __hip_atomic_store((GAS u32*)(ws + OFF_ANC + (b >> 3) * 128 + (wave >> 1) * 4),
                                   (call << 24) | ((u32)__builtin_amdgcn_s_memrealtime() & 0xFFFFFFu), __ATOMIC_RELAXED, __HIP_MEMORY_SCOPE_AGENT);
        }
        u32 qmeta;
        const u32 len = encode_quarter(dx, q * 2048 + q_img(lane), el + tl * USLOT, q, el + E_STAGE + wave * 2048, el + 2 * USLOT + wave * 64, lane, qmeta, [&](u32x4 v, bool again) {
            if (again) wait_vm<0>();
            st16<-4096>(xslot + q * 1024, voff, v);
            STAMP(61);
        });
        if (lane == 0) { lens[wave] = len; lens[8 + wave] = qmeta; }
        bar();
        if (q == 0) {
            Unit U;
            U.fetch(el + tl * USLOT, lens + tl * 4, lane);
            st16<0>(xslot, voff, U.p[4]);
            U.send_blocks(xslot, voff, lane);
            if (lane < 4) cnt[76 + tl * 4 + lane] = U.meta;
            STAMP(62);
        }
        wait_vm<0>();
        if (__builtin_amdgcn_readfirstlane(lds_add(cnt + 66 + tl, lane)) == 3) {
            put_flag(xslot + G_FLAG, call, cnt[76 + tl * 4 + (lane & 3)], lane);
            STAMP(63);
        }
    }
    const int j = b & 63;
    const int rb = (b >> 6) * 8 + wave;
    GAS u32* const tmo = (GAS u32*)(ws + OFF_TMO);
    const __amdgpu_buffer_rsrc_t rs = __builtin_amdgcn_make_buffer_rsrc(ws, 0, WS_NEED, 0x00020000);
    const __amdgpu_buffer_rsrc_t ro = __builtin_amdgcn_make_buffer_rsrc(P.out, 0, MS * GN * 4, 0x00020000);
    const float scale = __builtin_bit_cast(float, __builtin_amdgcn_readfirstlane(__builtin_bit_cast(u32, *P.sx * *P.sw)));
    if (lane == 0) __hip_atomic_store(cp, call, __ATOMIC_RELAXED, __HIP_MEMORY_SCOPE_AGENT);
    LAS char* const rp = lds + (lane >> 4) * 2048 + (lane & 15) * 16;
    const u32 txo = OFF_XD + (src_of(rank, j, lane & 31) * 16 + rb) * 16384 + (lane & 3) * 4096;
    f32x4 acc[2][4] = {};
    u32x4 ae[4] = {}, ao[4] = {};
    auto load_a = [&](u32x4 (&a)[4], int t) {
        const u32 so = __builtin_amdgcn_readlane(txo, t);
        u32 l16 = lane * 16;
        asm volatile("" : "+v"(l16));
#pragma unroll
        for (int i = 0; i < 4; ++i) a[i] = __builtin_amdgcn_raw_buffer_load_b128(rs, l16 + i * 1024, so, 17);
    };
    LAS char* const stash = lds + L_STA + wave * 8192 + lane * 16;
    auto tile = [&](u32x4 (&a)[4], int t, u32 need, int more) {
        lds_wait(cnt + t, need);
        LAS char* const bp = rp + (t & (NBUF - 1)) * 8192;
        u32x4 bq[8];
#pragma unroll
        for (int i = 0; i < 8; ++i) bq[i] = *(LAS u32x4*)(bp + (i >> 1) * 256 + (i & 1) * 1024);
        __builtin_amdgcn_sched_barrier(0);
#pragma unroll
        for (int c = 0; c < 4; ++c)
#pragma unroll
            for (int m = 0; m < 2; ++m) {
                const u32x4 a0 = a[2 * m], a1 = a[2 * m + 1], b0 = bq[2 * c], b1 = bq[2 * c + 1];
                const i32x8 af = {(int)a0[0], (int)a0[1], (int)a0[2], (int)a0[3], (int)a1[0], (int)a1[1], (int)a1[2], (int)a1[3]};
                const i32x8 bf = {(int)b0[0], (int)b0[1], (int)b0[2], (int)b0[3], (int)b1[0], (int)b1[1], (int)b1[2], (int)b1[3]};
                acc[m][c] = __builtin_amdgcn_mfma_scale_f32_16x16x128_f8f6f4(af, bf, acc[m][c], 1, 1, 0, 0x7F7F7F7F, 0, 0x7F7F7F7F);
            }
        asm volatile("" : "+v"(acc[0][0]), "+v"(acc[0][1]), "+v"(acc[0][2]), "+v"(acc[0][3]),
                          "+v"(acc[1][0]), "+v"(acc[1][1]), "+v"(acc[1][2]), "+v"(acc[1][3]));
        __builtin_amdgcn_sched_barrier(0);
        if (more == 1) {
            if (lane == 0) __hip_atomic_fetch_add(cnt + 32 + t, 1u, __ATOMIC_RELAXED, __HIP_MEMORY_SCOPE_WORKGROUP);
            load_a(a, t + 2);
        } else if (more == 2) {
#pragma unroll
            for (int i = 0; i < 4; ++i) a[i] = *(LAS u32x4*)(stash + (t & 1) * 4096 + i * 1024);
        }
    };
#pragma unroll 1
    for (int pass = 0; pass < 2; ++pass) {
        if (pass) {
            lds_wait(cnt + 68, 4);
            lds_wait(cnt + 69, 4);
            wait_flags<LF_SLEEP>(ws + OFF_LF + (((lane >> 1) & 7) * 32 + rb * 2 + (lane & 1)) * 128, 0xFFFF, call, tmo);
            STAMP(3);
            load_a(ae, 30);
            load_a(ao, 31);
            lds_wait(cnt + 66, 4);
            lds_wait(cnt + 67, 4);
#pragma unroll
            for (int i = 0; i < 4; ++i) { *(LAS u32x4*)(stash + i * 1024) = ae[i]; *(LAS u32x4*)(stash + 4096 + i * 1024) = ao[i]; }
            __builtin_amdgcn_sched_barrier(0);
            load_a(ae, 0);
            __builtin_amdgcn_sched_barrier(0);
            load_a(ao, 1);
#pragma unroll
            for (int i = 0; i < 8; ++i) acc[i >> 2][i & 3] = f32x4{0.f, 0.f, 0.f, 0.f};
#pragma unroll 1
            for (int t = 0; t < 28; t += 2) {
                tile(ae, t, 0x01010101u, 1);
                tile(ao, t + 1, 0x01010101u, 1);
                if (wave == 0) STAMP(30 + (t >> 1));
            }
        }
        tile(ae, 28, pass * 0x01010101u, 2);
        tile(ao, 29, pass * 0x01010101u, 2);
        if (pass && wave == 0) STAMP(44);
        tile(ae, 30, pass * 0x01010101u, 0);
        tile(ao, 31, pass * 0x01010101u, 0);
        const float sc = cnt[72] ? __builtin_nanf("") : scale;
        if (pass) STAMP(59);
        const u32 vo = pass ? (lane >> 4) * 4 * GN * 4 + (lane & 15) * 16 : OOB;
        const u32 so = (rb * 32 * GN + j * 64) * 4;
#pragma unroll
        for (int m = 0; m < 2; ++m)
#pragma unroll
            for (int r = 0; r < 4; ++r) {
                u32x4 v;
#pragma unroll
                for (int c = 0; c < 4; ++c)
                    v[c] = __builtin_bit_cast(u32, __builtin_elementwise_maximum(acc[m][c][r] * sc, 0.f));
                __builtin_amdgcn_raw_buffer_store_b128(v, ro, vo, so + (m * 16 + r) * GN * 4, OUT_AUX);
                __builtin_amdgcn_sched_barrier(0);
                asm volatile("s_nop 1");
                __builtin_amdgcn_sched_barrier(0);
            }
    }
    STAMP(60);
    __builtin_amdgcn_endpgm();
}

__device__ __forceinline__ void unpack_waves(const Params& P, int b, int wave, int lane, GAS u32* cp, LAS char* lds, u32 cv) {
    char* const ws = P.self;
    const int rank = P.rank;
    LAS u32* const cnt = (LAS u32*)(lds + L_CNT);
    if (wave == 8) { cnt[lane] = 0; cnt[64 + lane] = 0; }
    bar();
    const u32 call = __builtin_amdgcn_readfirstlane(cv) + 1;
    const int q = wave & 3, tl = (wave >> 2) & 1;
    const int j = b & 63;
    GAS u32* const tmo = (GAS u32*)(ws + OFF_TMO);
    const __amdgpu_buffer_rsrc_t rs = __builtin_amdgcn_make_buffer_rsrc(ws, 0, WS_NEED, 0x00020000);
    LAS char* const sym = lds + L_SYM + (wave & 7) * 2304;
    const int ux = 2 * b + tl, xsrc = ux & 7, xu = ux >> 3;
    constexpr int NJOB = 17;
    u32 juo, jfo;
    {
        const int i = lane < NJOB ? lane : 0, t = i ? 2 * (i - 1) + tl : 0;
        juo = i ? w_slot(src_of(rank, j, t), t & 3, j, call) : x_slot(xsrc, xu, call);
        jfo = lane < NJOB ? juo + G_FLAG + q * 8 : OFF_TMO;
    }
    const u32 img = q * 2048 + q_img(lane);
    LAS char* image = lds + L_XIM + tl * 8192;
    if (tl) __builtin_amdgcn_s_setprio(2);
    else __builtin_amdgcn_s_setprio(1);
    Packed pk;
    u64 look = 0;
    bool have = false;
    u32 fre = 8;
#pragma unroll 1
    for (int i = 0; i < NJOB; ++i) {
        const int t = 2 * (i - 1) + tl;
        if (__builtin_amdgcn_readfirstlane(fre) < 8) lds_wait(cnt + 32 + t - NBUF, 8);
        asm volatile("" : "+v"(look));
        if (!have) {
            u32 uo = __builtin_amdgcn_readlane(juo, i);
            const char* nf = ws + __builtin_amdgcn_readlane(jfo, i + 1);
            asm volatile("" : "+s"(uo), "+s"(nf));
            u32 meta = __builtin_amdgcn_readfirstlane((u32)look);
            if (__builtin_amdgcn_readfirstlane((u32)(look >> 32)) < call) {
                const char* f = ws + __builtin_amdgcn_readlane(jfo, i);
                if (i < EAGER_FROM) meta = __builtin_amdgcn_readfirstlane((u32)wait_flags<MID_SLEEP>(f, 1, call, tmo));
                else meta = wait_flag(f, call, tmo);
            }
            asm volatile("" ::: "memory");
            load_quarter(pk, rs, uo, meta, q, lane);
            look = __hip_atomic_load((GAS u64*)nf, __ATOMIC_RELAXED, __HIP_MEMORY_SCOPE_SYSTEM);
            if (i >= SJ) STAMP(4 + 3 * (i - 9));
        }
        decode_quarter(pk, rs, img, q, image, sym, lane, [&]() {
            if (t + 2 >= NBUF) fre = __hip_atomic_load(cnt + 32 + t + 2 - NBUF, __ATOMIC_RELAXED, __HIP_MEMORY_SCOPE_WORKGROUP);
            asm volatile("" : "+v"(look));
            Packed nx;
            if (i + 1 == NJOB) {
                if ((u32)look == call) cnt[72] = 1;
            } else {
                have = __builtin_amdgcn_readfirstlane((u32)(look >> 32)) >= call;
                asm volatile("" ::: "memory");
                if (have) load_quarter(nx, rs, __builtin_amdgcn_readlane(juo, i + 1), __builtin_amdgcn_readfirstlane((u32)look), q, lane);
                look = __hip_atomic_load((GAS u64*)(ws + __builtin_amdgcn_readlane(jfo, i + 1 + have)), __ATOMIC_RELAXED, __HIP_MEMORY_SCOPE_SYSTEM);
            }
            pk = nx;
        });
        if (i) {
            __hip_atomic_store((LAS unsigned char*)(cnt + t) + q, (unsigned char)1, __ATOMIC_RELEASE, __HIP_MEMORY_SCOPE_WORKGROUP);
        } else {
            lds_order();
            const u32x2 h0 = *(LAS u32x2*)(image + img), h1 = *(LAS u32x2*)(image + img + 256);
            const u32x2 h2 = *(LAS u32x2*)(image + img + 512), h3 = *(LAS u32x2*)(image + img + 768);
            const u32x4 v0 = {h0[0], h0[1], h1[0], h1[1]}, v1 = {h2[0], h2[1], h3[0], h3[1]};
            const char* xd = ws + OFF_XD + (xsrc * 32 + xu) * 8192 + q * 2048;
            st16<0>(xd, lane * 16, v0);
            st16<1024>(xd, lane * 16, v1);
            wait_vm<0>();
            if (__builtin_amdgcn_readfirstlane(lds_add(cnt + 68 + tl, lane)) == 3)
                put_flag(ws + OFF_LF + (xsrc * 32 + xu) * 128, call, 0, lane);
            STAMP(2);
        }
        image = lds + ((t + 2) & (NBUF - 1)) * 8192;
        if (i >= SJ - 1) STAMP(6 + 3 * (i - 9));
        else if (i < 9 && SJ == 9) STAMP(48 + i);
    }
    if (lane == 0) __hip_atomic_store(cp, call, __ATOMIC_RELAXED, __HIP_MEMORY_SCOPE_AGENT);
    __builtin_amdgcn_endpgm();
}

__global__ void __launch_bounds__(1024, 4) gemm_rs_kernel(Params P) {
    __shared__ __attribute__((aligned(16))) char smem[LDS_BYTES];
    const int lane = threadIdx.x & 63;
    const int wave = __builtin_amdgcn_readfirstlane(threadIdx.x >> 6);
    u32x4 xa, xc;
    {
        const u32 ux = 2 * (blockIdx.x - N_SEND_WG) + (wave >> 2), xdest = (ux + (ux >> 3)) & 7, xu = (ux >> 3) & 31;
        const __amdgpu_buffer_rsrc_t rx = __builtin_amdgcn_make_buffer_rsrc((void*)P.x, 0, 4096 * KS, 0x00020000);
        const u32 so = (xdest * MS + (xu >> 1) * 32 + (wave & 1) * 16) * KS + (xu & 1) * 256 + (wave & 2) * 64;
        const u32 vo = ((lane & 15) * KS + (lane >> 4) * 32) | (blockIdx.x >= N_SEND_WG && wave < 8 ? 0 : OOB);
        xa = __builtin_amdgcn_raw_buffer_load_b128(rx, vo, so, 0);
        xc = __builtin_amdgcn_raw_buffer_load_b128(rx, vo + 16, so, 0);
        asm volatile("" ::: "memory");
    }
    GAS u32* cp = (GAS u32*)(P.self + OFF_CNT) + blockIdx.x * 16 + wave;
    const u32 cv = __hip_atomic_load(cp, __ATOMIC_RELAXED, __HIP_MEMORY_SCOPE_AGENT);
    asm volatile("" ::: "memory");
    STAMP(0);
    if (blockIdx.x >= N_SEND_WG) {
        if (wave < 8) mma_waves(P, blockIdx.x - N_SEND_WG, wave, lane, cp, (LAS char*)smem, xa, xc, cv);
        else unpack_waves(P, blockIdx.x - N_SEND_WG, wave, lane, cp, (LAS char*)smem, cv);
    } else {
        if (wave < 8) sender(P, blockIdx.x, wave, lane, cp, (LAS char*)smem, cv);
        else if (wave == 8) anchor_wave(P.self, lane, cp, (LAS char*)smem, cv);
    }
}

extern "C" void kernel_launch(int rank, int nranks, ncclComm_t comm,
                              hipStream_t stream,
                              void* const* d_in, const int* in_sizes, int n_in,
                              void* d_out, int out_size,
                              const int* in_dtypes, int out_dtype,
                              void* const* d_out_all,
                              void* const* d_ws_all, size_t ws_bytes) {
    if (nranks != NR || n_in != 4 || ws_bytes < WS_NEED || (unsigned)rank >= (unsigned)NR ||
        in_sizes[0] != 4096 * KS || in_sizes[1] != KS * GN || out_size != MS * GN) {
        fprintf(stderr, "kernel_launch: unexpected problem shape\n");
        return;
    }
    Params P;
    P.x = (const uint8_t*)d_in[0];
    P.w = (const uint8_t*)d_in[1];
    P.sx = (const float*)d_in[2];
    P.sw = (const float*)d_in[3];
    P.out = (float*)d_out;
    for (int p = 0; p < NR; ++p) P.ws[p] = (char*)d_ws_all[p];
    P.self = P.ws[rank];
    P.rank = rank;
    P.pad = 0;
    gemm_rs_kernel<<<GRID, 1024, 0, stream>>>(P);
    HIP_CHECK(hipGetLastError());
}
```

```cpp
#include <hip/hip_runtime.h>
#include <rccl/rccl.h>
#include <cstdio>
#include <cstdint>

#define HIP_CHECK(expr) do { hipError_t _e = (expr); if (_e != hipSuccess) { \
    fprintf(stderr, "HIP error %s at %s:%d\n", hipGetErrorString(_e), __FILE__, __LINE__); \
    return; } } while (0)

typedef unsigned u32;
typedef unsigned long long u64;
typedef u32 u32x4 __attribute__((ext_vector_type(4)));
typedef u32 u32x2 __attribute__((ext_vector_type(2)));
typedef int i32x8 __attribute__((ext_vector_type(8)));
typedef float f32x4 __attribute__((ext_vector_type(4)));
#define GAS __attribute__((address_space(1)))
#define LAS __attribute__((address_space(3)))

#define PACE_T0 308
#define PACE_SLOT 363
#define PACE_C 4
#define USUAL16 42
#define LAM 70
#define WIN_SLOTS 5
#define DEAD 25
#define FORCE_RAW 0

constexpr int NR = 8;
constexpr int GN = 4096;
constexpr int KS = 512;
constexpr int MS = 512;
constexpr int NUNIT = 256;
constexpr int N_SEND_WG = 128;
constexpr int N_CONS_WG = 128;
constexpr int GRID = N_SEND_WG + N_CONS_WG;

constexpr u32 USLOT = 9216;
constexpr u32 GSLOT = 6144, NEAR_DW = 224, G_FLAG = 5120 + NEAR_DW * 4;
__host__ __device__ constexpr u32 w_slot(u32 src, u32 kc, u32 j, u32 call) {
    return (((((j & 7) * 8 + src) * 8 + (j >> 3)) * 4 + kc) * 2 + (call & 1)) * GSLOT;
}
__host__ __device__ constexpr u32 x_slot(u32 src, u32 xu, u32 call) { return (4096 + (src * 32 + xu) * 2 + (call & 1)) * GSLOT; }
constexpr u32 OFF_XD = x_slot(NR, 0, 0);
constexpr u32 OFF_LF = OFF_XD + NR * MS * KS;
constexpr u32 OFF_CNT = OFF_LF + NUNIT * 128;
constexpr u32 OFF_TMO = OFF_CNT + GRID * 16 * 4;
constexpr u32 OFF_ANC = OFF_TMO + 128;
constexpr u32 FAR = OFF_ANC + 16 * 128;
constexpr u32 WS_NEED = FAR + OFF_XD;
constexpr u32 DEADLINE_TICKS = 100000000u;
constexpr u32 OOB = 0x80000000u;
constexpr u32 RAW_DW = 256;
constexpr int NBUF = 8;
constexpr u32 L_SYM = NBUF * 8192;
constexpr u32 L_XIM = 4 * 8192;
constexpr u32 L_ENC = L_SYM + 8 * 2304;
constexpr u32 L_STA = L_ENC;
constexpr u32 L_CNT = L_STA + 8 * 8192;
constexpr int LDS_BYTES = L_CNT + 512;

struct Params {
    const uint8_t* x;
    const uint8_t* w;
    const float* sx;
    const float* sw;
    float* out;
    char* self;
    char* ws[NR];
    int rank;
    int pad;
};

template <int OFF>
__device__ __forceinline__ void st16(const char* base, u32 voff, u32x4 v) {
    asm volatile("global_store_dwordx4 %0, %1, %2 offset:%3 sc0 sc1\n\ts_nop 1"
                 :: "v"(voff), "v"(v), "s"(base), "i"(OFF) : "memory");
}
template <int N> __device__ __forceinline__ void wait_vm() { asm volatile("s_waitcnt vmcnt(%0)" :: "i"(N) : "memory"); }

__device__ __forceinline__ void put_flag(char* p, u32 call, u32 meta, int lane) {
    if (lane < 4)
        __hip_atomic_store((GAS u64*)p + lane, ((u64)call << 32) | meta, __ATOMIC_RELAXED, __HIP_MEMORY_SCOPE_SYSTEM);
}

template <int SLEEP>
__device__ __forceinline__ u64 wait_flags(const char* f, u64 need, u32 call, GAS u32* tmo, LAS u32* bad) {
    const u32 t0 = (u32)__builtin_amdgcn_s_memrealtime();
    u64 v;
    for (u32 n = 0;;) {
        v = __hip_atomic_load((GAS u64*)f, __ATOMIC_RELAXED, __HIP_MEMORY_SCOPE_SYSTEM);
        if ((__ballot((u32)(v >> 32) >= call) & need) == need) break;
        if ((++n & 15) == 0) {
            const u32 t = __hip_atomic_load(tmo, __ATOMIC_RELAXED, __HIP_MEMORY_SCOPE_AGENT);
            if (__builtin_amdgcn_readfirstlane(t) == call || (u32)__builtin_amdgcn_s_memrealtime() - t0 > DEADLINE_TICKS) {
                __hip_atomic_store(tmo, call, __ATOMIC_RELAXED, __HIP_MEMORY_SCOPE_AGENT);
                *bad = 1;
                break;
            }
        }
        __builtin_amdgcn_s_sleep(SLEEP);
    }
    asm volatile("" ::: "memory");
    return v;
}

__device__ __forceinline__ u32 wait_flag(const char* f, u32 call, GAS u32* tmo, LAS u32* bad) {
    const u32 t0 = (u32)__builtin_amdgcn_s_memrealtime();
    u32 meta;
    u64 va = __hip_atomic_load((GAS u64*)f, __ATOMIC_RELAXED, __HIP_MEMORY_SCOPE_SYSTEM);
    __builtin_amdgcn_s_sleep(6);
    for (u32 n = 1;; ++n) {
        const u64 vb = __hip_atomic_load((GAS u64*)f, __ATOMIC_RELAXED, __HIP_MEMORY_SCOPE_SYSTEM);
        meta = __builtin_amdgcn_readfirstlane((u32)va);
        if (__builtin_amdgcn_readfirstlane((u32)(va >> 32)) >= call) break;
        va = __hip_atomic_load((GAS u64*)f, __ATOMIC_RELAXED, __HIP_MEMORY_SCOPE_SYSTEM);
        meta = __builtin_amdgcn_readfirstlane((u32)vb);
        if (__builtin_amdgcn_readfirstlane((u32)(vb >> 32)) >= call) break;
        if ((n & 15) == 0) {
            const u32 t = __hip_atomic_load(tmo, __ATOMIC_RELAXED, __HIP_MEMORY_SCOPE_AGENT);
            if (__builtin_amdgcn_readfirstlane(t) == call || (u32)__builtin_amdgcn_s_memrealtime() - t0 > DEADLINE_TICKS) {
                __hip_atomic_store(tmo, call, __ATOMIC_RELAXED, __HIP_MEMORY_SCOPE_AGENT);
                *bad = 1;
                break;
            }
            va = __hip_atomic_load((GAS u64*)f, __ATOMIC_RELAXED, __HIP_MEMORY_SCOPE_SYSTEM);
            __builtin_amdgcn_s_sleep(6);
        }
    }
    asm volatile("" ::: "memory");
    return meta;
}

__device__ __forceinline__ u32 wave_scan(u32 v) {
    v += __builtin_amdgcn_update_dpp(0u, v, 0x111, 0xf, 0xf, true);
    v += __builtin_amdgcn_update_dpp(0u, v, 0x112, 0xf, 0xf, true);
    v += __builtin_amdgcn_update_dpp(0u, v, 0x114, 0xf, 0xf, true);
    v += __builtin_amdgcn_update_dpp(0u, v, 0x118, 0xf, 0xf, true);
    v += __builtin_amdgcn_update_dpp(0u, v, 0x142, 0xa, 0xf, false);
    v += __builtin_amdgcn_update_dpp(0u, v, 0x143, 0xc, 0xf, false);
    return v;
}
__device__ __forceinline__ u32 bfi(u32 m, u32 a, u32 b) { return (m & a) | (~m & b); }
__device__ __forceinline__ void lds_order() { __builtin_amdgcn_fence(__ATOMIC_ACQ_REL, "wavefront"); }

__device__ __forceinline__ u32 spread80(u32 m) { return (m << 1) - (m >> 7); }
template <class Early>
__device__ __forceinline__ u32 encode_quarter(const u32 (&d)[8], u32 img, LAS char* unit, int q, LAS char* stage,
                                              LAS char* lut, int lane, u32& meta, Early codes_out) {
    u32 mag[8], sum = 0;
#pragma unroll
    for (int k = 0; k < 8; ++k) { mag[k] = d[k] & 0x7F7F7F7Fu; sum = __builtin_amdgcn_sad_u8(mag[k], 0u, sum); }
    int b = ((int)__builtin_amdgcn_readlane(wave_scan(sum), 63) + 1024 - 14541) >> 11;
    b = b < 12 ? 12 : b;
    asm volatile("" : "+s"(b));
    const u32 B = b > 108 ? 108 : b;
    const u32 c2 = (126 - B) * 0x01010101u, c16 = (112 - B) * 0x01010101u;
    u32 nib[8], esc[8];
#pragma unroll
    for (int k = 0; k < 8; ++k) {
        const u32 u = mag[k] + c2, w = mag[k] + c16;
        const u32 dir = u & ~w & 0x80808080u;
        nib[k] = w & (((dir >> 3) - (dir >> 6)) | 0x01010101u);
        esc[k] = dir ^ 0x80808080u;
    }
    codes_out(u32x4{nib[0] | (nib[1] << 4), nib[2] | (nib[3] << 4), nib[4] | (nib[5] << 4), nib[6] | (nib[7] << 4)}, false);
    const u32 amax = B >= 40 ? 20 : B >> 1;
    const u32 cz = (129 + B) * 0x01010101u, clo = (128 - B + 2 * amax) * 0x01010101u, c20 = (108 - B) * 0x01010101u;
    u32 sel[8], off[8], cnt = 0, sg = 0, mb = 0;
#pragma unroll
    for (int k = 0; k < 8; ++k) sel[k] = *(LAS u32*)(lut + (__builtin_amdgcn_udot4(esc[k], 0x08040201u, 0u, false) >> 5));
#pragma unroll
    for (int k = 0; k < 8; ++k) {
        mb |= ((~(mag[k] + clo) | (mag[k] + c20)) & 0x80808080u) >> (7 - k);
        off[k] = cnt;
        cnt += __builtin_popcount(esc[k]);
        sg |= (d[k] >> (7 - k)) & (0x01010101u << k);
    }
    const u32 incl = wave_scan(cnt);
    const u32 nrec = (__builtin_amdgcn_readlane(incl, 63) + 7) >> 3, n1 = (3 * nrec + 3) >> 2;
    LAS char* blk = unit + 5120 + q * 1024;
    *(LAS u32*)(unit + 4096 + q * 256 + lane * 4) = sg;
    u32 n2 = 0, ne = 0;
    if (nrec <= 192 && !FORCE_RAW) {
        const u32 base = incl - cnt;
#pragma unroll
        for (int k = 0; k < 8; ++k) {
            const u32 v = ((cz - mag[k]) >> 1) & 0x3F3F3F3Fu, pos = base + off[k];
            const u64 x = (u64)__builtin_amdgcn_perm(0u, v, sel[k]) << (8 * (pos & 3));
            LAS u32* p = (LAS u32*)(stage + (pos & ~3u));
            __hip_atomic_fetch_or(p, (u32)x, __ATOMIC_RELAXED, __HIP_MEMORY_SCOPE_WORKGROUP);
            __hip_atomic_fetch_or(p + 1, (u32)(x >> 32), __ATOMIC_RELAXED, __HIP_MEMORY_SCOPE_WORKGROUP);
        }
        lds_order();
        const u32 cga = (127 - amax) * 0x01010101u;
        u32 nfar = 0;
        for (u32 r0 = 0; r0 < nrec; r0 += 64) {
            const u32x2 vv = *(LAS u32x2*)(stage + (r0 + lane) * 8);
            u32 c1[2], cf[2], far[2];
#pragma unroll
            for (int h = 0; h < 2; ++h) {
                const u32 v = vv[h];
                const u32 ge6 = v + 0x7A7A7A7Au, gta = v + cga, ge55 = v + 0x49494949u, ge56 = v + 0x48484848u;
                far[h] = ((ge6 & ~gta) | (ge55 & ~ge56)) & 0x80808080u;
                c1[h] = ((v & ~spread80(gta & 0x80808080u)) | (spread80(ge56 & 0x80808080u) & 0x06060606u) | spread80(far[h])) & 0x07070707u;
                const u32 t = (ge6 | spread80(ge55 & 0x80808080u)) & 0x0F0F0F0Fu;
                const u32 sl = *(LAS u32*)(lut + (__builtin_amdgcn_udot4(far[h], 0x08040201u, 0u, false) >> 5));
                const u32 y = __builtin_amdgcn_perm(0u, t, sl);
                cf[h] = __builtin_amdgcn_perm(0u, y | (y >> 4), 0x0C0C0200u);
            }
            const u32 x = c1[0] | (c1[1] << 3);
            const u32 rec = x | (__umul24(x >> 24, 0x41040u) & 0xC0C0C0u);
            if (r0 + lane < nrec) {
                LAS unsigned char* rp = (LAS unsigned char*)blk + 4 * n1 - 3 * (r0 + lane + 1);
                rp[0] = rec; rp[1] = rec >> 8; rp[2] = rec >> 16;
            }
            const u32 f0 = __builtin_popcount(far[0]), f = f0 + __builtin_popcount(far[1]), fi = wave_scan(f);
            const u32 pos = nfar + fi - f;
            nfar += __builtin_amdgcn_readlane(fi, 63);
            if (f != 0 && pos <= 256) {
                const u64 z = (u64)(cf[0] | (cf[1] << (4 * f0))) << (4 * (pos & 7));
                LAS u32* p = (LAS u32*)(blk + 4 * n1 + ((pos >> 1) & ~3u));
                __hip_atomic_fetch_or(p, (u32)z, __ATOMIC_RELAXED, __HIP_MEMORY_SCOPE_WORKGROUP);
                __hip_atomic_fetch_or(p + 1, (u32)(z >> 32), __ATOMIC_RELAXED, __HIP_MEMORY_SCOPE_WORKGROUP);
            }
        }
        n2 = (nfar + 7) >> 3;
        lds_order();
        for (u64 mk; (mk = __ballot(mb != 0)) != 0; ne += __builtin_popcountll(mk)) {
            if (mb != 0) {
                const u32 bit = __builtin_ctz(mb);
                mb &= mb - 1;
                const u32 k0 = (u32)((int)(bit << 31) >> 31), k1 = (u32)((int)(bit << 30) >> 31), k2 = (u32)((int)(bit << 29) >> 31);
                const u32 m0 = bfi(k2, mag[4], mag[0]), m1 = bfi(k2, mag[5], mag[1]), m2 = bfi(k2, mag[6], mag[2]), m3 = bfi(k2, mag[7], mag[3]);
                const u32 mg = (bfi(k0, bfi(k1, m3, m1), bfi(k1, m2, m0)) >> (bit & 24u)) & 127u;
                const u32 ad = img + (bit & 6u) * 128 + (k0 & 4u) + (bit >> 3);
                const u32 v = ((129 + B - mg) >> 1) & 63u;
                const u32 dflt = B - (v <= amax ? 2 * v : 0) + ((mg - B) & 1u);
                const u32 slot = n1 + n2 + ne + __builtin_amdgcn_mbcnt_hi((u32)(mk >> 32), __builtin_amdgcn_mbcnt_lo((u32)mk, 0u));
                *(LAS u32*)(blk + (slot < 255 ? slot : 255) * 4) = (ad << 8) | (mg ^ dflt);
            }
        }
    }
    if (nrec > 192 || n2 > 32 || ne > 31 || FORCE_RAW) {
        lds_order();
        codes_out(u32x4{d[0], d[1], d[2], d[3]}, true);
        *(LAS u32x4*)(blk + lane * 16) = u32x4{d[4], d[5], d[6], d[7]};
        meta = 3u << 11;
        return RAW_DW;
    }
    meta = n2 | (ne << 6) | ((nrec > 128 ? 2u : (nrec > 64 ? 1u : 0u)) << 11) | (B << 15) | (n1 << 22);
    return n1 + n2 + ne;
}

struct Packed {
    u32x4 n;
    u32 s, b1, b2, be;
    u32 meta, uo;
};
__device__ __forceinline__ u32 blk_off(u32 uo, u32 meta) { return uo + (meta & 0x4000u ? FAR : 5120) + (meta >> 22) * 4; }
__device__ __forceinline__ void load_quarter(Packed& p, char* ws, __amdgpu_buffer_rsrc_t rs, u32 uo, u32 meta, int q, int lane) {
    p.n = __builtin_amdgcn_raw_buffer_load_b128(rs, lane * 16 + q * 1024, uo, 17);
    __builtin_amdgcn_sched_barrier(0);
    p.meta = meta;
    p.uo = uo;
    const u32 bo = blk_off(uo, meta), n2 = meta & 63u;
    p.b1 = __builtin_amdgcn_raw_buffer_load_b32(rs, 189 - 3 * lane, bo - 192, 17);
    p.b2 = __builtin_amdgcn_raw_buffer_load_b16(__builtin_amdgcn_make_buffer_rsrc(ws + bo, 0, n2 * 4, 0x00020000), lane * 2, 0, 17);
    p.s = __builtin_amdgcn_raw_buffer_load_b32(rs, lane * 4 + 4096 + q * 256, uo, 17);
    p.be = __builtin_amdgcn_raw_buffer_load_b32(__builtin_amdgcn_make_buffer_rsrc(ws + bo + n2 * 4, 0, (meta >> 4) & 124u, 0x00020000), lane * 4, 0, 17);
}
struct Rec {
    u32 near[2], sel[2], f0, f;
    __device__ __forceinline__ Rec(u32 rec, u32 lut0, u32 lut1) {
        const u32 x = bfi(0xFF000000u, __umul24(rec & 0xC0C0C0u, 0x41040u), rec);
#pragma unroll
        for (int h = 0; h < 2; ++h) {
            const u32 c = (h ? x >> 3 : x) & 0x07070707u;
            near[h] = __builtin_amdgcn_perm(lut1, lut0, c);
            const u32 e10 = (c + 0x09090909u) & 0x10101010u;
            sel[h] = __umul24(e10, 0x101010u) + (e10 ^ 0x10101010u);
            if (h) f = f0 + __builtin_popcount(e10);
            else f0 = __builtin_popcount(e10);
        }
    }
    u32 ah[2], g0[2], g1[2];
    __device__ __forceinline__ void gather(u32 a) {
#pragma unroll
        for (int h = 0; h < 2; ++h) {
            ah[h] = h ? a + f0 : a;
            LAS u32* gp = (LAS u32*)(size_t)(ah[h] & ~3u);
            g0[h] = gp[0]; g1[h] = gp[1];
        }
    }
    __device__ __forceinline__ u32x2 bytes() const {
        u32x2 o;
#pragma unroll
        for (int h = 0; h < 2; ++h) {
            const u32 g = __builtin_amdgcn_alignbyte(g1[h], g0[h], ah[h]);
            o[h] = near[h] & __builtin_amdgcn_perm(g, g, sel[h]);
        }
        return o;
    }
};
template <class Mid>
__device__ __forceinline__ void decode_quarter(const Packed& p, __amdgpu_buffer_rsrc_t rs, u32 img, int q,
                                               LAS char* image, LAS char* sym, int lane, Mid mid) {
    const u32 uo = p.uo, bo = blk_off(uo, p.meta), cls = (p.meta >> 11) & 3u, B1 = ((p.meta >> 15) & 127u) * 0x01010101u;
    const u32 kx = ~B1, lut0 = (B1 - 0x06040200u) ^ kx, lut1 = ((B1 + 0x00100000u - 0x00000A08u) ^ kx) | 0xFF000000u;
    u32 nib[8], e10[8], off[8], cnt = 0;
#pragma unroll
    for (int k = 0; k < 8; ++k) {
        nib[k] = (k & 1 ? p.n[k >> 1] >> 4 : p.n[k >> 1]) & 0x0F0F0F0Fu;
        e10[k] = ~(nib[k] + 0x0E0E0E0Eu) & 0x10101010u;
        off[k] = cnt;
        cnt += __builtin_popcount(e10[k]);
    }
    __builtin_amdgcn_sched_barrier(0);
    {
        u32 t = (p.b2 | (p.b2 << 8)) & 0x00FF00FFu;
        t = (t | (t << 4)) & 0x0F0F0F0Fu;
        const u32 h = (t + 0x01010101u) & 0x10101010u;
        *(LAS u32*)(sym + 1600 + lane * 4) = (B1 - 0x0C0C0C0Cu + ((h << 2) - (h >> 2)) - (t << 1)) ^ kx;
    }
    Rec R(p.b1, lut0, lut1);
    const u32 both = cnt | (R.f << 16), inc = wave_scan(both), exc = inc - both;
    lds_order();
    R.gather((u32)(size_t)sym + 1600 + (exc >> 16));
    __builtin_amdgcn_sched_barrier(0);
    u32 ga[8], g0[8], g1[8], sel[8], o[8];
#pragma unroll
    for (int k = 0; k < 8; ++k) {
        off[k] += (u32)(size_t)sym + (exc & 0xFFFFu);
        ga[k] = off[k] & ~3u;
        nib[k] |= (p.s << (7 - k)) & 0x80808080u;
    }
    asm volatile("" : "+v"(ga[0]), "+v"(ga[1]), "+v"(ga[2]), "+v"(ga[3]), "+v"(ga[4]), "+v"(ga[5]), "+v"(ga[6]), "+v"(ga[7]),
                      "+v"(nib[0]), "+v"(nib[1]), "+v"(nib[2]), "+v"(nib[3]), "+v"(nib[4]), "+v"(nib[5]), "+v"(nib[6]), "+v"(nib[7]));
    __builtin_amdgcn_sched_barrier(0);
    *(LAS u32x2*)(sym + lane * 8) = R.bytes();
    if (cls == 1 || cls == 2) {
        u32 nf = __builtin_amdgcn_readlane(inc, 63) >> 16;
        for (u32 c = 1; c <= cls; ++c) {
            Rec Rc(__builtin_amdgcn_raw_buffer_load_b32(rs, 189 - 3 * lane, bo - 192 * (c + 1), 17), lut0, lut1);
            const u32 fi = wave_scan(Rc.f);
            Rc.gather((u32)(size_t)sym + 1600 + ((nf + fi - Rc.f) & 511u));
            *(LAS u32x2*)(sym + c * 512 + lane * 8) = Rc.bytes();
            nf += __builtin_amdgcn_readlane(fi, 63);
        }
    }
    lds_order();
#pragma unroll
    for (int k = 0; k < 8; ++k) {
        LAS u32* gp = (LAS u32*)(size_t)ga[k];
        g0[k] = gp[0]; g1[k] = gp[1];
    }
    __builtin_amdgcn_sched_barrier(0);
#pragma unroll
    for (int k = 0; k < 8; ++k)
        sel[k] = __umul24(e10[k], 0x101010u) + (e10[k] ^ 0x10101010u);
    u32 ev = (p.be & 127u) << ((p.be >> 5) & 24u);
    u32 ea = (u32)(size_t)image + ((p.be >> 8) & 0x1FFCu);
    u32 wa = (u32)(size_t)image + img;
    asm volatile("" : "+v"(ev), "+v"(ea), "+v"(wa), "+v"(sel[0]), "+v"(sel[1]), "+v"(sel[2]), "+v"(sel[3]),
                      "+v"(sel[4]), "+v"(sel[5]), "+v"(sel[6]), "+v"(sel[7]));
    mid();
    __builtin_amdgcn_sched_barrier(0);
#pragma unroll
    for (int c = 0; c < 4; ++c) {
#pragma unroll
        for (int k = 2 * c; k < 2 * c + 2; ++k) {
            const u32 g = __builtin_amdgcn_alignbyte(g1[k], g0[k], off[k]);
            o[k] = (__builtin_amdgcn_perm(g, g, sel[k]) ^ kx) + nib[k];
        }
        *(LAS u32x2*)(size_t)(wa + c * 256) = u32x2{o[2 * c], o[2 * c + 1]};
    }
    lds_order();
    if (ev) __hip_atomic_fetch_xor((LAS u32*)(size_t)ea, ev, __ATOMIC_RELAXED, __HIP_MEMORY_SCOPE_WORKGROUP);
    if (cls == 3) {
        const u32x4 l = __builtin_amdgcn_raw_buffer_load_b128(rs, lane * 16 + q * 1024, uo, 17);
        const u32x4 h = __builtin_amdgcn_raw_buffer_load_b128(rs, lane * 16, bo, 17);
        *(LAS u32x2*)(size_t)wa = u32x2{l[0], l[1]};
        *(LAS u32x2*)(size_t)(wa + 256) = u32x2{l[2], l[3]};
        *(LAS u32x2*)(size_t)(wa + 512) = u32x2{h[0], h[1]};
        *(LAS u32x2*)(size_t)(wa + 768) = u32x2{h[2], h[3]};
    }
}

__device__ __forceinline__ u32 q_img(int lane) {
    return (lane >> 5) * 1024 + ((lane & 7) | ((lane >> 1) & 8)) * 16 + (lane & 8);
}

struct Unit {
    u32x4 p[6];
    u32 meta;
    u32 c1, c2, c3, tot;
    u32 tot16;
    LAS char* lds;
    __device__ __forceinline__ u32x4 blk16(u32 i) const {
        u32x4 v;
#pragma unroll
        for (int k = 0; k < 4; ++k) {
            const u32 w = 4 * i + k;
            const u32 a = w + (w < c1 ? 0 : (w < c2 ? 256 - c1 : (w < c3 ? 512 - c2 : 768 - c3)));
            v[k] = *(LAS u32*)(lds + 5120 + (a & 1023u) * 4);
        }
        return v;
    }
    __device__ __forceinline__ void fetch(LAS char* unit, LAS u32* lens, int lane) {
        lds = unit;
        const u32 l = lens[lane & 3];
        c1 = __builtin_amdgcn_readlane(l, 0);
        c2 = c1 + __builtin_amdgcn_readlane(l, 1);
        c3 = c2 + __builtin_amdgcn_readlane(l, 2);
        tot = c3 + __builtin_amdgcn_readlane(l, 3);
        tot16 = (tot + 3) >> 2;
        meta = lens[8 + (lane & 3)] + (tot > NEAR_DW ? 0x4000u : 0u) + ((lane == 0 ? 0 : (lane == 1 ? c1 : (lane == 2 ? c2 : c3))) << 22);
#pragma unroll
        for (int i = 0; i < 5; ++i) p[i] = *(LAS u32x4*)(unit + i * 1024 + lane * 16);
        p[5] = blk16(lane);
    }
    __device__ __forceinline__ void send_blocks(const char* dst, u32 voff, int lane) const {
        if (tot > NEAR_DW) dst += FAR - 5120;
        for (u32 i = 64 + lane; i < tot16; i += 64) st16<1024>(dst, voff + (i - lane) * 16, blk16(i));
        if ((u32)lane < tot16) st16<1024>(dst, voff, p[5]);
    }
    __device__ __forceinline__ void send_all(const char* dst, u32 voff, int lane) const {
        st16<-4096>(dst, voff, p[0]); st16<-3072>(dst, voff, p[1]); st16<-2048>(dst, voff, p[2]);
        st16<-1024>(dst, voff, p[3]); st16<0>(dst, voff, p[4]);
        send_blocks(dst, voff, lane);
    }
};

__device__ __forceinline__ void pace(u32 due) {
    while ((int)((u32)__builtin_amdgcn_s_memrealtime() - due) < 0) __builtin_amdgcn_s_sleep(1);
}

__device__ __forceinline__ void bar() {
    asm volatile("s_waitcnt lgkmcnt(0)" ::: "memory");
    __builtin_amdgcn_s_barrier();
    asm volatile("" ::: "memory");
}
constexpr u32 E_STAGE = 2 * USLOT + 1024;
__device__ __forceinline__ void enc_init(LAS char* lds, int wave, int lane) {
    *(LAS u32x4*)(lds + (wave >> 2) * USLOT + 5120 + (wave & 3) * 1024 + lane * 16) = u32x4{0, 0, 0, 0};
    *(LAS u32x4*)(lds + E_STAGE + wave * 2048 + lane * 16) = u32x4{0, 0, 0, 0};
    *(LAS u32x4*)(lds + E_STAGE + wave * 2048 + 1024 + lane * 16) = u32x4{0, 0, 0, 0};
    u32 sel = 0x0C0C0C0Cu, n = 0;
#pragma unroll
    for (int i = 0; i < 4; ++i)
        if ((lane >> i) & 1) { sel = (sel & ~(0xFFu << n)) | ((u32)i << n); n += 8; }
    if (lane < 16) *(LAS u32*)(lds + 2 * USLOT + wave * 64 + lane * 4) = sel;
    lds_order();
}

__device__ __forceinline__ void anchor_wave(const char* self, int lane, GAS u32* cp, LAS char* lds, u32 cv) {
    const u32 call = __builtin_amdgcn_readfirstlane(cv) + 1;
    u32 v, now;
    u64 ok;
    int n = 0;
    __builtin_amdgcn_s_sleep(16);
    do {
        __builtin_amdgcn_s_sleep(4);
        v = __hip_atomic_load((GAS u32*)(self + OFF_ANC + (lane >> 2) * 128 + (lane & 3) * 4), __ATOMIC_RELAXED, __HIP_MEMORY_SCOPE_AGENT);
        ok = __ballot((v >> 24) == (call & 255u));
        now = (u32)__builtin_amdgcn_s_memrealtime();
    } while (ok != ~0ull && ++n < 8);
    u32 age = (now - v) & 0xFFFFFFu;
    age = (v >> 24) == (call & 255u) ? (age > 1000 ? 1000 : age) : 0;
    *(LAS u32*)(lds + 2 * USLOT + 512 + 64) = now - (__builtin_amdgcn_readlane(wave_scan(age), 63) >> 6);
    bar();
    if (lane == 0) __hip_atomic_store(cp, call, __ATOMIC_RELAXED, __HIP_MEMORY_SCOPE_AGENT);
    __builtin_amdgcn_endpgm();
}

__device__ __forceinline__ void sender(const Params& P, int b, int wave, int lane, GAS u32* cp, LAS char* lds, u32 cv) {
    const int rank = P.rank;
    char* const self = P.self;
    const int q = wave & 3, tl = wave >> 2;
    const int j = b & 63, kc0 = b >> 6;
    __builtin_amdgcn_s_sleep(8);
    u32 wr[8];
    {
        const u32 cg = (lane & 7) | ((lane >> 1) & 8), rg = ((lane >> 3) & 1) | ((lane >> 4) & 2);
        const uint8_t* wp = P.w + (size_t)((kc0 + 2 * tl) * 128 + q * 32 + rg * 8) * GN + j * 64 + cg * 4;
#pragma unroll
        for (int i = 0; i < 8; ++i) wr[i] = *(const GAS u32*)(wp + (size_t)i * GN);
    }
    enc_init(lds, wave, lane);
    LAS char* const unit = lds + tl * USLOT;
    LAS u32* const lens = (LAS u32*)(lds + 2 * USLOT + 512);
    u32 dw[8];
#pragma unroll
    for (int h = 0; h < 2; ++h) {
        const u32 r0 = wr[4 * h], r1 = wr[4 * h + 1], r2 = wr[4 * h + 2], r3 = wr[4 * h + 3];
        const u32 t0 = __builtin_amdgcn_perm(r1, r0, 0x05010400u), t1 = __builtin_amdgcn_perm(r1, r0, 0x07030602u);
        const u32 t2 = __builtin_amdgcn_perm(r3, r2, 0x05010400u), t3 = __builtin_amdgcn_perm(r3, r2, 0x07030602u);
        dw[0 + h] = __builtin_amdgcn_perm(t2, t0, 0x05040100u);
        dw[2 + h] = __builtin_amdgcn_perm(t2, t0, 0x07060302u);
        dw[4 + h] = __builtin_amdgcn_perm(t3, t1, 0x05040100u);
        dw[6 + h] = __builtin_amdgcn_perm(t3, t1, 0x07060302u);
    }
    u32 qmeta;
    const u32 len = encode_quarter(dw, q * 2048 + q_img(lane), unit, q, lds + E_STAGE + wave * 2048, lds + 2 * USLOT + wave * 64, lane, qmeta,
                                   [&](u32x4 v, bool) { *(LAS u32x4*)(unit + q * 1024 + lane * 16) = v; });
    if (lane == 0) { lens[wave] = len; lens[8 + wave] = qmeta; }
    const u32 call = __builtin_amdgcn_readfirstlane(cv) + 1;
    const u32 voff = lane * 16 + 4096;
    bar();
    if (q == 1 || q == 2) {
        const int kc = kc0 + 2 * tl;
        const u32 wo = w_slot(rank, kc, j, call), fo = wo + G_FLAG;
        Unit U;
        U.fetch(unit, lens + tl * 4, lane);
        if (q == 2) {
            U.send_all(self + wo, voff, lane);
            wait_vm<0>();
            put_flag(self + fo, call, U.meta, lane);
        } else {
            const int est = __builtin_amdgcn_readlane(wave_scan(lens[lane & 7]), 7) / 8 + PACE_SLOT - USUAL16;
            const int slot = est > PACE_SLOT + 6 || est < PACE_SLOT - 7 ? est + 2 : PACE_SLOT;
            const u32 anchor = __builtin_amdgcn_readfirstlane(lens[16]);
            const u32 t0 = anchor + PACE_T0 * slot / PACE_SLOT;
            u32 slip = 0;
            int pr = (kc + j) % 7;
#pragma unroll 1
            for (int s = 0; s < 7; ++s) {
                char* const peer = P.ws[(rank + 1 + pr) & 7];
                const char* const dst = peer + wo;
                pr = pr == 6 ? 0 : pr + 1;
                const int m0 = 4 * s + kc;
                int before = (m0 > 3 ? m0 - 3 : 0) * 64 * PACE_C;
#pragma unroll
                for (int k = -3; k < 4; ++k) {
                    const int v = PACE_C * j - 128 * k;
                    if ((u32)(m0 + k) < 28) before += v < 0 ? 0 : (v > 64 * PACE_C ? 64 * PACE_C : v);
                }
                const u32 due = t0 + before * slot / (256 * PACE_C);
                const bool win = s < WIN_SLOTS;
                pace(win ? due - LAM : due + slip);
                st16<-4096>(dst, voff, U.p[0]); __builtin_amdgcn_s_sleep(2);
                st16<-3072>(dst, voff, U.p[1]); __builtin_amdgcn_s_sleep(2);
                if (win) { wait_vm<0>(); pace(due); }
                st16<-2048>(dst, voff, U.p[2]); __builtin_amdgcn_s_sleep(2);
                st16<-1024>(dst, voff, U.p[3]); __builtin_amdgcn_s_sleep(2);
                if (win) { wait_vm<0>(); pace(due + LAM); const int late = (int)((u32)__builtin_amdgcn_s_memrealtime() - (due + LAM + DEAD)); slip = late > 0 ? late / 2 : 0; }
                st16<0>(dst, voff, U.p[4]); __builtin_amdgcn_s_sleep(2);
                U.send_blocks(dst, voff, lane);
                wait_vm<0>();
                put_flag(peer + fo, call, U.meta, lane);
            }
        }
    }
    if (lane == 0) __hip_atomic_store(cp, call, __ATOMIC_RELAXED, __HIP_MEMORY_SCOPE_AGENT);
    __builtin_amdgcn_endpgm();
}

__device__ __forceinline__ u32 lds_add(LAS u32* c, int lane) {
    u32 v = 0;
    if (lane == 0) v = __hip_atomic_fetch_add(c, 1u, __ATOMIC_ACQ_REL, __HIP_MEMORY_SCOPE_WORKGROUP);
    return v;
}
__device__ __forceinline__ void lds_wait(LAS u32* c, u32 n) {
    while (__builtin_amdgcn_readfirstlane(__hip_atomic_load(c, __ATOMIC_ACQUIRE, __HIP_MEMORY_SCOPE_WORKGROUP)) < n)
        __builtin_amdgcn_s_sleep(1);
}
__device__ __forceinline__ int src_of(int rank, int j, int t) {
    return t < 4 ? rank : (rank + 7 - ((t >> 2) - 1 + (t & 3) + j) % 7) & 7;
}

__device__ __forceinline__ void mma_waves(const Params& P, int b, int wave, int lane, GAS u32* cp, LAS char* lds, u32x4 a, u32x4 c, u32 cv) {
    char* const ws = P.self;
    const int rank = P.rank;
    u32 cnt_a = (u32)(size_t)(lds + L_CNT);
    asm volatile("" : "+v"(cnt_a));
    LAS u32* const cnt = (LAS u32*)(size_t)cnt_a;
    u32 call;
    {
        const int q = wave & 3, tl = wave >> 2, ux = 2 * b + tl;
        const int xdest = (ux + (ux >> 3)) & 7, xu = ux >> 3;
        char* const xdst = P.ws[xdest];
        u32 dx[8] = {a[0], a[1], a[2], a[3], c[0], c[1], c[2], c[3]};
        LAS char* const el = lds + L_ENC;
        enc_init(el, wave, lane);
        LAS u32* const lens = (LAS u32*)(el + 2 * USLOT + 512);
        call = __builtin_amdgcn_readfirstlane(cv) + 1;
        const u32 voff = lane * 16 + 4096;
        char* const xslot = xdst + x_slot(rank, xu, call);
        if ((b & 7) == (b >> 4) && (wave & 1)) {
            asm volatile("" : "+v"(dx[0]), "+v"(dx[7]));
            if (lane == 0)
                __hip_atomic_store((GAS u32*)(ws + OFF_ANC + (b >> 3) * 128 + (wave >> 1) * 4),
                                   (call << 24) | ((u32)__builtin_amdgcn_s_memrealtime() & 0xFFFFFFu), __ATOMIC_RELAXED, __HIP_MEMORY_SCOPE_AGENT);
        }
        u32 qmeta;
        const u32 len = encode_quarter(dx, q * 2048 + q_img(lane), el + tl * USLOT, q, el + E_STAGE + wave * 2048, el + 2 * USLOT + wave * 64, lane, qmeta, [&](u32x4 v, bool again) {
            if (again) wait_vm<0>();
            st16<-4096>(xslot + q * 1024, voff, v);
        });
        if (lane == 0) { lens[wave] = len; lens[8 + wave] = qmeta; }
        bar();
        if (q == 0) {
            Unit U;
            U.fetch(el + tl * USLOT, lens + tl * 4, lane);
            st16<0>(xslot, voff, U.p[4]);
            U.send_blocks(xslot, voff, lane);
            if (lane < 4) cnt[76 + tl * 4 + lane] = U.meta;
        }
        wait_vm<0>();
        if (__builtin_amdgcn_readfirstlane(lds_add(cnt + 66 + tl, lane)) == 3) {
            put_flag(xslot + G_FLAG, call, cnt[76 + tl * 4 + (lane & 3)], lane);
        }
    }
    const int j = b & 63;
    const int rb = (b >> 6) * 8 + wave;
    GAS u32* const tmo = (GAS u32*)(ws + OFF_TMO);
    const __amdgpu_buffer_rsrc_t rs = __builtin_amdgcn_make_buffer_rsrc(ws, 0, WS_NEED, 0x00020000);
    const __amdgpu_buffer_rsrc_t ro = __builtin_amdgcn_make_buffer_rsrc(P.out, 0, MS * GN * 4, 0x00020000);
    const float scale = __builtin_bit_cast(float, __builtin_amdgcn_readfirstlane(__builtin_bit_cast(u32, *P.sx * *P.sw)));
    if (lane == 0) __hip_atomic_store(cp, call, __ATOMIC_RELAXED, __HIP_MEMORY_SCOPE_AGENT);
    LAS char* const rp = lds + (lane >> 4) * 2048 + (lane & 15) * 16;
    const u32 txo = OFF_XD + (src_of(rank, j, lane & 31) * 16 + rb) * 16384 + (lane & 3) * 4096;
    f32x4 acc[2][4] = {};
    u32x4 ae[4] = {}, ao[4] = {};
    auto load_a = [&](u32x4 (&a)[4], int t) {
        const u32 so = __builtin_amdgcn_readlane(txo, t);
        u32 l16 = lane * 16;
        asm volatile("" : "+v"(l16));
#pragma unroll
        for (int i = 0; i < 4; ++i) a[i] = __builtin_amdgcn_raw_buffer_load_b128(rs, l16 + i * 1024, so, 17);
    };
    LAS char* const stash = lds + L_STA + wave * 8192 + lane * 16;
    auto tile = [&](u32x4 (&a)[4], int t, u32 need, int more) {
        lds_wait(cnt + t, need);
        LAS char* const bp = rp + (t & (NBUF - 1)) * 8192;
        u32x4 bq[8];
#pragma unroll
        for (int i = 0; i < 8; ++i) bq[i] = *(LAS u32x4*)(bp + (i >> 1) * 256 + (i & 1) * 1024);
        __builtin_amdgcn_sched_barrier(0);
#pragma unroll
        for (int c = 0; c < 4; ++c)
#pragma unroll
            for (int m = 0; m < 2; ++m) {
                const u32x4 a0 = a[2 * m], a1 = a[2 * m + 1], b0 = bq[2 * c], b1 = bq[2 * c + 1];
                const i32x8 af = {(int)a0[0], (int)a0[1], (int)a0[2], (int)a0[3], (int)a1[0], (int)a1[1], (int)a1[2], (int)a1[3]};
                const i32x8 bf = {(int)b0[0], (int)b0[1], (int)b0[2], (int)b0[3], (int)b1[0], (int)b1[1], (int)b1[2], (int)b1[3]};
                acc[m][c] = __builtin_amdgcn_mfma_scale_f32_16x16x128_f8f6f4(af, bf, acc[m][c], 1, 1, 0, 0x7F7F7F7F, 0, 0x7F7F7F7F);
            }
        asm volatile("" : "+v"(acc[0][0]), "+v"(acc[0][1]), "+v"(acc[0][2]), "+v"(acc[0][3]),
                          "+v"(acc[1][0]), "+v"(acc[1][1]), "+v"(acc[1][2]), "+v"(acc[1][3]));
        __builtin_amdgcn_sched_barrier(0);
        if (more == 1) {
            if (lane == 0) __hip_atomic_fetch_add(cnt + 32 + t, 1u, __ATOMIC_RELAXED, __HIP_MEMORY_SCOPE_WORKGROUP);
            load_a(a, t + 2);
        } else if (more == 2) {
#pragma unroll
            for (int i = 0; i < 4; ++i) a[i] = *(LAS u32x4*)(stash + (t & 1) * 4096 + i * 1024);
        }
    };
#pragma unroll 1
    for (int pass = 0; pass < 2; ++pass) {
        if (pass) {
            lds_wait(cnt + 68, 4);
            lds_wait(cnt + 69, 4);
            wait_flags<16>(ws + OFF_LF + (((lane >> 1) & 7) * 32 + rb * 2 + (lane & 1)) * 128, 0xFFFF, call, tmo, cnt + 72);
            load_a(ae, 30);
            load_a(ao, 31);
            lds_wait(cnt + 66, 4);
            lds_wait(cnt + 67, 4);
#pragma unroll
            for (int i = 0; i < 4; ++i) { *(LAS u32x4*)(stash + i * 1024) = ae[i]; *(LAS u32x4*)(stash + 4096 + i * 1024) = ao[i]; }
            __builtin_amdgcn_sched_barrier(0);
            load_a(ae, 0);
            __builtin_amdgcn_sched_barrier(0);
            load_a(ao, 1);
#pragma unroll
            for (int i = 0; i < 8; ++i) acc[i >> 2][i & 3] = f32x4{0.f, 0.f, 0.f, 0.f};
#pragma unroll 1
            for (int t = 0; t < 28; t += 2) {
                tile(ae, t, 0x01010101u, 1);
                tile(ao, t + 1, 0x01010101u, 1);
            }
        }
        tile(ae, 28, pass * 0x01010101u, 2);
        tile(ao, 29, pass * 0x01010101u, 2);
        tile(ae, 30, pass * 0x01010101u, 0);
        tile(ao, 31, pass * 0x01010101u, 0);
        const float sc = cnt[72] ? __builtin_nanf("") : scale;
        const u32 vo = pass ? (lane >> 4) * 4 * GN * 4 + (lane & 15) * 16 : OOB;
        const u32 so = (rb * 32 * GN + j * 64) * 4;
#pragma unroll
        for (int m = 0; m < 2; ++m)
#pragma unroll
            for (int r = 0; r < 4; ++r) {
                u32x4 v;
#pragma unroll
                for (int c = 0; c < 4; ++c)
                    v[c] = __builtin_bit_cast(u32, __builtin_elementwise_maximum(acc[m][c][r] * sc, 0.f));
                __builtin_amdgcn_raw_buffer_store_b128(v, ro, vo, so + (m * 16 + r) * GN * 4, 17);
                __builtin_amdgcn_sched_barrier(0);
                asm volatile("s_nop 1");
                __builtin_amdgcn_sched_barrier(0);
            }
    }
    __builtin_amdgcn_endpgm();
}

__device__ __forceinline__ void unpack_waves(const Params& P, int b, int wave, int lane, GAS u32* cp, LAS char* lds, u32 cv) {
    char* const ws = P.self;
    const int rank = P.rank;
    LAS u32* const cnt = (LAS u32*)(lds + L_CNT);
    if (wave == 8) { cnt[lane] = 0; cnt[64 + lane] = 0; }
    bar();
    const u32 call = __builtin_amdgcn_readfirstlane(cv) + 1;
    const int q = wave & 3, tl = (wave >> 2) & 1;
    const int j = b & 63;
    GAS u32* const tmo = (GAS u32*)(ws + OFF_TMO);
    const __amdgpu_buffer_rsrc_t rs = __builtin_amdgcn_make_buffer_rsrc(ws, 0, WS_NEED, 0x00020000);
    LAS char* const sym = lds + L_SYM + (wave & 7) * 2304;
    const int ux = 2 * b + tl, xsrc = ux & 7, xu = ux >> 3;
    constexpr int NJOB = 17;
    u32 juo, jfo;
    {
        const int i = lane < NJOB ? lane : 0, t = i ? 2 * (i - 1) + tl : 0;
        juo = i ? w_slot(src_of(rank, j, t), t & 3, j, call) : x_slot(xsrc, xu, call);
        jfo = lane < NJOB ? juo + G_FLAG + q * 8 : OFF_TMO;
    }
    const u32 img = q * 2048 + q_img(lane);
    LAS char* image = lds + L_XIM + tl * 8192;
    if (tl) __builtin_amdgcn_s_setprio(2);
    else __builtin_amdgcn_s_setprio(1);
    Packed pk;
    u64 look = 0;
    bool have = false;
    u32 fre = 8;
#pragma unroll 1
    for (int i = 0; i < NJOB; ++i) {
        const int t = 2 * (i - 1) + tl;
        if (__builtin_amdgcn_readfirstlane(fre) < 8) lds_wait(cnt + 32 + t - NBUF, 8);
        asm volatile("" : "+v"(look));
        if (!have) {
            u32 uo = __builtin_amdgcn_readlane(juo, i);
            const char* nf = ws + __builtin_amdgcn_readlane(jfo, i + 1);
            asm volatile("" : "+s"(uo), "+s"(nf));
            u32 meta = __builtin_amdgcn_readfirstlane((u32)look);
            if (__builtin_amdgcn_readfirstlane((u32)(look >> 32)) < call) {
                const char* f = ws + __builtin_amdgcn_readlane(jfo, i);
                if (i < NJOB - 2) meta = __builtin_amdgcn_readfirstlane((u32)wait_flags<8>(f, 1, call, tmo, cnt + 72));
                else meta = wait_flag(f, call, tmo, cnt + 72);
            }
            asm volatile("" ::: "memory");
            load_quarter(pk, ws, rs, uo, meta, q, lane);
            look = __hip_atomic_load((GAS u64*)nf, __ATOMIC_RELAXED, __HIP_MEMORY_SCOPE_SYSTEM);
        }
        decode_quarter(pk, rs, img, q, image, sym, lane, [&]() {
            if (t + 2 >= NBUF) fre = __hip_atomic_load(cnt + 32 + t + 2 - NBUF, __ATOMIC_RELAXED, __HIP_MEMORY_SCOPE_WORKGROUP);
            asm volatile("" : "+v"(look));
            Packed nx;
            if (i + 1 == NJOB) {
                if ((u32)look == call) cnt[72] = 1;
            } else {
                have = __builtin_amdgcn_readfirstlane((u32)(look >> 32)) >= call;
                asm volatile("" ::: "memory");
                if (have) load_quarter(nx, ws, rs, __builtin_amdgcn_readlane(juo, i + 1), __builtin_amdgcn_readfirstlane((u32)look), q, lane);
                look = __hip_atomic_load((GAS u64*)(ws + __builtin_amdgcn_readlane(jfo, i + 1 + have)), __ATOMIC_RELAXED, __HIP_MEMORY_SCOPE_SYSTEM);
            }
            pk = nx;
        });
        if (i) {
            __hip_atomic_store((LAS unsigned char*)(cnt + t) + q, (unsigned char)1, __ATOMIC_RELEASE, __HIP_MEMORY_SCOPE_WORKGROUP);
        } else {
            lds_order();
            const u32x2 h0 = *(LAS u32x2*)(image + img), h1 = *(LAS u32x2*)(image + img + 256);
            const u32x2 h2 = *(LAS u32x2*)(image + img + 512), h3 = *(LAS u32x2*)(image + img + 768);
            const u32x4 v0 = {h0[0], h0[1], h1[0], h1[1]}, v1 = {h2[0], h2[1], h3[0], h3[1]};
            const char* xd = ws + OFF_XD + (xsrc * 32 + xu) * 8192 + q * 2048;
            st16<0>(xd, lane * 16, v0);
            st16<1024>(xd, lane * 16, v1);
            wait_vm<0>();
            if (__builtin_amdgcn_readfirstlane(lds_add(cnt + 68 + tl, lane)) == 3)
                put_flag(ws + OFF_LF + (xsrc * 32 + xu) * 128, call, 0, lane);
        }
        image = lds + ((t + 2) & (NBUF - 1)) * 8192;
    }
    if (lane == 0) __hip_atomic_store(cp, call, __ATOMIC_RELAXED, __HIP_MEMORY_SCOPE_AGENT);
    __builtin_amdgcn_endpgm();
}

__global__ void __launch_bounds__(1024, 4) gemm_rs_kernel(Params P) {
    __shared__ __attribute__((aligned(16))) char smem[LDS_BYTES];
    const int lane = threadIdx.x & 63;
    const int wave = __builtin_amdgcn_readfirstlane(threadIdx.x >> 6);
    u32x4 xa, xc;
    {
        const u32 ux = 2 * (blockIdx.x - N_SEND_WG) + (wave >> 2), xdest = (ux + (ux >> 3)) & 7, xu = (ux >> 3) & 31;
        const __amdgpu_buffer_rsrc_t rx = __builtin_amdgcn_make_buffer_rsrc((void*)P.x, 0, 4096 * KS, 0x00020000);
        const u32 so = (xdest * MS + (xu >> 1) * 32 + (wave & 1) * 16) * KS + (xu & 1) * 256 + (wave & 2) * 64;
        const u32 vo = ((lane & 15) * KS + (lane >> 4) * 32) | (blockIdx.x >= N_SEND_WG && wave < 8 ? 0 : OOB);
        xa = __builtin_amdgcn_raw_buffer_load_b128(rx, vo, so, 0);
        xc = __builtin_amdgcn_raw_buffer_load_b128(rx, vo + 16, so, 0);
        asm volatile("" ::: "memory");
    }
    GAS u32* cp = (GAS u32*)(P.self + OFF_CNT) + blockIdx.x * 16 + wave;
    const u32 cv = __hip_atomic_load(cp, __ATOMIC_RELAXED, __HIP_MEMORY_SCOPE_AGENT);
    asm volatile("" ::: "memory");
    if (blockIdx.x >= N_SEND_WG) {
        if (wave < 8) mma_waves(P, blockIdx.x - N_SEND_WG, wave, lane, cp, (LAS char*)smem, xa, xc, cv);
        else unpack_waves(P, blockIdx.x - N_SEND_WG, wave, lane, cp, (LAS char*)smem, cv);
    } else {
        if (wave < 8) sender(P, blockIdx.x, wave, lane, cp, (LAS char*)smem, cv);
        else if (wave == 8) anchor_wave(P.self, lane, cp, (LAS char*)smem, cv);
    }
}

extern "C" void kernel_launch(int rank, int nranks, ncclComm_t comm,
                              hipStream_t stream,
                              void* const* d_in, const int* in_sizes, int n_in,
                              void* d_out, int out_size,
                              const int* in_dtypes, int out_dtype,
                              void* const* d_out_all,
                              void* const* d_ws_all, size_t ws_bytes) {
    if (nranks != NR || n_in != 4 || ws_bytes < WS_NEED || (unsigned)rank >= (unsigned)NR ||
        in_sizes[0] != 4096 * KS || in_sizes[1] != KS * GN || out_size != MS * GN) {
        fprintf(stderr, "kernel_launch: unexpected problem shape\n");
        return;
    }
    Params P;
    P.x = (const uint8_t*)d_in[0];
    P.w = (const uint8_t*)d_in[1];
    P.sx = (const float*)d_in[2];
    P.sw = (const float*)d_in[3];
    P.out = (float*)d_out;
    for (int p = 0; p < NR; ++p) P.ws[p] = (char*)d_ws_all[p];
    P.self = P.ws[rank];
    P.rank = rank;
    P.pad = 0;
    gemm_rs_kernel<<<GRID, 1024, 0, stream>>>(P);
    HIP_CHECK(hipGetLastError());
}
```
